```python
import numpy as np

kernel_name = "gpu_ag_gemm_m4096_k4096_n16384_f32_silu"


ABSMAX_RTOL = 0.001

COMPUTE_DTYPE = "float32"
EPILOGUE = "silu"
IN_QUANT = None
OUT_QUANT = None
IN_DTYPES = {"x": "float32", "w_mat": "float32"}
WORLD_SIZE = 8

def setup_inputs():

    x = np.random.randn(4096, 4096).astype(np.float32)
    w_mat = (np.random.randn(4096, 16384) / np.sqrt(4096)).astype(np.float32)
    return {"x": x, "w_mat": w_mat}

def _epilogue(y):
    return (y / (1.0 + np.exp(-np.clip(y, -60.0, 60.0)))).astype(np.float32)

def reference(x, w_mat):
    y = (x @ w_mat).astype(np.float32)
    return _epilogue(y)
```

```cpp
#include <hip/hip_runtime.h>
#include <rccl/rccl.h>
#include <cstdio>

#define HIP_CHECK(expr) do { hipError_t _e = (expr); if (_e != hipSuccess) { \
    fprintf(stderr, "HIP error %s at %s:%d\n", hipGetErrorString(_e), __FILE__, __LINE__); \
    return; } } while (0)

namespace {

#define GAS __attribute__((address_space(1)))
#define LAS __attribute__((address_space(3)))
typedef _Float16 h8 __attribute__((ext_vector_type(8)));
typedef _Float16 h2 __attribute__((ext_vector_type(2)));
typedef float f4 __attribute__((ext_vector_type(4)));
typedef unsigned u4 __attribute__((ext_vector_type(4)));
typedef unsigned u2 __attribute__((ext_vector_type(2)));

#define AG_WIN 12
#define AG_SLEEP_AFTER 24
#define AG_SLEEP_POLL 8
#define AG_NCONV 120
#define AG_SIG_EVERY 2
#define AG_SIG_FINE 56
#define AG_OUT_ST 2

constexpr int NR = 8;
constexpr int GM = 4096, GK = 4096, NLOC = 2048, MLOC = 512;
constexpr int BN = 128, BK = 64;
constexpr int NMT = 15, NNT = 16;
constexpr int GPT = 17;
constexpr int GCAP = 18;
constexpr int WG_ROWS = GCAP / 2;
constexpr int NKS = GK / BK;
constexpr int A_STAGE = GCAP * 16 * BK * 2;
constexpr int B_STAGE = BN * BK * 2;
constexpr int STAGE = A_STAGE + B_STAGE;
constexpr int NS = 3;
constexpr int CTRL_OFF = NS * STAGE;
constexpr int LDS_BYTES = CTRL_OFF + 256;
constexpr int NWG = 256, NTHREADS = 512;
constexpr int NCOMM = 16;
constexpr int C_CONS = 0;
constexpr int C_AVAIL = 16;
constexpr int C_BAD = 20;
constexpr int C_LD = 32;
constexpr int SIG_EVERY = AG_SIG_EVERY;
constexpr int SIG_FINE = AG_SIG_FINE;
constexpr int NCONV = AG_NCONV;

constexpr size_t FSTRIDE = 4096 + 256;
constexpr size_t WS_CALL = 0;
constexpr size_t WS_TMO = 49152;
constexpr size_t WS_XPROG = 65536;
constexpr size_t WS_WFLAG = 196608;
constexpr size_t WS_DBG = 524288;
constexpr size_t WS_WH = 1ull << 20;
constexpr size_t WS_XG = 32ull << 20;
constexpr size_t XG_BYTES = 36ull << 20;
constexpr size_t WS_NEED = WS_XG + 2 * XG_BYTES;
static_assert((size_t)NMT * NKS * A_STAGE <= XG_BYTES, "gather buffer");
static_assert(LDS_BYTES <= 160 * 1024, "LDS");

constexpr unsigned long long DEADLINE = 40000000ull;

struct Params {
    const float* x;
    const float* w;
    float* out;
    char* ws[NR];
    int rank;
};

__device__ __forceinline__ unsigned long long rt() { return __builtin_amdgcn_s_memrealtime(); }

__device__ __forceinline__ void st16_sys(void* p, u4 v) {
    asm volatile("global_store_dwordx4 %0, %1, off sc0 sc1\n\ts_nop 1" :: "v"(p), "v"(v) : "memory");
}
__device__ __forceinline__ void st16_dev(void* p, u4 v) {
    asm volatile("global_store_dwordx4 %0, %1, off sc1\n\ts_nop 1" :: "v"(p), "v"(v) : "memory");
}
__device__ __forceinline__ void st4_sys(void* p, unsigned v) {
    asm volatile("global_store_dword %0, %1, off sc0 sc1\n\ts_nop 1" :: "v"(p), "v"(v) : "memory");
}
__device__ __forceinline__ void lds_st(unsigned addr, unsigned v) {
    asm volatile("ds_write_b32 %0, %1" :: "v"(addr), "v"(v) : "memory");
}
__device__ __forceinline__ unsigned pk(float a, float b) {
    h2 h; h.x = (_Float16)a; h.y = (_Float16)b;
    return __builtin_bit_cast(unsigned, h);
}
__device__ __forceinline__ unsigned umin(unsigned a, unsigned b) { return a < b ? a : b; }
#define VM_WAIT(n) asm volatile("s_waitcnt vmcnt(" #n ")" ::: "memory")
#define SCHED_FENCE() __builtin_amdgcn_sched_barrier(0)

#define STAMP(i) do {} while (0)

__global__ void __launch_bounds__(NTHREADS, 2) ag_gemm_kernel(Params p) {
    extern __shared__ __attribute__((aligned(16))) unsigned char smem[];
    LAS unsigned char* lds = (LAS unsigned char*)smem;
    const int tid = threadIdx.x;
    const int wave = __builtin_amdgcn_readfirstlane(tid >> 6);
    const int lane = tid & 63;
    const int b = blockIdx.x;
    const int xcd = b & 7, idx = b >> 3;
    const bool is_comm = idx < 2;
    char* myws = p.ws[p.rank];

    constexpr int D = 3;
    f4 la[D + 1], lb[D + 1];
    const int cc = xcd * 2 + idx;
    const float* psrc = p.x + (size_t)((cc & 15) * 32 + (wave & 3) * 8 + (lane >> 3)) * GK + (lane & 7) * 8;
#define PLOAD(s_, slot_) do { \
    asm volatile("global_load_dwordx4 %0, %2, off\n\tglobal_load_dwordx4 %1, %2, off offset:16" \
                 : "=&v"(la[slot_]), "=&v"(lb[slot_]) : "v"(psrc + (size_t)(s_) * BK) : "memory"); } while (0)

    GAS unsigned* cctr = (GAS unsigned*)(myws + WS_CALL + (size_t)b * 128);
    unsigned cv;
    if (is_comm && wave < 4) {
        asm volatile("global_load_dwordx4 %0, %8, off\n\tglobal_load_dwordx4 %1, %8, off offset:16\n\t"
                     "global_load_dword %6, %7, off sc1\n\t"
                     "global_load_dwordx4 %2, %8, off offset:256\n\tglobal_load_dwordx4 %3, %8, off offset:272\n\t"
                     "global_load_dwordx4 %4, %8, off offset:512\n\tglobal_load_dwordx4 %5, %8, off offset:528\n\t"
                     "s_waitcnt vmcnt(0)"
                     : "=&v"(la[0]), "=&v"(lb[0]), "=&v"(la[1]), "=&v"(lb[1]), "=&v"(la[2]), "=&v"(lb[2]), "=&v"(cv)
                     : "v"(cctr), "v"(psrc) : "memory");
    } else {
        asm volatile("global_load_dword %0, %1, off sc1\n\ts_waitcnt vmcnt(0)" : "=&v"(cv) : "v"(cctr) : "memory");
    }
    const unsigned call = __builtin_amdgcn_readfirstlane(cv) + 1u;
    const unsigned ctrl = (unsigned)(size_t)(lds + CTRL_OFF);
    if (tid < 64) lds_st(ctrl + tid * 4, 0u);
    asm volatile("s_waitcnt lgkmcnt(0)" ::: "memory");
    __builtin_amdgcn_s_barrier();
    asm volatile("" ::: "memory");
    if (tid == 448) __hip_atomic_store(cctr, call, __ATOMIC_RELAXED, __HIP_MEMORY_SCOPE_AGENT);
    const unsigned pbase = (call - 1u) * (unsigned)NKS;
    const size_t xg_off = WS_XG + (size_t)(call & 1u) * XG_BYTES;
    GAS unsigned* tmo = (GAS unsigned*)(myws + WS_TMO);

    if (is_comm) {
        if (wave < 4) {
            const int c = lane & 7, rr = lane >> 3;
            const int row = cc * 32 + wave * 8 + rr;
            const int g = p.rank * 32 + (row >> 4);
            const int dmt = (g / GPT < NMT - 1) ? g / GPT : NMT - 1;
            const int rit = (g - dmt * GPT) * 16 + (row & 15);
            const size_t doff = xg_off + (size_t)dmt * NKS * A_STAGE + (size_t)rit * 128 + ((c ^ ((rit >> 1) & 7)) * 16);
            char* dst[NR];
#pragma unroll
            for (int j = 0; j < NR; ++j) dst[j] = p.ws[(p.rank + 1 + ((j + cc + 2 * wave) & 7)) & 7] + doff;
            if (wave == 0) STAMP(5);
            constexpr int X = AG_WIN;
            constexpr int DL = (X <= 2) ? 1 : (X - 2 + 9) / 10 + 1;
            static_assert(D == 3 && BK * 4 == 256 && X <= 22 && DL <= D, "window");
            for (int s4 = 0; s4 < NKS; s4 += 4) {
#pragma unroll
                for (int u = 0; u < 4; ++u) {
                    const int s = s4 + u;
                    const int sn = (s + D < NKS) ? s + D : NKS - 1;
                    PLOAD(sn, (u + D) & 3);
                    asm volatile("s_waitcnt vmcnt(%2)" : "+v"(la[u]), "+v"(lb[u]) : "i"(X) : "memory");
                    {
                        const int done = s - DL + 1;
                        lds_st(ctrl + C_CONS + wave * 4, (unsigned)(done > 0 ? done : 0));
                    }
                    u4 o;
                    o.x = pk(la[u][0], la[u][1]); o.y = pk(la[u][2], la[u][3]);
                    o.z = pk(lb[u][0], lb[u][1]); o.w = pk(lb[u][2], lb[u][3]);
                    const size_t so = (size_t)s * A_STAGE;
#pragma unroll
                    for (int j = 0; j < NR; ++j) {
                        asm volatile("s_waitcnt vmcnt(%0)" :: "i"(X) : "memory");
                        st16_sys(dst[j] + so, o);
                    }
                }
            }
            VM_WAIT(20); lds_st(ctrl + C_CONS + wave * 4, NKS - 2);
            VM_WAIT(10); lds_st(ctrl + C_CONS + wave * 4, NKS - 1);
            VM_WAIT(0);  lds_st(ctrl + C_CONS + wave * 4, NKS);
            if (wave == 0) STAMP(6);
        } else if (wave == 4) {
            const int g = p.rank * 32 + cc * 2 + ((lane >> 3) & 1);
            const int dmt = (g / GPT < NMT - 1) ? g / GPT : NMT - 1;
            char* pw = p.ws[lane & 7] + WS_XPROG + (size_t)dmt * FSTRIDE + (size_t)(g - dmt * GPT) * 4;
            unsigned last = 0;
            unsigned long long t0 = 0; unsigned sp = 0;
            while (last < (unsigned)NKS) {
                u4 c;
                asm volatile("ds_read_b128 %0, %1\n\ts_waitcnt lgkmcnt(0)" : "=&v"(c) : "v"(ctrl) : "memory");
                const unsigned p01 = umin(c.x, c.y), p23 = umin(c.z, c.w);
                const unsigned m = __builtin_amdgcn_readfirstlane(umin(p01, p23));
                if (m >= last + (unsigned)SIG_EVERY || (m > last && m >= (unsigned)SIG_FINE)) {
                    if (lane < 16) st4_sys(pw, pbase + (lane < 8 ? p01 : p23));
                    last = m;
                } else {
                    if ((++sp & 1023u) == 0u) { if (t0 == 0) t0 = rt(); else if (rt() - t0 > 2 * DEADLINE) break; }
                    __builtin_amdgcn_s_sleep(1);
                }
            }
        }
        return;
    }

    const int tile = xcd * 30 + (idx - 2);
    const int mt = tile >> 4, nt = tile & 15;
    const int gcnt = (mt == NMT - 1) ? GCAP : GPT;

    if (wave < 4) {
        if (wave == 0) STAMP(0);
        const int wm = wave >> 1, wn = wave & 1;
        const int r16 = lane & 15, q = lane >> 4;
        const unsigned sw = (unsigned)((q ^ (r16 >> 1)) * 16);
        const unsigned lbase = (unsigned)(size_t)lds;
        const unsigned offA = lbase + (unsigned)((wm * WG_ROWS * 16 + r16) * 128) + sw;
        const unsigned offB = lbase + (unsigned)(A_STAGE + (wn * 64 + r16) * 128) + sw;
        const unsigned ldw = ctrl + C_LD;
        f4 acc[WG_ROWS][4];
#pragma unroll
        for (int i = 0; i < WG_ROWS; ++i)
#pragma unroll
            for (int j = 0; j < 4; ++j) acc[i][j] = (f4){0.f, 0.f, 0.f, 0.f};
        h8 a[WG_ROWS], b0[4], b1[4];
#define DSR(dst_, addr_, off_) asm volatile("ds_read_b128 %0, %1 offset:%2" : "=v"(dst_) : "v"(addr_), "i"(off_) : "memory")
#define WAIT_LD(need) do { unsigned long long _t0 = 0; unsigned _sp = 0; \
        for (;;) { u2 _f; asm volatile("ds_read_b64 %0, %1\n\ts_waitcnt lgkmcnt(0)" : "=&v"(_f) : "v"(ldw) : "memory"); \
            if ((unsigned)__builtin_amdgcn_readfirstlane(umin(_f.x, _f.y)) >= (unsigned)(need)) break; \
            if ((++_sp & 1023u) == 0u) { if (_t0 == 0) _t0 = rt(); else if (rt() - _t0 > 2 * DEADLINE) break; } \
            __builtin_amdgcn_s_sleep(1); } \
        asm volatile("s_waitcnt lgkmcnt(0)" ::: "memory"); } while (0)
#define ROW(i_, BC_, na_, cnt_) do { \
        if ((i_) == 0) asm volatile("s_waitcnt lgkmcnt(%5)" : "+v"(a[i_]), "+v"(BC_[0]), "+v"(BC_[1]), "+v"(BC_[2]), "+v"(BC_[3]) : "i"(cnt_) : "memory"); \
        else asm volatile("s_waitcnt lgkmcnt(%1)" : "+v"(a[i_]) : "i"(cnt_) : "memory"); \
        SCHED_FENCE(); \
        _Pragma("unroll") for (int j = 0; j < 4; ++j) \
            acc[i_][j] = __builtin_amdgcn_mfma_f32_16x16x32_f16(BC_[j], a[i_], acc[i_][j], 0, 0, 0); \
        SCHED_FENCE(); \
        DSR(a[i_], na_, (i_) * 2048); } while (0)
        WAIT_LD(1);
        if (wave == 0) STAMP(2);
#pragma unroll
        for (int j = 0; j < 4; ++j) DSR(b0[j], offB, j * 2048);
#pragma unroll
        for (int i = 0; i < WG_ROWS; ++i) DSR(a[i], offA, i * 2048);
        asm volatile("s_waitcnt lgkmcnt(0)" ::: "memory");
        unsigned sb = 0;
        for (int t = 0; t < NKS; ++t) {
            {
                const unsigned na = (offA + sb) ^ 64u, nb = (offB + sb) ^ 64u;
                u2 f;
#pragma unroll
                for (int j = 0; j < 4; ++j) DSR(b1[j], nb, j * 2048);
                asm volatile("ds_read_b64 %0, %1" : "=v"(f) : "v"(ldw) : "memory");
                ROW(0, b0, na, 13); ROW(1, b0, na, 13); ROW(2, b0, na, 13); ROW(3, b0, na, 13); ROW(4, b0, na, 13);
                ROW(5, b0, na, 13); ROW(6, b0, na, 13); ROW(7, b0, na, 13); ROW(8, b0, na, 13);
                asm volatile("s_waitcnt lgkmcnt(9)" : "+v"(f) :: "memory");
                const unsigned fl = __builtin_amdgcn_readfirstlane(umin(f.x, f.y));
                const unsigned need = (t + 2 <= NKS) ? (unsigned)(t + 2) : (unsigned)NKS;
                if (fl < need) WAIT_LD(need);
            }
            sb = (sb == (unsigned)(2 * STAGE)) ? 0u : sb + (unsigned)STAGE;
            {
                const unsigned na = offA + sb, nb = offB + sb;
#pragma unroll
                for (int j = 0; j < 4; ++j) DSR(b0[j], nb, j * 2048);
                ROW(0, b1, na, 12); ROW(1, b1, na, 12); ROW(2, b1, na, 12);
                asm volatile("s_waitcnt lgkmcnt(7)" ::: "memory");
                lds_st(ctrl + C_CONS + wave * 4, (unsigned)(t + 1));
                ROW(3, b1, na, 12); ROW(4, b1, na, 12);
                ROW(5, b1, na, 12); ROW(6, b1, na, 12); ROW(7, b1, na, 12); ROW(8, b1, na, 12);
            }
        }
        asm volatile("s_waitcnt lgkmcnt(0)" ::: "memory");
        if (wave == 0) STAMP(3);
        unsigned bad;
        asm volatile("ds_read_b32 %0, %1 offset:%2\n\ts_waitcnt lgkmcnt(0)" : "=&v"(bad) : "v"(ctrl), "i"(C_BAD) : "memory");
        float* orow = p.out + (size_t)((mt * GPT + wm * WG_ROWS) * 16 + r16) * NLOC + nt * BN + wn * 64 + q * 4;
#pragma unroll
        for (int i = 0; i < WG_ROWS; ++i) {
            if (wm * WG_ROWS + i < gcnt) {
#pragma unroll
                for (int j = 0; j < 4; ++j) {
                    f4 y = acc[i][j], o;
#pragma unroll
                    for (int e = 0; e < 4; ++e) {
                        const float z = __builtin_amdgcn_fmed3f(y[e], -60.f, 60.f);
                        const float ex = __builtin_amdgcn_exp2f(z * -1.4426950408889634f);
                        o[e] = y[e] * __builtin_amdgcn_rcpf(1.0f + ex);
                    }
                    if (bad) o = (f4){__builtin_nanf(""), __builtin_nanf(""), __builtin_nanf(""), __builtin_nanf("")};
                    float* op = orow + (size_t)i * 16 * NLOC + j * 16;
                    st16_dev(op, __builtin_bit_cast(u4, o));
                }
            }
        }
        if (wave == 0) STAMP(4);
    } else if (wave < 6) {
        const int lw = wave - 4;
        const char* xsrc = myws + xg_off + (size_t)mt * NKS * A_STAGE + lane * 16;
        const char* wsrc = myws + WS_WH + (size_t)nt * NKS * B_STAGE + lane * 16;
        unsigned sb = 0;
        unsigned long long t0 = 0;
        for (int t = 0; t < NKS; ++t) {
            bool drained = false;
            unsigned sp = 0;
            for (;;) {
                u4 c; unsigned av;
                asm volatile("ds_read_b128 %0, %2\n\tds_read_b32 %1, %2 offset:16\n\ts_waitcnt lgkmcnt(0)"
                             : "=&v"(c), "=&v"(av) : "v"(ctrl) : "memory");
                const unsigned m0 = umin(umin(c.x, c.y), umin(c.z, c.w));
                const bool ok = (av > (unsigned)t) && ((int)m0 + NS > t);
                if (__builtin_amdgcn_readfirstlane(ok ? 1 : 0)) break;
                if (!drained) {
                    VM_WAIT(0);
                    lds_st(ctrl + C_LD + lw * 4, (unsigned)t);
                    drained = true;
                }
                if ((++sp & 1023u) == 0u) { if (t0 == 0) t0 = rt(); else if (rt() - t0 > 2 * DEADLINE) break; }
                __builtin_amdgcn_s_sleep(1);
            }
            const char* xa = xsrc + (size_t)t * A_STAGE;
            if (lw == 0) {
#pragma unroll
                for (int i = 0; i < 26; ++i)
                    __builtin_amdgcn_global_load_lds((const GAS unsigned*)(xa + i * 1024),
                                                     (LAS unsigned*)(lds + sb + i * 1024), 16, 0, 0);
            } else {
                const char* wa = wsrc + (size_t)t * B_STAGE;
#pragma unroll
                for (int i = 26; i < 36; ++i)
                    __builtin_amdgcn_global_load_lds((const GAS unsigned*)(xa + i * 1024),
                                                     (LAS unsigned*)(lds + sb + i * 1024), 16, 0, 0);
#pragma unroll
                for (int i = 0; i < 16; ++i)
                    __builtin_amdgcn_global_load_lds((const GAS unsigned*)(wa + i * 1024),
                                                     (LAS unsigned*)(lds + sb + A_STAGE + i * 1024), 16, 0, 0);
            }
            VM_WAIT(26);
            lds_st(ctrl + C_LD + lw * 4, (unsigned)t);
            sb = (sb == (unsigned)(2 * STAGE)) ? 0u : sb + (unsigned)STAGE;
        }
        VM_WAIT(0);
        lds_st(ctrl + C_LD + lw * 4, (unsigned)NKS);
    } else if (wave == 6) {
        __builtin_amdgcn_fence(__ATOMIC_ACQUIRE, "");
        VM_WAIT(0);
        GAS unsigned* xp = (GAS unsigned*)(myws + WS_XPROG + (size_t)mt * FSTRIDE) + ((lane & 31) < gcnt ? (lane & 31) : 0);
        GAS unsigned* wf = (GAS unsigned*)(myws + WS_WFLAG + (size_t)nt * FSTRIDE) + lane;
        unsigned pub = 0, wav = 0, npoll = 0;
        const unsigned long long t0 = rt();
        bool dead = false;
        while (pub < (unsigned)NKS) {
            unsigned xv = __hip_atomic_load(xp, __ATOMIC_RELAXED, __HIP_MEMORY_SCOPE_SYSTEM);
            if (wav < (unsigned)NKS) {
                const unsigned wv = __hip_atomic_load(wf, __ATOMIC_RELAXED, __HIP_MEMORY_SCOPE_AGENT);
                const unsigned long long notready = __ballot((int)(wv - call) < 0);
                wav = notready ? (unsigned)__builtin_ctzll(notready) : 64u;
            }
            int d = (int)(xv - pbase);
            d = d < 0 ? 0 : (d > NKS ? NKS : d);
#pragma unroll
            for (int o = 16; o >= 1; o >>= 1) { const int e = __shfl_xor(d, o); d = e < d ? e : d; }
            unsigned av = (unsigned)__builtin_amdgcn_readfirstlane(d);
            av = umin(av, wav);
            if (av > pub) {
                if (av == (unsigned)NKS) break;
                lds_st(ctrl + C_AVAIL, av);
                pub = av;
                if (av < (unsigned)SIG_FINE) __builtin_amdgcn_s_sleep(AG_SLEEP_AFTER);
            } else {
                if ((++npoll & 31u) == 0u) {
                    if (__builtin_amdgcn_readfirstlane(__hip_atomic_load(tmo, __ATOMIC_RELAXED, __HIP_MEMORY_SCOPE_AGENT)) == call
                        || rt() - t0 > DEADLINE) {
                        __hip_atomic_store(tmo, call, __ATOMIC_RELAXED, __HIP_MEMORY_SCOPE_AGENT);
                        dead = true;
                        break;
                    }
                }
                if (pub < (unsigned)SIG_FINE) __builtin_amdgcn_s_sleep(AG_SLEEP_POLL); else __builtin_amdgcn_s_sleep(2);
            }
        }
        if (dead) lds_st(ctrl + C_BAD, 1u);
        lds_st(ctrl + C_AVAIL, (unsigned)NKS);
        if (!dead && __builtin_amdgcn_readfirstlane(
                __hip_atomic_load(tmo, __ATOMIC_RELAXED, __HIP_MEMORY_SCOPE_AGENT)) == call)
            lds_st(ctrl + C_BAD, 1u);
    } else {
        constexpr int CSTEP = 240 / NCONV;
        const int cq = ((idx - 2) / CSTEP) * 8 + xcd;
        if ((idx - 2) % CSTEP == 0 && cq < NCONV) {
            const int kb = lane & 7, nbl = lane >> 3;
            for (int u = cq; u < NKS * 16; u += NCONV) {
                const int ks = u >> 4, cn = u & 15;
                const float* src = p.w + (size_t)(ks * BK + kb * 8) * NLOC + cn * BN + nbl * 4;
                char* dstw = myws + WS_WH + ((size_t)cn * NKS + ks) * B_STAGE;
                f4 v[4][8];
#pragma unroll
                for (int g = 0; g < 4; ++g)
#pragma unroll
                    for (int r = 0; r < 8; ++r)
                        v[g][r] = __builtin_nontemporal_load((const f4*)(src + (size_t)r * NLOC + g * 32));
#pragma unroll
                for (int g = 0; g < 4; ++g)
#pragma unroll
                    for (int c = 0; c < 4; ++c) {
                        const int n = g * 32 + nbl * 4 + c;
                        u4 o;
                        o.x = pk(v[g][0][c], v[g][1][c]); o.y = pk(v[g][2][c], v[g][3][c]);
                        o.z = pk(v[g][4][c], v[g][5][c]); o.w = pk(v[g][6][c], v[g][7][c]);
                        st16_dev(dstw + n * 128 + ((kb ^ ((n >> 1) & 7)) * 16), o);
                    }
                VM_WAIT(0);
                if (lane == 0)
                    __hip_atomic_store((GAS unsigned*)(myws + WS_WFLAG + (size_t)cn * FSTRIDE + ks * 4), call,
                                       __ATOMIC_RELAXED, __HIP_MEMORY_SCOPE_AGENT);
            }
            STAMP(1);
        }
    }
}

}

extern "C" void kernel_launch(int rank, int nranks, ncclComm_t comm,
                              hipStream_t stream,
                              void* const* d_in, const int* in_sizes, int n_in,
                              void* d_out, int out_size,
                              const int* in_dtypes, int out_dtype,
                              void* const* d_out_all,
                              void* const* d_ws_all, size_t ws_bytes) {
    (void)comm; (void)in_dtypes; (void)out_dtype; (void)d_out_all;
    if (nranks != NR || n_in < 2 || (unsigned)rank >= (unsigned)NR || ws_bytes < WS_NEED ||
        in_sizes[0] != MLOC * GK || in_sizes[1] != GK * NLOC || out_size != GM * NLOC) {
        fprintf(stderr, "ag_gemm: unexpected problem shape\n");
        return;
    }
    static bool attr_set = false;
    if (!attr_set) {
        HIP_CHECK(hipFuncSetAttribute((const void*)ag_gemm_kernel,
                                      hipFuncAttributeMaxDynamicSharedMemorySize, LDS_BYTES));
        attr_set = true;
    }
    Params p;
    p.x = (const float*)d_in[0];
    p.w = (const float*)d_in[1];
    p.out = (float*)d_out;
    for (int r = 0; r < NR; ++r) p.ws[r] = (char*)d_ws_all[r];
    p.rank = rank;
    hipLaunchKernelGGL(ag_gemm_kernel, dim3(NWG), dim3(NTHREADS), LDS_BYTES, stream, p);
    HIP_CHECK(hipGetLastError());
}
```

```cpp
#include <hip/hip_runtime.h>
#include <rccl/rccl.h>
#include <cstdio>

#define HIP_CHECK(expr) do { hipError_t _e = (expr); if (_e != hipSuccess) { \
    fprintf(stderr, "HIP error %s at %s:%d\n", hipGetErrorString(_e), __FILE__, __LINE__); \
    return; } } while (0)

namespace {

#define GAS __attribute__((address_space(1)))
#define LAS __attribute__((address_space(3)))
typedef _Float16 h8 __attribute__((ext_vector_type(8)));
typedef _Float16 h2 __attribute__((ext_vector_type(2)));
typedef float f4 __attribute__((ext_vector_type(4)));
typedef unsigned u4 __attribute__((ext_vector_type(4)));
typedef unsigned u2 __attribute__((ext_vector_type(2)));

#define AG_WIN 7
#define AG_SLEEP_AFTER 24
#define AG_SLEEP_POLL 8
#define AG_SLEEP_FINE 2
#define AG_NCONV 120
#define AG_SIG_FINE 48
#define AG_K2ROOM 3
#define AG_KA 57
#define AG_EIV 1
#define AG_XN 8
#define AG_LA 8
#define AG_PF 0
#define AG_XW 1
#define AG_NSTAT 6
#define AG_DIRECT (NITEMS - 1)

constexpr int NR = 8;
constexpr int GM = 4096, GK = 4096, NLOC = 2048, MLOC = 512;
constexpr int BN = 128, BK = 64;
constexpr int NMT = 15;
constexpr int GPT = 17;
constexpr int GCAP = 18;
constexpr int FR = 9, FA = 5, FB = 4;
constexpr int NKS = GK / BK;
constexpr int KA = AG_KA;
constexpr int NAB = NKS - KA;
constexpr int X_STEP = GCAP * 16 * BK * 2;
constexpr int B_STAGE = BN * BK * 2;
constexpr int X_RING = 3 * X_STEP;
constexpr int XA = 2 * FA * 2048, XB = 2 * FB * 2048;
constexpr int CTRL_OFF = X_RING + 3 * B_STAGE;
constexpr int LDS_BYTES = CTRL_OFF + 256;
constexpr int NWG = 256, NTHREADS = 512;
constexpr int NCOMM = 16;
constexpr int NITEMS = KA + 2 * NAB;
constexpr int KH = 40;
static_assert(KA % 3 == 0 && KH + NAB <= KA && NAB <= 7 && FA * AG_EIV <= NAB, "phases");
__host__ __device__ constexpr int next_slot(int off, int size, int nsize) { return off + size + nsize > X_RING ? 0 : off + size; }
constexpr unsigned long long slot_needs() {
    int off[3 + 2 * NAB] = {0, X_STEP, 2 * X_STEP}, end[3 + 2 * NAB] = {X_STEP, 2 * X_STEP, 3 * X_STEP};
    unsigned long long r = 0;
    for (int i = 0; i < 2 * NAB; ++i) {
        const int sz = i < NAB ? XA : XB;
        off[3 + i] = i ? next_slot(off[2 + i], end[2 + i] - off[2 + i], sz) : 0;
        end[3 + i] = off[3 + i] + sz;
        int n = 0;
        for (int j = 0; j < 3 + i; ++j) if (off[j] < end[3 + i] && off[3 + i] < end[j]) n = j;
        r |= (unsigned long long)n << (4 * i);
    }
    return r;
}
constexpr unsigned long long NEEDS = slot_needs();
constexpr unsigned long long ROT = 0x80605040302010ull;
constexpr int C_CONS = 0;
constexpr int C_AVAIL = 16;
constexpr int C_WAV = 20;
constexpr int C_BAD = 24;
constexpr int C_CALL = 28;
constexpr int C_LD = 32;
constexpr int C_HEAD = 16;
constexpr int C_LAND = 24;
constexpr int S_OFF = 4096;
constexpr int Q_OFF = 0;
constexpr int T_OFF = 256;
constexpr int NUNITS = NKS * 16;
constexpr int NSTAT = AG_NSTAT;
constexpr int SIG_FINE = AG_SIG_FINE;
constexpr int NCONV = AG_NCONV;
#define AG_EPOCHS 2, 4, 8, 12, 16, 20, 24, 28, 32, 36, 40, 42, 44, 46, 48, 49, 50, 51, 52, 53, 54, 55, 56, 57, \
                  58, 59, 60, 61, 62, 63, 64, 65, 66, 67, 68, 69, 70, 71
constexpr int EB[] = {AG_EPOCHS};
constexpr int NE = sizeof(EB) / sizeof(EB[0]);
constexpr int count_le(int v) { int n = 0; for (int i = 0; i < NE; ++i) n += EB[i] <= v; return n; }
constexpr int KD = count_le(AG_DIRECT);
static_assert(NE <= 64 && EB[NE - 1] == NITEMS && KD < NE && NITEMS < 255, "epochs");
constexpr float LOG2E = 1.4426950408889634f;

constexpr size_t FSTRIDE = 4096 + 256;
constexpr size_t WS_CALL = 0;
constexpr size_t WS_TMO = 49152;
constexpr size_t WS_CLAIM = 53248;
constexpr size_t WS_ECNT = 57344;
constexpr size_t WS_XPROG = 65536;
constexpr size_t WS_WFLAG = 196608;
constexpr size_t WS_DBG = 524288;
constexpr size_t WS_WH = 1ull << 20;
constexpr size_t WS_XG = 32ull << 20;
constexpr size_t XG_BYTES = 36ull << 20;
constexpr size_t WS_NEED = WS_XG + 2 * XG_BYTES;
static_assert((size_t)NMT * NKS * X_STEP <= XG_BYTES, "gather buffer");
static_assert(LDS_BYTES <= 160 * 1024, "LDS");

constexpr unsigned long long DEADLINE = 100000000ull;

struct Params {
    const float* x;
    const float* w;
    float* out;
    char* myws;
    char* ws[NR];
    int rank;
};

__device__ __forceinline__ unsigned long long rt() { return __builtin_amdgcn_s_memrealtime(); }

__device__ __forceinline__ void st16_sys(void* p, u4 v) {
    asm volatile("global_store_dwordx4 %0, %1, off sc0 sc1\n\ts_nop 1" :: "v"(p), "v"(v) : "memory");
}
__device__ __forceinline__ void st16_dev(void* p, u4 v) {
    asm volatile("global_store_dwordx4 %0, %1, off sc1\n\ts_nop 1" :: "v"(p), "v"(v) : "memory");
}
__device__ __forceinline__ void st4_sys(void* p, unsigned v) {
    asm volatile("global_store_dword %0, %1, off sc0 sc1\n\ts_nop 1" :: "v"(p), "v"(v) : "memory");
}
__device__ __forceinline__ void lds_st(unsigned addr, unsigned v) {
    asm volatile("ds_write_b32 %0, %1" :: "v"(addr), "v"(v) : "memory");
}
__device__ __forceinline__ void lds_st2(unsigned addr, u2 v) {
    asm volatile("ds_write_b64 %0, %1" :: "v"(addr), "v"(v) : "memory");
}
__device__ __forceinline__ unsigned lds_ld(unsigned addr) {
    unsigned v;
    asm volatile("ds_read_b32 %0, %1\n\ts_waitcnt lgkmcnt(0)" : "=&v"(v) : "v"(addr) : "memory");
    return v;
}
__device__ __forceinline__ unsigned ld_ctr(const void* p) {
    unsigned v;
    asm volatile("global_load_dword %0, %1, off sc1\n\ts_waitcnt vmcnt(0)" : "=&v"(v) : "v"(p) : "memory");
    return (unsigned)__builtin_amdgcn_readfirstlane(v);
}
__device__ __forceinline__ unsigned pk(float a, float b) {
    h2 h; h.x = (_Float16)a; h.y = (_Float16)b;
    return __builtin_bit_cast(unsigned, h);
}
__device__ __forceinline__ unsigned umin(unsigned a, unsigned b) { return a < b ? a : b; }
__device__ __forceinline__ unsigned rfl(unsigned v) { return (unsigned)__builtin_amdgcn_readfirstlane((int)v); }
__device__ __forceinline__ unsigned min4(u4 c) { return rfl(umin(umin(c.x, c.y), umin(c.z, c.w))); }
__device__ __forceinline__ unsigned epoch_items(int k) {
    unsigned e = (unsigned)NITEMS;
#pragma unroll
    for (int i = 0; i < NE; ++i) if (k == i) e = EB[i];
    return e;
}
__device__ __forceinline__ int vm_count() {
    const unsigned st = __builtin_amdgcn_s_getreg(7 | (31 << 11));
    return (int)((st & 15u) | ((st >> 18) & 48u));
}
#define VM_WAIT(n) asm volatile("s_waitcnt vmcnt(" #n ")" ::: "memory")
#define LGKM0() asm volatile("s_waitcnt lgkmcnt(0)" ::: "memory")
#define SCHED_FENCE() __builtin_amdgcn_sched_barrier(0)
#define SPIN_OUT(sp_, t0_) ((((++(sp_)) & 1023u) == 0u) && ((t0_) == 0 ? ((t0_) = rt(), false) : (rt() - (t0_) > 2 * DEADLINE)))

#define TRACE(k_, t_) do {} while (0)
#define STAMP(i) do {} while (0)

#define DSR(dst_, addr_, off_) asm volatile("ds_read_b128 %0, %1 offset:%2" : "+v"(dst_) : "v"(addr_), "i"(off_) : "memory")
#define WAIT_LD(need) do { unsigned long long _t0 = 0; unsigned _sp = 0; \
        for (;;) { u4 _f; asm volatile("ds_read_b128 %0, %1\n\ts_waitcnt lgkmcnt(0)" : "=&v"(_f) : "v"(ldw) : "memory"); \
            if (min4(_f) >= (unsigned)(need)) break; \
            if (SPIN_OUT(_sp, _t0)) break; \
            __builtin_amdgcn_s_sleep(1); } \
        LGKM0(); } while (0)

struct Frag { h8 a[FA]; h8 b[4]; };
__device__ __forceinline__ void wait_set(Frag& s) {
    asm volatile("s_waitcnt lgkmcnt(0)" : "+v"(s.a[0]), "+v"(s.a[1]), "+v"(s.a[2]), "+v"(s.a[3]), "+v"(s.a[4]),
                 "+v"(s.b[0]), "+v"(s.b[1]), "+v"(s.b[2]), "+v"(s.b[3]) :: "memory");
}
__device__ __forceinline__ void fetch(int g, Frag& s, unsigned pa, unsigned pb) {
    if (g == 0) DSR(s.b[0], pb, 0);
    if (g == 1) DSR(s.a[0], pa, 0);
    if (g == 2) DSR(s.b[1], pb, 2048);
    if (g == 3) DSR(s.b[2], pb, 2 * 2048);
    if (g == 4) DSR(s.b[3], pb, 3 * 2048);
    if (g == 5) DSR(s.a[1], pa, 2048);
    if (g == 6) DSR(s.a[2], pa, 2 * 2048);
    if (g == 7) DSR(s.a[3], pa, 3 * 2048);
    if (g == 8) DSR(s.a[4], pa, 4 * 2048);
}
struct Pos { unsigned pa, pb, qa, qb, rv, fl; int xo, ws; };
template <int F, int R0>
__device__ __forceinline__ void part_steps(f4 (&acc)[FR][4], Frag& s0, Frag& s1, Pos& P, unsigned tb, unsigned te,
                                           unsigned ldw, unsigned relw, int wm, char* trc) {
    for (unsigned t = tb; t < te; ++t) {
        const bool sw = t == (unsigned)(NKS - 1);
        const int xn = next_slot(P.xo, 2 * F * 2048, sw ? XB : 2 * F * 2048);
        const int da = xn - P.xo - (sw ? wm * (FA - FB) * 2048 : 0);
        const int db = P.ws == 2 ? -2 * B_STAGE : B_STAGE;
        P.xo = xn; P.ws = P.ws == 2 ? 0 : P.ws + 1;
        u4 f;
        if (P.fl <= t) {
            WAIT_LD(t + 1u);
#pragma unroll
            for (int g = 0; g < F + 4; ++g) fetch(g, s0, P.pa, P.pb);
            wait_set(s0);
            SCHED_FENCE();
        }
        if (threadIdx.x < 64) TRACE(1, t);
#pragma unroll
        for (int g = 0; g < 4 * F; ++g) {
            if (g == 4 * F - 1) { asm volatile("" : "+v"(f)); wait_set(s1); asm volatile("" : "+v"(f)); SCHED_FENCE(); }
            f4& c = acc[R0 + (g >> 2)][g & 3];
            c = __builtin_amdgcn_mfma_f32_16x16x32_f16(s0.b[g & 3], s0.a[g >> 2], c, 0, 0, 0);
            SCHED_FENCE();
            if (g < F + 4) fetch(g, s1, P.qa, P.qb);
            if (g == F + 4) { P.pa += da; asm volatile("" : "+v"(P.pa)); }
            if (g == F + 5) { P.pb += db; asm volatile("" : "+v"(P.pb)); }
            if (g == 4 * F - 6) asm volatile("ds_read_b128 %0, %1" : "=v"(f) : "v"(ldw) : "memory");
            SCHED_FENCE();
        }
        lds_st(relw, P.rv);
        const unsigned fl = umin(umin(f.x, f.y), umin(f.z, f.w));
        SCHED_FENCE();
#pragma unroll
        for (int g = 0; g < 4 * F; ++g) {
            if (g == 4 * F - 1) { wait_set(s0); SCHED_FENCE(); }
            f4& c = acc[R0 + (g >> 2)][g & 3];
            c = __builtin_amdgcn_mfma_f32_16x16x32_f16(s1.b[g & 3], s1.a[g >> 2], c, 0, 0, 0);
            SCHED_FENCE();
            if (g < F + 4) fetch(g, s0, P.pa, P.pb);
            if (g == F + 4) { P.qa += da; asm volatile("" : "+v"(P.qa)); }
            if (g == F + 5) { P.qb += db; asm volatile("" : "+v"(P.qb)); }
            if (g == F + 6) { P.rv += 1u; asm volatile("" : "+v"(P.rv)); }
            if (g == F + 7) { P.fl = rfl(fl); asm volatile("" : "+s"(P.fl)); }
            SCHED_FENCE();
        }
    }
}

__global__ void __launch_bounds__(NTHREADS, 2) ag_gemm_kernel(Params p) {
    extern __shared__ __attribute__((aligned(16))) unsigned char smem[];
    LAS unsigned char* lds = (LAS unsigned char*)smem;
    const int tid = threadIdx.x;
    const int wave = __builtin_amdgcn_readfirstlane(tid >> 6);
    const int lane = tid & 63;
    const int b = blockIdx.x;
    const int xcd = b & 7, idx = b >> 3;
    char* myws = p.myws;
    GAS unsigned* cctr = (GAS unsigned*)(myws + WS_CALL + (size_t)b * 128);
    const unsigned lbase = (unsigned)(size_t)lds;
    const unsigned ctrl = lbase + CTRL_OFF;

    if (idx < 2) {
        const int cc = xcd * 2 + idx;
        constexpr unsigned ENDD = 0xffffffffu;
        f4 la, lb;
        unsigned cv;
        if (wave < 4) {
            STAMP(20 + wave);
            const float* s0 = p.x + (size_t)(cc * 32 + wave * 8 + (lane >> 3)) * GK + (lane & 7) * 8;
            asm volatile("global_load_dwordx4 %0, %4, off\n\tglobal_load_dwordx4 %1, %4, off offset:16\n\t"
                         "global_load_dword %2, %3, off sc1"
                         : "=&v"(la), "=&v"(lb), "=&v"(cv) : "v"(cctr), "v"(s0) : "memory");
        }
        unsigned gi, ord, nA;
        {
            const unsigned l5 = (unsigned)lane & 31u, g = (unsigned)p.rank * 32u + l5;
            unsigned t = g / GPT; t = t < (unsigned)(NMT - 1) ? t : (unsigned)(NMT - 1);
            const unsigned gc = (t == NMT - 1) ? GCAP : GPT;
            unsigned sl = g - t * GPT + (unsigned)((ROT >> (4u * t)) & 15u);
            sl = sl >= gc ? sl - gc : sl;
            const unsigned mA = (unsigned)__ballot(sl < 2u * FA);
            const unsigned below = (1u << l5) - 1u;
            nA = (unsigned)__builtin_popcount(mA);
            ord = (sl < 2u * FA) ? (unsigned)__builtin_popcount(mA & below) : nA + (unsigned)__builtin_popcount(~mA & below);
            gi = l5 | sl << 11 | t << 16;
        }
        unsigned wlo = 0, whi = 0;
#pragma unroll
        for (int r = 0; r < NR; ++r) {
            const unsigned long long a = (unsigned long long)p.ws[r];
            if ((lane & 7) == r) { wlo = (unsigned)a; whi = (unsigned)(a >> 32); }
        }
        if (wave < 4) {
            const int h = wave >> 1;
            const int c = lane & 7, r16 = (wave & 1) * 8 + (lane >> 3);
            unsigned dq = (unsigned)__builtin_amdgcn_readlane((int)gi, cc * 2 + h);
            const size_t loff = WS_XG + (size_t)r16 * 128 + ((c ^ (r16 >> 1)) * 16);
            char* dst[NR];
#pragma unroll
            for (int j = 0; j < NR; ++j) {
                const int r = (p.rank + 1 + j + cc + 2 * wave) & 7;
                dst[j] = (char*)(((unsigned long long)(unsigned)__builtin_amdgcn_readlane((int)whi, r) << 32) |
                                 (unsigned)__builtin_amdgcn_readlane((int)wlo, r)) + loff;
            }
            const unsigned lsrc = lbase + S_OFF + (unsigned)((wave * 8 + (lane >> 3)) * 256 + c * 32);
            asm volatile("s_barrier\n\ts_waitcnt vmcnt(0)" : "+v"(la), "+v"(lb), "+v"(cv), "+v"(dst[0]), "+v"(dst[1]), "+v"(dst[2]),
                         "+v"(dst[3]), "+v"(dst[4]), "+v"(dst[5]), "+v"(dst[6]), "+v"(dst[7]) :: "memory");
            const unsigned xpar = ((rfl(cv) + 1u) & 1u) * (unsigned)XG_BYTES;
            if (wave == 0) STAMP(5);
            constexpr int X = AG_WIN;
            static_assert(X >= 2 && X <= 7 && AG_LA <= 14 && NSTAT >= 3 && NSTAT <= 14 && NSTAT <= KA, "window");
            unsigned long long t0 = 0;
            unsigned e = 0;
            for (;; ++e) {
                u4 o;
                o.x = pk(la[0], la[1]); o.y = pk(la[2], la[3]);
                o.z = pk(lb[0], lb[1]); o.w = pk(lb[2], lb[3]);
                const unsigned dmt = umin((dq >> 16) & 15u, NMT - 1), dsl = umin((dq >> 11) & 31u, GCAP - 1);
                const unsigned so = xpar + (dmt * NKS + ((dq >> 5) & 63u)) * (unsigned)X_STEP + dsl * 2048u;
                const unsigned qa = lbase + Q_OFF + ((e + 1u) & 15u) * 16u + h * 4, sa = lsrc + ((e + 1u) & 15u) * 8192u;
                unsigned ld, qv;
#define NEXT_ROWS "ds_read_b32 %0, %4 offset:%7\n\tds_read_b32 %1, %5\n\tds_read_b128 %2, %6\n\tds_read_b128 %3, %6 offset:16\n\t"
#pragma unroll
                for (int j = 0; j < NR; ++j) {
                    asm volatile("s_waitcnt vmcnt(%0)" :: "i"(X) : "memory");
                    if (j == X % 8 && e >= (unsigned)(X / 8)) lds_st(ctrl + C_CONS + wave * 4, e - X / 8);
                    if (j < NR - 1) st16_sys(dst[j] + so, o);
                }
                asm volatile(NEXT_ROWS "global_store_dwordx4 %8, %9, off sc0 sc1\n\ts_waitcnt lgkmcnt(0)"
                             : "=&v"(ld), "=&v"(qv), "=&v"(la), "=&v"(lb)
                             : "v"(ctrl + h * 4), "v"(qa), "v"(sa), "i"(C_LAND), "v"(dst[NR - 1] + so), "v"(o) : "memory");
                for (unsigned sp = 0;;) {
                    dq = rfl(qv);
                    if (rfl(ld) > e + 1u) break;
                    if (SPIN_OUT(sp, t0)) { dq = ENDD; break; }
                    if (vm_count() == 0) { VM_WAIT(0); lds_st(ctrl + C_CONS + wave * 4, e + 1u); }
                    __builtin_amdgcn_s_sleep(1);
                    asm volatile(NEXT_ROWS "s_waitcnt lgkmcnt(0)" : "=&v"(ld), "=&v"(qv), "=&v"(la), "=&v"(lb)
                                 : "v"(ctrl + h * 4), "v"(qa), "v"(sa), "i"(C_LAND) : "memory");
                }
                if (dq == ENDD) break;
            }
            if (X > 8 && vm_count() > 8) { VM_WAIT(8); lds_st(ctrl + C_CONS + wave * 4, e); }
            VM_WAIT(0); lds_st(ctrl + C_CONS + wave * 4, e + 1u);
            STAMP(6 + wave);
            return;
        }
        const unsigned nB = 32u - nA;
        if (wave >= 6) {
            const int h = wave - 6;
            const unsigned vo = (unsigned)((lane >> 4) * GK * 4 + (lane & 15) * 16);
            auto issue = [&](unsigned e, unsigned d) {
#pragma unroll
                for (int i = 0; i < 4; ++i) {
                    const char* src = (const char*)p.x + (size_t)((d & 31u) * 16u + (unsigned)i * 4u) * (GK * 4) + ((d >> 5) & 63u) * (BK * 4);
                    asm volatile("s_mov_b32 m0, %0\n\ts_nop 0\n\tglobal_load_lds_dwordx4 %1, %2"
                                 :: "s"(lbase + S_OFF + (e & 15u) * 8192u + h * 4096 + i * 1024), "v"(vo), "s"(src) : "memory");
                }
            };
            issue(1u, (unsigned)(cc * 2 + h) | 1u << 5);
            issue(2u, (unsigned)(cc * 2 + h) | 2u << 5);
            __builtin_amdgcn_s_barrier();
            unsigned il = 3, pv = 1;
            unsigned long long t0 = 0; unsigned sp = 0;
            for (;;) {
                unsigned hd, d;
                asm volatile("ds_read_b32 %0, %2 offset:%4\n\tds_read_b32 %1, %3\n\ts_waitcnt lgkmcnt(0)"
                             : "=&v"(hd), "=&v"(d) : "v"(ctrl), "v"(lbase + Q_OFF + (il & 15u) * 16u + h * 4), "i"(C_HEAD) : "memory");
                const unsigned fl = (unsigned)(vm_count() + 3) >> 2;
                const bool can = rfl(hd) > il, end = can && rfl(d) == ENDD;
                if (end && fl == 0u) break;
                if (il - fl > pv) {
                    switch (fl) {
                        case 0: VM_WAIT(0); break;
                        case 1: VM_WAIT(4); break;
                        case 2: VM_WAIT(8); break;
                        case 3: VM_WAIT(12); break;
                        case 4: VM_WAIT(16); break;
                        case 5: VM_WAIT(20); break;
                        case 6: VM_WAIT(24); break;
                        case 7: VM_WAIT(28); break;
                        case 8: VM_WAIT(32); break;
                        case 9: VM_WAIT(36); break;
                        case 10: VM_WAIT(40); break;
                        case 11: VM_WAIT(44); break;
                        case 12: VM_WAIT(48); break;
                        default: VM_WAIT(52); break;
                    }
                    pv = il - fl;
                    lds_st(ctrl + C_LAND + h * 4, pv);
                }
                if (can && !end) {
                    issue(il, rfl(d));
                    ++il;
                } else {
                    if (SPIN_OUT(sp, t0)) break;
                    __builtin_amdgcn_s_sleep(1);
                }
            }
            VM_WAIT(0);
            lds_st(ctrl + C_LAND + h * 4, il + 1u);
        } else if (wave == 4) {
            unsigned cw;
            asm volatile("global_load_dword %0, %1, off sc1" : "=&v"(cw) : "v"(cctr) : "memory");
            lds_st(ctrl + lane * 4, lane == C_HEAD / 4 ? (unsigned)NSTAT : 0u);
            {
                const unsigned g0 = (unsigned)__builtin_amdgcn_readlane((int)gi, cc * 2);
                const unsigned g1 = (unsigned)__builtin_amdgcn_readlane((int)gi, cc * 2 + 1);
                const unsigned en = (unsigned)lane >> 2;
                const unsigned q = (lane & 2) ? en * 16u + (unsigned)cc : ((lane & 1) ? g1 : g0) | en << 5;
                if (lane < 4 * NSTAT) lds_st(lbase + Q_OFF + lane * 4, q);
            }
            if (lane < 32) lds_st(lbase + T_OFF + ord * 4u, gi);
            LGKM0();
            __builtin_amdgcn_s_barrier();
            asm volatile("s_waitcnt vmcnt(0)" : "+v"(cw) :: "memory");
            const unsigned call = rfl(cw) + 1u;
            GAS unsigned* cl = (GAS unsigned*)(myws + WS_CLAIM + (size_t)(call & 1u) * 256);
            if (b == 0 && lane == 0)
                __hip_atomic_store((GAS unsigned*)(myws + WS_CLAIM + (size_t)((call + 1u) & 1u) * 256), 0u,
                                   __ATOMIC_RELAXED, __HIP_MEMORY_SCOPE_AGENT);
            const unsigned mgA = (65536u + nA - 1u) / nA, mgB = (65536u + nB - 1u) / nB;
            unsigned head = NSTAT;
            bool end = false;
            unsigned long long t0 = 0; unsigned sp = 0;
            while (!end) {
                u4 c;
                asm volatile("ds_read_b128 %0, %1\n\ts_waitcnt lgkmcnt(0)" : "=&v"(c) : "v"(ctrl) : "memory");
                const int room = AG_LA - (int)(head - min4(c));
                if (room > 0) {
                    const unsigned k = room > AG_K2ROOM ? 2u : 1u;
                    unsigned r = 0;
                    if (lane == 0) r = __hip_atomic_fetch_add(cl, k, __ATOMIC_RELAXED, __HIP_MEMORY_SCOPE_AGENT);
                    const unsigned u0 = rfl(r) + (unsigned)(NSTAT * NCOMM);
                    const unsigned u = u0 + (unsigned)((lane >> 2) & 1);
                    unsigned hu = umin(u, NUNITS - 1) * 2u + (unsigned)(lane & 1);
                    unsigned ks = hu >> 5, j = hu & 31u;
                    if (hu >= 32u * KA) {
                        hu -= 32u * KA;
                        const bool isB = hu >= nA * NAB;
                        hu = isB ? hu - nA * NAB : hu;
                        const unsigned kq = (hu * (isB ? mgB : mgA)) >> 16;
                        j = hu - kq * (isB ? nB : nA) + (isB ? nA : 0u);
                        ks = KA + kq;
                    }
                    unsigned q = lds_ld(lbase + T_OFF + (j & 31u) * 4u) | ks << 5;
                    if (u >= (unsigned)NUNITS) q = ENDD;
                    if (lane & 2) q = umin(u, NUNITS);
                    if ((unsigned)lane < 4u * k) lds_st(lbase + Q_OFF + ((head + (lane >> 2)) & 15u) * 16u + (lane & 3) * 4, q);
                    end = u0 + k > (unsigned)NUNITS;
                    head += k;
                    lds_st(ctrl + C_HEAD, head);
                } else {
                    if (SPIN_OUT(sp, t0)) break;
                    __builtin_amdgcn_s_sleep(2);
                }
            }
        } else if (wave == 5) {
            __builtin_amdgcn_s_barrier();
            const unsigned call = ld_ctr((const void*)cctr) + 1u;
            const unsigned it = epoch_items(lane);
            const unsigned thr = ((it <= (unsigned)KA ? it * 32u : it <= (unsigned)NKS ? KA * 32u + (it - KA) * nA
                                   : KA * 32u + NAB * nA + (it - NKS) * nB) + 1u) >> 1;
            const int mt0 = (p.rank * 32) / GPT;
            const int mt1 = ((p.rank * 32 + 31) / GPT < NMT - 1) ? (p.rank * 32 + 31) / GPT : NMT - 1;
            const int smt = mt0 + (lane >> 3);
            const bool son = smt <= mt1;
            const int slot = ((smt * GPT) / 32 == p.rank) ? 0 : 1;
            char* pw = (char*)(((unsigned long long)whi << 32) | wlo) + WS_XPROG + (size_t)(son ? smt : mt0) * FSTRIDE
                       + (size_t)(slot * 256);
            GAS unsigned* ec = (GAS unsigned*)(myws + WS_ECNT);
            int k = 0, alt = 0;
            unsigned long long t0 = 0; unsigned sp = 0;
            while (k < NE) {
                u4 c; unsigned hd;
                asm volatile("ds_read_b128 %0, %2\n\tds_read_b32 %1, %2 offset:%3\n\ts_waitcnt lgkmcnt(0)"
                             : "=&v"(c), "=&v"(hd) : "v"(ctrl), "i"(C_HEAD) : "memory");
                const unsigned m = min4(c);
                unsigned P = 0;
                if (rfl(hd) > m) P = rfl(lds_ld(lbase + Q_OFF + (m & 15u) * 16u + 8u));
                const unsigned long long rch = __ballot(lane < NE && thr <= P);
                const int k2 = ~rch ? __builtin_ctzll(~rch) : 64;
                if (k2 > k) {
                    if (k2 > KD) {
                        VM_WAIT(1);
                        if (son) st4_sys(pw + 128 + cc * 8 + alt, call * 256u + (unsigned)__builtin_amdgcn_readlane((int)it, k2 - 1));
                        alt ^= 4;
                    }
                    if (k < KD) {
                        unsigned old = 0;
                        const bool mine = lane >= k && lane < k2 && lane < KD;
                        if (mine) old = __hip_atomic_fetch_add(ec + lane, 1u, __ATOMIC_RELAXED, __HIP_MEMORY_SCOPE_AGENT);
                        const unsigned long long lastm = __ballot(mine && old == call * (unsigned)NCOMM - 1u);
                        if (lastm) {
                            const int e = 63 - __builtin_clzll(lastm);
                            if (son) st4_sys(pw + (e & 31) * 4, call * 256u + (unsigned)__builtin_amdgcn_readlane((int)it, e));
                        }
                    }
                    k = k2;
                } else {
                    if (SPIN_OUT(sp, t0)) break;
                    __builtin_amdgcn_s_sleep(1);
                }
            }
            if (lane == 0) __hip_atomic_store(cctr, call, __ATOMIC_RELAXED, __HIP_MEMORY_SCOPE_AGENT);
            STAMP(10);
            STAMP(11);
        }
        return;
    }

    constexpr int XC = 16 / AG_XN;
    const int tj = (xcd / XC) * 30 + (idx - 2);
    const int mt = tj / AG_XN, nt = (xcd % XC) * AG_XN + tj % AG_XN;
    const int gcnt = (mt == NMT - 1) ? GCAP : GPT;
    if (wave == 6) {
        const unsigned cl = ld_ctr((const void*)cctr) + 1u;
        if (lane == 0) __hip_atomic_store(cctr, cl, __ATOMIC_RELAXED, __HIP_MEMORY_SCOPE_AGENT);
        lds_st(ctrl + lane * 4, lane == C_CALL / 4 ? cl : lane == C_LD / 4 + 2 ? (unsigned)(AG_XW == 2 ? KA : NITEMS) : lane == C_LD / 4 + 3 ? (unsigned)KA : 0u);
        LGKM0();
    }
    __builtin_amdgcn_s_barrier();
    asm volatile("" ::: "memory");

    if (wave < 4) {
        if (wave == 0) STAMP(0);
        const int wm = wave >> 1, wn = wave & 1;
        const int r16 = lane & 15, q = lane >> 4;
        const unsigned sw = (unsigned)((q ^ (r16 >> 1)) * 16);
        const unsigned ldw = ctrl + C_LD;
        const unsigned relw = ctrl + C_CONS + wave * 4;
        const int rot = (int)((ROT >> (4 * mt)) & 15u);
        float* ocol = p.out + (size_t)(mt * GPT * 16 + r16) * NLOC + nt * BN + wn * 64 + q * 4;
        auto finish = [&](const f4 (&y4)[4], int slot, bool bad) {
            if (slot >= gcnt) return;
            const int pos = slot - rot < 0 ? slot - rot + gcnt : slot - rot;
            float* orow = ocol + (size_t)pos * 16 * NLOC;
            if (bad) {
#pragma unroll
                for (int j = 0; j < 4; ++j) st16_dev(orow + j * 16, (u4){0x7fc00000u, 0x7fc00000u, 0x7fc00000u, 0x7fc00000u});
                return;
            }
#pragma unroll
            for (int j = 0; j < 4; ++j) {
                const f4 y = y4[j];
                f4 o;
#pragma unroll
                for (int e = 0; e < 4; ++e) {
                    const float z = __builtin_amdgcn_fmed3f(y[e], -60.f * LOG2E, 60.f * LOG2E);
                    const float ex = __builtin_amdgcn_exp2f(-z);
                    o[e] = y[e] * __builtin_amdgcn_rcpf(__builtin_fmaf(ex, LOG2E, LOG2E));
                }
                st16_dev(orow + j * 16, __builtin_bit_cast(u4, o));
            }
        };
        f4 acc[FR][4];
#pragma unroll
        for (int i = 0; i < FR; ++i)
#pragma unroll
            for (int j = 0; j < 4; ++j) acc[i][j] = (f4){0.f, 0.f, 0.f, 0.f};
        {
        const unsigned offA = lbase + (unsigned)((wm * FR * 16 + r16) * 128) + sw;
        const unsigned offB = lbase + (unsigned)(X_RING + (wn * 64 + r16) * 128) + sw;
        h8 a[FR], b0[4], b1[4];
#pragma unroll
        for (int i = 0; i < FR; ++i) a[i] = (h8)(_Float16)0.f;
#pragma unroll
        for (int j = 0; j < 4; ++j) { b0[j] = (h8)(_Float16)0.f; b1[j] = (h8)(_Float16)0.f; }
#define MF(i_, j_, B_, A_) do { acc[i_][j_] = __builtin_amdgcn_mfma_f32_16x16x32_f16(B_, A_, acc[i_][j_], 0, 0, 0); SCHED_FENCE(); } while (0)
#define FILL(x_) do { x_; SCHED_FENCE(); } while (0)
#define ROW(i_, BC_, A_, F0_, F1_, F2_) do { \
        MF(i_, 0, BC_[0], A_); FILL(F0_); MF(i_, 1, BC_[1], A_); FILL(F1_); \
        MF(i_, 2, BC_[2], A_); FILL(F2_); MF(i_, 3, BC_[3], A_); } while (0)
#define NOP_ do {} while (0)
#define WA(i_, cnt_) asm volatile("s_waitcnt lgkmcnt(%1)" : "+v"(a[i_]) : "i"(cnt_) : "memory")
#define WAB(BC_, cnt_) asm volatile("s_waitcnt lgkmcnt(%8)" : "+v"(a[0]), "+v"(a[1]), "+v"(a[2]), "+v"(a[3]), \
                                    "+v"(BC_[0]), "+v"(BC_[1]), "+v"(BC_[2]), "+v"(BC_[3]) : "i"(cnt_) : "memory")
        WAIT_LD(1);
#pragma unroll
        for (int j = 0; j < 4; ++j) DSR(b0[j], offB, j * 2048);
#pragma unroll
        for (int i = 0; i < FR - 1; ++i) DSR(a[i], offA, i * 2048);
        WAB(b0, 0);
        SCHED_FENCE();
        unsigned sb = 0, sc = 0;
        char* trc = myws;
        for (int t = 0; t < KA; ++t) {
            u2 f;
            if (wave == 0) TRACE(1, t);
            {
                const unsigned ca = offA + sb, na = ca ^ 64u, nb = (offB + sc) ^ 64u;
                ROW(0, b0, a[0], DSR(a[8], ca, 8 * 2048), NOP_, NOP_);
                ROW(1, b0, a[1], DSR(a[0], na, 0), DSR(b1[0], nb, 0), NOP_);
                ROW(2, b0, a[2], DSR(a[1], na, 2048), DSR(b1[1], nb, 2048), NOP_);
                ROW(3, b0, a[3], DSR(a[2], na, 2 * 2048), DSR(b1[2], nb, 2 * 2048), WA(4, 10));
                ROW(4, b0, a[4], DSR(a[3], na, 3 * 2048), DSR(b1[3], nb, 3 * 2048), WA(5, 11));
                ROW(5, b0, a[5], DSR(a[4], na, 4 * 2048),
                    asm volatile("ds_read_b64 %0, %1" : "=v"(f) : "v"(ldw) : "memory"), WA(6, 11));
                ROW(6, b0, a[6], DSR(a[5], na, 5 * 2048), NOP_, WA(7, 11));
                ROW(7, b0, a[7], DSR(a[6], na, 6 * 2048), NOP_, WA(8, 11));
                ROW(8, b0, a[8], DSR(a[7], na, 7 * 2048), NOP_, WAB(b1, 4));
            }
            {
                const unsigned ca = (offA + sb) ^ 64u;
                sb = (sb == (unsigned)(2 * X_STEP)) ? 0u : sb + (unsigned)X_STEP;
                sc = (sc == (unsigned)(2 * B_STAGE)) ? 0u : sc + (unsigned)B_STAGE;
                const unsigned na = offA + sb, nb = offB + sc;
                ROW(0, b1, a[0], DSR(a[8], ca, 8 * 2048), NOP_,
                    asm volatile("s_waitcnt lgkmcnt(4)" : "+v"(f) :: "memory"));
                const unsigned fl = rfl(umin(f.x, f.y));
                const unsigned need = (t + 2 <= KA) ? (unsigned)(t + 2) : (unsigned)KA;
                if (fl < need) WAIT_LD(need);
                SCHED_FENCE();
                ROW(1, b1, a[1], DSR(a[0], na, 0), DSR(b0[0], nb, 0), NOP_);
                ROW(2, b1, a[2], DSR(a[1], na, 2048), DSR(b0[1], nb, 2048), NOP_);
                ROW(3, b1, a[3], DSR(a[2], na, 2 * 2048), DSR(b0[2], nb, 2 * 2048),
                    WA(4, 6); lds_st(relw, (unsigned)(t + 1)));
                ROW(4, b1, a[4], DSR(a[3], na, 3 * 2048), DSR(b0[3], nb, 3 * 2048), WA(5, 11));
                ROW(5, b1, a[5], DSR(a[4], na, 4 * 2048), NOP_, WA(6, 11));
                ROW(6, b1, a[6], DSR(a[5], na, 5 * 2048), NOP_, WA(7, 11));
                ROW(7, b1, a[7], DSR(a[6], na, 6 * 2048), NOP_, WA(8, 11));
                ROW(8, b1, a[8], DSR(a[7], na, 7 * 2048), NOP_, WAB(b0, 4));
            }
        }
        WAB(b0, 0);
        asm volatile("" : "+v"(a[4]), "+v"(a[5]), "+v"(a[6]), "+v"(a[7]));
        }
        if (wave == 0) STAMP(2);
        Frag s0, s1;
#pragma unroll
        for (int i = 0; i < FA; ++i) { s0.a[i] = (h8)(_Float16)0.f; s1.a[i] = (h8)(_Float16)0.f; }
#pragma unroll
        for (int j = 0; j < 4; ++j) { s0.b[j] = (h8)(_Float16)0.f; s1.b[j] = (h8)(_Float16)0.f; }
        Pos P;
        P.pa = lbase + (unsigned)((wm * FA * 16 + r16) * 128) + sw;
        P.pb = lbase + (unsigned)(X_RING + (wn * 64 + r16) * 128) + sw;
        P.qa = P.pa ^ 64u; P.qb = P.pb ^ 64u; P.rv = KA + 1; P.fl = 0; P.xo = 0; P.ws = 0;
        part_steps<FA, 0>(acc, s0, s1, P, KA, NKS, ldw, relw, wm, myws);
        if (wave == 0) STAMP(12);
#pragma unroll
        for (int i = 0; i < FA; ++i) {
            finish(acc[i], wm * FA + i, rfl(lds_ld(ctrl + C_BAD)) != 0u);
            SCHED_FENCE();
            part_steps<FB, FA>(acc, s0, s1, P, NKS + i * AG_EIV, i == FA - 1 ? NITEMS : NKS + (i + 1) * AG_EIV, ldw, relw, wm, myws);
        }
        if (wave == 0) STAMP(3);
        {
            const bool bad = rfl(lds_ld(ctrl + C_BAD)) != 0u;
#pragma unroll
            for (int i = 0; i < FB; ++i) finish(acc[FA + i], 2 * FA + wm * FB + i, bad);
        }
        if (wave == 0) STAMP(4);
    } else if (wave == 6) {
        __builtin_amdgcn_fence(__ATOMIC_ACQUIRE, "");
        VM_WAIT(0);
        const unsigned call = rfl(lds_ld(ctrl + C_CALL));
        GAS unsigned* tmo = (GAS unsigned*)(myws + WS_TMO);
        const bool two = (mt * GPT + gcnt - 1) / 32 != (mt * GPT) / 32;
        GAS unsigned long long* xp = (GAS unsigned long long*)(myws + WS_XPROG + (size_t)mt * FSTRIDE) + (lane & (two ? 63 : 31));
        GAS unsigned* wf = (GAS unsigned*)(myws + WS_WFLAG + (size_t)nt * FSTRIDE) + lane;
        unsigned pub = 0, wav = 0, npoll = 0;
        const unsigned long long t0 = rt();
        bool dead = false;
        while (pub < (unsigned)NITEMS || wav < (unsigned)NKS) {
            const unsigned long long xv = __hip_atomic_load(xp, __ATOMIC_RELAXED, __HIP_MEMORY_SCOPE_SYSTEM);
            bool adv = false;
            if (wav < (unsigned)NKS) {
                const unsigned wv = __hip_atomic_load(wf, __ATOMIC_RELAXED, __HIP_MEMORY_SCOPE_AGENT);
                const unsigned long long notready = __ballot((int)(wv - call) < 0);
                const unsigned w2 = notready ? (unsigned)__builtin_ctzll(notready) : 64u;
                if (w2 > wav) { wav = w2; lds_st(ctrl + C_WAV, wav); adv = true; }
            }
            const int xi = max((int)((unsigned)xv - call * 256u), (int)((unsigned)(xv >> 32) - call * 256u));
            int u = xi < 0 ? 0 : xi > NITEMS ? NITEMS : xi;
            u = (lane & 16) ? 255 - u : u;
            u = max(u, __builtin_amdgcn_update_dpp(0, u, 0x121, 15, 15, false));
            u = max(u, __builtin_amdgcn_update_dpp(0, u, 0x122, 15, 15, false));
            u = max(u, __builtin_amdgcn_update_dpp(0, u, 0x124, 15, 15, false));
            u = max(u, __builtin_amdgcn_update_dpp(0, u, 0x128, 15, 15, false));
            const unsigned av = (unsigned)min(max(__builtin_amdgcn_readlane(u, 0), 255 - __builtin_amdgcn_readlane(u, 16)),
                                              max(__builtin_amdgcn_readlane(u, 32), 255 - __builtin_amdgcn_readlane(u, 48)));
            if (av > pub) {
                lds_st(ctrl + C_AVAIL, av);
                pub = av;
                if (av < (unsigned)SIG_FINE && wav >= umin(av, NKS)) __builtin_amdgcn_s_sleep(AG_SLEEP_AFTER);
            } else {
                if (!adv && (++npoll & 31u) == 0u) {
                    if (rfl(__hip_atomic_load(tmo, __ATOMIC_RELAXED, __HIP_MEMORY_SCOPE_AGENT)) == call
                        || rt() - t0 > DEADLINE) {
                        __hip_atomic_store(tmo, call, __ATOMIC_RELAXED, __HIP_MEMORY_SCOPE_AGENT);
                        dead = true;
                        break;
                    }
                }
                if (pub < (unsigned)SIG_FINE) __builtin_amdgcn_s_sleep(AG_SLEEP_POLL); else __builtin_amdgcn_s_sleep(AG_SLEEP_FINE);
            }
        }
        if (dead) lds_st(ctrl + C_BAD, 1u);
        lds_st(ctrl + C_WAV, (unsigned)NKS);
        lds_st(ctrl + C_AVAIL, (unsigned)NITEMS);
    } else {
        const int hh = wave == 7, xh = wave == 5;
        const unsigned call = rfl(lds_ld(ctrl + C_CALL));
        const unsigned lo16 = (unsigned)lane * 16u;
        const char* wsrc = myws + WS_WH + (size_t)nt * NKS * B_STAGE;
        const char* xsrc = myws + WS_XG + (size_t)(call & 1u) * XG_BYTES + (size_t)mt * NKS * X_STEP;
        const bool full = gcnt == GCAP;
        unsigned long long t0 = 0;
        char* trc = myws;
        u4 hw[NAB][8];
#define GLDN(src_, dst_, n_) do { _Pragma("unroll") for (int _i = 0; _i < (n_); _i += 4) { \
            if (_i + 4 <= (n_)) asm volatile("s_mov_b32 m0, %0\n\ts_nop 0\n\t" \
                         "global_load_lds_dwordx4 %1, %2\n\tglobal_load_lds_dwordx4 %1, %2 offset:1024\n\t" \
                         "global_load_lds_dwordx4 %1, %2 offset:2048\n\tglobal_load_lds_dwordx4 %1, %2 offset:3072" \
                         :: "s"(lbase + (dst_) + _i * 1024), "v"(lo16), "s"((src_) + _i * 1024) : "memory"); \
            else asm volatile("s_mov_b32 m0, %0\n\ts_nop 0\n\t" \
                         "global_load_lds_dwordx4 %1, %2\n\tglobal_load_lds_dwordx4 %1, %2 offset:1024" \
                         :: "s"(lbase + (dst_) + _i * 1024), "v"(lo16), "s"((src_) + _i * 1024) : "memory"); } } while (0)
#define LD_READY(t_, word_, nav_, ncons_) do { bool _dr = false; unsigned _sp = 0; \
        for (;;) { u4 _c; unsigned _av; \
            asm volatile("ds_read_b128 %0, %2\n\tds_read_b32 %1, %2 offset:%3\n\ts_waitcnt lgkmcnt(0)" \
                         : "=&v"(_c), "=&v"(_av) : "v"(ctrl), "i"(word_) : "memory"); \
            if (rfl(_av) >= (unsigned)(nav_) && (int)min4(_c) >= (int)(ncons_)) break; \
            if (!_dr) { VM_WAIT(0); lds_st(ldw, (unsigned)(t_)); _dr = true; } \
            if (SPIN_OUT(_sp, t0)) break; \
            __builtin_amdgcn_s_sleep(1); } } while (0)
#define HOLD(j_) asm volatile("global_load_dwordx4 %0, %8, %9\n\tglobal_load_dwordx4 %1, %8, %9 offset:1024\n\t" \
                         "global_load_dwordx4 %2, %8, %9 offset:2048\n\tglobal_load_dwordx4 %3, %8, %9 offset:3072\n\t" \
                         "global_load_dwordx4 %4, %8, %10\n\tglobal_load_dwordx4 %5, %8, %10 offset:1024\n\t" \
                         "global_load_dwordx4 %6, %8, %10 offset:2048\n\tglobal_load_dwordx4 %7, %8, %10 offset:3072\n\t" \
                         "s_waitcnt vmcnt(0)" \
                         : "=&v"(hw[j_][0]), "=&v"(hw[j_][1]), "=&v"(hw[j_][2]), "=&v"(hw[j_][3]), \
                           "=&v"(hw[j_][4]), "=&v"(hw[j_][5]), "=&v"(hw[j_][6]), "=&v"(hw[j_][7]) \
                         : "v"(lo16), "s"(wsrc + (size_t)(KA + (j_)) * B_STAGE + hh * 8192), \
                           "s"(wsrc + (size_t)(KA + (j_)) * B_STAGE + hh * 8192 + 4096) : "memory")
        if (wave == 4) {
            const unsigned ldw = ctrl + C_LD;
            unsigned sb = 0;
            for (int t = 0; t < KA; ++t) {
                LD_READY(t, C_AVAIL, t + 1, t - 2);
                TRACE(0, t);
                const char* xa = xsrc + (size_t)t * X_STEP;
                GLDN(xa, sb, 2 * FA);
                GLDN(xa + 4 * FA * 1024, sb + 2 * FA * 1024, 2 * FB);
                GLDN(xa + 2 * FA * 1024, sb + 2 * FR * 1024, 2 * FA);
                GLDN(xa + (4 * FA + 2 * FB) * 1024, sb + (2 * FR + 2 * FA) * 1024, 2 * FB - 2);
                if (full) GLDN(xa + 34 * 1024, sb + 34 * 1024, 2);
                VM_WAIT(34);
                lds_st(ldw, (unsigned)t);
                sb = (sb == (unsigned)(2 * X_STEP)) ? 0u : sb + (unsigned)X_STEP;
            }
#pragma unroll
            for (int j = 0; j < NAB; ++j)
#pragma unroll
                for (int i = 0; i < 8; ++i) asm volatile("" : "=v"(hw[j][i]));
        } else if (wave == 5) {
            const unsigned ldw = ctrl + C_LD + 4;
            unsigned sc = 0;
#define W_STEP(t_, nav_) LD_READY(t_, C_WAV, nav_, (t_) - 2); TRACE(2, t_); \
                GLDN(wsrc + (size_t)(t_) * B_STAGE, X_RING + sc, 16); \
                sc = (sc == (unsigned)(2 * B_STAGE)) ? 0u : sc + (unsigned)B_STAGE
            for (int t = 0; t < KH; ++t) {
                W_STEP(t, t + 1);
                VM_WAIT(16);
                lds_st(ldw, (unsigned)t);
            }
#pragma unroll
            for (int j = 0; j < NAB; ++j) {
                W_STEP(KH + j, NKS);
                HOLD(j);
                lds_st(ldw, (unsigned)(KH + j + 1));
            }
            for (int t = KH + NAB; t < KA; ++t) {
                W_STEP(t, t + 1);
                VM_WAIT(16);
                lds_st(ldw, (unsigned)t);
            }
            VM_WAIT(0);
            lds_st(ldw, (unsigned)KA);
        } else {
            constexpr int CSTEP = 240 / NCONV;
            const int cq = ((idx - 2) / CSTEP) * 8 + xcd;
            if ((idx - 2) % CSTEP == 0 && cq < NCONV) {
                const int kb = lane & 7, nbl = lane >> 3;
                for (int u = cq; u < NKS * 16; u += NCONV) {
                    const int ks = u >> 4, cn = u & 15;
                    const float* src = p.w + (size_t)(ks * BK + kb * 8) * NLOC + cn * BN + nbl * 4;
                    char* dstw = myws + WS_WH + ((size_t)cn * NKS + ks) * B_STAGE;
                    f4 v[4][8];
#pragma unroll
                    for (int g = 0; g < 4; ++g)
#pragma unroll
                        for (int r = 0; r < 8; ++r)
                            v[g][r] = __builtin_nontemporal_load((const f4*)(src + (size_t)r * NLOC + g * 32)) * LOG2E;
#pragma unroll
                    for (int g = 0; g < 4; ++g)
#pragma unroll
                        for (int c = 0; c < 4; ++c) {
                            const int n = g * 32 + nbl * 4 + c;
                            u4 o;
                            o.x = pk(v[g][0][c], v[g][1][c]); o.y = pk(v[g][2][c], v[g][3][c]);
                            o.z = pk(v[g][4][c], v[g][5][c]); o.w = pk(v[g][6][c], v[g][7][c]);
                            st16_dev(dstw + n * 128 + ((kb ^ ((n >> 1) & 7)) * 16), o);
                        }
                    VM_WAIT(0);
                    if (lane == 0)
                        __hip_atomic_store((GAS unsigned*)(myws + WS_WFLAG + (size_t)cn * FSTRIDE + ks * 4), call,
                                           __ATOMIC_RELAXED, __HIP_MEMORY_SCOPE_AGENT);
                }
                STAMP(1);
            }
            for (unsigned sp = 0; rfl(lds_ld(ctrl + C_WAV)) < (unsigned)NKS;) {
                if (SPIN_OUT(sp, t0)) break;
                __builtin_amdgcn_s_sleep(32);
            }
#pragma unroll
            for (int j = 0; j < NAB; ++j) HOLD(j);
        }
        const int glo = max(mt * AG_XN, tj - (idx - 2)), ghi = min(mt * AG_XN + AG_XN, tj - (idx - 2) + 30);
        const int gn = ghi - glo, gme = tj - glo;
#define SHARE(nl_) __builtin_amdgcn_readfirstlane((nl_) * gme / gn | (nl_) * (gme + 1) / gn << 16)
        const int sh0 = SHARE(gcnt * 16), shA = SHARE(2 * FA * 16), shB = SHARE((gcnt - 2 * FA) * 16);
        int pf = (AG_PF && hh) ? (int)rfl(lds_ld(ctrl + C_AVAIL)) : NITEMS;
        unsigned sink = 0;
        constexpr int NA = 4 * FA / AG_XW, NB = 4 * FB / AG_XW;
        const int nb = ((xh || AG_XW == 1) && !full) ? NB - 2 : NB;
        const bool nox = AG_XW == 2 ? hh : wave != 4;
        int t = nox ? NITEMS : KA, pv = nox ? NITEMS : KA - 1 + xh, xo = 0;
        int left = 0;
        int tw = wave == 4 ? NITEMS : KA, jw = 0, ws = 0;
        unsigned wa = lbase + X_RING + hh * 8192 + lo16;
        unsigned sp = 0;
        while (pv < NITEMS || tw < NITEMS || pf < NITEMS) {
            u4 c; unsigned av_;
            asm volatile("ds_read_b128 %0, %2\n\tds_read_b32 %1, %2 offset:%3\n\ts_waitcnt lgkmcnt(0)"
                         : "=&v"(c), "=&v"(av_) : "v"(ctrl), "i"(C_AVAIL) : "memory");
            const int vm = vm_count(), av = (int)rfl(av_), cons = (int)min4(c);
            bool idle = true;
            for (; pf < av; ++pf) {
                const char* xa = xsrc + (pf < NKS ? (size_t)pf * X_STEP : (size_t)(pf - NAB) * X_STEP + XA);
                const int sh = pf < KA ? sh0 : pf < NKS ? shA : shB;
                for (int l = (sh & 0xffff) + lane; l < (sh >> 16); l += 64)
                    asm volatile("global_load_dword %0, %1, off" : "+v"(sink) : "v"(xa + l * 128) : "memory");
            }
            if (pv < t && vm <= left) {
                do { ++pv; left -= pv < NKS ? NA : nb; } while (pv < t && vm <= left);
                if (pv == t) left = 0;
                lds_st(ctrl + C_LD + xh * 8, (unsigned)pv);
            }
            if (tw < NITEMS && cons >= tw - 2) {
                if (wave == 5) TRACE(2, tw);
#define PUT(j_) case j_: if (j_ < NAB) asm volatile("ds_write_b128 %0, %1\n\tds_write_b128 %0, %2 offset:1024\n\t" \
                             "ds_write_b128 %0, %3 offset:2048\n\tds_write_b128 %0, %4 offset:3072\n\t" \
                             "ds_write_b128 %0, %5 offset:4096\n\tds_write_b128 %0, %6 offset:5120\n\t" \
                             "ds_write_b128 %0, %7 offset:6144\n\tds_write_b128 %0, %8 offset:7168" \
                             :: "v"(wa), "v"(hw[j_ % NAB][0]), "v"(hw[j_ % NAB][1]), "v"(hw[j_ % NAB][2]), "v"(hw[j_ % NAB][3]), \
                                "v"(hw[j_ % NAB][4]), "v"(hw[j_ % NAB][5]), "v"(hw[j_ % NAB][6]), "v"(hw[j_ % NAB][7]) : "memory"); break
                switch (jw) { PUT(0); PUT(1); PUT(2); PUT(3); PUT(4); PUT(5); PUT(6); }
                ++tw;
                lds_st(ctrl + C_LD + 4 + hh * 8, (unsigned)tw);
                jw = jw == NAB - 1 ? 0 : jw + 1;
                wa += ws == 2 ? -2 * B_STAGE : B_STAGE;
                ws = ws == 2 ? 0 : ws + 1;
                idle = false;
            }
            const int n = t < NKS ? NA : nb;
            if (t < NITEMS && av > t && vm + n <= 63 && cons >= KA - 2 + (int)((NEEDS >> (4 * (t - KA))) & 15u)) {
                if (wave == 4) TRACE(0, t);
                if (t < NKS) {
                    GLDN(xsrc + (size_t)t * X_STEP + xh * (XA / 2), xo + xh * (XA / 2), NA);
                    xo = next_slot(xo, XA, t == NKS - 1 ? XB : XA);
                } else {
                    const char* xa = xsrc + (size_t)(t - NAB) * X_STEP + XA + xh * (XB / 2);
                    GLDN(xa, xo + xh * (XB / 2), NB - 2);
                    if (nb == NB) GLDN(xa + (NB - 2) * 1024, xo + xh * (XB / 2) + (NB - 2) * 1024, 2);
                    xo = next_slot(xo, XB, XB);
                }
                if (pv < t) left += n;
                ++t;
                idle = false;
            }
            if (idle) {
                if (SPIN_OUT(sp, t0)) break;
                __builtin_amdgcn_s_sleep(1);
            }
        }
        asm volatile("s_waitcnt vmcnt(0)" : "+v"(sink) :: "memory");
        if (!hh) lds_st(ctrl + C_LD + xh * 8, (unsigned)NITEMS);
        if (wave != 4) lds_st(ctrl + C_LD + 4 + hh * 8, (unsigned)NITEMS);
        STAMP(9 + wave);
    }
}

}

extern "C" void kernel_launch(int rank, int nranks, ncclComm_t comm,
                              hipStream_t stream,
                              void* const* d_in, const int* in_sizes, int n_in,
                              void* d_out, int out_size,
                              const int* in_dtypes, int out_dtype,
                              void* const* d_out_all,
                              void* const* d_ws_all, size_t ws_bytes) {
    (void)comm; (void)in_dtypes; (void)out_dtype; (void)d_out_all;
    if (nranks != NR || n_in < 2 || (unsigned)rank >= (unsigned)NR || ws_bytes < WS_NEED ||
        in_sizes[0] != MLOC * GK || in_sizes[1] != GK * NLOC || out_size != GM * NLOC) {
        fprintf(stderr, "ag_gemm: unexpected problem shape\n");
        return;
    }
    static bool attr_set = false;
    if (!attr_set) {
        HIP_CHECK(hipFuncSetAttribute((const void*)ag_gemm_kernel,
                                      hipFuncAttributeMaxDynamicSharedMemorySize, LDS_BYTES));
        attr_set = true;
    }
    Params p;
    p.x = (const float*)d_in[0];
    p.w = (const float*)d_in[1];
    p.out = (float*)d_out;
    p.myws = (char*)d_ws_all[rank];
    for (int r = 0; r < NR; ++r) p.ws[r] = (char*)d_ws_all[r];
    p.rank = rank;
    hipLaunchKernelGGL(ag_gemm_kernel, dim3(NWG), dim3(NTHREADS), LDS_BYTES, stream, p);
    HIP_CHECK(hipGetLastError());
}
```

```cpp
#include <hip/hip_runtime.h>
#include <rccl/rccl.h>
#include <cstdio>

#define HIP_CHECK(expr) do { hipError_t _e = (expr); if (_e != hipSuccess) { \
    fprintf(stderr, "HIP error %s at %s:%d\n", hipGetErrorString(_e), __FILE__, __LINE__); \
    return; } } while (0)

namespace {

#define GAS __attribute__((address_space(1)))
#define LAS __attribute__((address_space(3)))
typedef _Float16 h8 __attribute__((ext_vector_type(8)));
typedef _Float16 h2 __attribute__((ext_vector_type(2)));
typedef float f4 __attribute__((ext_vector_type(4)));
typedef unsigned u4 __attribute__((ext_vector_type(4)));
typedef unsigned u2 __attribute__((ext_vector_type(2)));

#define AG_WIN 7
#define AG_SLEEP_AFTER 24
#define AG_SLEEP_POLL 8
#define AG_SLEEP_FINE 2
#define AG_NCONV 120
#define AG_SIG_FINE 48
#define AG_K2ROOM 3
#define AG_KA 57
#define AG_SPI 43
#define AG_XN 8
#define AG_LA 8
#define AG_POLLW 7
#define AG_HSLEEP 1
#define AG_PRIO 0
#define AG_NSTAT 6
#define AG_DIRECT (NITEMS - 1)

constexpr int NR = 8;
constexpr int GM = 4096, GK = 4096, NLOC = 2048, MLOC = 512;
constexpr int BN = 128, BK = 64;
constexpr int NMT = 15;
constexpr int GPT = 17;
constexpr int GCAP = 18;
constexpr int FR = 9, FA = 5, FB = 4;
constexpr int NKS = GK / BK;
constexpr int KA = AG_KA;
constexpr int NAB = NKS - KA;
constexpr int X_STEP = GCAP * 16 * BK * 2;
constexpr int B_STAGE = BN * BK * 2;
constexpr int X_RING = 3 * X_STEP;
constexpr int XA = 2 * FA * 2048, XB = 2 * FB * 2048;
constexpr int CTRL_OFF = X_RING + 3 * B_STAGE;
constexpr int LDS_BYTES = CTRL_OFF + 256;
constexpr int NWG = 256, NTHREADS = 512;
constexpr int NCOMM = 16;
constexpr int NITEMS = KA + 2 * NAB;
constexpr int KH = 40;
static_assert(KA % 3 == 0 && KH + NAB <= KA && NAB <= 7, "phases");
__host__ __device__ constexpr int next_slot(int off, int size, int nsize) { return off + size + nsize > X_RING ? 0 : off + size; }
constexpr unsigned long long slot_needs() {
    int off[3 + 2 * NAB] = {0, X_STEP, 2 * X_STEP}, end[3 + 2 * NAB] = {X_STEP, 2 * X_STEP, 3 * X_STEP};
    unsigned long long r = 0;
    for (int i = 0; i < 2 * NAB; ++i) {
        const int sz = i < NAB ? XA : XB;
        off[3 + i] = i ? next_slot(off[2 + i], end[2 + i] - off[2 + i], sz) : 0;
        end[3 + i] = off[3 + i] + sz;
        int n = 0;
        for (int j = 0; j < 3 + i; ++j) if (off[j] < end[3 + i] && off[3 + i] < end[j]) n = j;
        r |= (unsigned long long)n << (4 * i);
    }
    return r;
}
constexpr unsigned long long NEEDS = slot_needs();
constexpr unsigned long long ROT = 0x80605040302010ull;
constexpr int C_CONS = 0;
constexpr int C_AVAIL = 16;
constexpr int C_WAV = 20;
constexpr int C_BAD = 24;
constexpr int C_CALL = 28;
constexpr int C_LD = 32;
constexpr int C_HEAD = 16;
constexpr int C_LAND = 24;
constexpr int S_OFF = 4096;
constexpr int Q_OFF = 0;
constexpr int T_OFF = 256;
constexpr int NUNITS = NKS * 16;
constexpr int NSTAT = AG_NSTAT;
constexpr int SIG_FINE = AG_SIG_FINE;
constexpr int NCONV = AG_NCONV;
#define AG_EPOCHS 2, 4, 8, 12, 16, 20, 24, 28, 32, 36, 40, 42, 44, 46, 48, 49, 50, 51, 52, 53, 54, 55, 56, 57, \
                  58, 59, 60, 61, 62, 63, 64, 65, 66, 67, 68, 69, 70, 71
constexpr int EB[] = {AG_EPOCHS};
constexpr int NE = sizeof(EB) / sizeof(EB[0]);
constexpr int count_le(int v) { int n = 0; for (int i = 0; i < NE; ++i) n += EB[i] <= v; return n; }
constexpr int KD = count_le(AG_DIRECT);
static_assert(NE <= 64 && EB[NE - 1] == NITEMS && KD < NE && NITEMS < 255, "epochs");
constexpr float LOG2E = 1.4426950408889634f;

constexpr size_t FSTRIDE = 4096 + 256;
constexpr size_t WS_CALL = 0;
constexpr size_t WS_TMO = 49152;
constexpr size_t WS_CLAIM = 53248;
constexpr size_t WS_ECNT = 57344;
constexpr size_t WS_XPROG = 65536;
constexpr size_t WS_WFLAG = 196608;
constexpr size_t WS_TAB = 327680;
constexpr size_t WS_DBG = 524288;
constexpr size_t WS_WH = 1ull << 20;
constexpr size_t WS_XG = 32ull << 20;
constexpr size_t XG_BYTES = 36ull << 20;
constexpr size_t WS_NEED = WS_XG + 2 * XG_BYTES;
static_assert((size_t)NMT * NKS * X_STEP <= XG_BYTES, "gather buffer");
static_assert(LDS_BYTES <= 160 * 1024, "LDS");

constexpr unsigned long long DEADLINE = 100000000ull;

struct Params {
    const float* x;
    const float* w;
    float* out;
    char* myws;
    char* ws[NR];
    int rank;
};

__device__ __forceinline__ unsigned long long rt() { return __builtin_amdgcn_s_memrealtime(); }

__device__ __forceinline__ void st16_sys(void* p, u4 v) {
    asm volatile("global_store_dwordx4 %0, %1, off sc0 sc1\n\ts_nop 1" :: "v"(p), "v"(v) : "memory");
}
__device__ __forceinline__ void st16_dev(void* p, u4 v) {
    asm volatile("global_store_dwordx4 %0, %1, off sc1\n\ts_nop 1" :: "v"(p), "v"(v) : "memory");
}
__device__ __forceinline__ void st4_sys(void* p, unsigned v) {
    asm volatile("global_store_dword %0, %1, off sc0 sc1\n\ts_nop 1" :: "v"(p), "v"(v) : "memory");
}
__device__ __forceinline__ void lds_st(unsigned addr, unsigned v) {
    asm volatile("ds_write_b32 %0, %1" :: "v"(addr), "v"(v) : "memory");
}
__device__ __forceinline__ void lds_st2(unsigned addr, u2 v) {
    asm volatile("ds_write_b64 %0, %1" :: "v"(addr), "v"(v) : "memory");
}
__device__ __forceinline__ unsigned lds_ld(unsigned addr) {
    unsigned v;
    asm volatile("ds_read_b32 %0, %1\n\ts_waitcnt lgkmcnt(0)" : "=&v"(v) : "v"(addr) : "memory");
    return v;
}
__device__ __forceinline__ unsigned ld_ctr(const void* p) {
    unsigned v;
    asm volatile("global_load_dword %0, %1, off sc1\n\ts_waitcnt vmcnt(0)" : "=&v"(v) : "v"(p) : "memory");
    return (unsigned)__builtin_amdgcn_readfirstlane(v);
}
__device__ __forceinline__ unsigned pk(float a, float b) {
    h2 h; h.x = (_Float16)a; h.y = (_Float16)b;
    return __builtin_bit_cast(unsigned, h);
}
__device__ __forceinline__ unsigned umin(unsigned a, unsigned b) { return a < b ? a : b; }
__device__ __forceinline__ unsigned rfl(unsigned v) { return (unsigned)__builtin_amdgcn_readfirstlane((int)v); }
__device__ __forceinline__ unsigned min4(u4 c) { return rfl(umin(umin(c.x, c.y), umin(c.z, c.w))); }
__device__ __forceinline__ unsigned epoch_items(int k) {
    unsigned e = (unsigned)NITEMS;
#pragma unroll
    for (int i = 0; i < NE; ++i) if (k == i) e = EB[i];
    return e;
}
__device__ __forceinline__ int vm_count() {
    const unsigned st = __builtin_amdgcn_s_getreg(7 | (31 << 11));
    return (int)((st & 15u) | ((st >> 18) & 48u));
}
#define VM_WAIT(n) asm volatile("s_waitcnt vmcnt(" #n ")" ::: "memory")
#define LGKM0() asm volatile("s_waitcnt lgkmcnt(0)" ::: "memory")
#define SCHED_FENCE() __builtin_amdgcn_sched_barrier(0)
#define SPIN_OUT(sp_, t0_) ((((++(sp_)) & 1023u) == 0u) && ((t0_) == 0 ? ((t0_) = rt(), false) : (rt() - (t0_) > 2 * DEADLINE)))

#define TRACE(k_, t_) do {} while (0)
#define STAMP(i) do {} while (0)

#define DSR(dst_, addr_, off_) asm volatile("ds_read_b128 %0, %1 offset:%2" : "+v"(dst_) : "v"(addr_), "i"(off_) : "memory")
#define WAIT_LD(need) do { unsigned long long _t0 = 0; unsigned _sp = 0; \
        for (;;) { u4 _f; asm volatile("ds_read_b128 %0, %1\n\ts_waitcnt lgkmcnt(0)" : "=&v"(_f) : "v"(ldw) : "memory"); \
            if (min4(_f) >= (unsigned)(need)) break; \
            if (SPIN_OUT(_sp, _t0)) break; \
            __builtin_amdgcn_s_sleep(1); } \
        LGKM0(); } while (0)

struct Frag { h8 a[FA]; h8 b[4]; };
__device__ __forceinline__ void wait_set(Frag& s) {
    asm volatile("s_waitcnt lgkmcnt(0)" : "+v"(s.a[0]), "+v"(s.a[1]), "+v"(s.a[2]), "+v"(s.a[3]), "+v"(s.a[4]),
                 "+v"(s.b[0]), "+v"(s.b[1]), "+v"(s.b[2]), "+v"(s.b[3]) :: "memory");
}
__device__ __forceinline__ void fetch(int g, Frag& s, unsigned pa, unsigned pb) {
    if (g == 0) DSR(s.b[0], pb, 0);
    if (g == 1) DSR(s.a[0], pa, 0);
    if (g == 2) DSR(s.b[1], pb, 2048);
    if (g == 3) DSR(s.b[2], pb, 2 * 2048);
    if (g == 4) DSR(s.b[3], pb, 3 * 2048);
    if (g == 5) DSR(s.a[1], pa, 2048);
    if (g == 6) DSR(s.a[2], pa, 2 * 2048);
    if (g == 7) DSR(s.a[3], pa, 3 * 2048);
    if (g == 8) DSR(s.a[4], pa, 4 * 2048);
}
struct Pos { unsigned pa, pb, qa, qb, rv, fl; int xo, ws;
};
struct Epi {
    float t[4];
    f4 o;
    float lim, c;
    unsigned vo;
    const char* row[FA];
};
constexpr int NSLOT = 15 * 4 * FA;
template <int S>
__device__ __forceinline__ void epi_slot(f4 (&acc)[FR][4], Epi& E) {
    constexpr int q = S / 15, op = S % 15, k = (op & 1) * 2;
    if constexpr (S < NSLOT) {
        const f4& y = acc[q >> 2][q & 3];
        if constexpr (op < 2)
            asm volatile("v_med3_f32 %0, -%2, %4, -%4\n\tv_med3_f32 %1, -%3, %4, -%4"
                         : "=&v"(E.t[k]), "=&v"(E.t[k + 1]) : "v"(y[k]), "v"(y[k + 1]), "s"(E.lim), "v"(q ? E.o : y));
        else if constexpr (op < 6) asm volatile("v_exp_f32 %0, %0" : "+v"(E.t[op - 2]));
        else if constexpr (op < 8)
            asm volatile("v_fma_f32 %0, %0, %2, %2\n\tv_fma_f32 %1, %1, %2, %2" : "+v"(E.t[k]), "+v"(E.t[k + 1]) : "s"(E.c));
        else if constexpr (op < 12) asm volatile("v_rcp_f32 %0, %0" : "+v"(E.t[op - 8]));
        else if constexpr (op < 14) {
            float o0, o1;
            asm volatile("v_mul_f32 %0, %2, %4\n\tv_mul_f32 %1, %3, %5" : "=&v"(o0), "=v"(o1)
                         : "v"(y[k]), "v"(y[k + 1]), "v"(E.t[k]), "v"(E.t[k + 1]));
            E.o[k] = o0; E.o[k + 1] = o1;
        } else
            asm volatile("global_store_dwordx4 %0, %1, %2 offset:%3 sc1" :: "v"(E.vo), "v"(E.o), "s"(E.row[q >> 2]), "i"((q & 3) * 64) : "memory");
        if constexpr (S == NSLOT - 1) asm volatile("s_nop 1" :: "v"(E.o));
    }
}
constexpr int slots_before(int g, int n) {
    constexpr int ORD[16] = {13, 12, 14, 11, 10, 9, 8, 7, 6, 5, 4, 3, 2, 1, 0, 15};
    int c = 0;
    for (int i = 0; i < g; ++i) {
        int rank = 0;
        for (int j = 0; j < 16; ++j) if (ORD[j] == (i & 15)) rank = 2 * j + (i >> 4);
        c += 1 + (rank < n - 32);
    }
    return c < n ? c : n;
}
template <int S0, int NS, int G>
__device__ __forceinline__ void epi_gap(f4 (&acc)[FR][4], Epi& E) {
    if constexpr (NS > 0) {
        constexpr int b = slots_before(G, NS), e = slots_before(G + 1, NS);
        if constexpr (e > b) epi_slot<S0 + b>(acc, E);
        if constexpr (e > b + 1) epi_slot<S0 + b + 1>(acc, E);
    }
}
template <int F, int R0, int S0, int NS, int H, int G>
__device__ __forceinline__ void half_step(f4 (&acc)[FR][4], Frag& sc, Frag& sn, Pos& P, Epi& E, u4& f, unsigned fl,
                                          int da, int db, unsigned ldw) {
    if constexpr (G < 4 * F) {
        if (G == 4 * F - 1) {
            if (H == 0) asm volatile("" : "+v"(f));
            wait_set(sn);
            if (H == 0) asm volatile("" : "+v"(f));
            SCHED_FENCE();
        }
        f4& c = acc[R0 + (G >> 2)][G & 3];
        if (R0) asm volatile("v_mfma_f32_16x16x32_f16 %0, %1, %2, %0" : "+v"(c) : "v"(sc.b[G & 3]), "v"(sc.a[G >> 2]));
        else c = __builtin_amdgcn_mfma_f32_16x16x32_f16(sc.b[G & 3], sc.a[G >> 2], c, 0, 0, 0);
        SCHED_FENCE();
        unsigned& na = H ? P.pa : P.qa; unsigned& nb = H ? P.pb : P.qb;
        unsigned& ma = H ? P.qa : P.pa; unsigned& mb = H ? P.qb : P.pb;
        if (G < F + 4) fetch(G, sn, na, nb);
        if (G == F + 4) { ma += da; asm volatile("" : "+v"(ma)); }
        if (G == F + 5) { mb += db; asm volatile("" : "+v"(mb)); }
        if (H == 0 && G == 4 * F - 6) asm volatile("ds_read_b128 %0, %1" : "=v"(f) : "v"(ldw) : "memory");
        if (H == 1 && G == F + 6) { P.rv += 1u; asm volatile("" : "+v"(P.rv)); }
        if (H == 1 && G == F + 7) { P.fl = rfl(fl); asm volatile("" : "+s"(P.fl)); }
        epi_gap<S0, NS, H * 4 * F + G>(acc, E);
        SCHED_FENCE();
        half_step<F, R0, S0, NS, H, G + 1>(acc, sc, sn, P, E, f, fl, da, db, ldw);
    }
}
template <int F, int R0, int S0, int NS>
__device__ __forceinline__ void part_item(f4 (&acc)[FR][4], Frag& s0, Frag& s1, Pos& P, Epi& E, unsigned t,
                                          unsigned ldw, unsigned relw, int wm, char* trc) {
    const bool sw = t == (unsigned)(NKS - 1);
    const int xn = next_slot(P.xo, 2 * F * 2048, sw ? XB : 2 * F * 2048);
    const int da = xn - P.xo - (sw ? wm * (FA - FB) * 2048 : 0);
    const int db = P.ws == 2 ? -2 * B_STAGE : B_STAGE;
    P.xo = xn; P.ws = P.ws == 2 ? 0 : P.ws + 1;
    u4 f;
    if (P.fl <= t) {
        WAIT_LD(t + 1u);
#pragma unroll
        for (int g = 0; g < F + 4; ++g) fetch(g, s0, P.pa, P.pb);
        wait_set(s0);
        SCHED_FENCE();
    }
    if (threadIdx.x < 64) TRACE(1, t);
    half_step<F, R0, S0, NS, 0, 0>(acc, s0, s1, P, E, f, 0u, da, db, ldw);
    lds_st(relw, P.rv);
    const unsigned fl = umin(umin(f.x, f.y), umin(f.z, f.w));
    SCHED_FENCE();
    half_step<F, R0, S0, NS, 1, 0>(acc, s1, s0, P, E, f, fl, da, db, ldw);
}
template <int I>
__device__ __forceinline__ void part_b(f4 (&acc)[FR][4], Frag& s0, Frag& s1, Pos& P, Epi& E,
                                       unsigned ldw, unsigned relw, int wm, char* trc) {
    if constexpr (I < NAB) {
        constexpr int S0 = I * AG_SPI, NS = S0 >= NSLOT ? 0 : NSLOT - S0 < AG_SPI ? NSLOT - S0 : AG_SPI;
        part_item<FB, FA, S0, NS>(acc, s0, s1, P, E, NKS + I, ldw, relw, wm, trc);
        part_b<I + 1>(acc, s0, s1, P, E, ldw, relw, wm, trc);
    }
}
static_assert(AG_SPI * NAB >= NSLOT && AG_SPI <= 16 * FB, "slots");

__global__ void __launch_bounds__(NTHREADS, 2) ag_gemm_kernel(Params p) {
    extern __shared__ __attribute__((aligned(16))) unsigned char smem[];
    LAS unsigned char* lds = (LAS unsigned char*)smem;
    const int tid = threadIdx.x;
    const int wave = __builtin_amdgcn_readfirstlane(tid >> 6);
    const int lane = tid & 63;
    const int b = blockIdx.x;
    const int xcd = b & 7, idx = b >> 3;
    char* myws = p.myws;
    GAS unsigned* cctr = (GAS unsigned*)(myws + WS_CALL + (size_t)b * 128);
    const unsigned lbase = (unsigned)(size_t)lds;
    const unsigned ctrl = lbase + CTRL_OFF;

    if (idx < 2) {
        const int cc = xcd * 2 + idx;
        constexpr unsigned ENDD = 0xffffffffu;
        f4 la, lb;
        unsigned cv;
        if (wave < 4) {
            STAMP(20 + wave);
            const float* s0 = p.x + (size_t)(cc * 32 + wave * 8 + (lane >> 3)) * GK + (lane & 7) * 8;
            asm volatile("global_load_dwordx4 %0, %4, off\n\tglobal_load_dwordx4 %1, %4, off offset:16\n\t"
                         "global_load_dword %2, %3, off sc1"
                         : "=&v"(la), "=&v"(lb), "=&v"(cv) : "v"(cctr), "v"(s0) : "memory");
        }
        unsigned gi, ord, nA;
        {
            const unsigned l5 = (unsigned)lane & 31u, g = (unsigned)p.rank * 32u + l5;
            unsigned t = g / GPT; t = t < (unsigned)(NMT - 1) ? t : (unsigned)(NMT - 1);
            const unsigned gc = (t == NMT - 1) ? GCAP : GPT;
            unsigned sl = g - t * GPT + (unsigned)((ROT >> (4u * t)) & 15u);
            sl = sl >= gc ? sl - gc : sl;
            const unsigned mA = (unsigned)__ballot(sl < 2u * FA);
            const unsigned below = (1u << l5) - 1u;
            nA = (unsigned)__builtin_popcount(mA);
            ord = (sl < 2u * FA) ? (unsigned)__builtin_popcount(mA & below) : nA + (unsigned)__builtin_popcount(~mA & below);
            gi = l5 | sl << 11 | t << 16;
        }
        unsigned wlo = 0, whi = 0;
#pragma unroll
        for (int r = 0; r < NR; ++r) {
            const unsigned long long a = (unsigned long long)p.ws[r];
            if ((lane & 7) == r) { wlo = (unsigned)a; whi = (unsigned)(a >> 32); }
        }
        if (wave < 4) {
            const int h = wave >> 1;
            const int c = lane & 7, r16 = (wave & 1) * 8 + (lane >> 3);
            unsigned dq = (unsigned)__builtin_amdgcn_readlane((int)gi, cc * 2 + h);
            const size_t loff = WS_XG + (size_t)r16 * 128 + ((c ^ (r16 >> 1)) * 16);
            char* dst[NR];
#pragma unroll
            for (int j = 0; j < NR; ++j) {
                const int r = (p.rank + 1 + j + cc + 2 * wave) & 7;
                dst[j] = (char*)(((unsigned long long)(unsigned)__builtin_amdgcn_readlane((int)whi, r) << 32) |
                                 (unsigned)__builtin_amdgcn_readlane((int)wlo, r)) + loff;
            }
            const unsigned lsrc = lbase + S_OFF + (unsigned)((wave * 8 + (lane >> 3)) * 256 + c * 32);
            asm volatile("s_barrier\n\ts_waitcnt vmcnt(0)" : "+v"(la), "+v"(lb), "+v"(cv), "+v"(dst[0]), "+v"(dst[1]), "+v"(dst[2]),
                         "+v"(dst[3]), "+v"(dst[4]), "+v"(dst[5]), "+v"(dst[6]), "+v"(dst[7]) :: "memory");
            const unsigned xpar = ((rfl(cv) + 1u) & 1u) * (unsigned)XG_BYTES;
            if (wave == 0) STAMP(5);
            constexpr int X = AG_WIN;
            static_assert(X >= 2 && X <= 7 && AG_LA <= 14 && NSTAT >= 3 && NSTAT <= 14 && NSTAT <= KA, "window");
            unsigned long long t0 = 0;
            unsigned e = 0;
            for (;; ++e) {
                u4 o;
                o.x = pk(la[0], la[1]); o.y = pk(la[2], la[3]);
                o.z = pk(lb[0], lb[1]); o.w = pk(lb[2], lb[3]);
                const unsigned dmt = umin((dq >> 16) & 15u, NMT - 1), dsl = umin((dq >> 11) & 31u, GCAP - 1);
                const unsigned so = xpar + (dmt * NKS + ((dq >> 5) & 63u)) * (unsigned)X_STEP + dsl * 2048u;
                const unsigned qa = lbase + Q_OFF + ((e + 1u) & 15u) * 16u + h * 4, sa = lsrc + ((e + 1u) & 15u) * 8192u;
                unsigned ld, qv;
#define NEXT_ROWS "ds_read_b32 %0, %4 offset:%7\n\tds_read_b32 %1, %5\n\tds_read_b128 %2, %6\n\tds_read_b128 %3, %6 offset:16\n\t"
#pragma unroll
                for (int j = 0; j < NR; ++j) {
                    asm volatile("s_waitcnt vmcnt(%0)" :: "i"(X) : "memory");
                    if (j == X % 8 && e >= (unsigned)(X / 8)) lds_st(ctrl + C_CONS + wave * 4, e - X / 8);
                    if (j < NR - 1) st16_sys(dst[j] + so, o);
                }
                asm volatile(NEXT_ROWS "global_store_dwordx4 %8, %9, off sc0 sc1\n\ts_waitcnt lgkmcnt(0)"
                             : "=&v"(ld), "=&v"(qv), "=&v"(la), "=&v"(lb)
                             : "v"(ctrl + h * 4), "v"(qa), "v"(sa), "i"(C_LAND), "v"(dst[NR - 1] + so), "v"(o) : "memory");
                for (unsigned sp = 0;;) {
                    dq = rfl(qv);
                    if (rfl(ld) > e + 1u) break;
                    if (SPIN_OUT(sp, t0)) { dq = ENDD; break; }
                    if (vm_count() == 0) { VM_WAIT(0); lds_st(ctrl + C_CONS + wave * 4, e + 1u); }
                    __builtin_amdgcn_s_sleep(1);
                    asm volatile(NEXT_ROWS "s_waitcnt lgkmcnt(0)" : "=&v"(ld), "=&v"(qv), "=&v"(la), "=&v"(lb)
                                 : "v"(ctrl + h * 4), "v"(qa), "v"(sa), "i"(C_LAND) : "memory");
                }
                if (dq == ENDD) break;
            }
            if (X > 8 && vm_count() > 8) { VM_WAIT(8); lds_st(ctrl + C_CONS + wave * 4, e); }
            VM_WAIT(0); lds_st(ctrl + C_CONS + wave * 4, e + 1u);
            STAMP(6 + wave);
            return;
        }
        const unsigned nB = 32u - nA;
        if (wave >= 6) {
            const int h = wave - 6;
            const unsigned vo = (unsigned)((lane >> 4) * GK * 4 + (lane & 15) * 16);
            auto issue = [&](unsigned e, unsigned d) {
#pragma unroll
                for (int i = 0; i < 4; ++i) {
                    const char* src = (const char*)p.x + (size_t)((d & 31u) * 16u + (unsigned)i * 4u) * (GK * 4) + ((d >> 5) & 63u) * (BK * 4);
                    asm volatile("s_mov_b32 m0, %0\n\ts_nop 0\n\tglobal_load_lds_dwordx4 %1, %2"
                                 :: "s"(lbase + S_OFF + (e & 15u) * 8192u + h * 4096 + i * 1024), "v"(vo), "s"(src) : "memory");
                }
            };
            issue(1u, (unsigned)(cc * 2 + h) | 1u << 5);
            issue(2u, (unsigned)(cc * 2 + h) | 2u << 5);
            __builtin_amdgcn_s_barrier();
            unsigned il = 3, pv = 1;
            unsigned long long t0 = 0; unsigned sp = 0;
            for (;;) {
                unsigned hd, d;
                asm volatile("ds_read_b32 %0, %2 offset:%4\n\tds_read_b32 %1, %3\n\ts_waitcnt lgkmcnt(0)"
                             : "=&v"(hd), "=&v"(d) : "v"(ctrl), "v"(lbase + Q_OFF + (il & 15u) * 16u + h * 4), "i"(C_HEAD) : "memory");
                const unsigned fl = (unsigned)(vm_count() + 3) >> 2;
                const bool can = rfl(hd) > il, end = can && rfl(d) == ENDD;
                if (end && fl == 0u) break;
                if (il - fl > pv) {
                    switch (fl) {
                        case 0: VM_WAIT(0); break;
                        case 1: VM_WAIT(4); break;
                        case 2: VM_WAIT(8); break;
                        case 3: VM_WAIT(12); break;
                        case 4: VM_WAIT(16); break;
                        case 5: VM_WAIT(20); break;
                        case 6: VM_WAIT(24); break;
                        case 7: VM_WAIT(28); break;
                        case 8: VM_WAIT(32); break;
                        case 9: VM_WAIT(36); break;
                        case 10: VM_WAIT(40); break;
                        case 11: VM_WAIT(44); break;
                        case 12: VM_WAIT(48); break;
                        default: VM_WAIT(52); break;
                    }
                    pv = il - fl;
                    lds_st(ctrl + C_LAND + h * 4, pv);
                }
                if (can && !end) {
                    issue(il, rfl(d));
                    ++il;
                } else {
                    if (SPIN_OUT(sp, t0)) break;
                    __builtin_amdgcn_s_sleep(1);
                }
            }
            VM_WAIT(0);
            lds_st(ctrl + C_LAND + h * 4, il + 1u);
        } else if (wave == 4) {
            unsigned cw;
            asm volatile("global_load_dword %0, %1, off sc1" : "=&v"(cw) : "v"(cctr) : "memory");
            lds_st(ctrl + lane * 4, lane == C_HEAD / 4 ? (unsigned)NSTAT : 0u);
            {
                const unsigned g0 = (unsigned)__builtin_amdgcn_readlane((int)gi, cc * 2);
                const unsigned g1 = (unsigned)__builtin_amdgcn_readlane((int)gi, cc * 2 + 1);
                const unsigned en = (unsigned)lane >> 2;
                const unsigned q = (lane & 2) ? en * 16u + (unsigned)cc : ((lane & 1) ? g1 : g0) | en << 5;
                if (lane < 4 * NSTAT) lds_st(lbase + Q_OFF + lane * 4, q);
            }
            if (lane < 32) lds_st(lbase + T_OFF + ord * 4u, gi);
            LGKM0();
            __builtin_amdgcn_s_barrier();
            asm volatile("s_waitcnt vmcnt(0)" : "+v"(cw) :: "memory");
            const unsigned call = rfl(cw) + 1u;
            GAS unsigned* cl = (GAS unsigned*)(myws + WS_CLAIM + (size_t)(call & 1u) * 256);
            if (b == 0 && lane == 0)
                __hip_atomic_store((GAS unsigned*)(myws + WS_CLAIM + (size_t)((call + 1u) & 1u) * 256), 0u,
                                   __ATOMIC_RELAXED, __HIP_MEMORY_SCOPE_AGENT);
            const unsigned mgA = (65536u + nA - 1u) / nA, mgB = (65536u + nB - 1u) / nB;
            unsigned head = NSTAT;
            bool end = false;
            unsigned long long t0 = 0; unsigned sp = 0;
            while (!end) {
                u4 c;
                asm volatile("ds_read_b128 %0, %1\n\ts_waitcnt lgkmcnt(0)" : "=&v"(c) : "v"(ctrl) : "memory");
                const int room = AG_LA - (int)(head - min4(c));
                if (room > 0) {
                    const unsigned k = room > AG_K2ROOM ? 2u : 1u;
                    unsigned r = 0;
                    if (lane == 0) r = __hip_atomic_fetch_add(cl, k, __ATOMIC_RELAXED, __HIP_MEMORY_SCOPE_AGENT);
                    const unsigned u0 = rfl(r) + (unsigned)(NSTAT * NCOMM);
                    const unsigned u = u0 + (unsigned)((lane >> 2) & 1);
                    unsigned hu = umin(u, NUNITS - 1) * 2u + (unsigned)(lane & 1);
                    unsigned ks = hu >> 5, j = hu & 31u;
                    if (hu >= 32u * KA) {
                        hu -= 32u * KA;
                        const bool isB = hu >= nA * NAB;
                        hu = isB ? hu - nA * NAB : hu;
                        const unsigned kq = (hu * (isB ? mgB : mgA)) >> 16;
                        j = hu - kq * (isB ? nB : nA) + (isB ? nA : 0u);
                        ks = KA + kq;
                    }
                    unsigned q = lds_ld(lbase + T_OFF + (j & 31u) * 4u) | ks << 5;
                    if (u >= (unsigned)NUNITS) q = ENDD;
                    if (lane & 2) q = umin(u, NUNITS);
                    if ((unsigned)lane < 4u * k) lds_st(lbase + Q_OFF + ((head + (lane >> 2)) & 15u) * 16u + (lane & 3) * 4, q);
                    end = u0 + k > (unsigned)NUNITS;
                    head += k;
                    lds_st(ctrl + C_HEAD, head);
                } else {
                    if (SPIN_OUT(sp, t0)) break;
                    __builtin_amdgcn_s_sleep(2);
                }
            }
        } else if (wave == 5) {
            __builtin_amdgcn_s_barrier();
            const unsigned call = ld_ctr((const void*)cctr) + 1u;
            const unsigned it = epoch_items(lane);
            const unsigned thr = ((it <= (unsigned)KA ? it * 32u : it <= (unsigned)NKS ? KA * 32u + (it - KA) * nA
                                   : KA * 32u + NAB * nA + (it - NKS) * nB) + 1u) >> 1;
            const int mt0 = (p.rank * 32) / GPT;
            const int mt1 = ((p.rank * 32 + 31) / GPT < NMT - 1) ? (p.rank * 32 + 31) / GPT : NMT - 1;
            const int smt = mt0 + (lane >> 3);
            const bool son = smt <= mt1;
            const int slot = ((smt * GPT) / 32 == p.rank) ? 0 : 1;
            char* pw = (char*)(((unsigned long long)whi << 32) | wlo) + WS_XPROG + (size_t)(son ? smt : mt0) * FSTRIDE
                       + (size_t)(slot * 256);
            GAS unsigned* ec = (GAS unsigned*)(myws + WS_ECNT);
            int k = 0, alt = 0;
            unsigned long long t0 = 0; unsigned sp = 0;
            while (k < NE) {
                u4 c; unsigned hd;
                asm volatile("ds_read_b128 %0, %2\n\tds_read_b32 %1, %2 offset:%3\n\ts_waitcnt lgkmcnt(0)"
                             : "=&v"(c), "=&v"(hd) : "v"(ctrl), "i"(C_HEAD) : "memory");
                const unsigned m = min4(c);
                unsigned P = 0;
                if (rfl(hd) > m) P = rfl(lds_ld(lbase + Q_OFF + (m & 15u) * 16u + 8u));
                const unsigned long long rch = __ballot(lane < NE && thr <= P);
                const int k2 = ~rch ? __builtin_ctzll(~rch) : 64;
                if (k2 > k) {
                    if (k2 > KD) {
                        VM_WAIT(1);
                        if (son) st4_sys(pw + 128 + cc * 8 + alt, call * 256u + (unsigned)__builtin_amdgcn_readlane((int)it, k2 - 1));
                        alt ^= 4;
                    }
                    if (k < KD) {
                        unsigned old = 0;
                        const bool mine = lane >= k && lane < k2 && lane < KD;
                        if (mine) old = __hip_atomic_fetch_add(ec + lane, 1u, __ATOMIC_RELAXED, __HIP_MEMORY_SCOPE_AGENT);
                        const unsigned long long lastm = __ballot(mine && old == call * (unsigned)NCOMM - 1u);
                        if (lastm) {
                            const int e = 63 - __builtin_clzll(lastm);
                            if (son) st4_sys(pw + (e & 31) * 4, call * 256u + (unsigned)__builtin_amdgcn_readlane((int)it, e));
                        }
                    }
                    k = k2;
                } else {
                    if (SPIN_OUT(sp, t0)) break;
                    __builtin_amdgcn_s_sleep(1);
                }
            }
            if (lane == 0) __hip_atomic_store(cctr, call, __ATOMIC_RELAXED, __HIP_MEMORY_SCOPE_AGENT);
            STAMP(10);
            STAMP(11);
        }
        return;
    }

    constexpr int XC = 16 / AG_XN;
    const int tj = (xcd / XC) * 30 + (idx - 2);
    const int mt = tj / AG_XN, nt = (xcd % XC) * AG_XN + tj % AG_XN;
    const int gcnt = (mt == NMT - 1) ? GCAP : GPT;
    if (wave == AG_POLLW) {
        *(u2*)(myws + WS_TAB + (size_t)b * 512 + lane * 8) = (u2){2u * lane + 1u, 2u * lane + 2u};
        const unsigned cl = ld_ctr((const void*)cctr) + 1u;
        if (lane == 0) __hip_atomic_store(cctr, cl, __ATOMIC_RELAXED, __HIP_MEMORY_SCOPE_AGENT);
        lds_st(ctrl + lane * 4, lane == C_CALL / 4 ? cl : lane == C_LD / 4 + 2 ? (unsigned)NITEMS : lane == C_LD / 4 + 3 ? (unsigned)KA : 0u);
        LGKM0();
    }
    __builtin_amdgcn_s_barrier();
    asm volatile("" ::: "memory");

    if (wave < 4) {
        if (wave == 0) STAMP(0);
        const int wm = wave >> 1, wn = wave & 1;
        const int r16 = lane & 15, q = lane >> 4;
        const unsigned sw = (unsigned)((q ^ (r16 >> 1)) * 16);
        const unsigned ldw = ctrl + C_LD;
        const unsigned relw = ctrl + C_CONS + wave * 4;
        const int rot = (int)((ROT >> (4 * mt)) & 15u);
        float* ocol = p.out + (size_t)(mt * GPT * 16 + r16) * NLOC + nt * BN + wn * 64 + q * 4;
        auto finish = [&](const f4 (&y4)[4], int slot, bool bad) {
            if (slot >= gcnt) return;
            const int pos = slot - rot < 0 ? slot - rot + gcnt : slot - rot;
            float* orow = ocol + (size_t)pos * 16 * NLOC;
            if (bad) {
#pragma unroll
                for (int j = 0; j < 4; ++j) st16_dev(orow + j * 16, (u4){0x7fc00000u, 0x7fc00000u, 0x7fc00000u, 0x7fc00000u});
                return;
            }
#pragma unroll
            for (int j = 0; j < 4; ++j) {
                const f4 y = y4[j];
                f4 o;
#pragma unroll
                for (int e = 0; e < 4; ++e) {
                    const float z = __builtin_amdgcn_fmed3f(y[e], -60.f * LOG2E, 60.f * LOG2E);
                    const float ex = __builtin_amdgcn_exp2f(-z);
                    o[e] = y[e] * __builtin_amdgcn_rcpf(__builtin_fmaf(ex, LOG2E, LOG2E));
                }
                st16_dev(orow + j * 16, __builtin_bit_cast(u4, o));
            }
        };
        f4 acc[FR][4];
#pragma unroll
        for (int i = 0; i < FR; ++i)
#pragma unroll
            for (int j = 0; j < 4; ++j) acc[i][j] = (f4){0.f, 0.f, 0.f, 0.f};
        {
        const unsigned offA = lbase + (unsigned)((wm * FR * 16 + r16) * 128) + sw;
        const unsigned offB = lbase + (unsigned)(X_RING + (wn * 64 + r16) * 128) + sw;
        h8 a[FR], b0[4], b1[4];
#pragma unroll
        for (int i = 0; i < FR; ++i) a[i] = (h8)(_Float16)0.f;
#pragma unroll
        for (int j = 0; j < 4; ++j) { b0[j] = (h8)(_Float16)0.f; b1[j] = (h8)(_Float16)0.f; }
#define MF(i_, j_, B_, A_) do { acc[i_][j_] = __builtin_amdgcn_mfma_f32_16x16x32_f16(B_, A_, acc[i_][j_], 0, 0, 0); SCHED_FENCE(); } while (0)
#define FILL(x_) do { x_; SCHED_FENCE(); } while (0)
#define ROW(i_, BC_, A_, F0_, F1_, F2_) do { \
        MF(i_, 0, BC_[0], A_); FILL(F0_); MF(i_, 1, BC_[1], A_); FILL(F1_); \
        MF(i_, 2, BC_[2], A_); FILL(F2_); MF(i_, 3, BC_[3], A_); } while (0)
#define NOP_ do {} while (0)
#define WA(i_, cnt_) asm volatile("s_waitcnt lgkmcnt(%1)" : "+v"(a[i_]) : "i"(cnt_) : "memory")
#define WAB(BC_, cnt_) asm volatile("s_waitcnt lgkmcnt(%8)" : "+v"(a[0]), "+v"(a[1]), "+v"(a[2]), "+v"(a[3]), \
                                    "+v"(BC_[0]), "+v"(BC_[1]), "+v"(BC_[2]), "+v"(BC_[3]) : "i"(cnt_) : "memory")
        WAIT_LD(1);
#pragma unroll
        for (int j = 0; j < 4; ++j) DSR(b0[j], offB, j * 2048);
#pragma unroll
        for (int i = 0; i < FR - 1; ++i) DSR(a[i], offA, i * 2048);
        WAB(b0, 0);
        SCHED_FENCE();
        unsigned sb = 0, sc = 0;
        char* trc = myws;
        for (int t = 0; t < KA; ++t) {
            u2 f;
            if (wave == 0) TRACE(1, t);
            {
                const unsigned ca = offA + sb, na = ca ^ 64u, nb = (offB + sc) ^ 64u;
                ROW(0, b0, a[0], DSR(a[8], ca, 8 * 2048), NOP_, NOP_);
                ROW(1, b0, a[1], DSR(a[0], na, 0), DSR(b1[0], nb, 0), NOP_);
                ROW(2, b0, a[2], DSR(a[1], na, 2048), DSR(b1[1], nb, 2048), NOP_);
                ROW(3, b0, a[3], DSR(a[2], na, 2 * 2048), DSR(b1[2], nb, 2 * 2048), WA(4, 10));
                ROW(4, b0, a[4], DSR(a[3], na, 3 * 2048), DSR(b1[3], nb, 3 * 2048), WA(5, 11));
                ROW(5, b0, a[5], DSR(a[4], na, 4 * 2048),
                    asm volatile("ds_read_b64 %0, %1" : "=v"(f) : "v"(ldw) : "memory"), WA(6, 11));
                ROW(6, b0, a[6], DSR(a[5], na, 5 * 2048), NOP_, WA(7, 11));
                ROW(7, b0, a[7], DSR(a[6], na, 6 * 2048), NOP_, WA(8, 11));
                ROW(8, b0, a[8], DSR(a[7], na, 7 * 2048), NOP_, WAB(b1, 4));
            }
            {
                const unsigned ca = (offA + sb) ^ 64u;
                sb = (sb == (unsigned)(2 * X_STEP)) ? 0u : sb + (unsigned)X_STEP;
                sc = (sc == (unsigned)(2 * B_STAGE)) ? 0u : sc + (unsigned)B_STAGE;
                const unsigned na = offA + sb, nb = offB + sc;
                ROW(0, b1, a[0], DSR(a[8], ca, 8 * 2048), NOP_,
                    asm volatile("s_waitcnt lgkmcnt(4)" : "+v"(f) :: "memory"));
                const unsigned fl = rfl(umin(f.x, f.y));
                const unsigned need = (t + 2 <= KA) ? (unsigned)(t + 2) : (unsigned)KA;
                if (fl < need) WAIT_LD(need);
                SCHED_FENCE();
                ROW(1, b1, a[1], DSR(a[0], na, 0), DSR(b0[0], nb, 0), NOP_);
                ROW(2, b1, a[2], DSR(a[1], na, 2048), DSR(b0[1], nb, 2048), NOP_);
                ROW(3, b1, a[3], DSR(a[2], na, 2 * 2048), DSR(b0[2], nb, 2 * 2048),
                    WA(4, 6); lds_st(relw, (unsigned)(t + 1)));
                ROW(4, b1, a[4], DSR(a[3], na, 3 * 2048), DSR(b0[3], nb, 3 * 2048), WA(5, 11));
                ROW(5, b1, a[5], DSR(a[4], na, 4 * 2048), NOP_, WA(6, 11));
                ROW(6, b1, a[6], DSR(a[5], na, 5 * 2048), NOP_, WA(7, 11));
                ROW(7, b1, a[7], DSR(a[6], na, 6 * 2048), NOP_, WA(8, 11));
                ROW(8, b1, a[8], DSR(a[7], na, 7 * 2048), NOP_, WAB(b0, 4));
            }
        }
        WAB(b0, 0);
        asm volatile("" : "+v"(a[4]), "+v"(a[5]), "+v"(a[6]), "+v"(a[7]));
        }
        if (wave == 0) STAMP(2);
        Frag s0, s1;
#pragma unroll
        for (int i = 0; i < FA; ++i) { s0.a[i] = (h8)(_Float16)0.f; s1.a[i] = (h8)(_Float16)0.f; }
#pragma unroll
        for (int j = 0; j < 4; ++j) { s0.b[j] = (h8)(_Float16)0.f; s1.b[j] = (h8)(_Float16)0.f; }
        Pos P;
        P.pa = lbase + (unsigned)((wm * FA * 16 + r16) * 128) + sw;
        P.pb = lbase + (unsigned)(X_RING + (wn * 64 + r16) * 128) + sw;
        P.qa = P.pa ^ 64u; P.qb = P.pb ^ 64u; P.rv = KA + 1; P.fl = 0; P.xo = 0; P.ws = 0;
        Epi E;
        E.lim = 60.f * LOG2E;
        E.vo = (unsigned)((r16 * NLOC + q * 4) * 4);
#pragma unroll
        for (int i = 0; i < FA; ++i) {
            const int slot = wm * FA + i, pos = slot - rot < 0 ? slot - rot + gcnt : slot - rot;
            E.row[i] = (const char*)(p.out + (size_t)(mt * GPT + pos) * 16 * NLOC + nt * BN + wn * 64);
            asm volatile("" : "+s"(E.row[i]));
        }
        asm volatile("" : "+s"(E.lim), "+v"(E.vo));
#pragma unroll 1
        for (unsigned t = KA; t < (unsigned)NKS; ++t) part_item<FA, 0, 0, 0>(acc, s0, s1, P, E, t, ldw, relw, wm, myws);
        if (wave == 0) STAMP(12);
        E.c = __builtin_bit_cast(float, __builtin_bit_cast(unsigned, LOG2E) | rfl(lds_ld(ctrl + C_BAD)));
        SCHED_FENCE();
        part_b<0>(acc, s0, s1, P, E, ldw, relw, wm, myws);
        if (wave == 0) STAMP(3);
        {
            const bool bad = rfl(lds_ld(ctrl + C_BAD)) != 0u;
            asm volatile("s_nop 7\n\ts_nop 7" : "+v"(acc[FR - 1][0]), "+v"(acc[FR - 1][1]), "+v"(acc[FR - 1][2]), "+v"(acc[FR - 1][3]));
#pragma unroll
            for (int i = 0; i < FB; ++i) finish(acc[FA + i], 2 * FA + wm * FB + i, bad);
        }
        if (wave == 0) STAMP(4);
    } else if (wave == AG_POLLW) {
        __builtin_amdgcn_fence(__ATOMIC_ACQUIRE, "");
        VM_WAIT(0);
        const unsigned call = rfl(lds_ld(ctrl + C_CALL));
        GAS unsigned* tmo = (GAS unsigned*)(myws + WS_TMO);
        const bool two = (mt * GPT + gcnt - 1) / 32 != (mt * GPT) / 32;
        GAS unsigned long long* xp = (GAS unsigned long long*)(myws + WS_XPROG + (size_t)mt * FSTRIDE) + (lane & (two ? 63 : 31));
        GAS unsigned* wf = (GAS unsigned*)(myws + WS_WFLAG + (size_t)nt * FSTRIDE) + lane;
        unsigned pub = 0, wav = 0, npoll = 0;
        const unsigned long long t0 = rt();
        bool dead = false;
        while (pub < (unsigned)NITEMS || wav < (unsigned)NKS) {
            const unsigned long long xv = __hip_atomic_load(xp, __ATOMIC_RELAXED, __HIP_MEMORY_SCOPE_SYSTEM);
            bool adv = false;
            if (wav < (unsigned)NKS) {
                const unsigned wv = __hip_atomic_load(wf, __ATOMIC_RELAXED, __HIP_MEMORY_SCOPE_AGENT);
                const unsigned long long notready = __ballot((int)(wv - call) < 0);
                const unsigned w2 = notready ? (unsigned)__builtin_ctzll(notready) : 64u;
                if (w2 > wav) { wav = w2; lds_st(ctrl + C_WAV, wav); adv = true; }
            }
            const int xi = max((int)((unsigned)xv - call * 256u), (int)((unsigned)(xv >> 32) - call * 256u));
            int u = xi < 0 ? 0 : xi > NITEMS ? NITEMS : xi;
            u = (lane & 16) ? 255 - u : u;
            u = max(u, __builtin_amdgcn_update_dpp(0, u, 0x121, 15, 15, false));
            u = max(u, __builtin_amdgcn_update_dpp(0, u, 0x122, 15, 15, false));
            u = max(u, __builtin_amdgcn_update_dpp(0, u, 0x124, 15, 15, false));
            u = max(u, __builtin_amdgcn_update_dpp(0, u, 0x128, 15, 15, false));
            const unsigned av = (unsigned)min(max(__builtin_amdgcn_readlane(u, 0), 255 - __builtin_amdgcn_readlane(u, 16)),
                                              max(__builtin_amdgcn_readlane(u, 32), 255 - __builtin_amdgcn_readlane(u, 48)));
            if (av > pub) {
                lds_st(ctrl + C_AVAIL, av);
                pub = av;
                if (av < (unsigned)SIG_FINE && wav >= umin(av, NKS)) __builtin_amdgcn_s_sleep(AG_SLEEP_AFTER);
            } else {
                if (!adv && (++npoll & 31u) == 0u) {
                    if (rfl(__hip_atomic_load(tmo, __ATOMIC_RELAXED, __HIP_MEMORY_SCOPE_AGENT)) == call
                        || rt() - t0 > DEADLINE) {
                        __hip_atomic_store(tmo, call, __ATOMIC_RELAXED, __HIP_MEMORY_SCOPE_AGENT);
                        dead = true;
                        break;
                    }
                }
                if (pub < (unsigned)SIG_FINE) __builtin_amdgcn_s_sleep(AG_SLEEP_POLL); else __builtin_amdgcn_s_sleep(AG_SLEEP_FINE);
            }
        }
        if (dead) lds_st(ctrl + C_BAD, 0x7fc00000u);
        lds_st(ctrl + C_WAV, (unsigned)NKS);
        lds_st(ctrl + C_AVAIL, (unsigned)NITEMS);
    } else {
        const int hh = wave > 5;
        const unsigned call = rfl(lds_ld(ctrl + C_CALL));
        const unsigned lo16 = (unsigned)lane * 16u;
        const char* wsrc = myws + WS_WH + (size_t)nt * NKS * B_STAGE;
        const char* xsrc = myws + WS_XG + (size_t)(call & 1u) * XG_BYTES + (size_t)mt * NKS * X_STEP;
        const char* tab = myws + WS_TAB + (size_t)b * 512;
        const bool full = gcnt == GCAP;
        unsigned long long t0 = 0;
        char* trc = myws;
        int av = 0, cons = 0;
#define GLDN(src_, dst_, n_) do { _Pragma("unroll") for (int _i = 0; _i < (n_); _i += 4) { \
            if (_i + 4 <= (n_)) asm volatile("s_mov_b32 m0, %0\n\ts_waitcnt vmcnt(58)\n\t" \
                         "global_load_lds_dwordx4 %1, %2\n\tglobal_load_lds_dwordx4 %1, %2 offset:1024\n\t" \
                         "global_load_lds_dwordx4 %1, %2 offset:2048\n\tglobal_load_lds_dwordx4 %1, %2 offset:3072" \
                         :: "s"(lbase + (dst_) + _i * 1024), "v"(lo16), "s"((src_) + _i * 1024) : "memory"); \
            else asm volatile("s_mov_b32 m0, %0\n\ts_waitcnt vmcnt(58)\n\t" \
                         "global_load_lds_dwordx4 %1, %2\n\tglobal_load_lds_dwordx4 %1, %2 offset:1024" \
                         :: "s"(lbase + (dst_) + _i * 1024), "v"(lo16), "s"((src_) + _i * 1024) : "memory"); } } while (0)
#define MARK(t_) asm volatile("s_mov_b32 m0, %0\n\ts_mov_b64 exec, 1\n\tglobal_load_lds_dword %1, %2\n\ts_mov_b64 exec, -1" \
                         :: "s"(ldw), "v"(lo16), "s"(tab + (t_) * 4) : "memory")
#define READY(word_, nav_, ncons_) do { if (av < (nav_) || cons < (ncons_)) for (unsigned _sp = 0;;) { u4 _c; unsigned _av; \
            asm volatile("ds_read_b128 %0, %2\n\tds_read_b32 %1, %2 offset:%3\n\ts_waitcnt lgkmcnt(0)" \
                         : "=&v"(_c), "=&v"(_av) : "v"(ctrl), "i"(word_) : "memory"); \
            av = (int)rfl(_av); cons = (int)min4(_c); \
            if (av >= (nav_) && cons >= (ncons_)) break; \
            if (SPIN_OUT(_sp, t0)) break; \
            __builtin_amdgcn_s_sleep(1); } } while (0)
        if (wave == 4) {
            const unsigned ldw = ctrl + C_LD;
            unsigned sb = 0;
            const char* xa = xsrc;
            for (int t = 0; t < KA; ++t) {
                READY(C_AVAIL, t + 1, t - 2);
                TRACE(0, t);
                GLDN(xa, sb, 2 * FA);
                GLDN(xa + 4 * FA * 1024, sb + 2 * FA * 1024, 2 * FB);
                GLDN(xa + 2 * FA * 1024, sb + 2 * FR * 1024, 2 * FA);
                GLDN(xa + (4 * FA + 2 * FB) * 1024, sb + (2 * FR + 2 * FA) * 1024, 2 * FB - 2);
                if (full) GLDN(xa + 34 * 1024, sb + 34 * 1024, 2);
                MARK(t);
                sb = (sb == (unsigned)(2 * X_STEP)) ? 0u : sb + (unsigned)X_STEP;
                xa += X_STEP;
            }
            int xo = 0;
            unsigned long long nd = NEEDS;
#pragma unroll 1
            for (int t = KA; t < NKS; ++t) {
                READY(C_AVAIL, t + 1, KA - 2 + (int)(nd & 15u));
                TRACE(0, t);
                GLDN(xa, xo, 4 * FA);
                MARK(t);
                xo = next_slot(xo, XA, t == NKS - 1 ? XB : XA);
                xa += X_STEP;
                nd >>= 4;
            }
            xa += XA - NAB * X_STEP;
#pragma unroll 1
            for (int t = NKS; t < NITEMS; ++t) {
                READY(C_AVAIL, t + 1, KA - 2 + (int)(nd & 15u));
                TRACE(0, t);
                GLDN(xa, xo, 4 * FB - 2);
                if (full) GLDN(xa + (4 * FB - 2) * 1024, xo + (4 * FB - 2) * 1024, 2);
                MARK(t);
                xo = next_slot(xo, XB, XB);
                xa += X_STEP;
                nd >>= 4;
            }
            VM_WAIT(0);
            STAMP(13);
            return;
        }
        u4 hw[NAB][8];
#define HOLD(j_) asm volatile("global_load_dwordx4 %0, %8, %9\n\tglobal_load_dwordx4 %1, %8, %9 offset:1024\n\t" \
                         "global_load_dwordx4 %2, %8, %9 offset:2048\n\tglobal_load_dwordx4 %3, %8, %9 offset:3072\n\t" \
                         "global_load_dwordx4 %4, %8, %10\n\tglobal_load_dwordx4 %5, %8, %10 offset:1024\n\t" \
                         "global_load_dwordx4 %6, %8, %10 offset:2048\n\tglobal_load_dwordx4 %7, %8, %10 offset:3072\n\t" \
                         "s_waitcnt vmcnt(0)" \
                         : "=&v"(hw[j_][0]), "=&v"(hw[j_][1]), "=&v"(hw[j_][2]), "=&v"(hw[j_][3]), \
                           "=&v"(hw[j_][4]), "=&v"(hw[j_][5]), "=&v"(hw[j_][6]), "=&v"(hw[j_][7]) \
                         : "v"(lo16), "s"(wsrc + (size_t)(KA + (j_)) * B_STAGE + hh * 8192), \
                           "s"(wsrc + (size_t)(KA + (j_)) * B_STAGE + hh * 8192 + 4096) : "memory")
        if (wave == 5) {
            const unsigned ldw = ctrl + C_LD + 4;
            unsigned sc = 0;
#define W_STEP(t_, nav_) READY(C_WAV, nav_, (t_) - 2); TRACE(2, t_); \
                GLDN(wsrc + (size_t)(t_) * B_STAGE, X_RING + sc, 16); MARK(t_); \
                sc = (sc == (unsigned)(2 * B_STAGE)) ? 0u : sc + (unsigned)B_STAGE
            for (int t = 0; t < KH; ++t) { W_STEP(t, t + 1); }
#pragma unroll
            for (int j = 0; j < NAB; ++j) {
                W_STEP(KH + j, NKS);
                HOLD(j);
            }
            for (int t = KH + NAB; t < KA; ++t) { W_STEP(t, t + 1); }
            VM_WAIT(0);
        } else {
            constexpr int CSTEP = 240 / NCONV;
            const int cq = ((idx - 2) / CSTEP) * 8 + xcd;
            if ((idx - 2) % CSTEP == 0 && cq < NCONV) {
                const int kb = lane & 7, nbl = lane >> 3;
                for (int u = cq; u < NKS * 16; u += NCONV) {
                    const int ks = u >> 4, cn = u & 15;
                    const float* src = p.w + (size_t)(ks * BK + kb * 8) * NLOC + cn * BN + nbl * 4;
                    char* dstw = myws + WS_WH + ((size_t)cn * NKS + ks) * B_STAGE;
                    f4 v[4][8];
#pragma unroll
                    for (int g = 0; g < 4; ++g)
#pragma unroll
                        for (int r = 0; r < 8; ++r)
                            v[g][r] = __builtin_nontemporal_load((const f4*)(src + (size_t)r * NLOC + g * 32)) * LOG2E;
#pragma unroll
                    for (int g = 0; g < 4; ++g)
#pragma unroll
                        for (int c = 0; c < 4; ++c) {
                            const int n = g * 32 + nbl * 4 + c;
                            u4 o;
                            o.x = pk(v[g][0][c], v[g][1][c]); o.y = pk(v[g][2][c], v[g][3][c]);
                            o.z = pk(v[g][4][c], v[g][5][c]); o.w = pk(v[g][6][c], v[g][7][c]);
                            st16_dev(dstw + n * 128 + ((kb ^ ((n >> 1) & 7)) * 16), o);
                        }
                    VM_WAIT(0);
                    if (lane == 0)
                        __hip_atomic_store((GAS unsigned*)(myws + WS_WFLAG + (size_t)cn * FSTRIDE + ks * 4), call,
                                           __ATOMIC_RELAXED, __HIP_MEMORY_SCOPE_AGENT);
                }
                STAMP(1);
            }
            for (unsigned sp = 0; rfl(lds_ld(ctrl + C_WAV)) < (unsigned)NKS;) {
                if (SPIN_OUT(sp, t0)) break;
                __builtin_amdgcn_s_sleep(32);
            }
#pragma unroll
            for (int j = 0; j < NAB; ++j) HOLD(j);
            for (unsigned sp = 0; rfl(lds_ld(ctrl + C_CONS)) < (unsigned)(KA - 4);) {
                if (SPIN_OUT(sp, t0)) break;
                __builtin_amdgcn_s_sleep(8);
            }
        }
        int tw = KA;
        unsigned wa = lbase + X_RING + hh * 8192 + lo16;
        const unsigned wend = wa + 3 * B_STAGE, ldw = ctrl + C_LD + 4 + hh * 8;
#pragma unroll 1
        for (int pass = 0; pass < 2; ++pass) {
#pragma unroll
            for (int j = 0; j < NAB; ++j) {
                for (unsigned sp = 0;;) {
                    u4 c;
                    asm volatile("ds_read_b128 %0, %1\n\ts_waitcnt lgkmcnt(0)" : "=&v"(c) : "v"(ctrl) : "memory");
                    if ((int)min4(c) >= tw - 2) break;
                    if (SPIN_OUT(sp, t0)) break;
                    __builtin_amdgcn_s_sleep(AG_HSLEEP);
                }
                if (wave == 5) TRACE(2, tw);
                ++tw;
                asm volatile("ds_write_b128 %0, %1\n\tds_write_b128 %0, %2 offset:1024\n\t"
                             "ds_write_b128 %0, %3 offset:2048\n\tds_write_b128 %0, %4 offset:3072\n\t"
                             "ds_write_b128 %0, %5 offset:4096\n\tds_write_b128 %0, %6 offset:5120\n\t"
                             "ds_write_b128 %0, %7 offset:6144\n\tds_write_b128 %0, %8 offset:7168\n\t"
                             "ds_write_b32 %9, %10"
                             :: "v"(wa), "v"(hw[j][0]), "v"(hw[j][1]), "v"(hw[j][2]), "v"(hw[j][3]),
                                "v"(hw[j][4]), "v"(hw[j][5]), "v"(hw[j][6]), "v"(hw[j][7]), "v"(ldw), "v"(tw) : "memory");
                wa += B_STAGE;
                wa = wa == wend ? wa - 3 * B_STAGE : wa;
            }
        }
        STAMP(9 + wave);
    }
}

}

extern "C" void kernel_launch(int rank, int nranks, ncclComm_t comm,
                              hipStream_t stream,
                              void* const* d_in, const int* in_sizes, int n_in,
                              void* d_out, int out_size,
                              const int* in_dtypes, int out_dtype,
                              void* const* d_out_all,
                              void* const* d_ws_all, size_t ws_bytes) {
    (void)comm; (void)in_dtypes; (void)out_dtype; (void)d_out_all;
    if (nranks != NR || n_in < 2 || (unsigned)rank >= (unsigned)NR || ws_bytes < WS_NEED ||
        in_sizes[0] != MLOC * GK || in_sizes[1] != GK * NLOC || out_size != GM * NLOC) {
        fprintf(stderr, "ag_gemm: unexpected problem shape\n");
        return;
    }
    static bool attr_set = false;
    if (!attr_set) {
        HIP_CHECK(hipFuncSetAttribute((const void*)ag_gemm_kernel,
                                      hipFuncAttributeMaxDynamicSharedMemorySize, LDS_BYTES));
        attr_set = true;
    }
    Params p;
    p.x = (const float*)d_in[0];
    p.w = (const float*)d_in[1];
    p.out = (float*)d_out;
    p.myws = (char*)d_ws_all[rank];
    for (int r = 0; r < NR; ++r) p.ws[r] = (char*)d_ws_all[r];
    p.rank = rank;
    hipLaunchKernelGGL(ag_gemm_kernel, dim3(NWG), dim3(NTHREADS), LDS_BYTES, stream, p);
    HIP_CHECK(hipGetLastError());
}
```

```cpp
#include <hip/hip_runtime.h>
#include <rccl/rccl.h>
#include <cstdio>

#define HIP_CHECK(expr) do { hipError_t _e = (expr); if (_e != hipSuccess) { \
    fprintf(stderr, "HIP error %s at %s:%d\n", hipGetErrorString(_e), __FILE__, __LINE__); \
    return; } } while (0)

namespace {

#define GAS __attribute__((address_space(1)))
#define LAS __attribute__((address_space(3)))
typedef _Float16 h8 __attribute__((ext_vector_type(8)));
typedef _Float16 h2 __attribute__((ext_vector_type(2)));
typedef float f4 __attribute__((ext_vector_type(4)));
typedef unsigned u4 __attribute__((ext_vector_type(4)));
typedef unsigned u2 __attribute__((ext_vector_type(2)));

#define AG_PWIN 16
#define AG_WIN 8
#define AG_SLEEP_POLL 4
#define AG_SLEEP_FINE 1
#define AG_FINE 48
#define AG_NCONV 120
#define AG_KP0 30
#define AG_KPS 6
#define AG_SDELAY 0
#define AG_WNT 0
#define AG_SPI 45

constexpr int NR = 8;
constexpr int GM = 4096, GK = 4096, NLOC = 2048, MLOC = 512;
constexpr int BM = 256, BK = 64;
constexpr int FW = 9;
constexpr int NKS = GK / BK;
constexpr int K1 = 4;
constexpr int KP0 = AG_KP0, KPS = AG_KPS;
constexpr unsigned long long PMASK = (1ull | 1ull << KPS | 1ull << 2 * KPS | 1ull << 3 * KPS) << KP0;
constexpr int NAB = 6;
constexpr int KA = NKS - NAB;
constexpr int NITEMS = KA + 2 * NAB;
constexpr int X_STEP = BM * BK * 2;
constexpr int X_PART = X_STEP / 2;
constexpr int W_STEP = FW * 2048;
constexpr int W_ROW = NLOC * BK * 2;
constexpr int REG = X_PART + 2 * W_STEP;
constexpr int CTRL_OFF = 3 * REG;
constexpr int LDS_BYTES = CTRL_OFF + 512;
constexpr int NWG = 256, NTHREADS = 512;
static_assert(X_STEP + W_STEP <= REG && NAB == 6 && (NKS - K1) * 4 == NWG - 16 && KP0 > 0 && KPS > 1 && KP0 + 3 * KPS < KA - 1, "layout");
__host__ __device__ constexpr int part_w(int j) { return ((KA + j / 2) % 3) * REG + X_PART + (j & 1) * W_STEP; }
constexpr unsigned long long part_w_table() {
    unsigned long long r = 0;
    for (int j = 0; j <= NAB; ++j) r |= (unsigned long long)(part_w(j % NAB) >> 10) << (8 * j);
    return r;
}
constexpr int C_CONS = 0;
constexpr int C_AVAIL = 16;
constexpr int C_WAV = 20;
constexpr int C_STG = 24;
constexpr int C_CALL = 28;
constexpr int C_LD = 32;
constexpr int C_BAD = 40;
constexpr int NCONV = AG_NCONV;
constexpr float LOG2E = 1.4426950408889634f;

constexpr size_t FSTRIDE = 4096 + 256;
constexpr size_t WS_CALL = 0;
constexpr size_t WS_TMO = 49152;
constexpr size_t WS_SFLAG = 53248;
constexpr size_t WS_HINT = 65536;
constexpr size_t WS_WFLAG = 196608;
constexpr size_t WS_TAB = 327680;
constexpr size_t WS_DBG = 524288;
constexpr size_t WS_WH = 1ull << 20;
constexpr size_t WS_XS = 32ull << 20;
constexpr size_t WS_XG = 48ull << 20;
constexpr size_t WS_NEED = 64ull << 20;
static_assert(LDS_BYTES <= 160 * 1024, "LDS");

constexpr unsigned long long DEADLINE = 100000000ull;
constexpr unsigned long long SILENCE = 20000ull;

struct Params {
    const float* x;
    const float* w;
    float* out;
    char* myws;
    char* ws[NR];
    int rank;
};

__device__ __forceinline__ unsigned long long rt() { return __builtin_amdgcn_s_memrealtime(); }

__device__ __forceinline__ void st16_sys(void* p, u4 v) {
    asm volatile("global_store_dwordx4 %0, %1, off sc0 sc1\n\ts_nop 1" :: "v"(p), "v"(v) : "memory");
}
__device__ __forceinline__ void st16_dev(void* p, u4 v) {
    asm volatile("global_store_dwordx4 %0, %1, off sc1\n\ts_nop 1" :: "v"(p), "v"(v) : "memory");
}
__device__ __forceinline__ void st4_sys(void* p, unsigned v) {
    asm volatile("global_store_dword %0, %1, off sc0 sc1\n\ts_nop 1" :: "v"(p), "v"(v) : "memory");
}
__device__ __forceinline__ void post(const char* p, unsigned zero, unsigned v) {
    asm volatile("s_mov_b64 exec, 1\n\tglobal_store_dword %0, %1, %2\n\ts_mov_b64 exec, -1" :: "v"(zero), "v"(v), "s"(p) : "memory");
}
__device__ __forceinline__ void lds_st(unsigned addr, unsigned v) {
    asm volatile("ds_write_b32 %0, %1" :: "v"(addr), "v"(v) : "memory");
}
__device__ __forceinline__ unsigned lds_ld(unsigned addr) {
    unsigned v;
    asm volatile("ds_read_b32 %0, %1\n\ts_waitcnt lgkmcnt(0)" : "=&v"(v) : "v"(addr) : "memory");
    return v;
}
__device__ __forceinline__ unsigned ld_ctr(const void* p) {
    unsigned v;
    asm volatile("global_load_dword %0, %1, off sc1\n\ts_waitcnt vmcnt(0)" : "=&v"(v) : "v"(p) : "memory");
    return (unsigned)__builtin_amdgcn_readfirstlane(v);
}
__device__ __forceinline__ unsigned pk(float a, float b) {
    h2 h; h.x = (_Float16)a; h.y = (_Float16)b;
    return __builtin_bit_cast(unsigned, h);
}
__device__ __forceinline__ u4 pk8(f4 a, f4 b) { return (u4){pk(a[0], a[1]), pk(a[2], a[3]), pk(b[0], b[1]), pk(b[2], b[3])}; }
__device__ __forceinline__ unsigned umin(unsigned a, unsigned b) { return a < b ? a : b; }
__device__ __forceinline__ unsigned rfl(unsigned v) { return (unsigned)__builtin_amdgcn_readfirstlane((int)v); }
__device__ __forceinline__ unsigned min4(u4 c) { return rfl(umin(umin(c.x, c.y), umin(c.z, c.w))); }
__device__ __forceinline__ int row_min(int u) {
    u = min(u, __builtin_amdgcn_update_dpp(0, u, 0x121, 15, 15, false));
    u = min(u, __builtin_amdgcn_update_dpp(0, u, 0x122, 15, 15, false));
    u = min(u, __builtin_amdgcn_update_dpp(0, u, 0x124, 15, 15, false));
    u = min(u, __builtin_amdgcn_update_dpp(0, u, 0x128, 15, 15, false));
    return __builtin_amdgcn_readfirstlane(u);
}
__device__ __forceinline__ void arrivals(const char* sf, unsigned call, int lane, unsigned& staged, bool& stored) {
    u2 sv; unsigned pv;
    asm volatile("global_load_dwordx2 %0, %2, %4 sc0 sc1\n\tglobal_load_dword %1, %3, %4 offset:512 sc0 sc1\n\ts_waitcnt vmcnt(0)"
                 : "=&v"(sv), "=&v"(pv) : "v"(lane * 8), "v"((lane & 15) * 4), "s"(sf) : "memory");
    const unsigned long long miss = __ballot((int)(sv.x - call) < 0 || (int)(sv.y - call) < 0) & ~PMASK;
    staged = miss ? (unsigned)__builtin_ctzll(miss) : (unsigned)NKS;
    stored = __ballot((int)(pv - call) < 0) == 0;
}
#define VM_WAIT(n) asm volatile("s_waitcnt vmcnt(" #n ")" ::: "memory")
#define LGKM0() asm volatile("s_waitcnt lgkmcnt(0)" ::: "memory")
#define SCHED_FENCE() __builtin_amdgcn_sched_barrier(0)
#define SPIN_OUT(sp_, t0_) ((((++(sp_)) & 1023u) == 0u) && ((t0_) == 0 ? ((t0_) = rt(), false) : (rt() - (t0_) > 2 * DEADLINE)))

#define TRACE(k_, t_) do {} while (0)
#define TRDUMP(k_) do {} while (0)
#define TRZERO(k_) do {} while (0)
#define STAMP(i) do {} while (0)
#define STAMPC(i) do {} while (0)

#define DSR(dst_, addr_, off_) asm volatile("ds_read_b128 %0, %1 offset:%2" : "+v"(dst_) : "v"(addr_), "i"(off_) : "memory")
#define WAIT_LD(nx_, nw_) do { unsigned long long _t0 = 0; unsigned _sp = 0; \
        for (;;) { u2 _f; asm volatile("ds_read_b64 %0, %1\n\ts_waitcnt lgkmcnt(0)" : "=&v"(_f) : "v"(ldw) : "memory"); \
            if (rfl(_f.x) >= (unsigned)(nx_) && rfl(_f.y) >= (unsigned)(nw_)) break; \
            if (SPIN_OUT(_sp, _t0)) break; \
            __builtin_amdgcn_s_sleep(1); } } while (0)

struct Epi {
    float t[4];
    f4 o;
    float lim, c;
    unsigned vo;
    const char* row[2];
    unsigned long long last;
};
constexpr int NSLOT = 15 * 2 * FW;
template <int S>
__device__ __forceinline__ void epi_slot(f4 (&acc)[FW][4], Epi& E) {
    constexpr int q = S / 15, op = S % 15, k = (op & 1) * 2;
    if constexpr (S < NSLOT) {
        const f4& y = acc[q % FW][q / FW];
        if constexpr (op < 2)
            asm volatile("v_med3_f32 %0, -%2, %4, -%4\n\tv_med3_f32 %1, -%3, %4, -%4"
                         : "=&v"(E.t[k]), "=&v"(E.t[k + 1]) : "v"(y[k]), "v"(y[k + 1]), "s"(E.lim), "v"(q ? E.o : y));
        else if constexpr (op < 6) asm volatile("v_exp_f32 %0, %0" : "+v"(E.t[op - 2]));
        else if constexpr (op < 8)
            asm volatile("v_fma_f32 %0, %0, %2, %2\n\tv_fma_f32 %1, %1, %2, %2" : "+v"(E.t[k]), "+v"(E.t[k + 1]) : "s"(E.c));
        else if constexpr (op < 12) asm volatile("v_rcp_f32 %0, %0" : "+v"(E.t[op - 8]));
        else if constexpr (op < 14) {
            float o0, o1;
            asm volatile("v_mul_f32 %0, %2, %4\n\tv_mul_f32 %1, %3, %5" : "=&v"(o0), "=v"(o1)
                         : "v"(y[k]), "v"(y[k + 1]), "v"(E.t[k]), "v"(E.t[k + 1]));
            E.o[k] = o0; E.o[k + 1] = o1;
        } else if constexpr (q % FW < FW - 1)
            asm volatile("global_store_dwordx4 %0, %1, %2 offset:%3 sc1" :: "v"(E.vo), "v"(E.o), "s"(E.row[q / FW]), "i"((q % FW) * 64) : "memory");
        else
            asm volatile("s_mov_b64 exec, %4\n\tglobal_store_dwordx4 %0, %1, %2 offset:%3 sc1\n\ts_mov_b64 exec, -1"
                         :: "v"(E.vo), "v"(E.o), "s"(E.row[q / FW]), "i"((q % FW) * 64), "s"(E.last) : "memory");
        if constexpr (S == NSLOT - 1) asm volatile("s_nop 1" :: "v"(E.o));
    }
}
constexpr int slots_before(int g, int n) {
    constexpr int ORD[18] = {15, 11, 7, 3, 13, 9, 5, 16, 14, 12, 10, 8, 6, 4, 2, 17, 1, 0};
    int c = 0;
    for (int i = 0; i < g; ++i) {
        int rank = 0;
        for (int j = 0; j < 18; ++j) if (ORD[j] == i % 18) rank = 2 * j + i / 18;
        c += 1 + (rank < n - 36);
    }
    return c < n ? c : n;
}
template <int S0, int NS, int G>
__device__ __forceinline__ void epi_gap(f4 (&acc)[FW][4], Epi& E) {
    if constexpr (NS > 0) {
        constexpr int b = slots_before(G, NS), e = slots_before(G + 1, NS);
        if constexpr (e > b) epi_slot<S0 + b>(acc, E);
        if constexpr (e > b + 1) epi_slot<S0 + b + 1>(acc, E);
    }
}
constexpr int EXT[2][9] = {{0, 1, 1, 0, 0, 0, 0, 0, 0}, {2, 0, 0, 2, 0, 0, 0, 0, 0}};
constexpr int ops_after(int h, int k, int r) {
    int n = 0;
    if (k == 0) { for (int i = h ? 4 : 3; i <= r; ++i) n += 1 + EXT[h][i]; return n; }
    if (k == 8) { n = EXT[h][0]; for (int i = 1; i <= r; ++i) n += 1 + EXT[h][i]; return n; }
    n = EXT[1 - h][k + 1];
    for (int i = k + 2; i < 9; ++i) n += 1 + EXT[1 - h][i];
    for (int i = 0; i <= r; ++i) n += 1 + EXT[h][i];
    return n;
}
struct Pos { unsigned pa, qa, pb, qb, rv, wl; u4 f; };
template <int J0, int S0, int NS, int H, int G>
__device__ __forceinline__ void half_step(f4 (&acc)[FW][4], h8 (&a)[FW], h8 (&bc)[4], h8 (&bn)[4], Pos& P, Epi& E,
                                          unsigned nx, int da, int db, unsigned ldw, unsigned relw) {
    if constexpr (G < 2 * FW) {
        constexpr int r = G >> 1;
        f4& c = acc[r][J0 + (G & 1)];
        if (J0) asm volatile("v_mfma_f32_16x16x32_f16 %0, %1, %2, %0" : "+v"(c) : "v"(a[r]), "v"(bc[G & 1]));
        else c = __builtin_amdgcn_mfma_f32_16x16x32_f16(a[r], bc[G & 1], c, 0, 0, 0);
        SCHED_FENCE();
        unsigned& ca = H ? P.qa : P.pa; unsigned& na = H ? P.pa : P.qa;
        unsigned& cb = H ? P.qb : P.pb; unsigned& nb = H ? P.pb : P.qb;
        if constexpr (G == 0) DSR(a[FW - 1], ca, (FW - 1) * 2048);
        else if constexpr (!(G & 1)) DSR(a[r - 1], na, (r - 1) * 2048);
        if (H == 0 && G == 3) DSR(bn[0], nb, 0);
        if (H == 0 && G == 5) DSR(bn[1], nb, 2048);
        if (H == 1 && G == 1) {
            lds_st(relw, P.rv);
            asm volatile("ds_read_b128 %0, %1" : "=v"(P.f) : "v"(ldw) : "memory");
        }
        if (H == 1 && G == 7) {
            asm volatile("s_waitcnt lgkmcnt(3)" : "+v"(P.f) :: "memory");
            P.wl = rfl(P.f.y);
            if (rfl(P.f.x) < nx) WAIT_LD(nx, 0);
            SCHED_FENCE();
            DSR(bn[0], nb, 0);
            DSR(bn[1], nb, 2048);
        }
        if (G == 3) { ca += da; asm volatile("" : "+v"(ca)); }
        if (G == 11) { cb += db; asm volatile("" : "+v"(cb)); }
        if (H == 1 && G == 15) { P.rv += 1u; asm volatile("" : "+v"(P.rv)); }
        epi_gap<S0, NS, H * 2 * FW + G>(acc, E);
        if constexpr ((G & 3) == 1 && r < FW - 1)
            asm volatile("s_waitcnt lgkmcnt(%2)" : "+v"(a[r + 1]), "+v"(a[r + 2]) : "i"(ops_after(H, r + 2, r)) : "memory");
        if constexpr (G == 2 * FW - 1)
            asm volatile("s_waitcnt lgkmcnt(%3)" : "+v"(a[0]), "+v"(bn[0]), "+v"(bn[1]) : "i"(ops_after(H, 0, r)) : "memory");
        SCHED_FENCE();
        half_step<J0, S0, NS, H, G + 1>(acc, a, bc, bn, P, E, nx, da, db, ldw, relw);
    }
}
template <int J0, int S0, int NS>
__device__ __forceinline__ void part_item(f4 (&acc)[FW][4], h8 (&a)[FW], h8 (&b0)[4], h8 (&b1)[4], Pos& P, Epi& E, unsigned t,
                                          int da, int db, unsigned ldw, unsigned relw) {
    const unsigned nx = umin(t + 2u, NITEMS);
    half_step<J0, S0, NS, 0, 0>(acc, a, b0, b1, P, E, nx, da, db, ldw, relw);
    if (J0 == 0 && P.wl < nx) WAIT_LD(0, nx);
    SCHED_FENCE();
    half_step<J0, S0, NS, 1, 0>(acc, a, b1, b0, P, E, nx, da, db, ldw, relw);
}
template <int I>
__device__ __forceinline__ void part_b(f4 (&acc)[FW][4], h8 (&a)[FW], h8 (&b0)[4], h8 (&b1)[4], Pos& P, Epi& E,
                                       unsigned ldw, unsigned relw) {
    if constexpr (I < NAB) {
        constexpr int S0 = I * AG_SPI, NS = S0 >= NSLOT ? 0 : NSLOT - S0 < AG_SPI ? NSLOT - S0 : AG_SPI;
        part_item<2, S0, NS>(acc, a, b0, b1, P, E, NKS + I, part_w((I + 1) % NAB) - part_w(I),
                             (NKS + I) % 3 == 2 ? -2 * REG : REG, ldw, relw);
        part_b<I + 1>(acc, a, b0, b1, P, E, ldw, relw);
    }
}
static_assert(AG_SPI * NAB >= NSLOT && AG_SPI <= 72, "slots");

__global__ void __launch_bounds__(NTHREADS, 2) ag_gemm_kernel(Params p) {
    extern __shared__ __attribute__((aligned(16))) unsigned char smem[];
    LAS unsigned char* lds = (LAS unsigned char*)smem;
    const int tid = threadIdx.x;
    const int wave = __builtin_amdgcn_readfirstlane(tid >> 6);
    const int lane = tid & 63;
    const int b = blockIdx.x;
    const int src = b & 7, idx = b >> 3;
    char* myws = p.myws;
    GAS unsigned* cctr = (GAS unsigned*)(myws + WS_CALL + (size_t)b * 128);
    GAS unsigned* tmo = (GAS unsigned*)(myws + WS_TMO);
    const unsigned lbase = (unsigned)(size_t)lds;
    const unsigned ctrl = lbase + CTRL_OFF;
    const unsigned lo16 = (unsigned)lane * 16u;
    const int r8 = lane >> 3, c8 = lane & 7;
    const unsigned img = (unsigned)(r8 * 128 + ((c8 ^ (r8 >> 1)) * 16));
    const char* srcws = p.ws[0];
#pragma unroll
    for (int r = 1; r < NR; ++r) if (src == r) srcws = p.ws[r];

    if (idx < 2) {
        const int cc = src * 2 + idx;
        const char* hint = myws + WS_HINT + (size_t)cc * FSTRIDE;
        if (wave < 2) {
            STAMP(20 + wave);
            f4 v[K1][4];
            unsigned cv;
            {
                const unsigned s0 = (unsigned)((cc * 32 + wave * 16 + r8) * GK + c8 * 8) * 4u;
                const unsigned s1 = (unsigned)((cc * 32 + wave * 16 + 8 + r8) * GK + (c8 ^ 4) * 8) * 4u;
#pragma unroll
                for (int e = 0; e < K1; ++e)
                    asm volatile("global_load_dwordx4 %0, %4, %6\n\tglobal_load_dwordx4 %1, %4, %6 offset:16\n\t"
                                 "global_load_dwordx4 %2, %5, %6\n\tglobal_load_dwordx4 %3, %5, %6 offset:16"
                                 : "=&v"(v[e][0]), "=&v"(v[e][1]), "=&v"(v[e][2]), "=&v"(v[e][3])
                                 : "v"(s0), "v"(s1), "s"(p.x + (KP0 + e * KPS) * BK) : "memory");
                asm volatile("global_load_dword %0, %1, off sc1" : "=&v"(cv) : "v"(cctr) : "memory");
            }
            const char* dst[NR];
#pragma unroll
            for (int j = 0; j < NR; ++j) {
                const int r = (p.rank + 1 + j + cc + 4 * wave) & 7;
                dst[j] = p.ws[0];
#pragma unroll
                for (int q = 1; q < NR; ++q) if (r == q) dst[j] = p.ws[q];
                dst[j] += WS_XG + (size_t)((p.rank * 2 + (cc >> 3)) * K1) * X_STEP + (size_t)((cc & 7) * 2 + wave) * 2048;
                asm volatile("" : "+s"(dst[j]));
            }
            asm volatile("s_waitcnt vmcnt(0)" : "+v"(cv), "+v"(v[0][0]), "+v"(v[0][1]), "+v"(v[0][2]), "+v"(v[0][3]),
                         "+v"(v[1][0]), "+v"(v[1][1]), "+v"(v[1][2]), "+v"(v[1][3]), "+v"(v[2][0]), "+v"(v[2][1]), "+v"(v[2][2]),
                         "+v"(v[2][3]), "+v"(v[3][0]), "+v"(v[3][1]), "+v"(v[3][2]), "+v"(v[3][3]) :: "memory");
            const unsigned call = rfl(cv) + 1u;
            const unsigned vo = img + (call & 1u) * (unsigned)(NR * 2 * K1 * X_STEP);
            char* flag = myws + WS_SFLAG + (size_t)(cc >> 3) * FSTRIDE + 512 + ((cc & 7) * 2 + wave) * 4;
            if (wave == 0) STAMP(5);
            constexpr int W = AG_PWIN;
            static_assert(W >= 2 && W <= 32 && W % 2 == 0, "window");
#pragma unroll
            for (int e = 0; e < K1; ++e) {
                const u4 o0 = pk8(v[e][0], v[e][1]), o1 = pk8(v[e][2], v[e][3]);
                const unsigned ve = vo + (unsigned)(e * X_STEP);
#pragma unroll
                for (int j = 0; j < NR; ++j) {
                    if (e == 0 && j == 4) __builtin_amdgcn_s_barrier();
                    asm volatile("s_waitcnt vmcnt(%0)" :: "i"(W - 2) : "memory");
                    asm volatile("global_store_dwordx4 %0, %1, %3 sc0 sc1\n\tglobal_store_dwordx4 %0, %2, %3 offset:1024 sc0 sc1\n\ts_nop 1"
                                 :: "v"(ve), "v"(o0), "v"(o1), "s"(dst[j]) : "memory");
                }
            }
            VM_WAIT(0);
            if (lane == 0) st4_sys(flag, call);
            STAMP(14 + wave);
            return;
        }
        if (wave == 6) lds_st(ctrl + C_AVAIL, 0u);
        const unsigned call = ld_ctr((const void*)cctr) + 1u;
        LGKM0();
        __builtin_amdgcn_s_barrier();
        asm volatile("" ::: "memory");
        unsigned long long t0 = 0;
        if (wave < 6) {
            const int q = wave - 2;
            const char* hq = hint + q * 4;
            const char* xa = srcws + WS_XS + (size_t)((call & 1u) * 2u + (unsigned)idx) * NKS * X_STEP;
            unsigned hv = call * 256u + 1u, zero = 0u;
            asm volatile("s_mov_b32 m0, %2" : "+v"(hv), "+v"(zero) : "s"(lbase + q * 4096) : "memory");
            int av = 0;
#define STAGED(n_) do { if (av < (n_)) for (unsigned _sp = 0;;) { av = (int)rfl(lds_ld(ctrl + C_AVAIL)); \
                if (av >= (n_) || SPIN_OUT(_sp, t0)) break; \
                __builtin_amdgcn_s_sleep(1); } } while (0)
#define PULL4(a_) asm volatile("s_waitcnt vmcnt(%2)\n\tglobal_load_lds_dwordx4 %0, %1\n\t" \
                         "s_waitcnt vmcnt(%2)\n\tglobal_load_lds_dwordx4 %0, %1 offset:1024\n\t" \
                         "s_waitcnt vmcnt(%2)\n\tglobal_load_lds_dwordx4 %0, %1 offset:2048\n\t" \
                         "s_waitcnt vmcnt(%2)\n\tglobal_load_lds_dwordx4 %0, %1 offset:3072" \
                         :: "v"(lo16), "s"(a_), "i"(AG_WIN - 1) : "memory")
            if (src == p.rank) {
                while (av < NKS) {
                    const int need = av + 1;
                    STAGED(need);
                    if (av < need) break;
                    post(hq, zero, call * 256u + (unsigned)(av >= NKS ? NITEMS : min(av, KA)));
                }
                STAMP(4 + wave);
                return;
            }
#pragma unroll 1
            for (int t = 0; t < KA; ++t) {
                STAGED(t + 1);
                if (q == 0) TRACE(0, t);
                PULL4(xa + q * 8192);
                PULL4(xa + q * 8192 + 4096);
                if ((PMASK >> 1 >> t) & 1) { hv += 1u; xa += X_STEP; ++t; }
                post(hq, zero, hv);
                hv += 1u;
                xa += X_STEP;
            }
            STAGED(NKS);
            for (int t = KA; t < NITEMS; ++t) {
                if (q == 0) TRACE(0, t);
                PULL4(xa + q * 4096);
                post(hq, zero, hv);
                hv += 1u;
                xa += t == NKS - 1 ? X_PART - (NAB - 1) * X_STEP : X_STEP;
            }
            VM_WAIT(0);
            STAMP(4 + wave);
            if (q == 0) TRDUMP(0);
        } else if (wave == 6) {
            if (lane == 0) __hip_atomic_store(cctr, call, __ATOMIC_RELAXED, __HIP_MEMORY_SCOPE_AGENT);
            __builtin_amdgcn_fence(__ATOMIC_ACQUIRE, "");
            VM_WAIT(0);
            const char* sf = srcws + WS_SFLAG + (size_t)idx * FSTRIDE;
            unsigned staged = 0, npoll = 0;
            bool stored = false;
            post(hint + 16, 0u, call * 256u + KP0);
            t0 = rt();
            while (staged < (unsigned)NKS || !stored) {
                unsigned s2; bool p2;
                arrivals(sf, call, lane, s2, p2);
                if (s2 > staged) { if (!staged) STAMP(17); staged = s2; lds_st(ctrl + C_AVAIL, staged); if (s2 == NKS) STAMP(10); }
                if (p2 && !stored) { stored = true; post(hint + 16, 0u, call * 256u + NITEMS); }
                if ((++npoll & 31u) == 0u && (rfl(__hip_atomic_load(tmo, __ATOMIC_RELAXED, __HIP_MEMORY_SCOPE_AGENT)) == call
                                              || rt() - t0 > DEADLINE)) {
                    __hip_atomic_store(tmo, call, __ATOMIC_RELAXED, __HIP_MEMORY_SCOPE_AGENT);
                    post(hint + 16, 0u, call * 256u);
                    VM_WAIT(0);
                    lds_st(ctrl + C_AVAIL, (unsigned)NKS);
                    break;
                }
                if (npoll < 64u && staged < (unsigned)NKS) __builtin_amdgcn_s_sleep(2); else __builtin_amdgcn_s_sleep(32);
            }
            STAMP(11);
        }
        return;
    }

    const int mt = idx & 1, nt = (idx - 2) >> 1;
    const int f0 = (17 * nt) >> 1;
    f4 sv[4][2];
    const int g = b - 16;
    int sk = g >> 2;
#pragma unroll
    for (int e = 0; e < K1; ++e) sk += sk >= KP0 + e * KPS;
    if (wave < 4) {
        if (wave == 0) STAMP(0);
        const float* s0 = p.x + (size_t)((g & 3) * 128 + wave * 32 + r8) * GK + sk * BK;
        if (AG_SDELAY) __builtin_amdgcn_s_sleep(AG_SDELAY);
#pragma unroll
        for (int i = 0; i < 4; ++i)
            asm volatile("global_load_dwordx4 %0, %2, off\n\tglobal_load_dwordx4 %1, %2, off offset:16"
                         : "=&v"(sv[i][0]), "=&v"(sv[i][1]) : "v"(s0 + (size_t)i * 8 * GK + (c8 ^ ((i & 1) * 4)) * 8) : "memory");
    }
    if (wave == 7) {
        *(u2*)(myws + WS_TAB + (size_t)b * 512 + lane * 8) = (u2){2u * lane + 1u, 2u * lane + 2u};
        const unsigned cl = ld_ctr((const void*)cctr) + 1u;
        if (lane == 0) __hip_atomic_store(cctr, cl, __ATOMIC_RELAXED, __HIP_MEMORY_SCOPE_AGENT);
        lds_st(ctrl + lane * 4, lane == C_CALL / 4 ? cl : 0u);
        LGKM0();
    }
    __builtin_amdgcn_s_barrier();
    asm volatile("" ::: "memory");
    const unsigned call = rfl(lds_ld(ctrl + C_CALL));

    if (wave < 4) {
        {
            char* d0 = myws + WS_XS + ((size_t)((call & 1u) * 2u + (unsigned)((g >> 1) & 1)) * NKS + (size_t)sk) * X_STEP
                       + (size_t)((g & 1) * X_PART + wave * 4096) + img;
            asm volatile("s_waitcnt vmcnt(0)" : "+v"(sv[0][0]), "+v"(sv[0][1]), "+v"(sv[1][0]), "+v"(sv[1][1]),
                         "+v"(sv[2][0]), "+v"(sv[2][1]), "+v"(sv[3][0]), "+v"(sv[3][1]) :: "memory");
#pragma unroll
            for (int i = 0; i < 4; ++i) st16_sys(d0 + i * 1024, pk8(sv[i][0], sv[i][1]));
            unsigned n = 0;
            if (lane == 0) {
                asm volatile("s_waitcnt vmcnt(0)\n\tds_add_rtn_u32 %0, %1, %2\n\ts_waitcnt lgkmcnt(0)"
                             : "=&v"(n) : "v"(ctrl + C_STG), "v"(1u) : "memory");
                if (n == 3u) st4_sys(myws + WS_SFLAG + (size_t)((g >> 1) & 1) * FSTRIDE + sk * 8 + (g & 1) * 4, call);
            }
            if (wave == 0) STAMP(16);
        }
        const int r16 = lane & 15, q = lane >> 4;
        const unsigned sw = (unsigned)(r16 * 128 + (q ^ (r16 >> 1)) * 16);
        const unsigned ldw = ctrl + C_LD;
        const unsigned relw = ctrl + C_CONS + wave * 4;
        const bool dup = !(nt & 1) && nt != 14;
        f4 acc[FW][4];
#pragma unroll
        for (int i = 0; i < FW; ++i)
#pragma unroll
            for (int j = 0; j < 4; ++j) acc[i][j] = (f4){0.f, 0.f, 0.f, 0.f};
        const unsigned offA = lbase + sw;
        const unsigned offB = lbase + (unsigned)(wave * 4096) + sw;
        h8 a[FW], b0[4], b1[4];
#pragma unroll
        for (int i = 0; i < FW; ++i) a[i] = (h8)(_Float16)0.f;
#pragma unroll
        for (int j = 0; j < 4; ++j) { b0[j] = (h8)(_Float16)0.f; b1[j] = (h8)(_Float16)0.f; }
#define MF(i_, j_, B_, A_) do { acc[i_][j_] = __builtin_amdgcn_mfma_f32_16x16x32_f16(A_, B_, acc[i_][j_], 0, 0, 0); SCHED_FENCE(); } while (0)
#define FILL(x_) do { x_; SCHED_FENCE(); } while (0)
#define ROW(i_, BC_, A_, F0_, F1_, F2_) do { \
        MF(i_, 0, BC_[0], A_); FILL(F0_); MF(i_, 1, BC_[1], A_); FILL(F1_); \
        MF(i_, 2, BC_[2], A_); FILL(F2_); MF(i_, 3, BC_[3], A_); } while (0)
#define NOP_ do {} while (0)
#define XO(j_) (((j_) & 1) * 2048 + ((j_) >> 1) * X_PART)
#define WA(i_, cnt_) asm volatile("s_waitcnt lgkmcnt(%1)" : "+v"(a[i_]) : "i"(cnt_) : "memory")
#define WAB(BC_, cnt_) asm volatile("s_waitcnt lgkmcnt(%8)" : "+v"(a[0]), "+v"(a[1]), "+v"(a[2]), "+v"(a[3]), \
                                    "+v"(BC_[0]), "+v"(BC_[1]), "+v"(BC_[2]), "+v"(BC_[3]) : "i"(cnt_) : "memory")
        float* orow = p.out + (size_t)(src * MLOC + mt * BM + wave * 32) * NLOC + f0 * 16;
        Epi E;
        E.lim = 60.f * LOG2E;
        E.vo = (unsigned)((r16 * NLOC + q * 4) * 4);
        E.row[0] = (const char*)orow; E.row[1] = (const char*)(orow + 16 * NLOC);
        E.last = (unsigned long long)rfl(dup ? 0u : ~0u) * 0x100000001ull;
        asm volatile("" : "+s"(E.row[0]));
        asm volatile("" : "+s"(E.row[1]));
        asm volatile("" : "+s"(E.last));
        asm volatile("" : "+s"(E.lim));
        asm volatile("" : "+v"(E.vo));
        WAIT_LD(1, 1);
        if (wave == 0) { STAMP(18); STAMPC(30); }
#pragma unroll
        for (int j = 0; j < 4; ++j) DSR(b0[j], offB, XO(j));
#pragma unroll
        for (int i = 0; i < FW - 1; ++i) DSR(a[i], offA, X_STEP + i * 2048);
        WAB(b0, 0);
        __builtin_amdgcn_s_waitcnt(0xc07f);
        SCHED_FENCE();
        unsigned sb = 0, wl = 0;
        for (int t = 0; t < KA; ++t) {
            u2 f;
            {
                const unsigned ca = offA + sb + X_STEP, na = ca ^ 64u, nb = (offB + sb) ^ 64u;
                ROW(0, b0, a[0], DSR(a[8], ca, 8 * 2048), NOP_, NOP_);
                ROW(1, b0, a[1], DSR(a[0], na, 0), DSR(b1[0], nb, XO(0)), NOP_);
                ROW(2, b0, a[2], DSR(a[1], na, 2048), DSR(b1[1], nb, XO(1)), NOP_);
                ROW(3, b0, a[3], DSR(a[2], na, 2 * 2048), DSR(b1[2], nb, XO(2)), WA(4, 10));
                ROW(4, b0, a[4], DSR(a[3], na, 3 * 2048), DSR(b1[3], nb, XO(3)), WA(5, 11));
                ROW(5, b0, a[5], DSR(a[4], na, 4 * 2048),
                    asm volatile("ds_read_b64 %0, %1" : "=v"(f) : "v"(ldw) : "memory"), WA(6, 11));
                ROW(6, b0, a[6], DSR(a[5], na, 5 * 2048), NOP_, WA(7, 11));
                ROW(7, b0, a[7], DSR(a[6], na, 6 * 2048), NOP_, WA(8, 11));
                ROW(8, b0, a[8], DSR(a[7], na, 7 * 2048), NOP_, WAB(b1, 4));
            }
            {
                const unsigned ca = (offA + sb + X_STEP) ^ 64u;
                sb = (sb == (unsigned)(2 * REG)) ? 0u : sb + (unsigned)REG;
                const unsigned na = offA + (t == KA - 1 ? (unsigned)part_w(0) : sb + X_STEP), nb = offB + sb;
                ROW(0, b1, a[0], DSR(a[8], ca, 8 * 2048), NOP_,
                    asm volatile("s_waitcnt lgkmcnt(4)" : "+v"(f) :: "memory"));
                wl = rfl(f.y);
                if (umin(rfl(f.x), wl) < (unsigned)(t + 2)) {
                    LGKM0(); lds_st(relw, (unsigned)(t + 1)); WAIT_LD(t + 2, t + 2);
                }
                SCHED_FENCE();
                ROW(1, b1, a[1], DSR(a[0], na, 0), DSR(b0[0], nb, XO(0)), NOP_);
                ROW(2, b1, a[2], DSR(a[1], na, 2048), DSR(b0[1], nb, XO(1)), NOP_);
                ROW(3, b1, a[3], DSR(a[2], na, 2 * 2048), DSR(b0[2], nb, XO(2)),
                    WA(4, 6); lds_st(relw, (unsigned)(t + 1)));
                ROW(4, b1, a[4], DSR(a[3], na, 3 * 2048), DSR(b0[3], nb, XO(3)), WA(5, 11));
                ROW(5, b1, a[5], DSR(a[4], na, 4 * 2048), NOP_, WA(6, 11));
                ROW(6, b1, a[6], DSR(a[5], na, 5 * 2048), NOP_, WA(7, 11));
                ROW(7, b1, a[7], DSR(a[6], na, 6 * 2048), NOP_, WA(8, 11));
                ROW(8, b1, a[8], DSR(a[7], na, 7 * 2048), NOP_, WAB(b0, 4));
            }
        }
        if (wave == 0) { STAMP(2); STAMPC(31); }
        Pos P;
        P.pa = offA + (unsigned)part_w(0); P.pb = offB + (unsigned)((KA % 3) * REG);
        P.qa = P.pa ^ 64u; P.qb = P.pb ^ 64u; P.rv = KA + 1; P.wl = wl;
        {
            unsigned long long tw = part_w_table();
            int xr = KA % 3;
#pragma unroll 1
            for (unsigned t = KA; t < (unsigned)NKS; ++t) {
                const int da = ((int)((tw >> 8) & 255u) - (int)(tw & 255u)) * 1024, db = xr == 2 ? -2 * REG : REG;
                tw >>= 8; xr = xr == 2 ? 0 : xr + 1;
                part_item<0, 0, 0>(acc, a, b0, b1, P, E, t, da, db, ldw, relw);
            }
        }
        if (wave == 0) STAMP(12);
        E.c = __builtin_bit_cast(float, __builtin_bit_cast(unsigned, LOG2E) | rfl(P.f.z));
        SCHED_FENCE();
        part_b<0>(acc, a, b0, b1, P, E, ldw, relw);
        if (wave == 0) STAMP(3);
        {
            const float c = __builtin_bit_cast(float, __builtin_bit_cast(unsigned, LOG2E) | rfl(P.f.z));
            asm volatile("s_nop 7\n\ts_nop 7\n\ts_waitcnt lgkmcnt(0)" : "+v"(acc[FW - 1][2]), "+v"(acc[FW - 1][3]), "+v"(a[1]), "+v"(a[2]), "+v"(a[3]),
                         "+v"(a[4]), "+v"(a[5]), "+v"(a[6]), "+v"(a[7]));
            float* ob = (float*)(E.row[0] + (size_t)128 * NLOC * 4 + E.vo);
#pragma unroll
            for (int j = 0; j < 2; ++j)
#pragma unroll
                for (int i = 0; i < FW; ++i) {
                    if (i == FW - 1 && !E.last) continue;
                    const f4 y = acc[i][2 + j];
                    f4 o;
#pragma unroll
                    for (int e = 0; e < 4; ++e) {
                        const float z = __builtin_amdgcn_fmed3f(y[e], -60.f * LOG2E, 60.f * LOG2E);
                        const float ex = __builtin_amdgcn_exp2f(-z);
                        o[e] = y[e] * __builtin_amdgcn_rcpf(__builtin_fmaf(ex, c, c));
                    }
                    st16_dev(ob + (size_t)j * 16 * NLOC + i * 16, __builtin_bit_cast(u4, o));
                }
        }
        if (wave == 0) STAMP(4);
    } else if (wave == 7) {
        __builtin_amdgcn_fence(__ATOMIC_ACQUIRE, "");
        VM_WAIT(0);
        GAS unsigned* hp = (GAS unsigned*)(myws + WS_HINT + (size_t)(src * 2 + mt) * FSTRIDE) + c8;
        GAS unsigned* wf0 = (GAS unsigned*)(myws + WS_WFLAG + (size_t)(f0 >> 3) * FSTRIDE) + lane;
        GAS unsigned* wf1 = (GAS unsigned*)(myws + WS_WFLAG + (size_t)((f0 + FW - 1) >> 3) * FSTRIDE) + lane;
        unsigned pub = 0, wav = 0, npoll = 0;
        TRZERO(1);
        const unsigned long long t0 = rt();
        unsigned long long tall = 0;
        bool dead = false;
        while (pub < (unsigned)NITEMS || wav < (unsigned)NKS) {
            const unsigned hv = __hip_atomic_load(hp, __ATOMIC_RELAXED, __HIP_MEMORY_SCOPE_AGENT);
            bool adv = false;
            if (wav < (unsigned)NKS) {
                const unsigned w0 = __hip_atomic_load(wf0, __ATOMIC_RELAXED, __HIP_MEMORY_SCOPE_AGENT);
                const unsigned w1 = __hip_atomic_load(wf1, __ATOMIC_RELAXED, __HIP_MEMORY_SCOPE_AGENT);
                const unsigned long long notready = __ballot((int)(w0 - call) < 0 || (int)(w1 - call) < 0);
                const unsigned w2 = notready ? (unsigned)__builtin_ctzll(notready) : 64u;
                if (w2 > wav) { wav = w2; lds_st(ctrl + C_WAV, wav); adv = true; }
            }
            const int d = (int)(hv - call * 256u);
            const unsigned av = (unsigned)row_min(c8 < 5 ? max(min(d, NITEMS), 0) : NITEMS);
            if (av > pub) {
                lds_st(ctrl + C_AVAIL, av);
                pub = av;
                tall = 0;
                TRACE(1, av - 1);
            } else if (!adv && (++npoll & 63u) == 0u) {
                if (rt() - t0 > DEADLINE) { dead = true; break; }
                unsigned staged; bool stored;
                arrivals(srcws + WS_SFLAG + (size_t)mt * FSTRIDE, call, lane, staged, stored);
                if (staged == (unsigned)NKS && stored) {
                    if (tall == 0) tall = rt();
                    else if (rt() - tall > SILENCE) { pub = NITEMS; lds_st(ctrl + C_AVAIL, pub); }
                } else if (rfl(__hip_atomic_load(tmo, __ATOMIC_RELAXED, __HIP_MEMORY_SCOPE_AGENT)) == call) { dead = true; break; }
            }
            if (pub < (unsigned)AG_FINE) __builtin_amdgcn_s_sleep(AG_SLEEP_POLL); else __builtin_amdgcn_s_sleep(AG_SLEEP_FINE);
        }
        if (dead) {
            __hip_atomic_store(tmo, call, __ATOMIC_RELAXED, __HIP_MEMORY_SCOPE_AGENT);
            lds_st(ctrl + C_BAD, 0x7fc00000u);
        }
        lds_st(ctrl + C_WAV, (unsigned)NKS);
        lds_st(ctrl + C_AVAIL, (unsigned)NITEMS);
        TRDUMP(1);
    } else if (wave == 6) {
        constexpr int CSTEP = 240 / NCONV;
        const int cq = ((idx - 2) / CSTEP) * 8 + src;
        if ((idx - 2) % CSTEP == 0 && cq < NCONV) {
            for (int u = cq; u < NKS * 16; u += NCONV) {
                const int ks = u >> 4, cn = u & 15;
                const float* sw = p.w + (size_t)(ks * BK + c8 * 8) * NLOC + cn * 128 + r8 * 4;
                char* dstw = myws + WS_WH + (size_t)ks * W_ROW + (size_t)cn * 16384;
                f4 v[4][8];
#pragma unroll
                for (int g = 0; g < 4; ++g)
#pragma unroll
                    for (int r = 0; r < 8; ++r)
                        v[g][r] = __builtin_nontemporal_load((const f4*)(sw + (size_t)r * NLOC + g * 32));
                SCHED_FENCE();
#pragma unroll
                for (int g = 0; g < 4; ++g)
#pragma unroll
                    for (int r = 0; r < 8; ++r) v[g][r] *= LOG2E;
#pragma unroll
                for (int g = 0; g < 4; ++g)
#pragma unroll
                    for (int c = 0; c < 4; ++c) {
                        const int n = g * 32 + r8 * 4 + c;
                        u4 o;
                        o.x = pk(v[g][0][c], v[g][1][c]); o.y = pk(v[g][2][c], v[g][3][c]);
                        o.z = pk(v[g][4][c], v[g][5][c]); o.w = pk(v[g][6][c], v[g][7][c]);
                        st16_dev(dstw + n * 128 + ((c8 ^ ((n >> 1) & 7)) * 16), o);
                    }
                VM_WAIT(0);
                if (lane == 0)
                    __hip_atomic_store((GAS unsigned*)(myws + WS_WFLAG + (size_t)cn * FSTRIDE + ks * 4), call,
                                       __ATOMIC_RELAXED, __HIP_MEMORY_SCOPE_AGENT);
            }
            STAMP(1);
        }
    } else {
        const char* tab = myws + WS_TAB + (size_t)b * 512;
        const unsigned ldw = ctrl + C_LD + (wave - 4) * 4;
        unsigned long long t0 = 0;
        int av = 0, cons = 0;
        unsigned sb = 0;
#define GLDM(src_, dst_, n_, m_) do { _Pragma("unroll") for (int _i = 0; _i < (n_); _i += 4) { \
            if (_i + 4 <= (n_)) asm volatile("s_mov_b32 m0, %0\n\ts_waitcnt vmcnt(58)\n\t" \
                         "global_load_lds_dwordx4 %1, %2" m_ "\n\tglobal_load_lds_dwordx4 %1, %2 offset:1024" m_ "\n\t" \
                         "global_load_lds_dwordx4 %1, %2 offset:2048" m_ "\n\tglobal_load_lds_dwordx4 %1, %2 offset:3072" m_ \
                         :: "s"(lbase + (dst_) + _i * 1024), "v"(lo16), "s"((src_) + _i * 1024) : "memory"); \
            else asm volatile("s_mov_b32 m0, %0\n\ts_waitcnt vmcnt(58)\n\t" \
                         "global_load_lds_dwordx4 %1, %2" m_ "\n\tglobal_load_lds_dwordx4 %1, %2 offset:1024" m_ \
                         :: "s"(lbase + (dst_) + _i * 1024), "v"(lo16), "s"((src_) + _i * 1024) : "memory"); } } while (0)
#define GLDN(src_, dst_, n_) GLDM(src_, dst_, n_, "")
#define MARK(t_) asm volatile("s_mov_b32 m0, %0\n\ts_mov_b64 exec, 1\n\tglobal_load_lds_dword %1, %2\n\ts_mov_b64 exec, -1" \
                         :: "s"(ldw), "v"(lo16), "s"(tab + (t_) * 4) : "memory")
#define READY(word_, nav_, ncons_) do { if (av < (nav_) || cons < (ncons_)) for (unsigned _sp = 0;;) { u4 _c; unsigned _av; \
            asm volatile("ds_read_b128 %0, %2\n\tds_read_b32 %1, %2 offset:%3\n\ts_waitcnt lgkmcnt(0)" \
                         : "=&v"(_c), "=&v"(_av) : "v"(ctrl), "i"(word_) : "memory"); \
            av = (int)rfl(_av); cons = (int)min4(_c); \
            if (av >= (nav_) && cons >= (ncons_)) break; \
            if (SPIN_OUT(_sp, t0)) break; \
            __builtin_amdgcn_s_sleep(1); } } while (0)
#define NEXT_REG() sb = (sb == (unsigned)(2 * REG)) ? 0u : sb + (unsigned)REG
        if (wave == 4) {
            const char* xa = srcws + WS_XS + (size_t)((call & 1u) * 2u + (unsigned)mt) * NKS * X_STEP;
            const char* xg = myws + WS_XG + (size_t)((((call & 1u) * NR + (unsigned)src) * 2u + (unsigned)mt) * K1) * X_STEP;
            for (int t = 0; t < KA; ++t) {
                READY(C_AVAIL, t + 1, t - 2);
                TRACE(0, t);
                const bool st = (PMASK >> t) & 1;
                const char* xt = st ? xg : xa;
                GLDN(xt, sb, 32);
                MARK(t);
                NEXT_REG();
                xa += X_STEP;
                xg += st ? X_STEP : 0;
            }
#pragma unroll 1
            for (int t = KA; t < NITEMS; ++t) {
                READY(C_AVAIL, t + 1, t - 2);
                TRACE(0, t);
                GLDN(xa, sb, 16);
                MARK(t);
                NEXT_REG();
                xa += t == NKS - 1 ? X_PART - (NAB - 1) * X_STEP : X_STEP;
            }
            VM_WAIT(0);
            STAMP(13);
            TRDUMP(0);
        } else {
            const char* wa = myws + WS_WH + (size_t)f0 * 2048;
            for (int t = 0; t < KA; ++t) {
                READY(C_WAV, t + 1, t - 2);
                TRACE(2, t);
                if ((AG_WNT >> mt) & 1) GLDM(wa, sb + X_STEP, 18, " nt"); else GLDN(wa, sb + X_STEP, 18);
                MARK(t);
                NEXT_REG();
                wa += W_ROW;
            }
#pragma unroll 1
            for (int j = 0; j < NAB; ++j) {
                READY(C_WAV, NKS, KA + (j >> 1) - 2);
                TRACE(2, KA + j);
                GLDN(wa, sb + X_PART + (j & 1) * W_STEP, 18);
                MARK(j == NAB - 1 ? NITEMS - 1 : KA + j);
                if (j & 1) NEXT_REG();
                wa += W_ROW;
            }
            VM_WAIT(0);
            TRDUMP(2);
        }
    }
}

}

extern "C" void kernel_launch(int rank, int nranks, ncclComm_t comm,
                              hipStream_t stream,
                              void* const* d_in, const int* in_sizes, int n_in,
                              void* d_out, int out_size,
                              const int* in_dtypes, int out_dtype,
                              void* const* d_out_all,
                              void* const* d_ws_all, size_t ws_bytes) {
    (void)comm; (void)in_dtypes; (void)out_dtype; (void)d_out_all;
    if (nranks != NR || n_in < 2 || (unsigned)rank >= (unsigned)NR || ws_bytes < WS_NEED ||
        in_sizes[0] != MLOC * GK || in_sizes[1] != GK * NLOC || out_size != GM * NLOC) {
        fprintf(stderr, "ag_gemm: unexpected problem shape\n");
        return;
    }
    static bool attr_set = false;
    if (!attr_set) {
        HIP_CHECK(hipFuncSetAttribute((const void*)ag_gemm_kernel,
                                      hipFuncAttributeMaxDynamicSharedMemorySize, LDS_BYTES));
        attr_set = true;
    }
    Params p;
    p.x = (const float*)d_in[0];
    p.w = (const float*)d_in[1];
    p.out = (float*)d_out;
    p.myws = (char*)d_ws_all[rank];
    for (int r = 0; r < NR; ++r) p.ws[r] = (char*)d_ws_all[r];
    p.rank = rank;
    hipLaunchKernelGGL(ag_gemm_kernel, dim3(NWG), dim3(NTHREADS), LDS_BYTES, stream, p);
    HIP_CHECK(hipGetLastError());
}
```

```cpp
#include <hip/hip_runtime.h>
#include <rccl/rccl.h>
#include <cstdio>

#define HIP_CHECK(expr) do { hipError_t _e = (expr); if (_e != hipSuccess) { \
    fprintf(stderr, "HIP error %s at %s:%d\n", hipGetErrorString(_e), __FILE__, __LINE__); \
    return; } } while (0)

namespace {

#define GAS __attribute__((address_space(1)))
#define LAS __attribute__((address_space(3)))
typedef _Float16 h8 __attribute__((ext_vector_type(8)));
typedef _Float16 h2 __attribute__((ext_vector_type(2)));
typedef float f4 __attribute__((ext_vector_type(4)));
typedef unsigned u4 __attribute__((ext_vector_type(4)));
typedef unsigned u2 __attribute__((ext_vector_type(2)));

#define AG_PWIN 16
#define AG_WIN 8
#define AG_SLEEP_POLL 4
#define AG_SLEEP_FINE 1
#define AG_FINE 48
#define AG_NCONV 120
#define AG_KP0 30
#define AG_KPS 6
#define AG_SDELAY 8
#define AG_SEARLY 12
#define AG_SSLOPE 1
#define AG_AST 1
#define AST " sc1"
#define AG_LSLEEP 1
#define PACE ""
#define AG_EARLY0 2
#define AG_EARLY1 2
#define AG_WNT 0
#define AG_S30 54
#define AG_S31 54
#define AG_GE 27

constexpr int NR = 8;
constexpr int GM = 4096, GK = 4096, NLOC = 2048, MLOC = 512;
constexpr int BM = 256, BK = 64;
constexpr int FW = 9;
constexpr int NKS = GK / BK;
constexpr int K1 = 4;
constexpr int KP0 = AG_KP0, KPS = AG_KPS;
constexpr unsigned long long PMASK = (1ull | 1ull << KPS | 1ull << 2 * KPS | 1ull << 3 * KPS) << KP0;
constexpr int NAB = 6;
constexpr int KA = NKS - NAB;
constexpr int NITEMS = KA + 2 * NAB;
constexpr int X_STEP = BM * BK * 2;
constexpr int X_PART = X_STEP / 2;
constexpr int W_STEP = FW * 2048;
constexpr int W_ROW = NLOC * BK * 2;
constexpr int REG = X_PART + 2 * W_STEP;
constexpr int CTRL_OFF = 3 * REG;
constexpr int LDS_BYTES = CTRL_OFF + 512;
constexpr int NWG = 256, NTHREADS = 512;
static_assert(X_STEP + W_STEP <= REG && NAB == 6 && (NKS - K1) * 4 == NWG - 16 && KP0 > 0 && KPS > 1 && KP0 + 3 * KPS < KA - 1, "layout");
constexpr int C_CONS = 0;
constexpr int C_CONW = 48;
constexpr int C_AVAIL = 16;
constexpr int C_WAV = 20;
constexpr int C_STG = 24;
constexpr int C_CALL = 28;
constexpr int C_LD = 32;
constexpr int C_BAD = 40;
constexpr int NCONV = AG_NCONV;
constexpr float LOG2E = 1.4426950408889634f;

constexpr size_t FSTRIDE = 4096 + 256;
constexpr size_t WS_CALL = 0;
constexpr size_t WS_TMO = 49152;
constexpr size_t WS_SFLAG = 53248;
constexpr size_t WS_HINT = 65536;
constexpr size_t WS_WFLAG = 196608;
constexpr size_t WS_TAB = 327680;
constexpr size_t WS_DBG = 524288;
constexpr size_t WS_WH = 1ull << 20;
constexpr size_t WS_XS = 32ull << 20;
constexpr size_t WS_XG = 48ull << 20;
constexpr size_t WS_NEED = 64ull << 20;
static_assert(LDS_BYTES <= 160 * 1024, "LDS");

constexpr unsigned long long DEADLINE = 100000000ull;
constexpr unsigned long long SILENCE = 20000ull;

struct Params {
    const float* x;
    const float* w;
    float* out;
    char* myws;
    char* ws[NR];
    int rank;
};

__device__ __forceinline__ unsigned long long rt() { return __builtin_amdgcn_s_memrealtime(); }

__device__ __forceinline__ void st16_sys(void* p, u4 v) {
    asm volatile("global_store_dwordx4 %0, %1, off sc0 sc1\n\ts_nop 1" :: "v"(p), "v"(v) : "memory");
}
__device__ __forceinline__ void st16_dev(void* p, u4 v) {
    asm volatile("global_store_dwordx4 %0, %1, off sc1\n\ts_nop 1" :: "v"(p), "v"(v) : "memory");
}
__device__ __forceinline__ void st4_sys(void* p, unsigned v) {
    asm volatile("global_store_dword %0, %1, off sc0 sc1\n\ts_nop 1" :: "v"(p), "v"(v) : "memory");
}
__device__ __forceinline__ void post(const char* p, unsigned zero, unsigned v) {
    asm volatile("s_mov_b64 exec, 1\n\tglobal_store_dword %0, %1, %2\n\ts_mov_b64 exec, -1" :: "v"(zero), "v"(v), "s"(p) : "memory");
}
__device__ __forceinline__ void lds_st(unsigned addr, unsigned v) {
    asm volatile("ds_write_b32 %0, %1" :: "v"(addr), "v"(v) : "memory");
}
__device__ __forceinline__ unsigned lds_ld(unsigned addr) {
    unsigned v;
    asm volatile("ds_read_b32 %0, %1\n\ts_waitcnt lgkmcnt(0)" : "=&v"(v) : "v"(addr) : "memory");
    return v;
}
__device__ __forceinline__ unsigned ld_ctr(const void* p) {
    unsigned v;
    asm volatile("global_load_dword %0, %1, off sc1\n\ts_waitcnt vmcnt(0)" : "=&v"(v) : "v"(p) : "memory");
    return (unsigned)__builtin_amdgcn_readfirstlane(v);
}
__device__ __forceinline__ unsigned pk(float a, float b) {
    h2 h; h.x = (_Float16)a; h.y = (_Float16)b;
    return __builtin_bit_cast(unsigned, h);
}
__device__ __forceinline__ u4 pk8(f4 a, f4 b) { return (u4){pk(a[0], a[1]), pk(a[2], a[3]), pk(b[0], b[1]), pk(b[2], b[3])}; }
__device__ __forceinline__ unsigned umin(unsigned a, unsigned b) { return a < b ? a : b; }
__device__ __forceinline__ unsigned rfl(unsigned v) { return (unsigned)__builtin_amdgcn_readfirstlane((int)v); }
__device__ __forceinline__ unsigned min4(u4 c) { return rfl(umin(umin(c.x, c.y), umin(c.z, c.w))); }
__device__ __forceinline__ int row_min(int u) {
    u = min(u, __builtin_amdgcn_update_dpp(0, u, 0x121, 15, 15, false));
    u = min(u, __builtin_amdgcn_update_dpp(0, u, 0x122, 15, 15, false));
    u = min(u, __builtin_amdgcn_update_dpp(0, u, 0x124, 15, 15, false));
    u = min(u, __builtin_amdgcn_update_dpp(0, u, 0x128, 15, 15, false));
    return __builtin_amdgcn_readfirstlane(u);
}
__device__ __forceinline__ void arrivals(const char* sf, unsigned call, int lane, unsigned& staged, bool& stored) {
    u2 sv; unsigned pv;
    asm volatile("global_load_dwordx2 %0, %2, %4 sc0 sc1\n\tglobal_load_dword %1, %3, %4 offset:512 sc0 sc1\n\ts_waitcnt vmcnt(0)"
                 : "=&v"(sv), "=&v"(pv) : "v"(lane * 8), "v"((lane & 15) * 4), "s"(sf) : "memory");
    const unsigned long long miss = __ballot((int)(sv.x - call) < 0 || (int)(sv.y - call) < 0) & ~PMASK;
    staged = miss ? (unsigned)__builtin_ctzll(miss) : (unsigned)NKS;
    stored = __ballot((int)(pv - call) < 0) == 0;
}
#define VM_WAIT(n) asm volatile("s_waitcnt vmcnt(" #n ")" ::: "memory")
#define LGKM0() asm volatile("s_waitcnt lgkmcnt(0)" ::: "memory")
#define SCHED_FENCE() __builtin_amdgcn_sched_barrier(0)
#define SPIN_OUT(sp_, t0_) ((((++(sp_)) & 1023u) == 0u) && ((t0_) == 0 ? ((t0_) = rt(), false) : (rt() - (t0_) > 2 * DEADLINE)))

#define TRACE(k_, t_) do {} while (0)
#define TRDUMP(k_) do {} while (0)
#define TRZERO(k_) do {} while (0)
#define STAMP(i) do {} while (0)
#define STAMPC(i) do {} while (0)

#define DSR(dst_, addr_, off_) asm volatile("ds_read_b128 %0, %1 offset:%2" : "+v"(dst_) : "v"(addr_), "i"(off_) : "memory")
#define WAIT_LD(nx_, nw_) do { unsigned long long _t0 = 0; unsigned _sp = 0; \
        for (;;) { u2 _f; asm volatile("ds_read_b64 %0, %1\n\ts_waitcnt lgkmcnt(0)" : "=&v"(_f) : "v"(ldw) : "memory"); \
            if (rfl(_f.x) >= (unsigned)(nx_) && rfl(_f.y) >= (unsigned)(nw_)) break; \
            if (SPIN_OUT(_sp, _t0)) break; \
            __builtin_amdgcn_s_sleep(1); } } while (0)

struct Epi {
    float t[4];
    f4 o;
    float lim, c;
    unsigned vo;
    const char* row;
    unsigned long long last;
};
constexpr int NSLOT = 15 * FW;
template <int J, int S, int SE>
__device__ __forceinline__ void epi_slots(f4 (&acc)[FW][4], Epi& E) {
    constexpr int q = S / 15, op = S % 15, k = (op & 1) * 2;
    if constexpr (S < SE) {
        const f4& y = acc[q][J];
        if constexpr (op < 2)
            asm volatile("v_med3_f32 %0, -%2, %4, -%4\n\tv_med3_f32 %1, -%3, %4, -%4"
                         : "=&v"(E.t[k]), "=&v"(E.t[k + 1]) : "v"(y[k]), "v"(y[k + 1]), "s"(E.lim), "v"(q ? E.o : y));
        else if constexpr (op < 6) asm volatile("v_exp_f32 %0, %0" : "+v"(E.t[op - 2]));
        else if constexpr (op < 8)
            asm volatile("v_fma_f32 %0, %0, %2, %2\n\tv_fma_f32 %1, %1, %2, %2" : "+v"(E.t[k]), "+v"(E.t[k + 1]) : "s"(E.c));
        else if constexpr (op < 12) asm volatile("v_rcp_f32 %0, %0" : "+v"(E.t[op - 8]));
        else if constexpr (op < 14) {
            float o0, o1;
            asm volatile("v_mul_f32 %0, %2, %4\n\tv_mul_f32 %1, %3, %5" : "=&v"(o0), "=v"(o1)
                         : "v"(y[k]), "v"(y[k + 1]), "v"(E.t[k]), "v"(E.t[k + 1]));
            E.o[k] = o0; E.o[k + 1] = o1;
        }
        else if constexpr (q < FW - 1)
            asm volatile("global_store_dwordx4 %0, %1, %2 offset:%3" AST :: "v"(E.vo), "v"(E.o), "s"(E.row), "i"(q * 64) : "memory");
        else
            asm volatile("s_mov_b64 exec, %4\n\tglobal_store_dwordx4 %0, %1, %2 offset:%3" AST "\n\ts_mov_b64 exec, -1\n\ts_nop 1"
                         :: "v"(E.vo), "v"(E.o), "s"(E.row), "i"(q * 64), "s"(E.last) : "memory");
        epi_slots<J, S + 1, SE>(acc, E);
    }
}
constexpr int slots_before(int n, int g) {
    const int j = n % 3, c0 = n < 9 ? NSLOT / 3 : AG_S30, c1 = n < 9 ? NSLOT / 3 : AG_S31;
    const int b = j == 0 ? 0 : j == 1 ? c0 : c0 + c1, c = j == 0 ? c0 : j == 1 ? c1 : NSLOT - b;
    return b + (g >= AG_GE ? c : (c * g + AG_GE - 1) / AG_GE);
}
static_assert(AG_S30 + AG_S31 <= NSLOT, "slots");
constexpr int NPI = 2 * NAB;
constexpr int g_chk(int n) { return n < 3 ? 24 : 27; }
constexpr int g_rel(int n) { return n == 0 ? 12 : 3; }
constexpr int lds_ops(int n, int g) {
    if (g < 0) return n ? lds_ops(n - 1, g + 36) : 0;
    const bool nx = n < NPI - 1;
    return (g < 24 || nx) + (g == g_chk(n) - 10) + (g == g_rel(n)) + (nx ? 2 * (g == g_chk(n)) + (g == g_chk(n) + 1) + (g == 28) : 0);
}
constexpr int lds_wait(int n, int g) {
    int c = n == 0 && g < 9 ? (g == 2 ? 8 : g == 5 ? 5 : 1) : 0;
    for (int i = g - 8; i <= g; ++i) c += lds_ops(n, i);
    if (g == 35 && g_chk(n) == 27) c -= 1;
    return c < 15 ? c : 15;
}
struct Pos { unsigned pa, qa, pb, qb, rv, wl; u4 f; };
template <int N, int G>
__device__ __forceinline__ void part_items(f4 (&acc)[FW][4], h8 (&a)[FW], h8 (&b0)[4], h8 (&b1)[4], Pos& P, Epi& E,
                                           unsigned ldw, unsigned relw) {
    if constexpr (N < NPI) {
        constexpr int J = N / 3, d = (KA + N) % 3 == 2 ? -2 * REG : REG, F = (G + 12) % 36;
        constexpr bool nx = N < NPI - 1;
        h8& wf = G % 12 < FW ? a[G % 12 % FW] : b1[(G % 12 + 4) & 3];
        h8& xf = G < 9 ? b0[0] : G < 18 ? b1[0] : G < 27 ? b0[1] : b0[2 + (N & 1)];
        asm volatile("v_mfma_f32_16x16x32_f16 %0, %1, %2, %0" : "+v"(acc[G % FW][J]) : "v"(wf), "v"(xf));
        SCHED_FENCE();
        if (G == 0 && N % 3 == 0 && J) {
            E.c = __builtin_bit_cast(float, __builtin_bit_cast(unsigned, LOG2E) | rfl(P.f.z));
            if (J > 1) E.row += 64 * NLOC * 4;
            asm volatile("" : "+s"(E.c));
            asm volatile("" : "+s"(E.row));
        }
        if (G == g_rel(N)) lds_st(relw, P.rv);
        if (G == g_chk(N) - 10) asm volatile("ds_read_b128 %0, %1" : "=v"(P.f) : "v"(ldw) : "memory");
        if (nx && G == g_chk(N)) {
            constexpr unsigned nw = N < 2 ? KA + 2 * N + 4 : 0;
            if (__builtin_expect(rfl(N < 2 ? umin(P.f.x + (N + 2), P.f.y) : P.f.x) < (N < 2 ? nw : KA + N + 2), 0)) WAIT_LD(KA + N + 2, nw);
            SCHED_FENCE();
            DSR(b0[0], P.pb, 0);
            DSR(b1[0], P.qb, 0);
        }
        if (nx && G == g_chk(N) + 1) DSR(b0[3 - (N & 1)], P.qb, 8192);
        if (nx && G == 28) DSR(b0[1], P.pb, 8192);
        if (G < 24 || nx) DSR(wf, (F / 9) & 1 ? P.qa : P.pa, (F / 18) * W_STEP + (F % 9) * 2048);
        if (nx && G == 4) { P.pb += d; asm volatile("" : "+v"(P.pb)); }
        if (nx && G == 7) { P.qb += d; asm volatile("" : "+v"(P.qb)); }
        if (nx && G == 19) { P.pa += d; asm volatile("" : "+v"(P.pa)); }
        if (nx && G == 31) { P.qa += d; asm volatile("" : "+v"(P.qa)); }
        if (nx && G == 16) { P.rv += 1u; asm volatile("" : "+v"(P.rv)); }
        if constexpr (J > 0) epi_slots<J - 1, slots_before(N, G), slots_before(N, G + 1)>(acc, E);
        if (G % 3 == 2 && (nx || G < 35)) asm volatile("s_waitcnt lgkmcnt(%1)" : "+v"(P.f) : "i"(lds_wait(N, G)) : "memory");
        SCHED_FENCE();
        part_items<G == 35 ? N + 1 : N, (G + 1) % 36>(acc, a, b0, b1, P, E, ldw, relw);
    }
}

__global__ void __launch_bounds__(NTHREADS, 2) ag_gemm_kernel(Params p) {
    extern __shared__ __attribute__((aligned(16))) unsigned char smem[];
    LAS unsigned char* lds = (LAS unsigned char*)smem;
    const int tid = threadIdx.x;
    const int wave = __builtin_amdgcn_readfirstlane(tid >> 6);
    const int lane = tid & 63;
    const int b = blockIdx.x;
    const int src = b & 7, idx = b >> 3;
    char* myws = p.myws;
    GAS unsigned* cctr = (GAS unsigned*)(myws + WS_CALL + (size_t)b * 128);
    GAS unsigned* tmo = (GAS unsigned*)(myws + WS_TMO);
    const unsigned lbase = (unsigned)(size_t)lds;
    const unsigned ctrl = lbase + CTRL_OFF;
    const unsigned lo16 = (unsigned)lane * 16u;
    const int r8 = lane >> 3, c8 = lane & 7;
    const unsigned img = (unsigned)(r8 * 128 + ((c8 ^ (r8 >> 1)) * 16));
    const char* srcws = p.ws[0];
#pragma unroll
    for (int r = 1; r < NR; ++r) if (src == r) srcws = p.ws[r];

    if (idx < 2) {
        const int cc = src * 2 + idx;
        const char* hint = myws + WS_HINT + (size_t)cc * FSTRIDE;
        if (wave < 2) {
            STAMP(20 + wave);
            f4 v[K1][4];
            unsigned cv;
            {
                const unsigned s0 = (unsigned)((cc * 32 + wave * 16 + r8) * GK + c8 * 8) * 4u;
                const unsigned s1 = (unsigned)((cc * 32 + wave * 16 + 8 + r8) * GK + (c8 ^ 4) * 8) * 4u;
#pragma unroll
                for (int e = 0; e < K1; ++e)
                    asm volatile("global_load_dwordx4 %0, %4, %6\n\tglobal_load_dwordx4 %1, %4, %6 offset:16\n\t"
                                 "global_load_dwordx4 %2, %5, %6\n\tglobal_load_dwordx4 %3, %5, %6 offset:16"
                                 : "=&v"(v[e][0]), "=&v"(v[e][1]), "=&v"(v[e][2]), "=&v"(v[e][3])
                                 : "v"(s0), "v"(s1), "s"(p.x + (KP0 + e * KPS) * BK) : "memory");
                asm volatile("global_load_dword %0, %1, off sc1" : "=&v"(cv) : "v"(cctr) : "memory");
            }
            const char* dst[NR];
#pragma unroll
            for (int j = 0; j < NR; ++j) {
                const int r = (p.rank + 1 + j + cc + 4 * wave) & 7;
                dst[j] = p.ws[0];
#pragma unroll
                for (int q = 1; q < NR; ++q) if (r == q) dst[j] = p.ws[q];
                dst[j] += WS_XG + (size_t)((p.rank * 2 + (cc >> 3)) * K1) * X_STEP + (size_t)((cc & 7) * 2 + wave) * 2048;
                asm volatile("" : "+s"(dst[j]));
            }
            asm volatile("s_waitcnt vmcnt(0)" : "+v"(cv), "+v"(v[0][0]), "+v"(v[0][1]), "+v"(v[0][2]), "+v"(v[0][3]),
                         "+v"(v[1][0]), "+v"(v[1][1]), "+v"(v[1][2]), "+v"(v[1][3]), "+v"(v[2][0]), "+v"(v[2][1]), "+v"(v[2][2]),
                         "+v"(v[2][3]), "+v"(v[3][0]), "+v"(v[3][1]), "+v"(v[3][2]), "+v"(v[3][3]) :: "memory");
            const unsigned call = rfl(cv) + 1u;
            const unsigned vo = img + (call & 1u) * (unsigned)(NR * 2 * K1 * X_STEP);
            char* flag = myws + WS_SFLAG + (size_t)(cc >> 3) * FSTRIDE + 512 + ((cc & 7) * 2 + wave) * 4;
            if (wave == 0) STAMP(5);
            constexpr int W = AG_PWIN;
            static_assert(W >= 2 && W <= 32 && W % 2 == 0, "window");
#pragma unroll
            for (int e = 0; e < K1; ++e) {
                const u4 o0 = pk8(v[e][0], v[e][1]), o1 = pk8(v[e][2], v[e][3]);
                const unsigned ve = vo + (unsigned)(e * X_STEP);
#pragma unroll
                for (int j = 0; j < NR; ++j) {
                    if (e == 0 && j == 4) __builtin_amdgcn_s_barrier();
                    asm volatile("s_waitcnt vmcnt(%0)" :: "i"(W - 2) : "memory");
                    asm volatile("global_store_dwordx4 %0, %1, %3 sc0 sc1\n\tglobal_store_dwordx4 %0, %2, %3 offset:1024 sc0 sc1\n\ts_nop 1"
                                 :: "v"(ve), "v"(o0), "v"(o1), "s"(dst[j]) : "memory");
                }
            }
            VM_WAIT(0);
            if (lane == 0) st4_sys(flag, call);
            STAMP(14 + wave);
            return;
        }
        if (wave == 6) lds_st(ctrl + C_AVAIL, 0u);
        const unsigned call = ld_ctr((const void*)cctr) + 1u;
        LGKM0();
        __builtin_amdgcn_s_barrier();
        asm volatile("" ::: "memory");
        unsigned long long t0 = 0;
        if (wave < 6) {
            const int q = wave - 2;
            const char* hq = hint + q * 4;
            const char* xa = srcws + WS_XS + (size_t)((call & 1u) * 2u + (unsigned)idx) * NKS * X_STEP;
            unsigned hv = call * 256u + 1u, zero = 0u;
            asm volatile("s_mov_b32 m0, %2" : "+v"(hv), "+v"(zero) : "s"(lbase + q * 4096) : "memory");
            int av = 0;
#define STAGED(n_) do { if (av < (n_)) for (unsigned _sp = 0;;) { av = (int)rfl(lds_ld(ctrl + C_AVAIL)); \
                if (av >= (n_) || SPIN_OUT(_sp, t0)) break; \
                __builtin_amdgcn_s_sleep(1); } } while (0)
#define PULL4(a_) asm volatile("s_waitcnt vmcnt(%2)\n\tglobal_load_lds_dwordx4 %0, %1\n\t" \
                         "s_waitcnt vmcnt(%2)\n\tglobal_load_lds_dwordx4 %0, %1 offset:1024\n\t" \
                         "s_waitcnt vmcnt(%2)\n\tglobal_load_lds_dwordx4 %0, %1 offset:2048\n\t" \
                         "s_waitcnt vmcnt(%2)\n\tglobal_load_lds_dwordx4 %0, %1 offset:3072" \
                         :: "v"(lo16), "s"(a_), "i"(AG_WIN - 1) : "memory")
            if (src == p.rank) {
                while (av < NKS) {
                    const int need = av + 1;
                    STAGED(need);
                    if (av < need) break;
                    post(hq, zero, call * 256u + (unsigned)(av >= NKS ? NITEMS : min(av, KA)));
                }
                STAMP(4 + wave);
                return;
            }
#pragma unroll 1
            for (int t = 0; t < KA; ++t) {
                STAGED(t + 1);
                if (q == 0) TRACE(0, t);
                const char* xt = xa + q * 8192;
                if ((PMASK >> 1 >> t) & 1) { hv += 1u; xa += X_STEP; ++t; }
                const unsigned h2 = av < t + 2 ? hv : hv + 1u + (unsigned)((PMASK >> 2 >> t) & 1);
                if (AG_EARLY0 == 2) post(hq, zero, hv);
                if (AG_EARLY0 == 4) post(hq, zero, h2);
                PULL4(xt);
                if (AG_EARLY0 == 1) post(hq, zero, hv);
                if (AG_EARLY0 == 3) post(hq, zero, h2);
                PULL4(xt + 4096);
                if (AG_EARLY0 == 0) post(hq, zero, hv);
                hv += 1u;
                xa += X_STEP;
            }
            STAGED(NKS);
            xa += (q >> 1) * X_STEP + (q & 1) * 4096;
#pragma unroll
            for (int t = KA; t < NITEMS; ++t) {
                if (q == 0) TRACE(0, t);
                if (AG_EARLY1) post(hq, zero, hv + (AG_EARLY1 - 1));
                PULL4(xa + (t - KA) % 3 * 2 * X_STEP + (t - KA) / 3 * 8192);
                if (!AG_EARLY1) post(hq, zero, hv);
                hv += 1u;
            }
            VM_WAIT(0);
            STAMP(4 + wave);
            if (q == 0) TRDUMP(0);
        } else if (wave == 6) {
            if (lane == 0) __hip_atomic_store(cctr, call, __ATOMIC_RELAXED, __HIP_MEMORY_SCOPE_AGENT);
            __builtin_amdgcn_fence(__ATOMIC_ACQUIRE, "");
            VM_WAIT(0);
            const char* sf = srcws + WS_SFLAG + (size_t)idx * FSTRIDE;
            unsigned staged = 0, npoll = 0;
            bool stored = false;
            post(hint + 16, 0u, call * 256u + KP0);
            t0 = rt();
            while (staged < (unsigned)NKS || !stored) {
                unsigned s2; bool p2;
                arrivals(sf, call, lane, s2, p2);
                if (s2 > staged) { if (!staged) STAMP(17); staged = s2; lds_st(ctrl + C_AVAIL, staged); if (s2 == NKS) STAMP(10); }
                if (p2 && !stored) { stored = true; post(hint + 16, 0u, call * 256u + NITEMS); }
                if ((++npoll & 31u) == 0u && (rfl(__hip_atomic_load(tmo, __ATOMIC_RELAXED, __HIP_MEMORY_SCOPE_AGENT)) == call
                                              || rt() - t0 > DEADLINE)) {
                    __hip_atomic_store(tmo, call, __ATOMIC_RELAXED, __HIP_MEMORY_SCOPE_AGENT);
                    post(hint + 16, 0u, call * 256u);
                    VM_WAIT(0);
                    lds_st(ctrl + C_AVAIL, (unsigned)NKS);
                    break;
                }
                if (npoll < 64u && staged < (unsigned)NKS) __builtin_amdgcn_s_sleep(2); else __builtin_amdgcn_s_sleep(32);
            }
            STAMP(11);
        }
        return;
    }

    const int mt = idx & 1, nt = (idx - 2) >> 1;
    const int f0 = (17 * nt) >> 1;
    f4 sv[4][2];
    const int g = b - 16;
    int sk = g >> 2;
#pragma unroll
    for (int e = 0; e < K1; ++e) sk += sk >= KP0 + e * KPS;
    const bool late = wave < 4 && sk >= AG_SEARLY;
    if (wave < 4) {
        if (wave == 0) STAMP(0);
        const float* s0 = p.x + (size_t)((g & 3) * 128 + wave * 32 + r8) * GK + sk * BK;
        if (late) {
            __builtin_amdgcn_s_barrier();
            for (int i = AG_SEARLY; i < sk; ++i) __builtin_amdgcn_s_sleep(AG_SSLOPE);
        } else if (AG_SDELAY) __builtin_amdgcn_s_sleep(AG_SDELAY);
#pragma unroll
        for (int i = 0; i < 4; ++i)
            asm volatile("global_load_dwordx4 %0, %2, off\n\tglobal_load_dwordx4 %1, %2, off offset:16"
                         : "=&v"(sv[i][0]), "=&v"(sv[i][1]) : "v"(s0 + (size_t)i * 8 * GK + (c8 ^ ((i & 1) * 4)) * 8) : "memory");
    }
    if (wave == 7) {
        *(u2*)(myws + WS_TAB + (size_t)b * 512 + lane * 8) = (u2){2u * lane + 1u, 2u * lane + 2u};
        const unsigned cl = ld_ctr((const void*)cctr) + 1u;
        if (lane == 0) __hip_atomic_store(cctr, cl, __ATOMIC_RELAXED, __HIP_MEMORY_SCOPE_AGENT);
        lds_st(ctrl + lane * 4, lane == C_CALL / 4 ? cl : 0u);
        LGKM0();
    }
    if (!late) __builtin_amdgcn_s_barrier();
    asm volatile("" ::: "memory");
    const unsigned call = rfl(lds_ld(ctrl + C_CALL));

    if (wave < 4) {
        {
            char* d0 = myws + WS_XS + ((size_t)((call & 1u) * 2u + (unsigned)((g >> 1) & 1)) * NKS + (size_t)sk) * X_STEP
                       + (size_t)((g & 1) * X_PART + wave * 4096) + img;
            asm volatile("s_waitcnt vmcnt(0)" : "+v"(sv[0][0]), "+v"(sv[0][1]), "+v"(sv[1][0]), "+v"(sv[1][1]),
                         "+v"(sv[2][0]), "+v"(sv[2][1]), "+v"(sv[3][0]), "+v"(sv[3][1]) :: "memory");
#pragma unroll
            for (int i = 0; i < 4; ++i) st16_sys(d0 + i * 1024, pk8(sv[i][0], sv[i][1]));
            unsigned n = 0;
            if (lane == 0) {
                asm volatile("s_waitcnt vmcnt(0)\n\tds_add_rtn_u32 %0, %1, %2\n\ts_waitcnt lgkmcnt(0)"
                             : "=&v"(n) : "v"(ctrl + C_STG), "v"(1u) : "memory");
                if (n == 3u) st4_sys(myws + WS_SFLAG + (size_t)((g >> 1) & 1) * FSTRIDE + sk * 8 + (g & 1) * 4, call);
            }
            if (wave == 0) STAMP(16);
        }
        const int r16 = lane & 15, q = lane >> 4;
        const unsigned sw = (unsigned)(r16 * 128 + (q ^ (r16 >> 1)) * 16);
        unsigned ldw = ctrl + C_LD, relw = ctrl + C_CONS + wave * 4;
        const bool dup = !(nt & 1) && nt != 14;
        f4 acc[FW][4];
#pragma unroll
        for (int i = 0; i < FW; ++i)
#pragma unroll
            for (int j = 0; j < 4; ++j) acc[i][j] = (f4){0.f, 0.f, 0.f, 0.f};
        const unsigned offA = lbase + sw;
        const unsigned offB = lbase + (unsigned)(wave * 2048) + sw;
        h8 a[FW], b0[4], b1[4];
#pragma unroll
        for (int i = 0; i < FW; ++i) a[i] = (h8)(_Float16)0.f;
#pragma unroll
        for (int j = 0; j < 4; ++j) { b0[j] = (h8)(_Float16)0.f; b1[j] = (h8)(_Float16)0.f; }
#define MF(i_, j_, B_, A_) do { acc[i_][j_] = __builtin_amdgcn_mfma_f32_16x16x32_f16(A_, B_, acc[i_][j_], 0, 0, 0); SCHED_FENCE(); } while (0)
#define FILL(x_) do { x_; SCHED_FENCE(); } while (0)
#define ROW(i_, BC_, F0_, F1_, F2_, F3_) do { \
        MF(i_, 0, BC_[0], a[i_]); FILL(F0_); MF(i_, 1, BC_[1], a[i_]); FILL(F1_); \
        MF(i_, 2, BC_[2], a[i_]); FILL(F2_); MF(i_, 3, BC_[3], a[i_]); FILL(F3_); } while (0)
#define NOP_ do {} while (0)
#define XO(j_) ((j_) * 8192)
#define ADV(v_, d_) do { v_ += d_; asm volatile("" : "+v"(v_)); } while (0)
#define PIN(s_) asm volatile("" : "+s"(s_))
#define WHI(BC_, cnt_) asm volatile("s_waitcnt lgkmcnt(%8)" : "+v"(a[5]), "+v"(a[6]), "+v"(a[7]), "+v"(a[8]), \
                                    "+v"(BC_[0]), "+v"(BC_[1]), "+v"(BC_[2]), "+v"(BC_[3]) : "i"(cnt_) : "memory")
#define WLO(BC_, cnt_) asm volatile("s_waitcnt lgkmcnt(%9)" : "+v"(a[0]), "+v"(a[1]), "+v"(a[2]), "+v"(a[3]), "+v"(a[4]), \
                                    "+v"(BC_[0]), "+v"(BC_[1]), "+v"(BC_[2]), "+v"(BC_[3]) : "i"(cnt_) : "memory")
#define REL(off_) asm volatile("ds_write_b32 %0, %1 offset:%2" :: "v"(relw), "v"(P.rv), "i"(off_) : "memory")
        float* orow = p.out + (size_t)(src * MLOC + mt * BM + wave * 16) * NLOC + f0 * 16;
        Epi E;
        E.lim = 60.f * LOG2E;
        E.vo = (unsigned)((r16 * NLOC + q * 4) * 4);
        E.row = (const char*)orow;
        E.last = (unsigned long long)rfl(dup ? 0u : ~0u) * 0x100000001ull;
        asm volatile("" : "+s"(E.row));
        asm volatile("" : "+s"(E.last));
        asm volatile("" : "+s"(E.lim));
        asm volatile("" : "+v"(E.vo));
        WAIT_LD(1, 1);
        if (wave == 0) { STAMP(18); STAMPC(30); }
#pragma unroll
        for (int j = 0; j < 4; ++j) DSR(b0[j], offB, XO(j));
#pragma unroll
        for (int i = 0; i < FW - 1; ++i) DSR(a[i], offA, X_STEP + i * 2048);
        WLO(b0, 0);
        __builtin_amdgcn_s_waitcnt(0xc07f);
        Pos P;
        P.pa = offA + X_STEP; P.pb = offB; P.qa = P.pa ^ 64u; P.qb = P.pb ^ 64u; P.rv = 1u;
        asm volatile("" : "+v"(P.pa), "+v"(P.pb), "+v"(P.qa), "+v"(P.qb), "+v"(P.rv), "+v"(relw), "+v"(ldw));
        SCHED_FENCE();
        {
            u2 f;
            int da = REG, db = REG, xr = 1;
#pragma unroll 1
            for (unsigned need = 2; need < (unsigned)KA + 2u; ++need) {
                unsigned m;
                ROW(0, b0, DSR(a[8], P.pa, 8 * 2048), DSR(b1[0], P.qb, XO(0)), DSR(b1[1], P.qb, XO(1)), ADV(P.pa, da));
                ROW(1, b0, DSR(a[0], P.qa, 0), DSR(b1[2], P.qb, XO(2)), DSR(b1[3], P.qb, XO(3)), ADV(P.pb, db));
                ROW(2, b0, DSR(a[1], P.qa, 2048), ADV(P.qb, db), NOP_, NOP_);
                ROW(3, b0, DSR(a[2], P.qa, 2 * 2048), NOP_, NOP_, NOP_);
                ROW(4, b0, DSR(a[3], P.qa, 3 * 2048), WHI(b1, 3); REL(0), NOP_, NOP_);
                ROW(5, b0, DSR(a[4], P.qa, 4 * 2048), NOP_, NOP_, NOP_);
                ROW(6, b0, DSR(a[5], P.qa, 5 * 2048), NOP_, NOP_, NOP_);
                ROW(7, b0, DSR(a[6], P.qa, 6 * 2048), NOP_, NOP_, NOP_);
                ROW(8, b0, DSR(a[7], P.qa, 7 * 2048), NOP_, WLO(b1, 3), NOP_);
                ROW(0, b1, DSR(a[8], P.qa, 8 * 2048), ADV(P.qa, da), db = xr == 2 ? -2 * REG : REG; PIN(db), NOP_);
                ROW(1, b1, asm volatile("ds_read_b64 %0, %1" : "=v"(f) : "v"(ldw) : "memory"),
                    xr = xr == 2 ? 0 : xr + 1; PIN(xr), da = need == (unsigned)KA ? -X_PART : 0; PIN(da), da += db; PIN(da));
                ROW(2, b1, NOP_, NOP_, WHI(b0, 1); REL(C_CONW - C_CONS), ADV(P.rv, 1u));
                ROW(3, b1, asm volatile("s_waitcnt lgkmcnt(1)" : "+v"(f) :: "memory"); m = umin(f.x, f.y); asm volatile("" : "+v"(m)),
                    m = rfl(m); PIN(m), if (__builtin_expect(m < need, 0)) WAIT_LD(need, need), DSR(b0[0], P.pb, XO(0)));
                ROW(4, b1, DSR(a[0], P.pa, 0), DSR(b0[1], P.pb, XO(1)), DSR(b0[2], P.pb, XO(2)), DSR(b0[3], P.pb, XO(3)));
                ROW(5, b1, DSR(a[1], P.pa, 2048), DSR(a[2], P.pa, 2 * 2048), DSR(a[3], P.pa, 3 * 2048), DSR(a[4], P.pa, 4 * 2048));
                ROW(6, b1, DSR(a[5], P.pa, 5 * 2048), NOP_, NOP_, NOP_);
                ROW(7, b1, DSR(a[6], P.pa, 6 * 2048), NOP_, NOP_, NOP_);
                ROW(8, b1, DSR(a[7], P.pa, 7 * 2048), NOP_, WLO(b0, 3), NOP_);
            }
            P.wl = rfl(f.y);
        }
        if (wave == 0) { STAMP(2); STAMPC(31); }
        if (P.wl < KA + 2u) WAIT_LD(0, KA + 2);
        SCHED_FENCE();
        DSR(a[8], P.pa, 8 * 2048);
        DSR(b1[0], P.qb, 0);
        DSR(b1[1], P.qa, 0);
        DSR(b1[2], P.qa, 2048);
        DSR(b1[3], P.qa, 2 * 2048);
        DSR(b0[2], P.qb, 8192);
        P.f = (u4){0u, 0u, 0u, 0u};
        SCHED_FENCE();
        part_items<0, 0>(acc, a, b0, b1, P, E, ldw, relw);
        if (wave == 0) { STAMP(3); STAMPC(33); }
        {
            const float c = __builtin_bit_cast(float, __builtin_bit_cast(unsigned, LOG2E) | rfl(P.f.z));
            asm volatile("s_nop 7\n\ts_nop 7" : "+v"(acc[FW - 2][3]), "+v"(acc[FW - 1][3]));
            float* ob = (float*)(E.row + (size_t)64 * NLOC * 4 + E.vo);
#pragma unroll
            for (int i = 0; i < FW; ++i) {
                if (i == FW - 1 && !E.last) continue;
                const f4 y = acc[i][3];
                f4 o;
#pragma unroll
                for (int e = 0; e < 4; ++e) {
                    const float z = __builtin_amdgcn_fmed3f(y[e], -60.f * LOG2E, 60.f * LOG2E);
                    const float ex = __builtin_amdgcn_exp2f(-z);
                    o[e] = y[e] * __builtin_amdgcn_rcpf(__builtin_fmaf(ex, c, c));
                }
                st16_dev(ob + i * 16, __builtin_bit_cast(u4, o));
            }
        }
        if (wave == 0) { STAMP(4); STAMPC(34); }
    } else if (wave == 7) {
        __builtin_amdgcn_fence(__ATOMIC_ACQUIRE, "");
        VM_WAIT(0);
        GAS unsigned* hp = (GAS unsigned*)(myws + WS_HINT + (size_t)(src * 2 + mt) * FSTRIDE) + c8;
        GAS unsigned* wf0 = (GAS unsigned*)(myws + WS_WFLAG + (size_t)(f0 >> 3) * FSTRIDE) + lane;
        GAS unsigned* wf1 = (GAS unsigned*)(myws + WS_WFLAG + (size_t)((f0 + FW - 1) >> 3) * FSTRIDE) + lane;
        unsigned pub = 0, wav = 0, npoll = 0;
        TRZERO(1);
        const unsigned long long t0 = rt();
        unsigned long long tall = 0;
        bool dead = false;
        while (pub < (unsigned)NITEMS || wav < (unsigned)NKS) {
            const unsigned hv = __hip_atomic_load(hp, __ATOMIC_RELAXED, __HIP_MEMORY_SCOPE_AGENT);
            bool adv = false;
            if (wav < (unsigned)NKS) {
                const unsigned w0 = __hip_atomic_load(wf0, __ATOMIC_RELAXED, __HIP_MEMORY_SCOPE_AGENT);
                const unsigned w1 = __hip_atomic_load(wf1, __ATOMIC_RELAXED, __HIP_MEMORY_SCOPE_AGENT);
                const unsigned long long notready = __ballot((int)(w0 - call) < 0 || (int)(w1 - call) < 0);
                const unsigned w2 = notready ? (unsigned)__builtin_ctzll(notready) : 64u;
                if (w2 > wav) { wav = w2; lds_st(ctrl + C_WAV, wav); adv = true; }
            }
            const int d = (int)(hv - call * 256u);
            const unsigned av = (unsigned)row_min(c8 < 5 ? max(min(d, NITEMS), 0) : NITEMS);
            if (av > pub) {
                lds_st(ctrl + C_AVAIL, av);
                pub = av;
                tall = 0;
                TRACE(1, av - 1);
            } else if (!adv && (++npoll & 63u) == 0u) {
                if (rt() - t0 > DEADLINE) { dead = true; break; }
                unsigned staged; bool stored;
                arrivals(srcws + WS_SFLAG + (size_t)mt * FSTRIDE, call, lane, staged, stored);
                if (staged == (unsigned)NKS && stored) {
                    if (tall == 0) tall = rt();
                    else if (rt() - tall > SILENCE) { pub = NITEMS; lds_st(ctrl + C_AVAIL, pub); }
                } else if (rfl(__hip_atomic_load(tmo, __ATOMIC_RELAXED, __HIP_MEMORY_SCOPE_AGENT)) == call) { dead = true; break; }
            }
            if (pub < (unsigned)AG_FINE) __builtin_amdgcn_s_sleep(AG_SLEEP_POLL); else __builtin_amdgcn_s_sleep(AG_SLEEP_FINE);
        }
        if (dead) {
            __hip_atomic_store(tmo, call, __ATOMIC_RELAXED, __HIP_MEMORY_SCOPE_AGENT);
            lds_st(ctrl + C_BAD, 0x7fc00000u);
        }
        lds_st(ctrl + C_WAV, (unsigned)NKS);
        lds_st(ctrl + C_AVAIL, (unsigned)NITEMS);
        TRDUMP(1);
    } else if (wave == 6) {
        constexpr int CSTEP = 240 / NCONV;
        const int cq = ((idx - 2) / CSTEP) * 8 + src;
        if ((idx - 2) % CSTEP == 0 && cq < NCONV) {
            for (int u = cq; u < NKS * 16; u += NCONV) {
                const int ks = u >> 4, cn = u & 15;
                const float* sw = p.w + (size_t)(ks * BK + c8 * 8) * NLOC + cn * 128 + r8 * 4;
                char* dstw = myws + WS_WH + (size_t)ks * W_ROW + (size_t)cn * 16384;
                f4 v[4][8];
#pragma unroll
                for (int g = 0; g < 4; ++g)
#pragma unroll
                    for (int r = 0; r < 8; ++r)
                        v[g][r] = __builtin_nontemporal_load((const f4*)(sw + (size_t)r * NLOC + g * 32));
                SCHED_FENCE();
#pragma unroll
                for (int g = 0; g < 4; ++g)
#pragma unroll
                    for (int r = 0; r < 8; ++r) v[g][r] *= LOG2E;
#pragma unroll
                for (int g = 0; g < 4; ++g)
#pragma unroll
                    for (int c = 0; c < 4; ++c) {
                        const int n = g * 32 + r8 * 4 + c;
                        u4 o;
                        o.x = pk(v[g][0][c], v[g][1][c]); o.y = pk(v[g][2][c], v[g][3][c]);
                        o.z = pk(v[g][4][c], v[g][5][c]); o.w = pk(v[g][6][c], v[g][7][c]);
                        st16_dev(dstw + n * 128 + ((c8 ^ ((n >> 1) & 7)) * 16), o);
                    }
                VM_WAIT(0);
                if (lane == 0)
                    __hip_atomic_store((GAS unsigned*)(myws + WS_WFLAG + (size_t)cn * FSTRIDE + ks * 4), call,
                                       __ATOMIC_RELAXED, __HIP_MEMORY_SCOPE_AGENT);
            }
            STAMP(1);
        }
    } else {
        const char* tab = myws + WS_TAB + (size_t)b * 512;
        const unsigned ldw = ctrl + C_LD + (wave - 4) * 4;
        unsigned long long t0 = 0;
        int av = 0, cons = 0;
        unsigned sb = 0;
#define GLDM(src_, dst_, n_, m_) do { _Pragma("unroll") for (int _i = 0; _i < (n_); _i += 4) { \
            if (_i + 4 <= (n_)) asm volatile("s_mov_b32 m0, %0\n\ts_waitcnt vmcnt(58)\n\t" \
                         "global_load_lds_dwordx4 %1, %2" m_ PACE "\n\tglobal_load_lds_dwordx4 %1, %2 offset:1024" m_ PACE "\n\t" \
                         "global_load_lds_dwordx4 %1, %2 offset:2048" m_ PACE "\n\tglobal_load_lds_dwordx4 %1, %2 offset:3072" m_ PACE \
                         :: "s"(lbase + (dst_) + _i * 1024), "v"(lo16), "s"((src_) + _i * 1024) : "memory"); \
            else asm volatile("s_mov_b32 m0, %0\n\ts_waitcnt vmcnt(58)\n\t" \
                         "global_load_lds_dwordx4 %1, %2" m_ "\n\tglobal_load_lds_dwordx4 %1, %2 offset:1024" m_ \
                         :: "s"(lbase + (dst_) + _i * 1024), "v"(lo16), "s"((src_) + _i * 1024) : "memory"); } } while (0)
#define GLDN(src_, dst_, n_) GLDM(src_, dst_, n_, "")
#define MARK(t_) asm volatile("s_mov_b32 m0, %0\n\ts_mov_b64 exec, 1\n\tglobal_load_lds_dword %1, %2\n\ts_mov_b64 exec, -1" \
                         :: "s"(ldw), "v"(lo16), "s"(tab + (t_) * 4) : "memory")
#define READY(word_, nav_, ncons_) do { if (av < (nav_) || cons < (ncons_)) for (unsigned _sp = 0;;) { u4 _c; unsigned _av; \
            asm volatile("ds_read_b128 %0, %2 offset:%4\n\tds_read_b32 %1, %2 offset:%3\n\ts_waitcnt lgkmcnt(0)" \
                         : "=&v"(_c), "=&v"(_av) : "v"(ctrl), "i"(word_), "i"((word_) == C_WAV ? C_CONW : C_CONS) : "memory"); \
            av = (int)rfl(_av); cons = (int)min4(_c); \
            if (av >= (nav_) && cons >= (ncons_)) break; \
            if (SPIN_OUT(_sp, t0)) break; \
            __builtin_amdgcn_s_sleep(AG_LSLEEP); } } while (0)
#define NEXT_REG() sb = (sb == (unsigned)(2 * REG)) ? 0u : sb + (unsigned)REG
        if (wave == 4) {
            const char* xa = srcws + WS_XS + (size_t)((call & 1u) * 2u + (unsigned)mt) * NKS * X_STEP;
            const char* xg = myws + WS_XG + (size_t)((((call & 1u) * NR + (unsigned)src) * 2u + (unsigned)mt) * K1) * X_STEP;
            for (int t = 0; t < KA; ++t) {
                READY(C_AVAIL, t + 1, t - 2);
                TRACE(0, t);
                const bool st = (PMASK >> t) & 1;
                const char* xt = st ? xg : xa;
                GLDN(xt, sb, 32);
                MARK(t);
                NEXT_REG();
                xa += X_STEP;
                xg += st ? X_STEP : 0;
            }
#pragma unroll 1
            for (int t = KA; t < NITEMS; ++t) {
                READY(C_AVAIL, t + 1, t - 2);
                TRACE(0, t);
                GLDN(xa, sb, 8);
                GLDN(xa + X_STEP, sb + 8192, 8);
                MARK(t);
                NEXT_REG();
                xa += sb == (unsigned)(KA % 3 * REG) ? 8192 - 4 * X_STEP : 2 * X_STEP;
            }
            VM_WAIT(0);
            STAMP(13);
            TRDUMP(0);
        } else {
            const char* wa = myws + WS_WH + (size_t)f0 * 2048;
            for (int t = 0; t < KA; ++t) {
                READY(C_WAV, t + 1, t - 2);
                TRACE(2, t);
                if ((AG_WNT >> mt) & 1) GLDM(wa, sb + X_STEP, 18, " nt"); else GLDN(wa, sb + X_STEP, 18);
                MARK(t);
                NEXT_REG();
                wa += W_ROW;
            }
#pragma unroll 1
            for (int j = 0; j < NAB; ++j) {
                READY(C_WAV, NKS, KA + (j >> 1) - 2);
                TRACE(2, KA + j);
                GLDN(wa, sb + X_PART + (j & 1) * W_STEP, 18);
                MARK(KA + j);
                if (j & 1) NEXT_REG();
                wa += W_ROW;
            }
            VM_WAIT(0);
            TRDUMP(2);
        }
    }
}

}

extern "C" void kernel_launch(int rank, int nranks, ncclComm_t comm,
                              hipStream_t stream,
                              void* const* d_in, const int* in_sizes, int n_in,
                              void* d_out, int out_size,
                              const int* in_dtypes, int out_dtype,
                              void* const* d_out_all,
                              void* const* d_ws_all, size_t ws_bytes) {
    (void)comm; (void)in_dtypes; (void)out_dtype; (void)d_out_all;
    if (nranks != NR || n_in < 2 || (unsigned)rank >= (unsigned)NR || ws_bytes < WS_NEED ||
        in_sizes[0] != MLOC * GK || in_sizes[1] != GK * NLOC || out_size != GM * NLOC) {
        fprintf(stderr, "ag_gemm: unexpected problem shape\n");
        return;
    }
    static bool attr_set = false;
    if (!attr_set) {
        HIP_CHECK(hipFuncSetAttribute((const void*)ag_gemm_kernel,
                                      hipFuncAttributeMaxDynamicSharedMemorySize, LDS_BYTES));
        attr_set = true;
    }
    Params p;
    p.x = (const float*)d_in[0];
    p.w = (const float*)d_in[1];
    p.out = (float*)d_out;
    p.myws = (char*)d_ws_all[rank];
    for (int r = 0; r < NR; ++r) p.ws[r] = (char*)d_ws_all[r];
    p.rank = rank;
    hipLaunchKernelGGL(ag_gemm_kernel, dim3(NWG), dim3(NTHREADS), LDS_BYTES, stream, p);
    HIP_CHECK(hipGetLastError());
}
```

```cpp
#include <hip/hip_runtime.h>
#include <rccl/rccl.h>
#include <cstdio>

#define HIP_CHECK(expr) do { hipError_t _e = (expr); if (_e != hipSuccess) { \
    fprintf(stderr, "HIP error %s at %s:%d\n", hipGetErrorString(_e), __FILE__, __LINE__); \
    return; } } while (0)

namespace {

#define GAS __attribute__((address_space(1)))
#define LAS __attribute__((address_space(3)))
typedef _Float16 h8 __attribute__((ext_vector_type(8)));
typedef _Float16 h2 __attribute__((ext_vector_type(2)));
typedef float f4 __attribute__((ext_vector_type(4)));
typedef unsigned u4 __attribute__((ext_vector_type(4)));
typedef unsigned u2 __attribute__((ext_vector_type(2)));

#define AG_PWIN 16
#define AG_WIN 8
#define AG_SLEEP_POLL 4
#define AG_SLEEP_FINE 1
#define AG_FINE 48
#define AG_NCONV 120
#define AG_KP0 30
#define AG_KPS 6
#define AG_SDELAY 8
#define AG_SEARLY 12
#define AG_SSLOPE 1
#define AG_LSLEEP 1
#define AG_EARLY0 2
#define AG_EARLY1 2
#define AG_EARLY2 0
#define AG_E2T 4
#define AG_EPRIO 0
#define AG_LPRIO 0
#define AG_RW 9
#define AG_WG 3
#define AG_NAB 8
#define AG_OWN 85
#define AG_CA 2
#define AG_AST 2
#define AG_TRYM 0x90
#define AST " nt"

constexpr int NR = 8;
constexpr int GM = 4096, GK = 4096, NLOC = 2048, MLOC = 512;
constexpr int BM = 256, BK = 64;
constexpr int FW = 9;
constexpr int NKS = GK / BK;
constexpr int K1 = 4;
constexpr int KP0 = AG_KP0, KPS = AG_KPS;
constexpr unsigned long long PMASK = (1ull | 1ull << KPS | 1ull << 2 * KPS | 1ull << 3 * KPS) << KP0;
constexpr int NAB = AG_NAB, NI = NAB / 2;
constexpr int KA = NKS - NAB;
constexpr int NITEMS = KA + 4 * NI;
constexpr int X_STEP = BM * BK * 2;
constexpr int X_PART = X_STEP / 2;
constexpr int W_BLK = 2048 + 64;
constexpr int W_STEP = FW * W_BLK - 64;
constexpr int W_ROW = NLOC * BK * 2;
constexpr int NW = 8;
constexpr int SCR_OFF = NW * W_STEP;
constexpr int CTRL_OFF = SCR_OFF + 8192;
constexpr int LDS_BYTES = CTRL_OFF + 256;
constexpr int NWG = 256, NTHREADS = 704;
static_assert(LDS_BYTES <= 160 * 1024 && NAB <= NW && NAB % 2 == 0 && NAB >= 4 && NKS % NW == 0, "layout");
static_assert((NKS - K1) * 4 == NWG - 16 && KP0 > 1 && KPS > 1 && KP0 + 3 * KPS < KA - 1, "layout");
constexpr int C_CONS = 0;
constexpr int C_WAV = 68;
constexpr int C_STG = 72;
constexpr int C_CALL = 76;
constexpr int C_LD = 84;
constexpr int C_BAD = 88;
constexpr int C_AVAIL = 92;
constexpr int NCONV = AG_NCONV;
constexpr float LOG2E = 1.4426950408889634f;

constexpr size_t FSTRIDE = 4096 + 256;
constexpr size_t WS_CALL = 0;
constexpr size_t WS_TMO = 49152;
constexpr size_t WS_SFLAG = 53248;
constexpr size_t WS_HINT = 65536;
constexpr size_t WS_WFLAG = 196608;
constexpr size_t WS_TAB = 327680;
constexpr size_t WS_DBG = 524288;
constexpr size_t WS_WH = 1ull << 20;
constexpr size_t WS_XS = 32ull << 20;
constexpr size_t WS_XG = 48ull << 20;
constexpr size_t WS_NEED = 64ull << 20;

constexpr unsigned long long DEADLINE = 100000000ull;
constexpr unsigned long long SILENCE = 20000ull;

struct Params {
    const float* x;
    const float* w;
    float* out;
    char* myws;
    char* ws[NR];
    int rank;
};

__device__ __forceinline__ unsigned long long rt() { return __builtin_amdgcn_s_memrealtime(); }

__device__ __forceinline__ void st16_sys(void* p, u4 v) {
    asm volatile("global_store_dwordx4 %0, %1, off sc0 sc1\n\ts_nop 1" :: "v"(p), "v"(v) : "memory");
}
__device__ __forceinline__ void st16_dev(void* p, u4 v) {
    asm volatile("global_store_dwordx4 %0, %1, off sc1\n\ts_nop 1" :: "v"(p), "v"(v) : "memory");
}
__device__ __forceinline__ void st4_sys(void* p, unsigned v) {
    asm volatile("global_store_dword %0, %1, off sc0 sc1\n\ts_nop 1" :: "v"(p), "v"(v) : "memory");
}
__device__ __forceinline__ void post(const char* p, unsigned zero, unsigned v) {
    asm volatile("s_mov_b64 exec, 1\n\tglobal_store_dword %0, %1, %2\n\ts_mov_b64 exec, -1" :: "v"(zero), "v"(v), "s"(p) : "memory");
}
__device__ __forceinline__ void lds_st(unsigned addr, unsigned v) {
    asm volatile("ds_write_b32 %0, %1" :: "v"(addr), "v"(v) : "memory");
}
__device__ __forceinline__ unsigned lds_ld(unsigned addr) {
    unsigned v;
    asm volatile("ds_read_b32 %0, %1\n\ts_waitcnt lgkmcnt(0)" : "=&v"(v) : "v"(addr) : "memory");
    return v;
}
__device__ __forceinline__ unsigned ld_ctr(const void* p) {
    unsigned v;
    asm volatile("global_load_dword %0, %1, off sc1\n\ts_waitcnt vmcnt(0)" : "=&v"(v) : "v"(p) : "memory");
    return (unsigned)__builtin_amdgcn_readfirstlane(v);
}
__device__ __forceinline__ unsigned pk(float a, float b) {
    h2 h; h.x = (_Float16)a; h.y = (_Float16)b;
    return __builtin_bit_cast(unsigned, h);
}
__device__ __forceinline__ u4 pk8(f4 a, f4 b) { return (u4){pk(a[0], a[1]), pk(a[2], a[3]), pk(b[0], b[1]), pk(b[2], b[3])}; }
__device__ __forceinline__ unsigned umin(unsigned a, unsigned b) { return a < b ? a : b; }
__device__ __forceinline__ unsigned rfl(unsigned v) { return (unsigned)__builtin_amdgcn_readfirstlane((int)v); }
__device__ __forceinline__ unsigned min8(u4 c, u4 d) {
    return rfl(umin(umin(umin(c.x, c.y), umin(c.z, c.w)), umin(umin(d.x, d.y), umin(d.z, d.w))));
}
__device__ __forceinline__ int row_min(int u) {
    u = min(u, __builtin_amdgcn_update_dpp(0, u, 0x121, 15, 15, false));
    u = min(u, __builtin_amdgcn_update_dpp(0, u, 0x122, 15, 15, false));
    u = min(u, __builtin_amdgcn_update_dpp(0, u, 0x124, 15, 15, false));
    u = min(u, __builtin_amdgcn_update_dpp(0, u, 0x128, 15, 15, false));
    return __builtin_amdgcn_readfirstlane(u);
}
__device__ __forceinline__ void arrivals(const char* sf, unsigned call, int lane, unsigned& staged, bool& stored) {
    u2 sv; unsigned pv;
    asm volatile("global_load_dwordx2 %0, %2, %4 sc0 sc1\n\tglobal_load_dword %1, %3, %4 offset:512 sc0 sc1\n\ts_waitcnt vmcnt(0)"
                 : "=&v"(sv), "=&v"(pv) : "v"(lane * 8), "v"((lane & 15) * 4), "s"(sf) : "memory");
    const unsigned long long miss = __ballot((int)(sv.x - call) < 0 || (int)(sv.y - call) < 0) & ~PMASK;
    staged = miss ? (unsigned)__builtin_ctzll(miss) : (unsigned)NKS;
    stored = __ballot((int)(pv - call) < 0) == 0;
}
#define VM_WAIT(n) asm volatile("s_waitcnt vmcnt(" #n ")" ::: "memory")
#define LGKM0() asm volatile("s_waitcnt lgkmcnt(0)" ::: "memory")
#define SCHED_FENCE() __builtin_amdgcn_sched_barrier(0)
#define SPIN_OUT(sp_, t0_) ((((++(sp_)) & 1023u) == 0u) && ((t0_) == 0 ? ((t0_) = rt(), false) : ((unsigned)(rt() - (t0_)) > 2u * (unsigned)DEADLINE)))

#define TRACE(k_, t_) do {} while (0)
#define TRDUMP(k_) do {} while (0)
#define TRZERO(k_) do {} while (0)
#define STAMP(i) do {} while (0)
#define STAMPC(i) do {} while (0)

#define DSR(dst_, addr_, off_) asm volatile("ds_read_b128 %0, %1 offset:%2" : "+v"(dst_) : "v"(addr_), "i"(off_) : "memory")

__device__ __forceinline__ f4 silu4(float z0, float z1, float z2, float z3, float lim, float c, f4 keep) {
    float o0, o1, o2, o3;
    asm volatile("v_med3_f32 %0, -%4, %8, -%8\n\tv_med3_f32 %1, -%5, %8, -%8\n\tv_med3_f32 %2, -%6, %8, -%8\n\tv_med3_f32 %3, -%7, %8, -%8\n\t"
                 "v_exp_f32 %0, %0\n\tv_exp_f32 %1, %1\n\tv_exp_f32 %2, %2\n\tv_exp_f32 %3, %3\n\t"
                 "v_fma_f32 %0, %0, %9, %9\n\tv_fma_f32 %1, %1, %9, %9\n\tv_fma_f32 %2, %2, %9, %9\n\tv_fma_f32 %3, %3, %9, %9\n\t"
                 "v_rcp_f32 %0, %0\n\tv_rcp_f32 %1, %1\n\tv_rcp_f32 %2, %2\n\tv_rcp_f32 %3, %3\n\t"
                 "v_mul_f32 %0, %4, %0\n\tv_mul_f32 %1, %5, %1\n\tv_mul_f32 %2, %6, %2\n\tv_mul_f32 %3, %7, %3"
                 : "=&v"(o0), "=&v"(o1), "=&v"(o2), "=&v"(o3) : "v"(z0), "v"(z1), "v"(z2), "v"(z3), "s"(lim), "s"(c), "v"(keep));
    return (f4){o0, o1, o2, o3};
}
#define REL() asm volatile("s_mov_b32 m0, %0\n\ts_mov_b64 exec, 1\n\tglobal_load_lds_dword %1, %2\n\ts_mov_b64 exec, -1" \
                           :: "s"(P.relw), "v"(P.rv), "s"(P.tab) : "memory")
#define TT(i_) do {} while (0)
struct Pos { unsigned w, n, x, rv, need, relw, ldw; const char* tab; const char* p[2]; u4 f; };
#define ADV(v_, d_) do { v_ += d_; asm volatile("" : "+v"(v_)); } while (0)
#define PIN(s_) asm volatile("" : "+s"(s_))
#define FLAGS() asm volatile("ds_read_b128 %0, %1" : "=v"(P.f) : "v"(P.ldw) : "memory")
#define WAIT_AV(nx_, nw_) do { unsigned long long _t0 = 0; unsigned _sp = 0; \
        for (;;) { asm volatile("ds_read_b128 %0, %1\n\ts_waitcnt lgkmcnt(0)" : "=&v"(P.f) : "v"(P.ldw) : "memory"); \
            if ((int)((rfl(P.f.w) - (unsigned)(nx_)) | (rfl(P.f.y) - (unsigned)(nw_))) >= 0) break; \
            if (SPIN_OUT(_sp, _t0)) break; \
            __builtin_amdgcn_s_sleep(1); } } while (0)
#define CHECK_AV(nx_, nw_) do { if (__builtin_expect((int)((rfl(P.f.w) - (unsigned)(nx_)) | (rfl(P.f.y) - (unsigned)(nw_))) < 0, 0)) \
            WAIT_AV(nx_, nw_); SCHED_FENCE(); } while (0)
#define MFMA(c_, x_, a_) asm volatile("v_mfma_f32_16x16x32_f16 %0, %1, %2, %0" : "+v"(c_) : "v"(x_), "v"(a_))
#define MFMA_R(c_, x_, a_, ad_, off_) asm volatile("v_mfma_f32_16x16x32_f16 %0, %2, %1, %0\n\tds_read_b128 %1, %3 offset:%4" \
                                                   : "+v"(c_), "+v"(a_) : "v"(x_), "v"(ad_), "i"(off_) : "memory")
#define XLD(x_, k_) asm volatile("global_load_dwordx4 %0, %1, %2 offset:%3" : "+v"(x_) : "v"(P.x), "s"(P.p[(k_) & 1]), "i"(((k_) & 2) * 512) : "memory")
constexpr int wcol(int i) { return i < 8 ? (i >> 2) * 4 * W_BLK + (i & 3) * 64 : 0; }

constexpr int RW = AG_RW, G_A = AG_CA;
struct Loop { const char* xa; const char* xg; unsigned t, set; };
template <int B, int G>
__device__ __forceinline__ void k_step(f4 (&acc)[FW][2], h8 (&a)[RW], h8 (&x)[8], Pos& P, Loop& L) {
    if constexpr (G < 36) {
        constexpr int h = G / 18, i = G % 18 / 2;
        if (G % 18 == 0) asm volatile("s_waitcnt vmcnt(%0) lgkmcnt(7)" :: "i"(h ? 5 : 3) : "memory");
        else if (G == G_A) asm volatile("s_waitcnt lgkmcnt(%1)" : "+v"(P.f) : "i"(3 + G_A / 2) : "memory");
        else if (G % 18 % 4 == 0) asm volatile("s_waitcnt lgkmcnt(7)" ::: "memory");
        if (G & 1) MFMA_R(acc[i][1], x[B * 4 + h * 2 + 1], a[i], i < 8 ? P.w : P.n, (h ? 1 - B : B) * W_STEP + (1 - h) * 1024 + wcol(i));
        else MFMA(acc[i][0], x[B * 4 + h * 2], a[i]);
        SCHED_FENCE();
        if (G == G_A) CHECK_AV(P.need, L.t + 1u);
        if (G > G_A && G <= G_A + 8 && !(G & 1)) XLD(x[4 - B * 4 + (G - G_A) / 2 - 1], (G - G_A) / 2 - 1);
        if (B && G == 18) { const int d = (L.t & (NW - 1)) ? 2 * W_STEP : (2 - NW) * W_STEP; ADV(P.w, d); ADV(P.n, d); }
        if (G == 20) { L.t += 1u; PIN(L.t); L.xa += X_STEP; PIN(L.xa); }
        if (G == 22) {
            const bool st = (PMASK >> L.t) & 1;
            P.p[0] = st ? L.xg : L.xa; PIN(P.p[0]);
            L.xg += st ? X_STEP : 0; PIN(L.xg);
        }
        if (G == 24) {
            P.need = L.t + 1u;
            if (B) P.p[1] = P.p[0] + 8192;
            else {
                const bool last = L.t == (unsigned)KA;
                P.p[0] -= last && L.set ? X_STEP : 0;
                P.p[1] = P.p[0] + (last && !L.set ? X_STEP : 8192);
                P.need = last && L.set ? 0u : P.need;
                PIN(P.p[0]);
            }
            PIN(P.p[1]); PIN(P.need);
        }
        if (G == 30) FLAGS();
        if (G == 32) ADV(P.rv, 4u);
        if (G == 34) REL();
        SCHED_FENCE();
        k_step<B, G + 1>(acc, a, x, P, L);
    }
}
constexpr int NT = 36 * NI, WG = AG_WG, G_C = 35 - RW;
constexpr bool flags(int t) { return t % 36 == (t % NT < NT - 36 ? G_C - 9 : 26); }
constexpr int xfetch(int t) {
    for (int k = 0; k < 4; ++k) if (t % 36 == (G_C + 1 + k > 9 * k + 8 ? G_C + 1 + k : 9 * k + 8)) return t % NT < NT - 72 ? k : -1;
    return -1;
}
struct Waits { int l[2 * NT + 1], v[NT / 9]; };
constexpr Waits waits() {
    Waits r{};
    int pw[2 * NT + 16] = {}, pf = 0, c = 0, done = -1, xs = 8;
    for (int k = 0; k < RW; ++k) pw[k] = c++;
    for (int t = 0; t <= 2 * NT; ++t) {
        int need = t % WG || t == 2 * NT ? -1 : pw[t + WG - 1];
        if (t % 36 == G_C || (t && t % NT == 0)) need = pf > need ? pf : need;
        r.l[t] = need <= done ? -1 : c - 1 - need < 15 ? c - 1 - need : 15;
        done = need > done ? need : done;
        if (t + RW < 2 * NT) pw[t + RW] = c++;
        if (flags(t)) pf = c++;
        if (t < NT && t % 9 == 0) r.v[t / 9] = xs - 1 - t / 9;
        if (xfetch(t) >= 0) ++xs;
    }
    return r;
}
static_assert(RW >= FW && RW > WG && NT % WG == 0, "parts");
constexpr int woff(int k) { return (k % 36 / 9 & 1) * W_STEP + k % 36 / 18 * 1024 + wcol(k % 9); }
template <int T, int E>
__device__ __forceinline__ void part_items(f4 (&acc)[FW][2], h8 (&a)[RW], h8 (&x)[8], Pos& P) {
    constexpr int left = waits().l[T];
    if constexpr (left >= 0 && (T > E - NT || T == 0)) asm volatile("s_waitcnt lgkmcnt(%0)" :: "i"(left) : "memory");
    if (T == E || T % 36 == G_C) asm volatile("" : "+v"(P.f));
    if constexpr (T < E) {
        constexpr int J = T / NT, N = T % NT / 36, G = T % 36, K = T + RW, X = (N & 1) * 4;
        if (G % 9 == 0 && (!J || N)) asm volatile("s_waitcnt vmcnt(%0)" :: "i"(waits().v[T % NT / 9]) : "memory");
        if constexpr (K < 2 * NT) MFMA_R(acc[T % FW][J], x[X + G / 9], a[T % RW], K % 9 < 8 ? P.w : P.n, woff(K));
        else MFMA(acc[T % FW][J], x[X + G / 9], a[T % RW]);
        SCHED_FENCE();
        if (G == 0 && N >= NI - 3) TT(J * 6 + 4 + N - NI);
        if (K < 2 * NT && K % 36 >= 34) {
            constexpr int d = K % NT / 36 == NI - 1 ? (2 - NAB) * W_STEP : 2 * W_STEP;
            if (K % 36 == 34) ADV(P.w, d); else ADV(P.n, d);
        }
        if (flags(T)) FLAGS();
        if (G == G_C && N < (J ? NI - 2 : NI - 1)) CHECK_AV(N < NI - 2 ? P.need + (N + 2) : 0u, J ? 0 : KA + 2 * N + 4);
        if (G == G_C - 2 && N < NI - 2) { P.p[0] += 2 * X_STEP; PIN(P.p[0]); P.p[1] += 2 * X_STEP; PIN(P.p[1]); }
        if constexpr (xfetch(T) >= 0) XLD(x[X + xfetch(T)], xfetch(T));
        SCHED_FENCE();
        part_items<T + 1, E>(acc, a, x, P);
    }
}
__device__ __forceinline__ void item_x(h8* x, unsigned vx, const char* p, unsigned lo, unsigned hi) {
    asm volatile("s_cmp_gt_u32 %7, %8\n\ts_cbranch_scc1 1f\n\t"
                 "global_load_dwordx4 %0, %4, %5\n\tglobal_load_dwordx4 %1, %4, %6\n\t"
                 "global_load_dwordx4 %2, %4, %5 offset:1024\n\tglobal_load_dwordx4 %3, %4, %6 offset:1024\n1:"
                 : "+v"(x[0]), "+v"(x[1]), "+v"(x[2]), "+v"(x[3])
                 : "v"(vx), "s"(p), "s"(p + X_STEP), "s"(lo), "s"(hi) : "memory", "scc");
}
__device__ __forceinline__ void to_frags(u4 o0, u4 o1, unsigned wr, unsigned rd, u4& f0, u4& f1) {
    asm volatile("ds_write_b128 %2, %4\n\tds_write_b128 %2, %5 offset:1024\n\tds_read_b128 %0, %3\n\tds_read_b128 %1, %6"
                 : "=&v"(f0), "=&v"(f1) : "v"(wr), "v"(rd), "v"(o0), "v"(o1), "v"(rd ^ 64u) : "memory");
}

__global__ void __launch_bounds__(NTHREADS, 3) ag_gemm_kernel(Params p) {
    extern __shared__ __attribute__((aligned(16))) unsigned char smem[];
    LAS unsigned char* lds = (LAS unsigned char*)smem;
    const int tid = threadIdx.x;
    const int wave = __builtin_amdgcn_readfirstlane(tid >> 6);
    const int lane = tid & 63;
    const int b = blockIdx.x;
    const int src = b & 7, idx = b >> 3;
    char* myws = p.myws;
    GAS unsigned* cctr = (GAS unsigned*)(myws + WS_CALL + (size_t)b * 128);
    GAS unsigned* tmo = (GAS unsigned*)(myws + WS_TMO);
    const unsigned lbase = (unsigned)(size_t)lds;
    const unsigned ctrl = lbase + CTRL_OFF;
    const unsigned lo16 = (unsigned)lane * 16u;
    const int r8 = lane >> 3, c8 = lane & 7;
    const int r16 = lane & 15, q = lane >> 4;
    const unsigned img = lbase + (unsigned)(r8 * 128 + (c8 ^ (r8 >> 1)) * 16);
    const unsigned frg = lbase + (unsigned)(r16 * 128 + (q ^ (r16 >> 1)) * 16);
    const char* srcws = p.ws[0];
#pragma unroll
    for (int r = 1; r < NR; ++r) if (src == r) srcws = p.ws[r];

    if (idx < 2) {
        const int cc = src * 2 + idx;
        const char* hint = myws + WS_HINT + (size_t)cc * FSTRIDE;
        if (wave < 2) {
            STAMP(20 + wave);
            f4 v[K1][4];
            unsigned cv;
            {
                const unsigned s0 = (unsigned)((cc * 32 + wave * 16 + r8) * GK + c8 * 8) * 4u;
                const unsigned s1 = (unsigned)((cc * 32 + wave * 16 + 8 + r8) * GK + (c8 ^ 4) * 8) * 4u;
                asm volatile("global_load_dword %0, %1, off sc1" : "=&v"(cv) : "v"(cctr) : "memory");
#pragma unroll
                for (int e = 0; e < K1; ++e)
                    asm volatile("global_load_dwordx4 %0, %4, %6\n\tglobal_load_dwordx4 %1, %4, %6 offset:16\n\t"
                                 "global_load_dwordx4 %2, %5, %6\n\tglobal_load_dwordx4 %3, %5, %6 offset:16"
                                 : "=&v"(v[e][0]), "=&v"(v[e][1]), "=&v"(v[e][2]), "=&v"(v[e][3])
                                 : "v"(s0), "v"(s1), "s"(p.x + (KP0 + e * KPS) * BK) : "memory");
            }
            const char* dst[NR];
#pragma unroll
            for (int j = 0; j < NR; ++j) {
                const int r = (p.rank + 1 + j + cc + 4 * wave) & 7;
                dst[j] = p.ws[0];
#pragma unroll
                for (int k = 1; k < NR; ++k) if (r == k) dst[j] = p.ws[k];
                dst[j] += WS_XG + (size_t)((p.rank * 2 + (cc >> 3)) * K1) * X_STEP + (size_t)((cc & 7) * 2 + wave) * 2048;
                asm volatile("" : "+s"(dst[j]));
            }
            u4 o[K1][2];
#pragma unroll
            for (int e = 0; e < K1; ++e) {
                asm volatile("s_waitcnt vmcnt(%5)" : "+v"(cv), "+v"(v[e][0]), "+v"(v[e][1]), "+v"(v[e][2]), "+v"(v[e][3])
                             : "i"(4 * (K1 - 1 - e)) : "memory");
                to_frags(pk8(v[e][0], v[e][1]), pk8(v[e][2], v[e][3]), img + 32768u + wave * 2048u, frg + 32768u + wave * 2048u, o[e][0], o[e][1]);
            }
            const unsigned call = rfl(cv) + 1u;
            const unsigned vo = lo16 + (call & 1u) * (unsigned)(NR * 2 * K1 * X_STEP);
            char* flag = myws + WS_SFLAG + (size_t)(cc >> 3) * FSTRIDE + 512 + ((cc & 7) * 2 + wave) * 4;
            if (wave == 0) STAMP(5);
            constexpr int W = AG_PWIN;
            static_assert(W >= 2 && W <= 32 && W % 2 == 0, "window");
#pragma unroll
            for (int e = 0; e < K1; ++e) {
                asm volatile("s_waitcnt lgkmcnt(%2)" : "+v"(o[e][0]), "+v"(o[e][1]) : "i"(4 * (K1 - 1 - e)) : "memory");
                const unsigned ve = vo + (unsigned)(e * X_STEP);
#pragma unroll
                for (int j = 0; j < NR; ++j) {
                    if (e == 0 && j == 4) __builtin_amdgcn_s_barrier();
                    asm volatile("s_waitcnt vmcnt(%0)" :: "i"(W - 2) : "memory");
                    asm volatile("global_store_dwordx4 %0, %1, %3 sc0 sc1\n\tglobal_store_dwordx4 %0, %2, %3 offset:1024 sc0 sc1\n\ts_nop 1"
                                 :: "v"(ve), "v"(o[e][0]), "v"(o[e][1]), "s"(dst[j]) : "memory");
                }
            }
            VM_WAIT(0);
            if (lane == 0) st4_sys(flag, call);
            STAMP(14 + wave);
            return;
        }
        if (wave == 6) lds_st(ctrl + C_AVAIL, 0u);
        const unsigned call = ld_ctr((const void*)cctr) + 1u;
        LGKM0();
        __builtin_amdgcn_s_barrier();
        asm volatile("" ::: "memory");
        unsigned long long t0 = 0;
        if (wave < 6) {
            const int qt = wave - 2;
            const char* hq = hint + qt * 4;
            const char* xa = srcws + WS_XS + (size_t)((call & 1u) * 2u + (unsigned)idx) * NKS * X_STEP;
            unsigned hv = call * 256u + 1u, zero = 0u;
            asm volatile("s_mov_b32 m0, %2" : "+v"(hv), "+v"(zero) : "s"(lbase + qt * 4096) : "memory");
            int av = 0;
#define STAGED(n_) do { if (av < (n_)) for (unsigned _sp = 0;;) { av = (int)rfl(lds_ld(ctrl + C_AVAIL)); \
                if (av >= (n_) || SPIN_OUT(_sp, t0)) break; \
                __builtin_amdgcn_s_sleep(1); } } while (0)
#define PULL4(a_) asm volatile("s_waitcnt vmcnt(%2)\n\tglobal_load_lds_dwordx4 %0, %1\n\t" \
                         "s_waitcnt vmcnt(%2)\n\tglobal_load_lds_dwordx4 %0, %1 offset:1024\n\t" \
                         "s_waitcnt vmcnt(%2)\n\tglobal_load_lds_dwordx4 %0, %1 offset:2048\n\t" \
                         "s_waitcnt vmcnt(%2)\n\tglobal_load_lds_dwordx4 %0, %1 offset:3072" \
                         :: "v"(lo16), "s"(a_), "i"(AG_WIN - 1) : "memory")
            unsigned long long own = src == p.rank ? rt() : 0;
#define PACE(n_) do { if (own) { own += n_; while (rt() < own) __builtin_amdgcn_s_sleep(2); } } while (0)
#pragma unroll 1
            for (int t = 0; t < KA; ++t) {
                STAGED(t + 1);
                PACE(AG_OWN);
                if (qt == 0) TRACE(0, t);
                const char* xt = xa + qt * 8192;
                if ((PMASK >> 1 >> t) & 1) { hv += 1u; xa += X_STEP; ++t; }
                const unsigned h2 = av < t + 2 ? hv : hv + 1u + (unsigned)((PMASK >> 2 >> t) & 1);
                if (AG_EARLY0 == 2) post(hq, zero, hv);
                if (AG_EARLY0 == 4) post(hq, zero, h2);
                PULL4(xt);
                if (AG_EARLY0 == 1) post(hq, zero, hv);
                if (AG_EARLY0 == 3) post(hq, zero, h2);
                PULL4(xt + 4096);
                if (AG_EARLY0 == 0) post(hq, zero, hv);
                hv += 1u;
                xa += X_STEP;
            }
            STAGED(NKS);
            xa += (qt >> 1) * X_STEP + (qt & 1) * 4096;
#pragma unroll
            for (int t = KA; t < NITEMS; ++t) {
                PACE(AG_OWN / 2);
                if (qt == 0) TRACE(0, t);
                if (AG_EARLY1) post(hq, zero, hv + (AG_EARLY1 - 1) + (t >= NITEMS - AG_E2T ? AG_EARLY2 : 0));
                PULL4(xa + (t - KA) % NI * 2 * X_STEP + ((t - KA) / NI % 2 * 2 + (t - KA) / NAB) * 8192);
                if (!AG_EARLY1) post(hq, zero, hv);
                hv += 1u;
            }
            VM_WAIT(0);
            STAMP(4 + wave);
            if (qt == 0) TRDUMP(0);
        } else if (wave == 6) {
            if (lane == 0) __hip_atomic_store(cctr, call, __ATOMIC_RELAXED, __HIP_MEMORY_SCOPE_AGENT);
            __builtin_amdgcn_fence(__ATOMIC_ACQUIRE, "");
            VM_WAIT(0);
            const char* sf = srcws + WS_SFLAG + (size_t)idx * FSTRIDE;
            unsigned staged = 0, npoll = 0;
            bool stored = false;
            post(hint + 16, 0u, call * 256u + KP0);
            t0 = rt();
            while (staged < (unsigned)NKS || !stored) {
                unsigned s2; bool p2;
                arrivals(sf, call, lane, s2, p2);
                if (s2 > staged) { if (!staged) STAMP(17); staged = s2; lds_st(ctrl + C_AVAIL, staged); if (s2 == NKS) STAMP(10); }
                if (p2 && !stored) { stored = true; post(hint + 16, 0u, call * 256u + NITEMS); }
                if ((++npoll & 31u) == 0u && (rfl(__hip_atomic_load(tmo, __ATOMIC_RELAXED, __HIP_MEMORY_SCOPE_AGENT)) == call
                                              || rt() - t0 > DEADLINE)) {
                    __hip_atomic_store(tmo, call, __ATOMIC_RELAXED, __HIP_MEMORY_SCOPE_AGENT);
                    post(hint + 16, 0u, call * 256u);
                    VM_WAIT(0);
                    lds_st(ctrl + C_AVAIL, (unsigned)NKS);
                    break;
                }
                if (npoll < 64u && staged < (unsigned)NKS) __builtin_amdgcn_s_sleep(2); else __builtin_amdgcn_s_sleep(32);
            }
            STAMP(11);
        }
        return;
    }

    const int mt = idx & 1, nt = (idx - 2) >> 1;
    const int f0 = (17 * nt) >> 1;
    if (wave < 8) {
        f4 sv[4][2];
        const int g = b - 16;
        int sk = g >> 2;
#pragma unroll
        for (int e = 0; e < K1; ++e) sk += sk >= KP0 + e * KPS;
        const bool late = wave < 4 && sk >= AG_SEARLY;
        if (wave < 4) {
            if (wave == 0) STAMP(0);
            const float* s0 = p.x + (size_t)((g & 3) * 128 + wave * 32 + r8) * GK + sk * BK;
            if (late) {
                __builtin_amdgcn_s_barrier();
                for (int i = AG_SEARLY; i < sk; ++i) __builtin_amdgcn_s_sleep(AG_SSLOPE);
            } else if (AG_SDELAY) __builtin_amdgcn_s_sleep(AG_SDELAY);
#pragma unroll
            for (int i = 0; i < 4; ++i)
                asm volatile("global_load_dwordx4 %0, %2, off\n\tglobal_load_dwordx4 %1, %2, off offset:16"
                             : "=&v"(sv[i][0]), "=&v"(sv[i][1]) : "v"(s0 + (size_t)i * 8 * GK + (c8 ^ ((i & 1) * 4)) * 8) : "memory");
        }
        if (!late) __builtin_amdgcn_s_barrier();
        asm volatile("" ::: "memory");
        const unsigned call = rfl(lds_ld(ctrl + C_CALL));
        if (wave < 4) {
            char* d0 = myws + WS_XS + ((size_t)((call & 1u) * 2u + (unsigned)((g >> 1) & 1)) * NKS + (size_t)sk) * X_STEP
                       + (size_t)((g & 1) * X_PART + wave * 4096) + lo16;
            asm volatile("s_waitcnt vmcnt(0)" : "+v"(sv[0][0]), "+v"(sv[0][1]), "+v"(sv[1][0]), "+v"(sv[1][1]),
                         "+v"(sv[2][0]), "+v"(sv[2][1]), "+v"(sv[3][0]), "+v"(sv[3][1]) :: "memory");
            u4 o[4];
#pragma unroll
            for (int i = 0; i < 4; i += 2)
                to_frags(pk8(sv[i][0], sv[i][1]), pk8(sv[i + 1][0], sv[i + 1][1]), img + (unsigned)(SCR_OFF + wave * 2048),
                         frg + (unsigned)(SCR_OFF + wave * 2048), o[i], o[i + 1]);
            asm volatile("s_waitcnt lgkmcnt(0)" : "+v"(o[0]), "+v"(o[1]), "+v"(o[2]), "+v"(o[3]) :: "memory");
#pragma unroll
            for (int i = 0; i < 4; ++i) st16_sys(d0 + i * 1024, o[i]);
            unsigned n = 0;
            if (lane == 0) {
                asm volatile("s_waitcnt vmcnt(0)\n\tds_add_rtn_u32 %0, %1, %2\n\ts_waitcnt lgkmcnt(0)"
                             : "=&v"(n) : "v"(ctrl + C_STG), "v"(1u) : "memory");
                if (n == 3u) st4_sys(myws + WS_SFLAG + (size_t)((g >> 1) & 1) * FSTRIDE + sk * 8 + (g & 1) * 4, call);
            }
            if (wave == 0) STAMP(16);
        }
        const int set = wave >> 2, w4 = wave & 3;
        const bool dup = !(nt & 1) && nt != 14;
        f4 acc[FW][2];
#pragma unroll
        for (int i = 0; i < FW; ++i) {
            acc[i][0] = (f4){0.f, 0.f, 0.f, 0.f}; acc[i][1] = (f4){0.f, 0.f, 0.f, 0.f};
            asm volatile("" : "+v"(acc[i][0]), "+v"(acc[i][1]));
        }
        h8 a[RW], x[8];
#pragma unroll
        for (int i = 0; i < RW; ++i) asm volatile("" : "=v"(a[i]));
#pragma unroll
        for (int j = 0; j < 8; ++j) asm volatile("" : "=v"(x[j]));
        Pos P;
        Loop L;
        P.w = lbase + (unsigned)((r16 >> 2) * W_BLK + q * 256 + (r16 & 3) * 16);
        P.n = lbase + (unsigned)(8 * W_BLK + q * 256 + (r16 & 3) * 64 + (r16 >> 2) * 16);
        P.x = lo16 + (unsigned)(set * X_PART + w4 * 2048);
        P.ldw = ctrl + C_LD - 4;
        P.rv = -4u; P.relw = ctrl + C_CONS + wave * 4; P.tab = myws + WS_TAB + (size_t)b * 512;
        unsigned vo = (unsigned)(q * NLOC + r16) * 16u, vn = (unsigned)(q * 4 * NLOC + r16) * 4u;
        asm volatile("" : "+v"(P.w), "+v"(P.n), "+v"(vn), "+v"(P.x), "+v"(P.rv), "+v"(P.ldw), "+v"(vo));
        L.xa = srcws + WS_XS + (size_t)((call & 1u) * 2u + (unsigned)mt) * NKS * X_STEP;
        L.xg = myws + WS_XG + (size_t)((((call & 1u) * NR + (unsigned)src) * 2u + (unsigned)mt) * K1) * X_STEP;
        L.set = set;
        const char* ol = (const char*)(p.out + (size_t)(src * MLOC + mt * BM + set * 128 + w4 * 16) * NLOC + f0 * 16);
        P.p[0] = L.xa; P.p[1] = L.xa + 8192;
        PIN(ol); PIN(P.relw); PIN(P.tab); PIN(L.xg); PIN(L.set); PIN(P.p[0]); PIN(P.p[1]);
        WAIT_AV(1, 0);
#pragma unroll
        for (int k = 0; k < 4; ++k) XLD(x[k], k);
        L.t = 1u; P.need = 2u; L.xa += X_STEP; P.p[0] = L.xa; P.p[1] = L.xa + 8192;
        PIN(L.t); PIN(P.need); PIN(L.xa); PIN(P.p[0]); PIN(P.p[1]);
        WAIT_AV(0, 1);
        if (wave == 0) { STAMP(18); STAMPC(30); }
        FLAGS();
#pragma unroll
        for (int i = 0; i < FW; ++i) DSR(a[i], i < 8 ? P.w : P.n, wcol(i));
        __builtin_amdgcn_s_waitcnt(0x0070);
        SCHED_FENCE();
#pragma unroll 1
        do {
            k_step<0, 0>(acc, a, x, P, L);
            k_step<1, 0>(acc, a, x, P, L);
        } while (L.t <= (unsigned)KA);
        if (wave == 0) { STAMP(2); STAMPC(31); }
        P.need = (unsigned)(KA + 1 + NI * set);
        const char* xp = L.xa - X_STEP;
        unsigned pre = P.need - set;
#define TRY() do { const unsigned _p = rfl(P.f.w); item_x(x, P.x, xp, pre < P.need ? P.need : ~0u, _p); \
                   item_x(x + 4, P.x, xp + 2 * X_STEP, pre <= P.need ? P.need + 1u : ~0u, _p); pre = _p; } while (0)
#define SILU(r_, J_, keep_) silu4(acc[(r_) & 4][J_][(r_) & 3], acc[((r_) & 4) + 1][J_][(r_) & 3], acc[((r_) & 4) + 2][J_][(r_) & 3], \
                                  acc[((r_) & 4) + 3][J_][(r_) & 3], 60.f * LOG2E, c, keep_)
#define STORE(r_, o_) asm volatile("global_store_dwordx4 %0, %1, %2 offset:%3" AST \
                                   :: "v"(vo), "v"(o_), "s"(ol + ((r_) & 3) * NLOC * 4), "i"(((r_) & 4) * 64) : "memory")
#define PART(J_) do { \
        SCHED_FENCE(); \
        TT(J_ * 6); \
        if (pre < P.need) WAIT_AV(P.need, 0); \
        item_x(x, P.x, xp, pre + 1u, P.need); \
        if (pre <= P.need) WAIT_AV(P.need + 1u, KA + 2); \
        P.p[0] = xp + 2 * X_STEP; P.p[1] = xp + 3 * X_STEP; PIN(P.p[0]); PIN(P.p[1]); \
        item_x(x + 4, P.x, P.p[0], pre, P.need); \
        if (!J_) _Pragma("unroll") for (int k = FW; k < RW; ++k) DSR(a[k], P.w, woff(k)); \
        if (J_) { if (pre < P.need) VM_WAIT(4); else VM_WAIT(8); } \
        if (J_ && AG_LPRIO && set) __builtin_amdgcn_s_setprio(AG_LPRIO); \
        SCHED_FENCE(); \
        part_items<J_ * NT, J_ * NT + NT>(acc, a, x, P); \
        TT(J_ * 6 + 4); \
        if (!J_) { xp += 8192; PIN(xp); P.need += NAB; pre = 0u; TRY(); } \
        if (!J_ && AG_EPRIO && set) __builtin_amdgcn_s_setprio(AG_EPRIO); \
        if (wave == 4 && J_) { STAMP(3); STAMPC(33); } \
        const float c = __builtin_bit_cast(float, __builtin_bit_cast(unsigned, LOG2E) | rfl(P.f.z)); \
        asm volatile("s_nop 7\n\ts_nop 7" : "+v"(acc[FW - 3][J_]), "+v"(acc[FW - 2][J_]), "+v"(acc[FW - 1][J_])); \
        f4 o = acc[FW - 1][J_], od[4]; \
        _Pragma("unroll") for (int r = 0; r < 4; ++r) { o = SILU(r, J_, o); STORE(r, o); } \
        if (!dup) { \
            const f4 on = silu4(acc[FW - 1][J_][0], acc[FW - 1][J_][1], acc[FW - 1][J_][2], acc[FW - 1][J_][3], 60.f * LOG2E, c, o); \
            _Pragma("unroll") for (int r = 0; r < 4; ++r) \
                asm volatile("global_store_dword %0, %1, %2 offset:512" AST :: "v"(vn), "v"(on[r]), "s"(ol + r * NLOC * 4) : "memory"); \
        } \
        _Pragma("unroll") for (int r = 4; r < 8; ++r) {        \
            if (!J_ && (AG_TRYM >> r & 1)) FLAGS(); \
            o = od[r - 4] = SILU(r, J_, o); \
            if (J_) STORE(r, o); \
            if (!J_ && (AG_TRYM >> r & 1)) { asm volatile("s_waitcnt lgkmcnt(0)" : "+v"(P.f) :: "memory"); TRY(); } \
        } \
        if (!J_) _Pragma("unroll") for (int r = 4; r < 8; ++r) STORE(r, od[r - 4]); \
        if (!J_ && AG_EPRIO) __builtin_amdgcn_s_setprio(0); \
        TT(J_ * 6 + 5); \
        asm volatile("s_nop 1" ::: "memory"); } while (0)
        PART(0);
        ADV(vo, 64 * NLOC * 4); ADV(vn, 64 * NLOC * 4);
        PART(1);
        if (wave == 4) { STAMP(4); STAMPC(34); }
        __builtin_amdgcn_endpgm();
    }
    if (wave == 10) {
        *(u2*)(myws + WS_TAB + (size_t)b * 512 + lane * 8) = (u2){2u * lane + 1u, 2u * lane + 2u};
        const unsigned cl = ld_ctr((const void*)cctr) + 1u;
        if (lane == 0) __hip_atomic_store(cctr, cl, __ATOMIC_RELAXED, __HIP_MEMORY_SCOPE_AGENT);
        lds_st(ctrl + lane * 4, lane == C_CALL / 4 ? cl : 0u);
        LGKM0();
    }
    __builtin_amdgcn_s_barrier();
    asm volatile("" ::: "memory");
    const unsigned call = rfl(lds_ld(ctrl + C_CALL));
    if (wave == 10) {
        __builtin_amdgcn_fence(__ATOMIC_ACQUIRE, "");
        VM_WAIT(0);
        GAS unsigned* hp = (GAS unsigned*)(myws + WS_HINT + (size_t)(src * 2 + mt) * FSTRIDE) + c8;
        GAS unsigned* wf0 = (GAS unsigned*)(myws + WS_WFLAG + (size_t)(f0 >> 3) * FSTRIDE) + lane;
        GAS unsigned* wf1 = (GAS unsigned*)(myws + WS_WFLAG + (size_t)((f0 + FW - 1) >> 3) * FSTRIDE) + lane;
        unsigned pub = 0, wav = 0, npoll = 0;
        TRZERO(1);
        const unsigned long long t0 = rt();
        unsigned long long tall = 0;
        bool dead = false;
        while (pub < (unsigned)NITEMS || wav < (unsigned)NKS) {
            const unsigned hv = __hip_atomic_load(hp, __ATOMIC_RELAXED, __HIP_MEMORY_SCOPE_AGENT);
            bool adv = false;
            if (wav < (unsigned)NKS) {
                const unsigned w0 = __hip_atomic_load(wf0, __ATOMIC_RELAXED, __HIP_MEMORY_SCOPE_AGENT);
                const unsigned w1 = __hip_atomic_load(wf1, __ATOMIC_RELAXED, __HIP_MEMORY_SCOPE_AGENT);
                const unsigned long long notready = __ballot((int)(w0 - call) < 0 || (int)(w1 - call) < 0);
                const unsigned w2 = notready ? (unsigned)__builtin_ctzll(notready) : 64u;
                if (w2 > wav) { wav = w2; lds_st(ctrl + C_WAV, wav); adv = true; }
            }
            const int d = (int)(hv - call * 256u);
            const unsigned av = (unsigned)row_min(c8 < 5 ? max(min(d, NITEMS), 0) : NITEMS);
            if (av > pub) {
                lds_st(ctrl + C_AVAIL, av);
                pub = av;
                tall = 0;
                TRACE(1, av - 1);
            } else if (!adv && (++npoll & 63u) == 0u) {
                if (rt() - t0 > DEADLINE) { dead = true; break; }
                unsigned staged; bool stored;
                arrivals(srcws + WS_SFLAG + (size_t)mt * FSTRIDE, call, lane, staged, stored);
                if (staged == (unsigned)NKS && stored) {
                    if (tall == 0) tall = rt();
                    else if (rt() - tall > SILENCE) { pub = NITEMS; lds_st(ctrl + C_AVAIL, pub); }
                } else if (rfl(__hip_atomic_load(tmo, __ATOMIC_RELAXED, __HIP_MEMORY_SCOPE_AGENT)) == call) { dead = true; break; }
            }
            if (pub < (unsigned)AG_FINE) __builtin_amdgcn_s_sleep(AG_SLEEP_POLL); else __builtin_amdgcn_s_sleep(AG_SLEEP_FINE);
        }
        if (dead) {
            __hip_atomic_store(tmo, call, __ATOMIC_RELAXED, __HIP_MEMORY_SCOPE_AGENT);
            lds_st(ctrl + C_BAD, 0x7fc00000u);
        }
        lds_st(ctrl + C_WAV, (unsigned)NKS);
        lds_st(ctrl + C_AVAIL, (unsigned)NITEMS);
        TRDUMP(1);
    } else if (wave == 9) {
        constexpr int CSTEP = 240 / NCONV;
        const int cq = ((idx - 2) / CSTEP) * 8 + src;
        if ((idx - 2) % CSTEP == 0 && cq < NCONV) {
            const unsigned so = (unsigned)(r8 * 8 * NLOC + c8 * 4) * 4u;
            const unsigned dof = (unsigned)((c8 >> 2) * 2048 + r8 * 256 + (c8 & 3) * 16);
            for (int u = cq; u < NKS * 16; u += NCONV) {
                const int ks = u >> 4, cn = u & 15;
                const float* sw = p.w + (size_t)ks * BK * NLOC + cn * 128;
                const char* dstw = myws + WS_WH + (size_t)ks * W_ROW + (size_t)cn * 16384;
                f4 v[4][8];
#pragma unroll
                for (int g = 0; g < 4; ++g)
#pragma unroll
                    for (int r = 0; r < 8; ++r)
                        asm volatile("global_load_dwordx4 %0, %1, %2 offset:%3 nt" : "=v"(v[g][r]) : "v"(so), "s"(sw + r * NLOC), "i"(g * 128) : "memory");
                u4 o[4][4];
#pragma unroll
                for (int g = 0; g < 4; ++g) {
                    asm volatile("s_waitcnt vmcnt(%8)" : "+v"(v[g][0]), "+v"(v[g][1]), "+v"(v[g][2]), "+v"(v[g][3]),
                                 "+v"(v[g][4]), "+v"(v[g][5]), "+v"(v[g][6]), "+v"(v[g][7]) : "i"(24 - 8 * g) : "memory");
#pragma unroll
                    for (int r = 0; r < 8; ++r) v[g][r] *= LOG2E;
#pragma unroll
                    for (int c = 0; c < 4; ++c)
                        o[g][c] = (u4){pk(v[g][0][c], v[g][1][c]), pk(v[g][2][c], v[g][3][c]), pk(v[g][4][c], v[g][5][c]), pk(v[g][6][c], v[g][7][c])};
                    SCHED_FENCE();
                }
#pragma unroll
                for (int g = 0; g < 4; ++g)
#pragma unroll
                    for (int c = 0; c < 4; ++c)
                        asm volatile("global_store_dwordx4 %0, %1, %2 offset:%3 sc1\n\ts_nop 1"
                                     :: "v"(dof), "v"(o[g][c]), "s"(dstw + g * 4096), "i"(c * 64) : "memory");
                VM_WAIT(0);
                if (lane == 0)
                    __hip_atomic_store((GAS unsigned*)(myws + WS_WFLAG + (size_t)cn * FSTRIDE + ks * 4), call,
                                       __ATOMIC_RELAXED, __HIP_MEMORY_SCOPE_AGENT);
            }
            STAMP(1);
        }
    } else {
        const char* tab = myws + WS_TAB + (size_t)b * 512;
        const char* wa = myws + WS_WH + (size_t)f0 * 2048;
        unsigned long long t0 = 0;
        int av = 0, cons = 0;
#pragma unroll 1
        for (int t = 0; t < NKS; ++t) {
            if (av <= t || cons <= t - NW) for (unsigned sp = 0;;) {
                u4 c, d; unsigned v;
                asm volatile("ds_read_b128 %0, %3\n\tds_read_b128 %1, %3 offset:16\n\tds_read_b32 %2, %3 offset:%4\n\t"
                             "s_waitcnt lgkmcnt(0)" : "=&v"(c), "=&v"(d), "=&v"(v) : "v"(ctrl + C_CONS), "i"(C_WAV - C_CONS) : "memory");
                av = (int)rfl(v); cons = (int)min8(c, d);
                if ((av > t && cons > t - NW) || SPIN_OUT(sp, t0)) break;
                __builtin_amdgcn_s_sleep(AG_LSLEEP);
            }
            TRACE(2, t);
            const unsigned sb = lbase + (unsigned)(t & (NW - 1)) * W_STEP;
#pragma unroll
            for (int i = 0; i < FW; i += 2) {
                asm volatile("s_mov_b32 m0, %0\n\ts_waitcnt vmcnt(58)\n\t"
                             "global_load_lds_dwordx4 %1, %2\n\tglobal_load_lds_dwordx4 %1, %2 offset:1024"
                             :: "s"(sb + i * W_BLK), "v"(lo16), "s"(wa + i * 2048) : "memory");
                if (i + 1 < FW) asm volatile("s_mov_b32 m0, %0\n\ts_nop 0\n\t"
                             "global_load_lds_dwordx4 %1, %2 offset:2048\n\tglobal_load_lds_dwordx4 %1, %2 offset:3072"
                             :: "s"(sb + i * W_BLK + 64), "v"(lo16), "s"(wa + i * 2048) : "memory");
            }
            asm volatile("s_mov_b32 m0, %0\n\ts_mov_b64 exec, 1\n\tglobal_load_lds_dword %1, %2\n\ts_mov_b64 exec, -1"
                         :: "s"(ctrl + C_LD), "v"(lo16), "s"(tab + t * 4) : "memory");
            wa += W_ROW;
        }
        VM_WAIT(0);
    }
}

}

extern "C" void kernel_launch(int rank, int nranks, ncclComm_t comm,
                              hipStream_t stream,
                              void* const* d_in, const int* in_sizes, int n_in,
                              void* d_out, int out_size,
                              const int* in_dtypes, int out_dtype,
                              void* const* d_out_all,
                              void* const* d_ws_all, size_t ws_bytes) {
    (void)comm; (void)in_dtypes; (void)out_dtype; (void)d_out_all;
    if (nranks != NR || n_in < 2 || (unsigned)rank >= (unsigned)NR || ws_bytes < WS_NEED ||
        in_sizes[0] != MLOC * GK || in_sizes[1] != GK * NLOC || out_size != GM * NLOC) {
        fprintf(stderr, "ag_gemm: unexpected problem shape\n");
        return;
    }
    static bool attr_set = false;
    if (!attr_set) {
        HIP_CHECK(hipFuncSetAttribute((const void*)ag_gemm_kernel,
                                      hipFuncAttributeMaxDynamicSharedMemorySize, LDS_BYTES));
        attr_set = true;
    }
    Params p;
    p.x = (const float*)d_in[0];
    p.w = (const float*)d_in[1];
    p.out = (float*)d_out;
    p.myws = (char*)d_ws_all[rank];
    for (int r = 0; r < NR; ++r) p.ws[r] = (char*)d_ws_all[r];
    p.rank = rank;
    hipLaunchKernelGGL(ag_gemm_kernel, dim3(NWG), dim3(NTHREADS), LDS_BYTES, stream, p);
    HIP_CHECK(hipGetLastError());
}
```

```cpp
#include <hip/hip_runtime.h>
#include <rccl/rccl.h>
#include <cstdio>
#include <cstddef>

#define HIP_CHECK(expr) do { hipError_t _e = (expr); if (_e != hipSuccess) { \
    fprintf(stderr, "HIP error %s at %s:%d\n", hipGetErrorString(_e), __FILE__, __LINE__); \
    return; } } while (0)

namespace {

#define GAS __attribute__((address_space(1)))
#define LAS __attribute__((address_space(3)))
typedef _Float16 h8 __attribute__((ext_vector_type(8)));
typedef _Float16 h2 __attribute__((ext_vector_type(2)));
typedef float f4 __attribute__((ext_vector_type(4)));
typedef unsigned u4 __attribute__((ext_vector_type(4)));
typedef unsigned u2 __attribute__((ext_vector_type(2)));

#define AG_PWIN 16
#define AG_WIN 8
#define AG_EFENCE 0
#define AG_HSTEP 14
#define AG_HN 4
#define AG_CFIRST -17
#define AG_CDELAY 20
#define AG_WNT 32
#define AG_WNT0 8
#define AG_NOCHK 3
#define AG_PAIR 0
#define AG_PUB 0
#define AG_PUBG 0
#define AG_RELG 34
#define AG_MAP 1
#define AG_SLEEP_POLL 4
#define AG_SLEEP_FINE 1
#define AG_FINE 48
#define AG_NCONV 120
#define AG_KP0 30
#define AG_KPS 6
#define AG_SDELAY 8
#define AG_SEARLY 12
#define AG_SSLOPE 1
#define AG_LSLEEP 1
#define AG_EARLY0 2
#define AG_EARLY1 2
#define AG_EARLY2 0
#define AG_E2T 4
#define AG_EPRIO 0
#define AG_LPRIO 0
#define AG_RW 9
#define AG_WG 3
#define AG_NAB 8
#define AG_OWN 85
#define AG_CA 2
#define AG_AST 2
#define AG_TRYM 0x90
#define AST " nt"

#define CST " sc1 nt"
#define WLD ""
#define CLD " nt"
#define XLP ""
#define PLD ""
#define SST " nt"
#define PST ""
#define SLD " nt"
constexpr int NR = 8;
constexpr int GM = 4096, GK = 4096, NLOC = 2048, MLOC = 512;
constexpr int BM = 256, BK = 64;
constexpr int FW = 9;
constexpr int NKS = GK / BK;
constexpr int K1 = 4;
constexpr int KP0 = AG_KP0, KPS = AG_KPS;
constexpr unsigned long long PMASK = (1ull | 1ull << KPS | 1ull << 2 * KPS | 1ull << 3 * KPS) << KP0;
constexpr int NAB = AG_NAB, NI = NAB / 2;
constexpr int KA = NKS - NAB;
constexpr int NITEMS = KA + 4 * NI;
constexpr int X_STEP = BM * BK * 2;
constexpr int X_PART = X_STEP / 2;
constexpr int W_BLK = 2048 + 64;
constexpr int W_STEP = FW * W_BLK - 64;
constexpr int W_ROW = NLOC * BK * 2;
constexpr int NW = 8;
constexpr int SCR_OFF = NW * W_STEP;
constexpr int CTRL_OFF = SCR_OFF + 8192;
constexpr int LDS_BYTES = CTRL_OFF + 512;
constexpr int NWG = 256, NTHREADS = 704;
static_assert(LDS_BYTES <= 160 * 1024 && NAB <= NW && NAB % 2 == 0 && NAB >= 4 && NKS % NW == 0, "layout");
static_assert((NKS - K1) * 4 == NWG - 16 && KP0 > 1 && KPS > 1 && KP0 + 3 * KPS < KA - 1, "layout");
constexpr int C_CONS = 0;
constexpr int C_WAV = 68;
constexpr int C_STG = 72;
constexpr int C_CALL = 76;
constexpr int C_LD = 84;
constexpr int C_BAD = 88;
constexpr int C_AVAIL = 92;
constexpr int NCONV = AG_NCONV;
constexpr float LOG2E = 1.4426950408889634f;

constexpr size_t FSTRIDE = 4096 + 256;
constexpr size_t WS_CALL = 0;
constexpr size_t WS_TMO = 49152;
constexpr size_t WS_SFLAG = 53248;
constexpr size_t WS_HINT = 65536;
constexpr size_t WS_WFLAG = 196608;
constexpr size_t WS_TAB = 327680;
constexpr size_t WS_DBG = 524288;
constexpr size_t WS_WH = 1ull << 20;
constexpr size_t WS_XS = 32ull << 20;
constexpr size_t WS_XG = 48ull << 20;
constexpr size_t WS_NEED = 64ull << 20;

constexpr unsigned long long DEADLINE = 100000000ull;
constexpr unsigned long long SILENCE = 20000ull;

struct Params {
    const float* x;
    char* myws;
    const float* w;
    float* out;
    char* ws[NR];
    char* xg[2 * NR];
    unsigned srcs;
};
#define AG_ROTATED
typedef const char* __attribute__((address_space(4))) const* KPTR;

__device__ __forceinline__ unsigned long long rt() { return __builtin_amdgcn_s_memrealtime(); }

__device__ __forceinline__ void st16_sys(void* p, u4 v) {
    asm volatile("global_store_dwordx4 %0, %1, off sc0 sc1" SST "\n\ts_nop 1" :: "v"(p), "v"(v) : "memory");
}
__device__ __forceinline__ void st16_dev(void* p, u4 v) {
    asm volatile("global_store_dwordx4 %0, %1, off sc1\n\ts_nop 1" :: "v"(p), "v"(v) : "memory");
}
__device__ __forceinline__ void st4_sys(void* p, unsigned v) {
    asm volatile("global_store_dword %0, %1, off sc0 sc1\n\ts_nop 1" :: "v"(p), "v"(v) : "memory");
}
__device__ __forceinline__ void post(const char* p, unsigned zero, unsigned v) {
    asm volatile("s_mov_b64 exec, 1\n\tglobal_store_dword %0, %1, %2\n\ts_mov_b64 exec, -1" :: "v"(zero), "v"(v), "s"(p) : "memory");
}
__device__ __forceinline__ void lds_st(unsigned addr, unsigned v) {
    asm volatile("ds_write_b32 %0, %1" :: "v"(addr), "v"(v) : "memory");
}
__device__ __forceinline__ unsigned lds_ld(unsigned addr) {
    unsigned v;
    asm volatile("ds_read_b32 %0, %1\n\ts_waitcnt lgkmcnt(0)" : "=&v"(v) : "v"(addr) : "memory");
    return v;
}
__device__ __forceinline__ unsigned ld_ctr(const void* p) {
    unsigned v;
    asm volatile("global_load_dword %0, %1, off sc1\n\ts_waitcnt vmcnt(0)" : "=&v"(v) : "v"(p) : "memory");
    return (unsigned)__builtin_amdgcn_readfirstlane(v);
}
__device__ __forceinline__ unsigned pk(float a, float b) {
    h2 h; h.x = (_Float16)a; h.y = (_Float16)b;
    return __builtin_bit_cast(unsigned, h);
}
__device__ __forceinline__ u4 pk8(f4 a, f4 b) { return (u4){pk(a[0], a[1]), pk(a[2], a[3]), pk(b[0], b[1]), pk(b[2], b[3])}; }
__device__ __forceinline__ unsigned umin(unsigned a, unsigned b) { return a < b ? a : b; }
__device__ __forceinline__ unsigned rfl(unsigned v) { return (unsigned)__builtin_amdgcn_readfirstlane((int)v); }
__device__ __forceinline__ unsigned min8(u4 c, u4 d) {
    return rfl(umin(umin(umin(c.x, c.y), umin(c.z, c.w)), umin(umin(d.x, d.y), umin(d.z, d.w))));
}
__device__ __forceinline__ int row_min(int u) {
    u = min(u, __builtin_amdgcn_update_dpp(0, u, 0x121, 15, 15, false));
    u = min(u, __builtin_amdgcn_update_dpp(0, u, 0x122, 15, 15, false));
    u = min(u, __builtin_amdgcn_update_dpp(0, u, 0x124, 15, 15, false));
    u = min(u, __builtin_amdgcn_update_dpp(0, u, 0x128, 15, 15, false));
    return __builtin_amdgcn_readfirstlane(u);
}
__device__ __forceinline__ void arrivals(const char* sf, unsigned call, int lane, unsigned& staged, bool& stored) {
    u2 sv; unsigned pv;
    asm volatile("global_load_dwordx2 %0, %2, %4 sc0 sc1\n\tglobal_load_dword %1, %3, %4 offset:512 sc0 sc1\n\ts_waitcnt vmcnt(0)"
                 : "=&v"(sv), "=&v"(pv) : "v"(lane * 8), "v"((lane & 15) * 4), "s"(sf) : "memory");
    const unsigned long long miss = __ballot((int)(sv.x - call) < 0 || (int)(sv.y - call) < 0) & ~PMASK;
    staged = miss ? (unsigned)__builtin_ctzll(miss) : (unsigned)NKS;
    stored = __ballot((int)(pv - call) < 0) == 0;
}
#define VM_WAIT(n) asm volatile("s_waitcnt vmcnt(" #n ")" ::: "memory")
#define LGKM0() asm volatile("s_waitcnt lgkmcnt(0)" ::: "memory")
#define SCHED_FENCE() __builtin_amdgcn_sched_barrier(0)
#define SPIN_OUT(sp_, t0_) ((((++(sp_)) & 1023u) == 0u) && ((t0_) == 0 ? ((t0_) = rt(), false) : ((unsigned)(rt() - (t0_)) > 2u * (unsigned)DEADLINE)))

#define TRACE(k_, t_) do {} while (0)
#define TRDUMP(k_) do {} while (0)
#define TRZERO(k_) do {} while (0)
#define STAMP(i) do {} while (0)
#define STAMPC(i) do {} while (0)

#define DSR(dst_, addr_, off_) asm volatile("ds_read_b128 %0, %1 offset:%2" : "+v"(dst_) : "v"(addr_), "i"(off_) : "memory")

__device__ __forceinline__ f4 silu4(float z0, float z1, float z2, float z3, float lim, float c, f4 keep) {
    float o0, o1, o2, o3;
    asm volatile("v_med3_f32 %0, -%4, %8, -%8\n\tv_med3_f32 %1, -%5, %8, -%8\n\tv_med3_f32 %2, -%6, %8, -%8\n\tv_med3_f32 %3, -%7, %8, -%8\n\t"
                 "v_exp_f32 %0, %0\n\tv_exp_f32 %1, %1\n\tv_exp_f32 %2, %2\n\tv_exp_f32 %3, %3\n\t"
                 "v_fma_f32 %0, %0, %9, %9\n\tv_fma_f32 %1, %1, %9, %9\n\tv_fma_f32 %2, %2, %9, %9\n\tv_fma_f32 %3, %3, %9, %9\n\t"
                 "v_rcp_f32 %0, %0\n\tv_rcp_f32 %1, %1\n\tv_rcp_f32 %2, %2\n\tv_rcp_f32 %3, %3\n\t"
                 "v_mul_f32 %0, %4, %0\n\tv_mul_f32 %1, %5, %1\n\tv_mul_f32 %2, %6, %2\n\tv_mul_f32 %3, %7, %3"
                 : "=&v"(o0), "=&v"(o1), "=&v"(o2), "=&v"(o3) : "v"(z0), "v"(z1), "v"(z2), "v"(z3), "s"(lim), "s"(c), "v"(keep));
    return (f4){o0, o1, o2, o3};
}
#define REL() asm volatile("s_mov_b32 m0, %0\n\ts_mov_b64 exec, 1\n\tglobal_load_lds_dword %1, %2\n\ts_mov_b64 exec, -1" \
                           :: "s"(P.relw), "v"(P.rv), "s"(P.tab) : "memory")
#define TT(i_) do {} while (0)
struct Pos { unsigned w, n, x, rv, need, relw, ldw; const char* tab; const char* p[2]; u4 f; };
#define TTJ(j_, i_) TT(i_)
#define ADV(v_, d_) do { v_ += d_; asm volatile("" : "+v"(v_)); } while (0)
#define PIN(s_) asm volatile("" : "+s"(s_))
#define FLAGS() asm volatile("ds_read_b128 %0, %1" : "=v"(P.f) : "v"(P.ldw) : "memory")
#define WAIT_AV(nx_, nw_) do { unsigned long long _t0 = 0; unsigned _sp = 0; \
        for (;;) { asm volatile("ds_read_b128 %0, %1\n\ts_waitcnt lgkmcnt(0)" : "+v"(P.f) : "v"(P.ldw) : "memory"); \
            if ((int)((rfl(P.f.w) - (unsigned)(nx_)) | (rfl(P.f.y) - (unsigned)(nw_))) >= 0) break; \
            if (SPIN_OUT(_sp, _t0)) break; \
            __builtin_amdgcn_s_sleep(1); } } while (0)
#define CHECK_AV(nx_, nw_) do { if (__builtin_expect((int)((rfl(P.f.w) - (unsigned)(nx_)) | (rfl(P.f.y) - (unsigned)(nw_))) < 0, 0)) \
            WAIT_AV(nx_, nw_); SCHED_FENCE(); } while (0)
#define MFMA(c_, x_, a_) asm volatile("v_mfma_f32_16x16x32_f16 %0, %1, %2, %0" : "+v"(c_) : "v"(x_), "v"(a_))
#define MFMA_R(c_, x_, a_, ad_, off_) asm volatile("v_mfma_f32_16x16x32_f16 %0, %2, %1, %0\n\tds_read_b128 %1, %3 offset:%4" \
                                                   : "+v"(c_), "+v"(a_) : "v"(x_), "v"(ad_), "i"(off_) : "memory")
#define XLD(x_, k_) asm volatile("global_load_dwordx4 %0, %1, %2 offset:%3" : "+v"(x_) : "v"(P.x), "s"(P.p[(k_) & 1]), "i"(((k_) & 2) * 512) : "memory")
constexpr int wcol(int i) { return i < 8 ? (i >> 2) * 4 * W_BLK + (i & 3) * 64 : 0; }

constexpr int RW = AG_RW, G_A = AG_CA;
struct Loop { const char* xa; const char* xg; unsigned t, set; };
template <int B, int G>
__device__ __forceinline__ void k_step(f4 (&acc)[FW][2], h8 (&a)[RW], h8 (&x)[8], Pos& P, Loop& L) {
    if constexpr (G < 36) {
        constexpr int h = G / 18, i = G % 18 / 2;
        if (G % 18 == 0) asm volatile("s_waitcnt vmcnt(%0) lgkmcnt(7)" :: "i"(h ? 5 : 3) : "memory");
        else if (G == G_A) asm volatile("s_waitcnt lgkmcnt(%1)" : "+v"(P.f) : "i"(3 + G_A / 2) : "memory");
        else if (G == AG_RELG && G <= 30) asm volatile("s_waitcnt lgkmcnt(%0)" :: "i"((G - 18) / 2) : "memory");
        else if (G % 18 % 4 == 0 && !(G > AG_RELG && G > 18)) asm volatile("s_waitcnt lgkmcnt(7)" ::: "memory");
        if (G & 1) MFMA_R(acc[i][1], x[B * 4 + h * 2 + 1], a[i], i < 8 ? P.w : P.n, (h ? 1 - B : B) * W_STEP + (1 - h) * 1024 + wcol(i));
        else MFMA(acc[i][0], x[B * 4 + h * 2], a[i]);
        SCHED_FENCE();
        if (G == G_A) CHECK_AV(P.need, L.t + 1u);
        if (G > G_A && G <= G_A + 8 && !(G & 1)) XLD(x[4 - B * 4 + (G - G_A) / 2 - 1], (G - G_A) / 2 - 1);
        if (B && G == 18) { const int d = (L.t & (NW - 1)) ? 2 * W_STEP : (2 - NW) * W_STEP; ADV(P.w, d); ADV(P.n, d); }
        if (G == 20) { L.t += 1u; PIN(L.t); L.xa += X_STEP; PIN(L.xa); }
        if (G == 22) {
            const bool st = (PMASK >> L.t) & 1;
            P.p[0] = st ? L.xg : L.xa; PIN(P.p[0]);
            L.xg += st ? X_STEP : 0; PIN(L.xg);
        }
        if (G == 24) {
            P.need = L.t + 1u;
            if (B) P.p[1] = P.p[0] + 8192;
            else {
                const bool last = L.t == (unsigned)KA;
                P.p[0] -= last && L.set ? X_STEP : 0;
                P.p[1] = P.p[0] + (last && !L.set ? X_STEP : 8192);
                P.need = last && L.set ? 0u : P.need;
                PIN(P.p[0]);
            }
            PIN(P.p[1]); PIN(P.need);
        }
        if (G == 30) FLAGS();
        if (G == (AG_RELG < 34 ? 14 : 32)) ADV(P.rv, 4u);
        if (G == AG_RELG) REL();
        SCHED_FENCE();
        k_step<B, G + 1>(acc, a, x, P, L);
    }
}
constexpr int NT = 36 * NI, WG = AG_WG, G_C = 35 - RW;
constexpr bool flags(int t) { return t % 36 == (t % NT < NT - 36 ? G_C - 9 : 26); }
constexpr int xfetch(int t) {
    for (int k = 0; k < 4; ++k) if (t % 36 == (G_C + 1 + k > 9 * k + 8 ? G_C + 1 + k : 9 * k + 8)) return t % NT < NT - 72 ? k : -1;
    return -1;
}
constexpr bool pub(int t) { return AG_PUB && t % 36 == AG_PUBG && t % NT / 36 + AG_PUB >= NI; }
struct Waits { int l[2 * NT + 1], v[NT / 9]; };
constexpr Waits waits() {
    Waits r{};
    int pw[2 * NT + 16] = {}, pf = 0, c = 0, done = -1, xs = 8;
    for (int k = 0; k < RW; ++k) pw[k] = c++;
    for (int t = 0; t <= 2 * NT; ++t) {
        int need = t % WG || t == 2 * NT ? -1 : pw[t + WG - 1];
        if (t % 36 == G_C || (t && t % NT == 0)) need = pf > need ? pf : need;
        r.l[t] = need <= done ? -1 : c - 1 - need < 15 ? c - 1 - need : 15;
        done = need > done ? need : done;
        if (t + RW < 2 * NT) pw[t + RW] = c++;
        if (flags(t)) pf = c++;
        if (pub(t)) ++c;
        if (t < NT && t % 9 == 0) r.v[t / 9] = xs - 1 - t / 9;
        if (xfetch(t) >= 0) ++xs;
    }
    return r;
}
static_assert(RW >= FW && RW > WG && NT % WG == 0, "parts");
constexpr int woff(int k) { return (k % 36 / 9 & 1) * W_STEP + k % 36 / 18 * 1024 + wcol(k % 9); }
template <int T, int E>
__device__ __forceinline__ void part_items(f4 (&acc)[FW][2], h8 (&a)[RW], h8 (&x)[8], Pos& P) {
    constexpr int left = waits().l[T];
    if constexpr (left >= 0 && (T > E - NT || T == 0)) asm volatile("s_waitcnt lgkmcnt(%0)" :: "i"(left) : "memory");
    if (T == E || T % 36 == G_C) asm volatile("" : "+v"(P.f));
    if constexpr (T < E) {
        constexpr int J = T / NT, N = T % NT / 36, G = T % 36, K = T + RW, X = (N & 1) * 4;
        if (G % 9 == 0 && (!J || N)) asm volatile("s_waitcnt vmcnt(%0)" :: "i"(waits().v[T % NT / 9]) : "memory");
        if constexpr (K < 2 * NT) MFMA_R(acc[T % FW][J], x[X + G / 9], a[T % RW], K % 9 < 8 ? P.w : P.n, woff(K));
        else MFMA(acc[T % FW][J], x[X + G / 9], a[T % RW]);
        SCHED_FENCE();
        if (G == 0 && N >= NI - 3) TT(J * 6 + 4 + N - NI);
        if (K < 2 * NT && K % 36 >= 34) {
            constexpr int d = K % NT / 36 == NI - 1 ? (2 - NAB) * W_STEP : 2 * W_STEP;
            if (K % 36 == 34) ADV(P.w, d); else ADV(P.n, d);
        }
        if (flags(T)) FLAGS();
        if (pub(T)) asm volatile("v_mov_b32 %0, %2\n\ts_mov_b64 exec, 1\n\tds_max_u32 %1, %0 offset:12\n\ts_mov_b64 exec, -1"
                                 : "+v"(P.rv) : "v"(P.ldw), "s"(P.need + (N + AG_PUB)) : "memory");
        if (G == G_C && N < (J ? NI - 2 : NI - 1) && !(J && (AG_NOCHK & 1)))
            CHECK_AV(N < NI - 2 && !(AG_NOCHK & 2) ? P.need + (N + 2) : 0u, J ? 0 : KA + 2 * N + 4);
        if (G == G_C - 2 && N < NI - 2) { P.p[0] += 2 * X_STEP; PIN(P.p[0]); P.p[1] += 2 * X_STEP; PIN(P.p[1]); }
        if constexpr (xfetch(T) >= 0) XLD(x[X + xfetch(T)], xfetch(T));
        SCHED_FENCE();
        part_items<T + 1, E>(acc, a, x, P);
    }
}
__device__ __forceinline__ void item_x(h8* x, unsigned vx, const char* p, unsigned lo, unsigned hi) {
    asm volatile("s_cmp_gt_u32 %7, %8\n\ts_cbranch_scc1 1f\n\t"
                 "global_load_dwordx4 %0, %4, %5\n\tglobal_load_dwordx4 %1, %4, %6\n\t"
                 "global_load_dwordx4 %2, %4, %5 offset:1024\n\tglobal_load_dwordx4 %3, %4, %6 offset:1024\n1:"
                 : "+v"(x[0]), "+v"(x[1]), "+v"(x[2]), "+v"(x[3])
                 : "v"(vx), "s"(p), "s"(p + X_STEP), "s"(lo), "s"(hi) : "memory", "scc");
}
__device__ __forceinline__ void to_frags(u4 o0, u4 o1, unsigned wr, unsigned rd, u4& f0, u4& f1) {
    asm volatile("ds_write_b128 %2, %4\n\tds_write_b128 %2, %5 offset:1024\n\tds_read_b128 %0, %3\n\tds_read_b128 %1, %6"
                 : "=&v"(f0), "=&v"(f1) : "v"(wr), "v"(rd), "v"(o0), "v"(o1), "v"(rd ^ 64u) : "memory");
}

__global__ void __launch_bounds__(NTHREADS, 3) ag_gemm_kernel(Params p) {
    extern __shared__ __attribute__((aligned(16))) unsigned char smem[];
#define STAMPE(i) do {} while (0)
    LAS unsigned char* lds = (LAS unsigned char*)smem;
    const int tid = threadIdx.x;
    const int wave = __builtin_amdgcn_readfirstlane(tid >> 6);
    const int lane = tid & 63;
    const int b = blockIdx.x;
    const int xr = b & 7, idx = b >> 3;
    char* myws = p.myws;
    GAS unsigned* cctr = (GAS unsigned*)(myws + WS_CALL + (size_t)b * 128);
    GAS unsigned* tmo = (GAS unsigned*)(myws + WS_TMO);
    const unsigned lbase = (unsigned)(size_t)lds;
    const unsigned ctrl = lbase + CTRL_OFF;
    const unsigned lo16 = (unsigned)lane * 16u;
    const int r8 = lane >> 3, c8 = lane & 7;
    const int r16 = lane & 15, q = lane >> 4;
    const unsigned img = lbase + (unsigned)(r8 * 128 + (c8 ^ (r8 >> 1)) * 16);
    const unsigned frg = lbase + (unsigned)(r16 * 128 + (q ^ (r16 >> 1)) * 16);
    const char* __attribute__((address_space(4))) const* ka = (KPTR)__builtin_amdgcn_kernarg_segment_ptr();

    if (__builtin_expect(idx < 2 && wave < 2, 1)) {
        const int pid = (xr * 2 + idx) * 2 + wave;
        if (wave < 2) STAMP(20 + wave);
        f4 v[K1][4];
        unsigned cv;
        {
            const unsigned s0 = (unsigned)((pid * 16 + r8) * GK + c8 * 8) * 4u;
            const unsigned s1 = (unsigned)((pid * 16 + 8 + r8) * GK + (c8 ^ 4) * 8) * 4u;
#pragma unroll
            for (int e = 0; e < K1; ++e) {
                asm volatile("global_load_dwordx4 %0, %4, %6" PLD "\n\tglobal_load_dwordx4 %1, %4, %6 offset:16" PLD "\n\t"
                             "global_load_dwordx4 %2, %5, %6" PLD "\n\tglobal_load_dwordx4 %3, %5, %6 offset:16" PLD
                             : "=&v"(v[e][0]), "=&v"(v[e][1]), "=&v"(v[e][2]), "=&v"(v[e][3])
                             : "v"(s0), "v"(s1), "s"(p.x + (KP0 + e * KPS) * BK) : "memory");
                if (e == 0) asm volatile("global_load_dword %0, %1, %2 sc1" : "=&v"(cv) : "v"(b * 128), "s"(myws + WS_CALL) : "memory");
            }
        }
        SCHED_FENCE();
        const char* dst[NR];
#pragma unroll
        for (int j = 0; j < NR; ++j) dst[j] = ka[offsetof(Params, xg) / 8 + ((xr * 2 + idx + 4 * wave) & 7) + j];
        unsigned vo = lo16 + (unsigned)((pid >> 4) * K1 * X_STEP + (pid & 15) * 2048), call = 0;
        char* flag = myws + WS_SFLAG + (size_t)(pid >> 4) * FSTRIDE + 512 + (pid & 15) * 4;
        constexpr int W = AG_PWIN;
        static_assert(W >= 2 && W <= 32 && W % 2 == 0, "window");
        u4 o[K1][2];
#pragma unroll
        for (int e = 0; e < K1; ++e) {
            if (e < 2) {
#pragma unroll
                for (int k = e; k < (e ? K1 : 1); ++k) {
                    asm volatile("s_waitcnt vmcnt(%5)" : "+v"(cv), "+v"(v[k][0]), "+v"(v[k][1]), "+v"(v[k][2]), "+v"(v[k][3])
                                 : "i"(4 * (K1 - 1 - k) + e * 2 * NR) : "memory");
                    to_frags(pk8(v[k][0], v[k][1]), pk8(v[k][2], v[k][3]), img + 32768u + wave * 2048u, frg + 32768u + wave * 2048u, o[k][0], o[k][1]);
                }
            }
            if (e == 0) {
                call = rfl(cv) + 1u;
                vo += (call & 1u) * (unsigned)(NR * 2 * K1 * X_STEP);
                if (wave == 0) STAMP(5);
            }
            if (e == 1) __builtin_amdgcn_s_barrier();
            asm volatile("s_waitcnt lgkmcnt(%2)" : "+v"(o[e][0]), "+v"(o[e][1]) : "i"(e ? 4 * (K1 - 1 - e) : 0) : "memory");
            const unsigned ve = vo + (unsigned)(e * X_STEP);
#pragma unroll
            for (int j = 0; j < NR; ++j) {
                if (e) asm volatile("s_waitcnt vmcnt(%0)" :: "i"(W - 2) : "memory");
                asm volatile("global_store_dwordx4 %0, %1, %3 sc0 sc1" PST "\n\tglobal_store_dwordx4 %0, %2, %3 offset:1024 sc0 sc1" PST "\n\ts_nop 1"
                             :: "v"(ve), "v"(o[e][0]), "v"(o[e][1]), "s"(dst[j]) : "memory");
            }
        }
        VM_WAIT(0);
        if (lane == 0) st4_sys(flag, call);
        if (wave < 2) STAMP(14 + wave);
        if (wave == 0) STAMPE(22);
        __builtin_amdgcn_endpgm();
    }
    const char* srcws = ka[offsetof(Params, ws) / 8 + xr];
    if (idx < 2) {
        const char* hint = myws + WS_HINT + (size_t)(xr * 2 + idx) * FSTRIDE;
        if (wave == 7) lds_st(ctrl + C_AVAIL, 0u);
        const unsigned call = ld_ctr((const void*)cctr) + 1u;
        LGKM0();
        __builtin_amdgcn_s_barrier();
        asm volatile("" ::: "memory");
        unsigned long long t0 = 0;
        if (wave >= 3 && wave < 7) {
            const int qt = wave - 3;
            const char* hq = hint + qt * 4;
            const char* xa = srcws + WS_XS + (size_t)((call & 1u) * 2u + (unsigned)idx) * NKS * X_STEP;
            unsigned hv = call * 256u + 1u, zero = 0u;
            asm volatile("s_mov_b32 m0, %2" : "+v"(hv), "+v"(zero) : "s"(lbase + qt * 4096) : "memory");
            int av = 0;
#define STAGED(n_) do { if (av < (n_)) for (unsigned _sp = 0;;) { av = (int)rfl(lds_ld(ctrl + C_AVAIL)); \
                if (av >= (n_) || SPIN_OUT(_sp, t0)) break; \
                __builtin_amdgcn_s_sleep(1); } } while (0)
#define PULL4(a_) asm volatile("s_waitcnt vmcnt(%2)\n\tglobal_load_lds_dwordx4 %0, %1\n\t" \
                         "s_waitcnt vmcnt(%2)\n\tglobal_load_lds_dwordx4 %0, %1 offset:1024\n\t" \
                         "s_waitcnt vmcnt(%2)\n\tglobal_load_lds_dwordx4 %0, %1 offset:2048\n\t" \
                         "s_waitcnt vmcnt(%2)\n\tglobal_load_lds_dwordx4 %0, %1 offset:3072" \
                         :: "v"(lo16), "s"(a_), "i"(AG_WIN - 1) : "memory")
            unsigned long long own = srcws == myws ? rt() : 0;
#define PACE(n_) do { if (own) { own += n_; while (rt() < own) __builtin_amdgcn_s_sleep(2); } } while (0)
#pragma unroll 1
            for (int t = 0; t < KA; ++t) {
                STAGED(t + 1);
                PACE(AG_OWN);
                if (qt == 0) TRACE(0, t);
                if (qt == 0 && (t & 7) == 1) STAMP(54 + t / 8);

                if (qt == 0 && t < 12) STAMP(36 + t);
                const char* xt = xa + qt * 8192;
                if ((PMASK >> 1 >> t) & 1) { hv += 1u; xa += X_STEP; ++t; }
                post(hq, zero, hv);
                PULL4(xt);
                PULL4(xt + 4096);
                hv += 1u;
                xa += X_STEP;
            }
            STAGED(NKS);
            xa += (qt >> 1) * X_STEP + (qt & 1) * 4096;
#pragma unroll
            for (int t = KA; t < NITEMS; ++t) {
                PACE(AG_OWN / 2);
                if (qt == 0) TRACE(0, t);
                hv += 1u;
                post(hq, zero, hv);
                PULL4(xa + (t - KA) % NI * 2 * X_STEP + ((t - KA) / NI % 2 * 2 + (t - KA) / NAB) * 8192);
            }
            VM_WAIT(0);
            STAMP(6 + qt);
        } else if (wave == 7) {
            if (lane == 0) __hip_atomic_store(cctr, call, __ATOMIC_RELAXED, __HIP_MEMORY_SCOPE_AGENT);
            __builtin_amdgcn_fence(__ATOMIC_ACQUIRE, "");
            VM_WAIT(0);
            post(hint + 16, 0u, call * 256u + KP0);
            const char* sf = srcws + WS_SFLAG + (size_t)idx * FSTRIDE;
            unsigned staged = 0, npoll = 0;
            bool stored = false;
            t0 = rt();
            STAMP(19);
            while (staged < (unsigned)NKS || !stored) {
                unsigned s2; bool p2;
                arrivals(sf, call, lane, s2, p2);
                if (!npoll) STAMP(27);
                if (AG_EFENCE && !npoll) post(hint + 16, 0u, call * 256u + KP0);
                if (s2 > staged) {
                    asm volatile("ds_max_u32 %0, %1" :: "v"(lane ? ctrl + 128 + lane * 4 : ctrl + C_AVAIL), "v"(s2) : "memory");
                    if (!staged) STAMP(17);
                    staged = s2;
                    if (s2 == NKS) STAMP(10);
                }
                if (p2 && !stored) { stored = true; post(hint + 16, 0u, call * 256u + NITEMS); }
                if ((++npoll & 31u) == 0u && (rfl(__hip_atomic_load(tmo, __ATOMIC_RELAXED, __HIP_MEMORY_SCOPE_AGENT)) == call
                                              || rt() - t0 > DEADLINE)) {
                    __hip_atomic_store(tmo, call, __ATOMIC_RELAXED, __HIP_MEMORY_SCOPE_AGENT);
                    post(hint + 16, 0u, call * 256u);
                    VM_WAIT(0);
                    lds_st(ctrl + C_AVAIL, (unsigned)NKS);
                    break;
                }
                if (npoll < 64u && staged < (unsigned)NKS) __builtin_amdgcn_s_sleep(2); else __builtin_amdgcn_s_sleep(32);
            }
            STAMP(11);
        }
        return;
    }

    const int mt = idx & 1, nt = (idx - 2) >> 1;
    const int f0 = (17 * nt) >> 1;
    if (wave < 8) {
        f4 sv[4][2];
        const int g = b - 16;
        int sk = g >> 2;
#pragma unroll
        for (int e = 0; e < K1; ++e) sk += sk >= KP0 + e * KPS;
        const bool late = wave < 4 && sk >= AG_SEARLY;
        if (wave < 4) {
            if (wave == 0) STAMP(0);
            const float* s0 = p.x + (size_t)((g & 3) * 128 + wave * 32 + r8) * GK + sk * BK;
            if (late) {
                __builtin_amdgcn_s_barrier();
                for (int i = AG_SEARLY; i < sk; ++i) __builtin_amdgcn_s_sleep(AG_SSLOPE);
            } else if (AG_SDELAY) __builtin_amdgcn_s_sleep(AG_SDELAY);
#pragma unroll
            for (int i = 0; i < 4; ++i)
                asm volatile("global_load_dwordx4 %0, %2, off" SLD "\n\tglobal_load_dwordx4 %1, %2, off offset:16" SLD
                             : "=&v"(sv[i][0]), "=&v"(sv[i][1]) : "v"(s0 + (size_t)i * 8 * GK + (c8 ^ ((i & 1) * 4)) * 8) : "memory");
        }
        if (!late) __builtin_amdgcn_s_barrier();
        asm volatile("" ::: "memory");
        const unsigned call = rfl(lds_ld(ctrl + C_CALL));
        if (wave < 4) {
            char* d0 = myws + WS_XS + ((size_t)((call & 1u) * 2u + (unsigned)((g >> 1) & 1)) * NKS + (size_t)sk) * X_STEP
                       + (size_t)((g & 1) * X_PART + wave * 4096) + lo16;
            asm volatile("s_waitcnt vmcnt(0)" : "+v"(sv[0][0]), "+v"(sv[0][1]), "+v"(sv[1][0]), "+v"(sv[1][1]),
                         "+v"(sv[2][0]), "+v"(sv[2][1]), "+v"(sv[3][0]), "+v"(sv[3][1]) :: "memory");
            if (wave == 0) STAMP(24);
            u4 o[4];
#pragma unroll
            for (int i = 0; i < 4; i += 2)
                to_frags(pk8(sv[i][0], sv[i][1]), pk8(sv[i + 1][0], sv[i + 1][1]), img + (unsigned)(SCR_OFF + wave * 2048),
                         frg + (unsigned)(SCR_OFF + wave * 2048), o[i], o[i + 1]);
            asm volatile("s_waitcnt lgkmcnt(0)" : "+v"(o[0]), "+v"(o[1]), "+v"(o[2]), "+v"(o[3]) :: "memory");
#pragma unroll
            for (int i = 0; i < 4; ++i)
                st16_sys(d0 + i * 1024, o[i]);
            if (wave == 0) STAMP(25);
            unsigned n = 0;
            if (lane == 0) {
                asm volatile("s_waitcnt vmcnt(0)\n\tds_add_rtn_u32 %0, %1, %2\n\ts_waitcnt lgkmcnt(0)"
                             : "=&v"(n) : "v"(ctrl + C_STG), "v"(1u) : "memory");
                if (n == 3u) st4_sys(myws + WS_SFLAG + (size_t)((g >> 1) & 1) * FSTRIDE + sk * 8 + (g & 1) * 4, call);
            }
            if (wave == 0) { STAMP(16); STAMPE(23); }
        }
        const int set = wave >> 2, w4 = wave & 3;
        const bool dup = !(nt & 1) && nt != 14;
        f4 acc[FW][2];
#pragma unroll
        for (int i = 0; i < FW; ++i) {
            acc[i][0] = (f4){0.f, 0.f, 0.f, 0.f}; acc[i][1] = (f4){0.f, 0.f, 0.f, 0.f};
            asm volatile("" : "+v"(acc[i][0]), "+v"(acc[i][1]));
        }
        h8 a[RW], x[8];
#pragma unroll
        for (int i = 0; i < RW; ++i) asm volatile("" : "=v"(a[i]));
#pragma unroll
        for (int j = 0; j < 8; ++j) asm volatile("" : "=v"(x[j]));
        Pos P;
        Loop L;
        asm volatile("" : "=v"(P.f));
        P.w = lbase + (unsigned)((r16 >> 2) * W_BLK + q * 256 + (r16 & 3) * 16);
        P.n = lbase + (unsigned)(8 * W_BLK + q * 256 + (r16 & 3) * 64 + (r16 >> 2) * 16);
        P.x = lo16 + (unsigned)(set * X_PART + w4 * 2048);
        P.ldw = ctrl + C_LD - 4;
        P.rv = -4u; P.relw = ctrl + C_CONS + wave * 4; P.tab = myws + WS_TAB + (size_t)b * 512;
        unsigned vo = (unsigned)(q * NLOC + r16) * 16u, vn = (unsigned)(q * 4 * NLOC + r16) * 4u;
        asm volatile("" : "+v"(P.w), "+v"(P.n), "+v"(vn), "+v"(P.x), "+v"(P.rv), "+v"(P.ldw), "+v"(vo));
        L.xa = srcws + WS_XS + (size_t)((call & 1u) * 2u + (unsigned)mt) * NKS * X_STEP;
        const int src = (p.srcs >> 4 * xr) & 7;
        L.xg = myws + WS_XG + (size_t)((((call & 1u) * NR + (unsigned)src) * 2u + (unsigned)mt) * K1) * X_STEP;
        L.set = set;
        const char* ol = (const char*)(p.out + (size_t)(src * MLOC + mt * BM + set * 128 + w4 * 16) * NLOC + f0 * 16);
        P.p[0] = L.xa; P.p[1] = L.xa + 8192;
        PIN(ol); PIN(P.relw); PIN(P.tab); PIN(L.xg); PIN(L.set); PIN(P.p[0]); PIN(P.p[1]);
        WAIT_AV(1, 0);
#pragma unroll
        for (int k = 0; k < 4; ++k) XLD(x[k], k);
        L.t = 1u; P.need = 2u; L.xa += X_STEP; P.p[0] = L.xa; P.p[1] = L.xa + 8192;
        PIN(L.t); PIN(P.need); PIN(L.xa); PIN(P.p[0]); PIN(P.p[1]);
        WAIT_AV(0, 1);
        if (wave == 0) { STAMP(18); STAMPC(30); }
        FLAGS();
#pragma unroll
        for (int i = 0; i < FW; ++i) DSR(a[i], i < 8 ? P.w : P.n, wcol(i));
        __builtin_amdgcn_s_waitcnt(0x0070);
        SCHED_FENCE();
#pragma unroll 1
        do {
            k_step<0, 0>(acc, a, x, P, L);
            k_step<1, 0>(acc, a, x, P, L);
        } while (L.t <= (unsigned)KA);
        if (wave == 0) { STAMP(2); STAMPC(31); }
        P.need = (unsigned)(KA + 1 + NI * set);
        const char* xp = L.xa - X_STEP;
        unsigned pre = P.need - set;
#define TRY() do { const unsigned _p = AG_PAIR ? (rfl(P.f.w) >= P.need ? P.need + 1u : 0u) : rfl(P.f.w); item_x(x, P.x, xp, pre < P.need ? P.need : ~0u, _p); \
                   item_x(x + 4, P.x, xp + 2 * X_STEP, pre <= P.need ? P.need + 1u : ~0u, _p); pre = _p; } while (0)
#define SILU(r_, J_, keep_) silu4(acc[(r_) & 4][J_][(r_) & 3], acc[((r_) & 4) + 1][J_][(r_) & 3], acc[((r_) & 4) + 2][J_][(r_) & 3], \
                                  acc[((r_) & 4) + 3][J_][(r_) & 3], 60.f * LOG2E, c, keep_)
#define STORE(r_, o_) asm volatile("global_store_dwordx4 %0, %1, %2 offset:%3" AST \
                                   :: "v"(vo), "v"(o_), "s"(ol + ((r_) & 3) * NLOC * 4), "i"(((r_) & 4) * 64) : "memory")
#define PART(J_) do { \
        SCHED_FENCE(); \
        TTJ(J_, J_ * 6); \
        if (pre < P.need) WAIT_AV(P.need, 0); \
        item_x(x, P.x, xp, pre + 1u, P.need); \
        if (AG_PAIR >> !J_ & 1) { if (!J_) WAIT_AV(P.need, KA + 2); } else if (pre <= P.need) WAIT_AV(P.need + 1u, KA + 2); \
        P.p[0] = xp + 2 * X_STEP; P.p[1] = xp + 3 * X_STEP; PIN(P.p[0]); PIN(P.p[1]); \
        item_x(x + 4, P.x, P.p[0], pre, P.need); \
        if (!J_) _Pragma("unroll") for (int k = FW; k < RW; ++k) DSR(a[k], P.w, woff(k)); \
        if (J_) { if (pre < P.need) VM_WAIT(4); else VM_WAIT(8); } \
        if (J_ && AG_LPRIO && set) __builtin_amdgcn_s_setprio(AG_LPRIO); \
        SCHED_FENCE(); \
        part_items<J_ * NT, J_ * NT + NT>(acc, a, x, P); \
        TTJ(J_, J_ * 6 + 4); \
        if (!J_) { xp += 8192; PIN(xp); P.need += NAB; pre = 0u; TRY(); } \
        if (!J_ && AG_EPRIO && set) __builtin_amdgcn_s_setprio(AG_EPRIO); \
        if (wave == 4 && J_) { STAMP(3); STAMPC(33); } \
        const float c = __builtin_bit_cast(float, __builtin_bit_cast(unsigned, LOG2E) | rfl(P.f.z)); \
        asm volatile("s_nop 7\n\ts_nop 7" : "+v"(acc[FW - 3][J_]), "+v"(acc[FW - 2][J_]), "+v"(acc[FW - 1][J_])); \
        f4 o = acc[FW - 1][J_], od[4]; \
        _Pragma("unroll") for (int r = 0; r < 4; ++r) { o = SILU(r, J_, o); STORE(r, o); } \
        if (!dup) { \
            const f4 on = silu4(acc[FW - 1][J_][0], acc[FW - 1][J_][1], acc[FW - 1][J_][2], acc[FW - 1][J_][3], 60.f * LOG2E, c, o); \
            _Pragma("unroll") for (int r = 0; r < 4; ++r) \
                asm volatile("global_store_dword %0, %1, %2 offset:512" AST :: "v"(vn), "v"(on[r]), "s"(ol + r * NLOC * 4) : "memory"); \
        } \
        _Pragma("unroll") for (int r = 4; r < 8; ++r) {        \
            if (!J_ && (AG_TRYM >> r & 1)) FLAGS(); \
            o = od[r - 4] = SILU(r, J_, o); \
            if (J_) STORE(r, o); \
            if (!J_ && (AG_TRYM >> r & 1)) { asm volatile("s_waitcnt lgkmcnt(0)" : "+v"(P.f) :: "memory"); TRY(); } \
        } \
        if (!J_) _Pragma("unroll") for (int r = 4; r < 8; ++r) STORE(r, od[r - 4]); \
        if (!J_ && AG_EPRIO) __builtin_amdgcn_s_setprio(0); \
        TTJ(J_, J_ * 6 + 5); \
        asm volatile("s_nop 1" ::: "memory"); } while (0)
        PART(0);
        ADV(vo, 64 * NLOC * 4); ADV(vn, 64 * NLOC * 4);
        PART(1);
        if (wave == 4) { STAMP(4); STAMPC(34); }
        __builtin_amdgcn_endpgm();
    }
    if (wave == 10) {
        *(u2*)(myws + WS_TAB + (size_t)b * 512 + lane * 8) = (u2){2u * lane + 1u, 2u * lane + 2u};
        const unsigned cl = ld_ctr((const void*)cctr) + 1u;
        if (lane == 0) __hip_atomic_store(cctr, cl, __ATOMIC_RELAXED, __HIP_MEMORY_SCOPE_AGENT);
        lds_st(ctrl + lane * 4, lane == C_CALL / 4 ? cl : 0u);
        LGKM0();
    }
    __builtin_amdgcn_s_barrier();
    asm volatile("" ::: "memory");
    const unsigned call = rfl(lds_ld(ctrl + C_CALL));
    if (wave == 10) {
        __builtin_amdgcn_fence(__ATOMIC_ACQUIRE, "");
        VM_WAIT(0);
        GAS unsigned* hp = (GAS unsigned*)(myws + WS_HINT + (size_t)(xr * 2 + mt) * FSTRIDE) + c8;
        GAS unsigned* wf0 = (GAS unsigned*)(myws + WS_WFLAG + (size_t)(f0 >> 3) * FSTRIDE) + lane;
        GAS unsigned* wf1 = (GAS unsigned*)(myws + WS_WFLAG + (size_t)((f0 + FW - 1) >> 3) * FSTRIDE) + lane;
        unsigned pub = 0, wav = 0, npoll = 0;
        TRZERO(1);
        const unsigned long long t0 = rt();
        unsigned long long tall = 0;
        bool dead = false;
        while (pub < (unsigned)NITEMS || wav < (unsigned)NKS) {
            const unsigned hv = __hip_atomic_load(hp, __ATOMIC_RELAXED, __HIP_MEMORY_SCOPE_AGENT);
            bool adv = false;
            if (wav < (unsigned)NKS) {
                const unsigned w0 = __hip_atomic_load(wf0, __ATOMIC_RELAXED, __HIP_MEMORY_SCOPE_AGENT);
                const unsigned w1 = __hip_atomic_load(wf1, __ATOMIC_RELAXED, __HIP_MEMORY_SCOPE_AGENT);
                const unsigned long long notready = __ballot((int)(w0 - call) < 0 || (int)(w1 - call) < 0);
                const unsigned w2 = notready ? (unsigned)__builtin_ctzll(notready) : 64u;
                if (w2 > wav) { if (!wav) STAMP(29); wav = w2; lds_st(ctrl + C_WAV, wav); adv = true; }
            }
            const int d = (int)(hv - call * 256u);
            const unsigned av = (unsigned)row_min(c8 < 5 ? max(min(d, NITEMS), 0) : NITEMS);
            if (av > pub) {
                asm volatile("ds_max_u32 %0, %1" :: "v"(lane ? ctrl + 128 + lane * 4 : ctrl + C_AVAIL), "v"(av) : "memory");
                pub = av;
                tall = 0;
                TRACE(1, av - 1);
            } else if (!adv && (++npoll & 63u) == 0u) {
                if (rt() - t0 > DEADLINE) { dead = true; break; }
                unsigned staged; bool stored;
                arrivals(srcws + WS_SFLAG + (size_t)mt * FSTRIDE, call, lane, staged, stored);
                if (staged == (unsigned)NKS && stored) {
                    if (tall == 0) tall = rt();
                    else if (rt() - tall > SILENCE) { pub = NITEMS; lds_st(ctrl + C_AVAIL, pub); }
                } else if (rfl(__hip_atomic_load(tmo, __ATOMIC_RELAXED, __HIP_MEMORY_SCOPE_AGENT)) == call) { dead = true; break; }
            }
            if (pub < (unsigned)AG_FINE) __builtin_amdgcn_s_sleep(AG_SLEEP_POLL); else __builtin_amdgcn_s_sleep(AG_SLEEP_FINE);
        }
        if (dead) {
            __hip_atomic_store(tmo, call, __ATOMIC_RELAXED, __HIP_MEMORY_SCOPE_AGENT);
            lds_st(ctrl + C_BAD, 0x7fc00000u);
        }
        lds_st(ctrl + C_WAV, (unsigned)NKS);
        lds_st(ctrl + C_AVAIL, (unsigned)NITEMS);
        TRDUMP(1);
    } else if (wave == 9) {
        constexpr int CSTEP = 240 / NCONV;
        const int cq = ((idx - 2) / CSTEP) * 8 + xr;
        if ((idx - 2) % CSTEP == 0 && cq < NCONV) {
            const unsigned so = (unsigned)(r8 * 8 * NLOC + c8 * 4) * 4u;
            const unsigned dof = (unsigned)((c8 >> 2) * 2048 + r8 * 256 + (c8 & 3) * 16);
            for (int i = AG_CFIRST; i < cq; i += 16) __builtin_amdgcn_s_sleep(AG_CDELAY);
            for (int u = cq; u < NKS * 16; u += NCONV) {
                const int ks = u >> 4, cn = u & 15;
                const float* sw = p.w + (size_t)ks * BK * NLOC + cn * 128;
                const char* dstw = myws + WS_WH + (size_t)ks * W_ROW + (size_t)cn * 16384;
                f4 v[4][8];
                if (u < 16) STAMP(12);
#pragma unroll
                for (int g = 0; g < 4; ++g)
#pragma unroll
                    for (int r = 0; r < 8; ++r)
                        asm volatile("global_load_dwordx4 %0, %1, %2 offset:%3" CLD : "=v"(v[g][r]) : "v"(so), "s"(sw + r * NLOC), "i"(g * 128) : "memory");
                u4 o[4][4];
#pragma unroll
                for (int g = 0; g < 4; ++g) {
                    asm volatile("s_waitcnt vmcnt(%8)" : "+v"(v[g][0]), "+v"(v[g][1]), "+v"(v[g][2]), "+v"(v[g][3]),
                                 "+v"(v[g][4]), "+v"(v[g][5]), "+v"(v[g][6]), "+v"(v[g][7]) : "i"(24 - 8 * g) : "memory");
#pragma unroll
                    for (int r = 0; r < 8; ++r) v[g][r] *= LOG2E;
#pragma unroll
                    for (int c = 0; c < 4; ++c)
                        o[g][c] = (u4){pk(v[g][0][c], v[g][1][c]), pk(v[g][2][c], v[g][3][c]), pk(v[g][4][c], v[g][5][c]), pk(v[g][6][c], v[g][7][c])};
                    SCHED_FENCE();
                }
                if (u < 16) STAMP(13);
#pragma unroll
                for (int g = 0; g < 4; ++g)
#pragma unroll
                    for (int c = 0; c < 4; ++c)
                        asm volatile("global_store_dwordx4 %0, %1, %2 offset:%3" CST "\n\ts_nop 1"
                                     :: "v"(dof), "v"(o[g][c]), "s"(dstw + g * 4096), "i"(c * 64) : "memory");
                VM_WAIT(0);
                if (lane == 0)
                    __hip_atomic_store((GAS unsigned*)(myws + WS_WFLAG + (size_t)cn * FSTRIDE + ks * 4), call,
                                       __ATOMIC_RELAXED, __HIP_MEMORY_SCOPE_AGENT);
                if (u < 16) STAMP(28);
                if (u < NCONV) STAMP(35);
            }
            STAMP(1);
        }
    } else {
        const char* tab = myws + WS_TAB + (size_t)b * 512;
        const char* wa = myws + WS_WH + (size_t)f0 * 2048;
        unsigned long long t0 = 0;
        int av = 0, cons = 0;
#pragma unroll 1
        for (int t = 0; t < NKS; ++t) {
            if (av <= t || cons <= t - NW) for (unsigned sp = 0;;) {
                u4 c, d; unsigned v;
                asm volatile("ds_read_b128 %0, %3\n\tds_read_b128 %1, %3 offset:16\n\tds_read_b32 %2, %3 offset:%4\n\t"
                             "s_waitcnt lgkmcnt(0)" : "=&v"(c), "=&v"(d), "=&v"(v) : "v"(ctrl + C_CONS), "i"(C_WAV - C_CONS) : "memory");
                av = (int)rfl(v); cons = (int)min8(c, d);
                if ((av > t && cons > t - NW) || SPIN_OUT(sp, t0)) break;
                __builtin_amdgcn_s_sleep(AG_LSLEEP);
            }
            TRACE(2, t);
            if (t == 0) STAMP(32);
            const unsigned sb = lbase + (unsigned)(t & (NW - 1)) * W_STEP;
#define WLOADS(P_) _Pragma("unroll") for (int i = 0; i < FW; i += 2) { \
                asm volatile("s_mov_b32 m0, %0\n\ts_waitcnt vmcnt(58)\n\t" \
                             "global_load_lds_dwordx4 %1, %2" P_ "\n\tglobal_load_lds_dwordx4 %1, %2 offset:1024" P_ \
                             :: "s"(sb + i * W_BLK), "v"(lo16), "s"(wa + i * 2048) : "memory"); \
                if (i + 1 < FW) asm volatile("s_mov_b32 m0, %0\n\ts_nop 0\n\t" \
                             "global_load_lds_dwordx4 %1, %2 offset:2048" P_ "\n\tglobal_load_lds_dwordx4 %1, %2 offset:3072" P_ \
                             :: "s"(sb + i * W_BLK + 64), "v"(lo16), "s"(wa + i * 2048) : "memory"); \
            }
            if (mt == 0 && t >= AG_WNT0 && t < AG_WNT) { WLOADS(" nt") } else
            { WLOADS(WLD) }
            asm volatile("s_mov_b32 m0, %0\n\ts_mov_b64 exec, 1\n\tglobal_load_lds_dword %1, %2\n\ts_mov_b64 exec, -1"
                         :: "s"(ctrl + C_LD), "v"(lo16), "s"(tab + t * 4) : "memory");
            wa += W_ROW;
        }
        VM_WAIT(0);
    }
}

}

extern "C" void kernel_launch(int rank, int nranks, ncclComm_t comm,
                              hipStream_t stream,
                              void* const* d_in, const int* in_sizes, int n_in,
                              void* d_out, int out_size,
                              const int* in_dtypes, int out_dtype,
                              void* const* d_out_all,
                              void* const* d_ws_all, size_t ws_bytes) {
    (void)comm; (void)in_dtypes; (void)out_dtype; (void)d_out_all;
    if (nranks != NR || n_in < 2 || (unsigned)rank >= (unsigned)NR || ws_bytes < WS_NEED ||
        in_sizes[0] != MLOC * GK || in_sizes[1] != GK * NLOC || out_size != GM * NLOC) {
        fprintf(stderr, "ag_gemm: unexpected problem shape\n");
        return;
    }
    static bool attr_set = false;
    if (!attr_set) {
        HIP_CHECK(hipFuncSetAttribute((const void*)ag_gemm_kernel,
                                      hipFuncAttributeMaxDynamicSharedMemorySize, LDS_BYTES));
        attr_set = true;
    }
    Params p;
    p.x = (const float*)d_in[0];
    p.w = (const float*)d_in[1];
    p.out = (float*)d_out;
    p.myws = (char*)d_ws_all[rank];
    p.srcs = 0;
    {
        int order[NR], n = 0;
        const unsigned half = 0xb2;
        for (int far = 0; far < 2; ++far)
            for (int r = 0; r < NR; ++r) { const int s = (rank + r) & 7; if ((int)((half >> s ^ half >> rank) & 1) == far) order[n++] = s; }
        for (int r = 0; r < NR; ++r) { p.ws[r] = (char*)d_ws_all[order[r]]; p.srcs |= (unsigned)order[r] << 4 * r; }
    }
    for (int r = 0; r < 2 * NR; ++r) p.xg[r] = (char*)d_ws_all[(rank + 1 + r) & 7] + WS_XG + (size_t)rank * 2 * K1 * X_STEP;
    hipLaunchKernelGGL(ag_gemm_kernel, dim3(NWG), dim3(NTHREADS), LDS_BYTES, stream, p);
    HIP_CHECK(hipGetLastError());
}
```

```cpp
#include <hip/hip_runtime.h>
#include <rccl/rccl.h>
#include <cstdio>
#include <cstddef>

#define HIP_CHECK(expr) do { hipError_t _e = (expr); if (_e != hipSuccess) { \
    fprintf(stderr, "HIP error %s at %s:%d\n", hipGetErrorString(_e), __FILE__, __LINE__); \
    return; } } while (0)

namespace {

#define GAS __attribute__((address_space(1)))
#define LAS __attribute__((address_space(3)))
typedef _Float16 h8 __attribute__((ext_vector_type(8)));
typedef _Float16 h2 __attribute__((ext_vector_type(2)));
typedef float f4 __attribute__((ext_vector_type(4)));
typedef unsigned u4 __attribute__((ext_vector_type(4)));
typedef unsigned u2 __attribute__((ext_vector_type(2)));

#define AG_PWIN 16
#define AG_WIN 8
#define AG_EFENCE 0
#define AG_HSTEP 14
#define AG_HN 4
#define AG_CFIRST -17
#define AG_CDELAY 20
#define AG_WNT 32
#define AG_WNT0 8
#define AG_NOCHK 3
#define AG_PAIR 0
#define AG_PUB 0
#define AG_PUBG 0
#define AG_RELG 34
#define AG_MAP 0
#define AG_SLEEP_POLL 4
#define AG_SLEEP_FINE 1
#define AG_FINE 48
#define AG_NCONV 120
#define AG_KP0 30
#define AG_KPS 6
#define AG_SDELAY 8
#define AG_SEARLY 12
#define AG_SSLOPE 1
#define AG_LSLEEP 1
#define AG_EARLY0 2
#define AG_EARLY1 2
#define AG_EARLY2 0
#define AG_E2T 4
#define AG_EPRIO 0
#define AG_LPRIO 0
#define AG_RW 9
#define AG_WG 3
#define AG_NAB 8
#define AG_OWN 85
#define AG_CA 2
#define AG_AST 2
#define AG_TRYM 0x90
#define AST " nt"

#define CST " sc1 nt"
#define WLD ""
#define CLD " nt"
#define XLP ""
#define PLD ""
#define SST " nt"
#define PST ""
#define SLD " nt"
constexpr int NR = 8;
constexpr int GM = 4096, GK = 4096, NLOC = 2048, MLOC = 512;
constexpr int BM = 256, BK = 64;
constexpr int FW = 9;
constexpr int NKS = GK / BK;
constexpr int K1 = 4;
constexpr int KP0 = AG_KP0, KPS = AG_KPS;
constexpr unsigned long long PMASK = (1ull | 1ull << KPS | 1ull << 2 * KPS | 1ull << 3 * KPS) << KP0;
constexpr int NAB = AG_NAB, NI = NAB / 2;
constexpr int KA = NKS - NAB;
constexpr int NITEMS = KA + 4 * NI;
constexpr int X_STEP = BM * BK * 2;
constexpr int X_PART = X_STEP / 2;
constexpr int W_BLK = 2048 + 64;
constexpr int W_STEP = FW * W_BLK - 64;
constexpr int W_ROW = NLOC * BK * 2;
constexpr int NW = 8;
constexpr int SCR_OFF = NW * W_STEP;
constexpr int CTRL_OFF = SCR_OFF + 8192;
constexpr int LDS_BYTES = CTRL_OFF + 512;
constexpr int NWG = 256, NTHREADS = 704;
static_assert(LDS_BYTES <= 160 * 1024 && NAB <= NW && NAB % 2 == 0 && NAB >= 4 && NKS % NW == 0, "layout");
static_assert((NKS - K1) * 4 == NWG - 16 && KP0 > 1 && KPS > 1 && KP0 + 3 * KPS < KA - 1, "layout");
constexpr int C_CONS = 0;
constexpr int C_WAV = 68;
constexpr int C_STG = 72;
constexpr int C_CALL = 76;
constexpr int C_LD = 84;
constexpr int C_BAD = 88;
constexpr int C_AVAIL = 92;
constexpr int NCONV = AG_NCONV;
constexpr float LOG2E = 1.4426950408889634f;

constexpr size_t FSTRIDE = 4096 + 256;
constexpr size_t WS_CALL = 0;
constexpr size_t WS_TMO = 49152;
constexpr size_t WS_SFLAG = 53248;
constexpr size_t WS_HINT = 65536;
constexpr size_t WS_WFLAG = 196608;
constexpr size_t WS_TAB = 327680;
constexpr size_t WS_DBG = 524288;
constexpr size_t WS_WH = 1ull << 20;
constexpr size_t WS_XS = 32ull << 20;
constexpr size_t WS_XG = 48ull << 20;
constexpr size_t WS_NEED = 64ull << 20;

constexpr unsigned long long DEADLINE = 100000000ull;
constexpr unsigned long long SILENCE = 20000ull;

struct Params {
    const float* x;
    char* myws;
    const float* w;
    float* out;
    char* ws[NR];
    char* xg[2 * NR];
    unsigned srcs;
};
typedef const char* __attribute__((address_space(4))) const* KPTR;

__device__ __forceinline__ unsigned long long rt() { return __builtin_amdgcn_s_memrealtime(); }

__device__ __forceinline__ void st16_sys(void* p, u4 v) {
    asm volatile("global_store_dwordx4 %0, %1, off sc0 sc1" SST "\n\ts_nop 1" :: "v"(p), "v"(v) : "memory");
}
__device__ __forceinline__ void st16_dev(void* p, u4 v) {
    asm volatile("global_store_dwordx4 %0, %1, off sc1\n\ts_nop 1" :: "v"(p), "v"(v) : "memory");
}
__device__ __forceinline__ void st4_sys(void* p, unsigned v) {
    asm volatile("global_store_dword %0, %1, off sc0 sc1\n\ts_nop 1" :: "v"(p), "v"(v) : "memory");
}
__device__ __forceinline__ void post(const char* p, unsigned zero, unsigned v) {
    asm volatile("s_mov_b64 exec, 1\n\tglobal_store_dword %0, %1, %2\n\ts_mov_b64 exec, -1" :: "v"(zero), "v"(v), "s"(p) : "memory");
}
__device__ __forceinline__ void lds_st(unsigned addr, unsigned v) {
    asm volatile("ds_write_b32 %0, %1" :: "v"(addr), "v"(v) : "memory");
}
__device__ __forceinline__ unsigned lds_ld(unsigned addr) {
    unsigned v;
    asm volatile("ds_read_b32 %0, %1\n\ts_waitcnt lgkmcnt(0)" : "=&v"(v) : "v"(addr) : "memory");
    return v;
}
__device__ __forceinline__ unsigned ld_ctr(const void* p) {
    unsigned v;
    asm volatile("global_load_dword %0, %1, off sc1\n\ts_waitcnt vmcnt(0)" : "=&v"(v) : "v"(p) : "memory");
    return (unsigned)__builtin_amdgcn_readfirstlane(v);
}
__device__ __forceinline__ unsigned pk(float a, float b) {
    h2 h; h.x = (_Float16)a; h.y = (_Float16)b;
    return __builtin_bit_cast(unsigned, h);
}
__device__ __forceinline__ u4 pk8(f4 a, f4 b) { return (u4){pk(a[0], a[1]), pk(a[2], a[3]), pk(b[0], b[1]), pk(b[2], b[3])}; }
__device__ __forceinline__ unsigned umin(unsigned a, unsigned b) { return a < b ? a : b; }
__device__ __forceinline__ unsigned rfl(unsigned v) { return (unsigned)__builtin_amdgcn_readfirstlane((int)v); }
__device__ __forceinline__ unsigned min8(u4 c, u4 d) {
    return rfl(umin(umin(umin(c.x, c.y), umin(c.z, c.w)), umin(umin(d.x, d.y), umin(d.z, d.w))));
}
__device__ __forceinline__ int row_min(int u) {
    u = min(u, __builtin_amdgcn_update_dpp(0, u, 0x121, 15, 15, false));
    u = min(u, __builtin_amdgcn_update_dpp(0, u, 0x122, 15, 15, false));
    u = min(u, __builtin_amdgcn_update_dpp(0, u, 0x124, 15, 15, false));
    u = min(u, __builtin_amdgcn_update_dpp(0, u, 0x128, 15, 15, false));
    return __builtin_amdgcn_readfirstlane(u);
}
__device__ __forceinline__ void arrivals(const char* sf, unsigned call, int lane, unsigned& staged, bool& stored) {
    u2 sv; unsigned pv;
    asm volatile("global_load_dwordx2 %0, %2, %4 sc0 sc1\n\tglobal_load_dword %1, %3, %4 offset:512 sc0 sc1\n\ts_waitcnt vmcnt(0)"
                 : "=&v"(sv), "=&v"(pv) : "v"(lane * 8), "v"((lane & 15) * 4), "s"(sf) : "memory");
    const unsigned long long miss = __ballot((int)(sv.x - call) < 0 || (int)(sv.y - call) < 0) & ~PMASK;
    staged = miss ? (unsigned)__builtin_ctzll(miss) : (unsigned)NKS;
    stored = __ballot((int)(pv - call) < 0) == 0;
}
#define VM_WAIT(n) asm volatile("s_waitcnt vmcnt(" #n ")" ::: "memory")
#define LGKM0() asm volatile("s_waitcnt lgkmcnt(0)" ::: "memory")
#define SCHED_FENCE() __builtin_amdgcn_sched_barrier(0)
#define SPIN_OUT(sp_, t0_) ((((++(sp_)) & 1023u) == 0u) && ((t0_) == 0 ? ((t0_) = rt(), false) : ((unsigned)(rt() - (t0_)) > 2u * (unsigned)DEADLINE)))

#define TRACE(k_, t_) do {} while (0)
#define TRDUMP(k_) do {} while (0)
#define TRZERO(k_) do {} while (0)
#define STAMP(i) do {} while (0)
#define STAMPC(i) do {} while (0)

#define DSR(dst_, addr_, off_) asm volatile("ds_read_b128 %0, %1 offset:%2" : "+v"(dst_) : "v"(addr_), "i"(off_) : "memory")

__device__ __forceinline__ f4 silu4(float z0, float z1, float z2, float z3, float lim, float c, f4 keep) {
    float o0, o1, o2, o3;
    asm volatile("v_med3_f32 %0, -%4, %8, -%8\n\tv_med3_f32 %1, -%5, %8, -%8\n\tv_med3_f32 %2, -%6, %8, -%8\n\tv_med3_f32 %3, -%7, %8, -%8\n\t"
                 "v_exp_f32 %0, %0\n\tv_exp_f32 %1, %1\n\tv_exp_f32 %2, %2\n\tv_exp_f32 %3, %3\n\t"
                 "v_fma_f32 %0, %0, %9, %9\n\tv_fma_f32 %1, %1, %9, %9\n\tv_fma_f32 %2, %2, %9, %9\n\tv_fma_f32 %3, %3, %9, %9\n\t"
                 "v_rcp_f32 %0, %0\n\tv_rcp_f32 %1, %1\n\tv_rcp_f32 %2, %2\n\tv_rcp_f32 %3, %3\n\t"
                 "v_mul_f32 %0, %4, %0\n\tv_mul_f32 %1, %5, %1\n\tv_mul_f32 %2, %6, %2\n\tv_mul_f32 %3, %7, %3"
                 : "=&v"(o0), "=&v"(o1), "=&v"(o2), "=&v"(o3) : "v"(z0), "v"(z1), "v"(z2), "v"(z3), "s"(lim), "s"(c), "v"(keep));
    return (f4){o0, o1, o2, o3};
}
#define REL() asm volatile("s_mov_b32 m0, %0\n\ts_mov_b64 exec, 1\n\tglobal_load_lds_dword %1, %2\n\ts_mov_b64 exec, -1" \
                           :: "s"(P.relw), "v"(P.rv), "s"(P.tab) : "memory")
#define TT(i_) do {} while (0)
struct Pos { unsigned w, n, x, rv, need, relw, ldw; const char* tab; const char* p[2]; u4 f; };
#define TTJ(j_, i_) TT(i_)
#define ADV(v_, d_) do { v_ += d_; asm volatile("" : "+v"(v_)); } while (0)
#define PIN(s_) asm volatile("" : "+s"(s_))
#define FLAGS() asm volatile("ds_read_b128 %0, %1" : "=v"(P.f) : "v"(P.ldw) : "memory")
#define WAIT_AV(nx_, nw_) do { unsigned long long _t0 = 0; unsigned _sp = 0; \
        for (;;) { asm volatile("ds_read_b128 %0, %1\n\ts_waitcnt lgkmcnt(0)" : "+v"(P.f) : "v"(P.ldw) : "memory"); \
            if ((int)((rfl(P.f.w) - (unsigned)(nx_)) | (rfl(P.f.y) - (unsigned)(nw_))) >= 0) break; \
            if (SPIN_OUT(_sp, _t0)) break; \
            __builtin_amdgcn_s_sleep(1); } } while (0)
#define CHECK_AV(nx_, nw_) do { if (__builtin_expect((int)((rfl(P.f.w) - (unsigned)(nx_)) | (rfl(P.f.y) - (unsigned)(nw_))) < 0, 0)) \
            WAIT_AV(nx_, nw_); SCHED_FENCE(); } while (0)
#define MFMA(c_, x_, a_) asm volatile("v_mfma_f32_16x16x32_f16 %0, %1, %2, %0" : "+v"(c_) : "v"(x_), "v"(a_))
#define MFMA_R(c_, x_, a_, ad_, off_) asm volatile("v_mfma_f32_16x16x32_f16 %0, %2, %1, %0\n\tds_read_b128 %1, %3 offset:%4" \
                                                   : "+v"(c_), "+v"(a_) : "v"(x_), "v"(ad_), "i"(off_) : "memory")
#define XLD(x_, k_) asm volatile("global_load_dwordx4 %0, %1, %2 offset:%3" : "+v"(x_) : "v"(P.x), "s"(P.p[(k_) & 1]), "i"(((k_) & 2) * 512) : "memory")
constexpr int wcol(int i) { return i < 8 ? (i >> 2) * 4 * W_BLK + (i & 3) * 64 : 0; }

constexpr int RW = AG_RW, G_A = AG_CA;
struct Loop { const char* xa; const char* xg; unsigned t, set; };
template <int B, int G>
__device__ __forceinline__ void k_step(f4 (&acc)[FW][2], h8 (&a)[RW], h8 (&x)[8], Pos& P, Loop& L) {
    if constexpr (G < 36) {
        constexpr int h = G / 18, i = G % 18 / 2;
        if (G % 18 == 0) asm volatile("s_waitcnt vmcnt(%0) lgkmcnt(7)" :: "i"(h ? 5 : 3) : "memory");
        else if (G == G_A) asm volatile("s_waitcnt lgkmcnt(%1)" : "+v"(P.f) : "i"(3 + G_A / 2) : "memory");
        else if (G == AG_RELG && G <= 30) asm volatile("s_waitcnt lgkmcnt(%0)" :: "i"((G - 18) / 2) : "memory");
        else if (G % 18 % 4 == 0 && !(G > AG_RELG && G > 18)) asm volatile("s_waitcnt lgkmcnt(7)" ::: "memory");
        if (G & 1) MFMA_R(acc[i][1], x[B * 4 + h * 2 + 1], a[i], i < 8 ? P.w : P.n, (h ? 1 - B : B) * W_STEP + (1 - h) * 1024 + wcol(i));
        else MFMA(acc[i][0], x[B * 4 + h * 2], a[i]);
        SCHED_FENCE();
        if (G == G_A) CHECK_AV(P.need, L.t + 1u);
        if (G > G_A && G <= G_A + 8 && !(G & 1)) XLD(x[4 - B * 4 + (G - G_A) / 2 - 1], (G - G_A) / 2 - 1);
        if (B && G == 18) { const int d = (L.t & (NW - 1)) ? 2 * W_STEP : (2 - NW) * W_STEP; ADV(P.w, d); ADV(P.n, d); }
        if (G == 20) { L.t += 1u; PIN(L.t); L.xa += X_STEP; PIN(L.xa); }
        if (G == 22) {
            const bool st = (PMASK >> L.t) & 1;
            P.p[0] = st ? L.xg : L.xa; PIN(P.p[0]);
            L.xg += st ? X_STEP : 0; PIN(L.xg);
        }
        if (G == 24) {
            P.need = L.t + 1u;
            if (B) P.p[1] = P.p[0] + 8192;
            else {
                const bool last = L.t == (unsigned)KA;
                P.p[0] -= last && L.set ? X_STEP : 0;
                P.p[1] = P.p[0] + (last && !L.set ? X_STEP : 8192);
                P.need = last && L.set ? 0u : P.need;
                PIN(P.p[0]);
            }
            PIN(P.p[1]); PIN(P.need);
        }
        if (G == 30) FLAGS();
        if (G == (AG_RELG < 34 ? 14 : 32)) ADV(P.rv, 4u);
        if (G == AG_RELG) REL();
        SCHED_FENCE();
        k_step<B, G + 1>(acc, a, x, P, L);
    }
}
constexpr int NT = 36 * NI, WG = AG_WG, G_C = 35 - RW;
constexpr bool flags(int t) { return t % 36 == (t % NT < NT - 36 ? G_C - 9 : 26); }
constexpr int xfetch(int t) {
    for (int k = 0; k < 4; ++k) if (t % 36 == (G_C + 1 + k > 9 * k + 8 ? G_C + 1 + k : 9 * k + 8)) return t % NT < NT - 72 ? k : -1;
    return -1;
}
constexpr bool pub(int t) { return AG_PUB && t % 36 == AG_PUBG && t % NT / 36 + AG_PUB >= NI; }
struct Waits { int l[2 * NT + 1], v[NT / 9]; };
constexpr Waits waits() {
    Waits r{};
    int pw[2 * NT + 16] = {}, pf = 0, c = 0, done = -1, xs = 8;
    for (int k = 0; k < RW; ++k) pw[k] = c++;
    for (int t = 0; t <= 2 * NT; ++t) {
        int need = t % WG || t == 2 * NT ? -1 : pw[t + WG - 1];
        if (t % 36 == G_C || (t && t % NT == 0)) need = pf > need ? pf : need;
        r.l[t] = need <= done ? -1 : c - 1 - need < 15 ? c - 1 - need : 15;
        done = need > done ? need : done;
        if (t + RW < 2 * NT) pw[t + RW] = c++;
        if (flags(t)) pf = c++;
        if (pub(t)) ++c;
        if (t < NT && t % 9 == 0) r.v[t / 9] = xs - 1 - t / 9;
        if (xfetch(t) >= 0) ++xs;
    }
    return r;
}
static_assert(RW >= FW && RW > WG && NT % WG == 0, "parts");
constexpr int woff(int k) { return (k % 36 / 9 & 1) * W_STEP + k % 36 / 18 * 1024 + wcol(k % 9); }
template <int T, int E>
__device__ __forceinline__ void part_items(f4 (&acc)[FW][2], h8 (&a)[RW], h8 (&x)[8], Pos& P) {
    constexpr int left = waits().l[T];
    if constexpr (left >= 0 && (T > E - NT || T == 0)) asm volatile("s_waitcnt lgkmcnt(%0)" :: "i"(left) : "memory");
    if (T == E || T % 36 == G_C) asm volatile("" : "+v"(P.f));
    if constexpr (T < E) {
        constexpr int J = T / NT, N = T % NT / 36, G = T % 36, K = T + RW, X = (N & 1) * 4;
        if (G % 9 == 0 && (!J || N)) asm volatile("s_waitcnt vmcnt(%0)" :: "i"(waits().v[T % NT / 9]) : "memory");
        if constexpr (K < 2 * NT) MFMA_R(acc[T % FW][J], x[X + G / 9], a[T % RW], K % 9 < 8 ? P.w : P.n, woff(K));
        else MFMA(acc[T % FW][J], x[X + G / 9], a[T % RW]);
        SCHED_FENCE();
        if (G == 0 && N >= NI - 3) TT(J * 6 + 4 + N - NI);
        if (K < 2 * NT && K % 36 >= 34) {
            constexpr int d = K % NT / 36 == NI - 1 ? (2 - NAB) * W_STEP : 2 * W_STEP;
            if (K % 36 == 34) ADV(P.w, d); else ADV(P.n, d);
        }
        if (flags(T)) FLAGS();
        if (pub(T)) asm volatile("v_mov_b32 %0, %2\n\ts_mov_b64 exec, 1\n\tds_max_u32 %1, %0 offset:12\n\ts_mov_b64 exec, -1"
                                 : "+v"(P.rv) : "v"(P.ldw), "s"(P.need + (N + AG_PUB)) : "memory");
        if (G == G_C && N < (J ? NI - 2 : NI - 1) && !(J && (AG_NOCHK & 1)))
            CHECK_AV(N < NI - 2 && !(AG_NOCHK & 2) ? P.need + (N + 2) : 0u, J ? 0 : KA + 2 * N + 4);
        if (G == G_C - 2 && N < NI - 2) { P.p[0] += 2 * X_STEP; PIN(P.p[0]); P.p[1] += 2 * X_STEP; PIN(P.p[1]); }
        if constexpr (xfetch(T) >= 0) XLD(x[X + xfetch(T)], xfetch(T));
        SCHED_FENCE();
        part_items<T + 1, E>(acc, a, x, P);
    }
}
__device__ __forceinline__ void item_x(h8* x, unsigned vx, const char* p, unsigned lo, unsigned hi) {
    asm volatile("s_cmp_gt_u32 %7, %8\n\ts_cbranch_scc1 1f\n\t"
                 "global_load_dwordx4 %0, %4, %5\n\tglobal_load_dwordx4 %1, %4, %6\n\t"
                 "global_load_dwordx4 %2, %4, %5 offset:1024\n\tglobal_load_dwordx4 %3, %4, %6 offset:1024\n1:"
                 : "+v"(x[0]), "+v"(x[1]), "+v"(x[2]), "+v"(x[3])
                 : "v"(vx), "s"(p), "s"(p + X_STEP), "s"(lo), "s"(hi) : "memory", "scc");
}
__device__ __forceinline__ void to_frags(u4 o0, u4 o1, unsigned wr, unsigned rd, u4& f0, u4& f1) {
    asm volatile("ds_write_b128 %2, %4\n\tds_write_b128 %2, %5 offset:1024\n\tds_read_b128 %0, %3\n\tds_read_b128 %1, %6"
                 : "=&v"(f0), "=&v"(f1) : "v"(wr), "v"(rd), "v"(o0), "v"(o1), "v"(rd ^ 64u) : "memory");
}

__global__ void __launch_bounds__(NTHREADS, 3) ag_gemm_kernel(Params p) {
    extern __shared__ __attribute__((aligned(16))) unsigned char smem[];
#define STAMPE(i) do {} while (0)
    LAS unsigned char* lds = (LAS unsigned char*)smem;
    const int tid = threadIdx.x;
    const int wave = __builtin_amdgcn_readfirstlane(tid >> 6);
    const int lane = tid & 63;
    const int b = blockIdx.x;
    const int xr = b & 7, idx = b >> 3;
    char* myws = p.myws;
    GAS unsigned* cctr = (GAS unsigned*)(myws + WS_CALL + (size_t)b * 128);
    GAS unsigned* tmo = (GAS unsigned*)(myws + WS_TMO);
    const unsigned lbase = (unsigned)(size_t)lds;
    const unsigned ctrl = lbase + CTRL_OFF;
    const unsigned lo16 = (unsigned)lane * 16u;
    const int r8 = lane >> 3, c8 = lane & 7;
    const int r16 = lane & 15, q = lane >> 4;
    const unsigned img = lbase + (unsigned)(r8 * 128 + (c8 ^ (r8 >> 1)) * 16);
    const unsigned frg = lbase + (unsigned)(r16 * 128 + (q ^ (r16 >> 1)) * 16);
    const char* __attribute__((address_space(4))) const* ka = (KPTR)__builtin_amdgcn_kernarg_segment_ptr();

    if (__builtin_expect(idx < 2 && wave < 2, 1)) {
        const int pid = (xr * 2 + idx) * 2 + wave;
        if (wave < 2) STAMP(20 + wave);
        f4 v[K1][4];
        unsigned cv;
        {
            const unsigned s0 = (unsigned)((pid * 16 + r8) * GK + c8 * 8) * 4u;
            const unsigned s1 = (unsigned)((pid * 16 + 8 + r8) * GK + (c8 ^ 4) * 8) * 4u;
#pragma unroll
            for (int e = 0; e < K1; ++e) {
                asm volatile("global_load_dwordx4 %0, %4, %6" PLD "\n\tglobal_load_dwordx4 %1, %4, %6 offset:16" PLD "\n\t"
                             "global_load_dwordx4 %2, %5, %6" PLD "\n\tglobal_load_dwordx4 %3, %5, %6 offset:16" PLD
                             : "=&v"(v[e][0]), "=&v"(v[e][1]), "=&v"(v[e][2]), "=&v"(v[e][3])
                             : "v"(s0), "v"(s1), "s"(p.x + (KP0 + e * KPS) * BK) : "memory");
                if (e == 0) asm volatile("global_load_dword %0, %1, %2 sc1" : "=&v"(cv) : "v"(b * 128), "s"(myws + WS_CALL) : "memory");
            }
        }
        SCHED_FENCE();
        const char* dst[NR];
#pragma unroll
        for (int j = 0; j < NR; ++j) dst[j] = ka[offsetof(Params, xg) / 8 + ((xr * 2 + idx + 4 * wave) & 7) + j];
        unsigned vo = lo16 + (unsigned)((pid >> 4) * K1 * X_STEP + (pid & 15) * 2048), call = 0;
        char* flag = myws + WS_SFLAG + (size_t)(pid >> 4) * FSTRIDE + 512 + (pid & 15) * 4;
        constexpr int W = AG_PWIN;
        static_assert(W >= 2 && W <= 32 && W % 2 == 0, "window");
        u4 o[K1][2];
#pragma unroll
        for (int e = 0; e < K1; ++e) {
            if (e < 2) {
#pragma unroll
                for (int k = e; k < (e ? K1 : 1); ++k) {
                    asm volatile("s_waitcnt vmcnt(%5)" : "+v"(cv), "+v"(v[k][0]), "+v"(v[k][1]), "+v"(v[k][2]), "+v"(v[k][3])
                                 : "i"(4 * (K1 - 1 - k) + e * 2 * NR) : "memory");
                    to_frags(pk8(v[k][0], v[k][1]), pk8(v[k][2], v[k][3]), img + 32768u + wave * 2048u, frg + 32768u + wave * 2048u, o[k][0], o[k][1]);
                }
            }
            if (e == 0) {
                call = rfl(cv) + 1u;
                vo += (call & 1u) * (unsigned)(NR * 2 * K1 * X_STEP);
                if (wave == 0) STAMP(5);
            }
            if (e == 1) __builtin_amdgcn_s_barrier();
            asm volatile("s_waitcnt lgkmcnt(%2)" : "+v"(o[e][0]), "+v"(o[e][1]) : "i"(e ? 4 * (K1 - 1 - e) : 0) : "memory");
            const unsigned ve = vo + (unsigned)(e * X_STEP);
#pragma unroll
            for (int j = 0; j < NR; ++j) {
                if (e) asm volatile("s_waitcnt vmcnt(%0)" :: "i"(W - 2) : "memory");
                asm volatile("global_store_dwordx4 %0, %1, %3 sc0 sc1" PST "\n\tglobal_store_dwordx4 %0, %2, %3 offset:1024 sc0 sc1" PST "\n\ts_nop 1"
                             :: "v"(ve), "v"(o[e][0]), "v"(o[e][1]), "s"(dst[j]) : "memory");
            }
        }
        VM_WAIT(0);
        if (lane == 0) st4_sys(flag, call);
        if (wave < 2) STAMP(14 + wave);
        if (wave == 0) STAMPE(22);
        __builtin_amdgcn_endpgm();
    }
    const char* srcws = ka[offsetof(Params, ws) / 8 + xr];
    if (idx < 2) {
        const char* hint = myws + WS_HINT + (size_t)(xr * 2 + idx) * FSTRIDE;
        if (wave == 7) lds_st(ctrl + C_AVAIL, 0u);
        const unsigned call = ld_ctr((const void*)cctr) + 1u;
        LGKM0();
        __builtin_amdgcn_s_barrier();
        asm volatile("" ::: "memory");
        unsigned long long t0 = 0;
        if (wave >= 3 && wave < 7) {
            const int qt = wave - 3;
            const char* hq = hint + qt * 4;
            const char* xa = srcws + WS_XS + (size_t)((call & 1u) * 2u + (unsigned)idx) * NKS * X_STEP;
            unsigned hv = call * 256u + 1u, zero = 0u;
            asm volatile("s_mov_b32 m0, %2" : "+v"(hv), "+v"(zero) : "s"(lbase + qt * 4096) : "memory");
            int av = 0;
#define STAGED(n_) do { if (av < (n_)) for (unsigned _sp = 0;;) { av = (int)rfl(lds_ld(ctrl + C_AVAIL)); \
                if (av >= (n_) || SPIN_OUT(_sp, t0)) break; \
                __builtin_amdgcn_s_sleep(1); } } while (0)
#define PULL4(a_) asm volatile("s_waitcnt vmcnt(%2)\n\tglobal_load_lds_dwordx4 %0, %1\n\t" \
                         "s_waitcnt vmcnt(%2)\n\tglobal_load_lds_dwordx4 %0, %1 offset:1024\n\t" \
                         "s_waitcnt vmcnt(%2)\n\tglobal_load_lds_dwordx4 %0, %1 offset:2048\n\t" \
                         "s_waitcnt vmcnt(%2)\n\tglobal_load_lds_dwordx4 %0, %1 offset:3072" \
                         :: "v"(lo16), "s"(a_), "i"(AG_WIN - 1) : "memory")
            unsigned long long own = srcws == myws ? rt() : 0;
#define PACE(n_) do { if (own) { own += n_; while (rt() < own) __builtin_amdgcn_s_sleep(2); } } while (0)
#pragma unroll 1
            for (int t = 0; t < KA; ++t) {
                STAGED(t + 1);
                PACE(AG_OWN);
                if (qt == 0) TRACE(0, t);
                if (qt == 0 && (t & 7) == 1) STAMP(54 + t / 8);

                if (qt == 0 && t < 12) STAMP(36 + t);
                const char* xt = xa + qt * 8192;
                if ((PMASK >> 1 >> t) & 1) { hv += 1u; xa += X_STEP; ++t; }
                post(hq, zero, hv);
                PULL4(xt);
                PULL4(xt + 4096);
                hv += 1u;
                xa += X_STEP;
            }
            STAGED(NKS);
            xa += (qt >> 1) * X_STEP + (qt & 1) * 4096;
#pragma unroll
            for (int t = KA; t < NITEMS; ++t) {
                PACE(AG_OWN / 2);
                if (qt == 0) TRACE(0, t);
                hv += 1u;
                post(hq, zero, hv);
                PULL4(xa + (t - KA) % NI * 2 * X_STEP + ((t - KA) / NI % 2 * 2 + (t - KA) / NAB) * 8192);
            }
            VM_WAIT(0);
            STAMP(6 + qt);
        } else if (wave == 7) {
            if (lane == 0) __hip_atomic_store(cctr, call, __ATOMIC_RELAXED, __HIP_MEMORY_SCOPE_AGENT);
            __builtin_amdgcn_fence(__ATOMIC_ACQUIRE, "");
            VM_WAIT(0);
            post(hint + 16, 0u, call * 256u + KP0);
            const char* sf = srcws + WS_SFLAG + (size_t)idx * FSTRIDE;
            unsigned staged = 0, npoll = 0;
            bool stored = false;
            t0 = rt();
            STAMP(19);
            while (staged < (unsigned)NKS || !stored) {
                unsigned s2; bool p2;
                arrivals(sf, call, lane, s2, p2);
                if (!npoll) STAMP(27);
                if (AG_EFENCE && !npoll) post(hint + 16, 0u, call * 256u + KP0);
                if (s2 > staged) {
                    asm volatile("ds_max_u32 %0, %1" :: "v"(lane ? ctrl + 128 + lane * 4 : ctrl + C_AVAIL), "v"(s2) : "memory");
                    if (!staged) STAMP(17);
                    staged = s2;
                    if (s2 == NKS) STAMP(10);
                }
                if (p2 && !stored) { stored = true; post(hint + 16, 0u, call * 256u + NITEMS); }
                if ((++npoll & 31u) == 0u && (rfl(__hip_atomic_load(tmo, __ATOMIC_RELAXED, __HIP_MEMORY_SCOPE_AGENT)) == call
                                              || rt() - t0 > DEADLINE)) {
                    __hip_atomic_store(tmo, call, __ATOMIC_RELAXED, __HIP_MEMORY_SCOPE_AGENT);
                    post(hint + 16, 0u, call * 256u);
                    VM_WAIT(0);
                    lds_st(ctrl + C_AVAIL, (unsigned)NKS);
                    break;
                }
                if (npoll < 64u && staged < (unsigned)NKS) __builtin_amdgcn_s_sleep(2); else __builtin_amdgcn_s_sleep(32);
            }
            STAMP(11);
        }
        return;
    }

    const int mt = idx & 1, nt = (idx - 2) >> 1;
    const int f0 = (17 * nt) >> 1;
    if (wave < 8) {
        f4 sv[4][2];
        const int g = b - 16;
        int sk = g >> 2;
#pragma unroll
        for (int e = 0; e < K1; ++e) sk += sk >= KP0 + e * KPS;
        const bool late = wave < 4 && sk >= AG_SEARLY;
        if (wave < 4) {
            if (wave == 0) STAMP(0);
            const float* s0 = p.x + (size_t)((g & 3) * 128 + wave * 32 + r8) * GK + sk * BK;
            if (late) {
                __builtin_amdgcn_s_barrier();
                for (int i = AG_SEARLY; i < sk; ++i) __builtin_amdgcn_s_sleep(AG_SSLOPE);
            } else if (AG_SDELAY) __builtin_amdgcn_s_sleep(AG_SDELAY);
#pragma unroll
            for (int i = 0; i < 4; ++i)
                asm volatile("global_load_dwordx4 %0, %2, off" SLD "\n\tglobal_load_dwordx4 %1, %2, off offset:16" SLD
                             : "=&v"(sv[i][0]), "=&v"(sv[i][1]) : "v"(s0 + (size_t)i * 8 * GK + (c8 ^ ((i & 1) * 4)) * 8) : "memory");
        }
        if (!late) __builtin_amdgcn_s_barrier();
        asm volatile("" ::: "memory");
        const unsigned call = rfl(lds_ld(ctrl + C_CALL));
        if (wave < 4) {
            char* d0 = myws + WS_XS + ((size_t)((call & 1u) * 2u + (unsigned)((g >> 1) & 1)) * NKS + (size_t)sk) * X_STEP
                       + (size_t)((g & 1) * X_PART + wave * 4096) + lo16;
            asm volatile("s_waitcnt vmcnt(0)" : "+v"(sv[0][0]), "+v"(sv[0][1]), "+v"(sv[1][0]), "+v"(sv[1][1]),
                         "+v"(sv[2][0]), "+v"(sv[2][1]), "+v"(sv[3][0]), "+v"(sv[3][1]) :: "memory");
            if (wave == 0) STAMP(24);
            u4 o[4];
#pragma unroll
            for (int i = 0; i < 4; i += 2)
                to_frags(pk8(sv[i][0], sv[i][1]), pk8(sv[i + 1][0], sv[i + 1][1]), img + (unsigned)(SCR_OFF + wave * 2048),
                         frg + (unsigned)(SCR_OFF + wave * 2048), o[i], o[i + 1]);
            asm volatile("s_waitcnt lgkmcnt(0)" : "+v"(o[0]), "+v"(o[1]), "+v"(o[2]), "+v"(o[3]) :: "memory");
#pragma unroll
            for (int i = 0; i < 4; ++i)
                st16_sys(d0 + i * 1024, o[i]);
            if (wave == 0) STAMP(25);
            unsigned n = 0;
            if (lane == 0) {
                asm volatile("s_waitcnt vmcnt(0)\n\tds_add_rtn_u32 %0, %1, %2\n\ts_waitcnt lgkmcnt(0)"
                             : "=&v"(n) : "v"(ctrl + C_STG), "v"(1u) : "memory");
                if (n == 3u) st4_sys(myws + WS_SFLAG + (size_t)((g >> 1) & 1) * FSTRIDE + sk * 8 + (g & 1) * 4, call);
            }
            if (wave == 0) { STAMP(16); STAMPE(23); }
        }
        const int set = wave >> 2, w4 = wave & 3;
        const bool dup = !(nt & 1) && nt != 14;
        f4 acc[FW][2];
#pragma unroll
        for (int i = 0; i < FW; ++i) {
            acc[i][0] = (f4){0.f, 0.f, 0.f, 0.f}; acc[i][1] = (f4){0.f, 0.f, 0.f, 0.f};
            asm volatile("" : "+v"(acc[i][0]), "+v"(acc[i][1]));
        }
        h8 a[RW], x[8];
#pragma unroll
        for (int i = 0; i < RW; ++i) asm volatile("" : "=v"(a[i]));
#pragma unroll
        for (int j = 0; j < 8; ++j) asm volatile("" : "=v"(x[j]));
        Pos P;
        Loop L;
        asm volatile("" : "=v"(P.f));
        P.w = lbase + (unsigned)((r16 >> 2) * W_BLK + q * 256 + (r16 & 3) * 16);
        P.n = lbase + (unsigned)(8 * W_BLK + q * 256 + (r16 & 3) * 64 + (r16 >> 2) * 16);
        P.x = lo16 + (unsigned)(set * X_PART + w4 * 2048);
        P.ldw = ctrl + C_LD - 4;
        P.rv = -4u; P.relw = ctrl + C_CONS + wave * 4; P.tab = myws + WS_TAB + (size_t)b * 512;
        unsigned vo = (unsigned)(q * NLOC + r16) * 16u, vn = (unsigned)(q * 4 * NLOC + r16) * 4u;
        asm volatile("" : "+v"(P.w), "+v"(P.n), "+v"(vn), "+v"(P.x), "+v"(P.rv), "+v"(P.ldw), "+v"(vo));
        L.xa = srcws + WS_XS + (size_t)((call & 1u) * 2u + (unsigned)mt) * NKS * X_STEP;
        const int src = (p.srcs >> 4 * xr) & 7;
        L.xg = myws + WS_XG + (size_t)((((call & 1u) * NR + (unsigned)src) * 2u + (unsigned)mt) * K1) * X_STEP;
        L.set = set;
        const char* ol = (const char*)(p.out + (size_t)(src * MLOC + mt * BM + set * 128 + w4 * 16) * NLOC + f0 * 16);
        P.p[0] = L.xa; P.p[1] = L.xa + 8192;
        PIN(ol); PIN(P.relw); PIN(P.tab); PIN(L.xg); PIN(L.set); PIN(P.p[0]); PIN(P.p[1]);
        WAIT_AV(1, 0);
#pragma unroll
        for (int k = 0; k < 4; ++k) XLD(x[k], k);
        L.t = 1u; P.need = 2u; L.xa += X_STEP; P.p[0] = L.xa; P.p[1] = L.xa + 8192;
        PIN(L.t); PIN(P.need); PIN(L.xa); PIN(P.p[0]); PIN(P.p[1]);
        WAIT_AV(0, 1);
        if (wave == 0) { STAMP(18); STAMPC(30); }
        FLAGS();
#pragma unroll
        for (int i = 0; i < FW; ++i) DSR(a[i], i < 8 ? P.w : P.n, wcol(i));
        __builtin_amdgcn_s_waitcnt(0x0070);
        SCHED_FENCE();
#pragma unroll 1
        do {
            k_step<0, 0>(acc, a, x, P, L);
            k_step<1, 0>(acc, a, x, P, L);
        } while (L.t <= (unsigned)KA);
        if (wave == 0) { STAMP(2); STAMPC(31); }
        P.need = (unsigned)(KA + 1 + NI * set);
        const char* xp = L.xa - X_STEP;
        unsigned pre = P.need - set;
#define TRY() do { const unsigned _p = AG_PAIR ? (rfl(P.f.w) >= P.need ? P.need + 1u : 0u) : rfl(P.f.w); item_x(x, P.x, xp, pre < P.need ? P.need : ~0u, _p); \
                   item_x(x + 4, P.x, xp + 2 * X_STEP, pre <= P.need ? P.need + 1u : ~0u, _p); pre = _p; } while (0)
#define SILU(r_, J_, keep_) silu4(acc[(r_) & 4][J_][(r_) & 3], acc[((r_) & 4) + 1][J_][(r_) & 3], acc[((r_) & 4) + 2][J_][(r_) & 3], \
                                  acc[((r_) & 4) + 3][J_][(r_) & 3], 60.f * LOG2E, c, keep_)
#define STORE(r_, o_) asm volatile("global_store_dwordx4 %0, %1, %2 offset:%3" AST \
                                   :: "v"(vo), "v"(o_), "s"(ol + ((r_) & 3) * NLOC * 4), "i"(((r_) & 4) * 64) : "memory")
#define PART(J_) do { \
        SCHED_FENCE(); \
        TTJ(J_, J_ * 6); \
        if (pre < P.need) WAIT_AV(P.need, 0); \
        item_x(x, P.x, xp, pre + 1u, P.need); \
        if (AG_PAIR >> !J_ & 1) { if (!J_) WAIT_AV(P.need, KA + 2); } else if (pre <= P.need) WAIT_AV(P.need + 1u, KA + 2); \
        P.p[0] = xp + 2 * X_STEP; P.p[1] = xp + 3 * X_STEP; PIN(P.p[0]); PIN(P.p[1]); \
        item_x(x + 4, P.x, P.p[0], pre, P.need); \
        if (!J_) _Pragma("unroll") for (int k = FW; k < RW; ++k) DSR(a[k], P.w, woff(k)); \
        if (J_) { if (pre < P.need) VM_WAIT(4); else VM_WAIT(8); } \
        if (J_ && AG_LPRIO && set) __builtin_amdgcn_s_setprio(AG_LPRIO); \
        SCHED_FENCE(); \
        part_items<J_ * NT, J_ * NT + NT>(acc, a, x, P); \
        TTJ(J_, J_ * 6 + 4); \
        if (!J_) { xp += 8192; PIN(xp); P.need += NAB; pre = 0u; TRY(); } \
        if (!J_ && AG_EPRIO && set) __builtin_amdgcn_s_setprio(AG_EPRIO); \
        if (wave == 4 && J_) { STAMP(3); STAMPC(33); } \
        const float c = __builtin_bit_cast(float, __builtin_bit_cast(unsigned, LOG2E) | rfl(P.f.z)); \
        asm volatile("s_nop 7\n\ts_nop 7" : "+v"(acc[FW - 3][J_]), "+v"(acc[FW - 2][J_]), "+v"(acc[FW - 1][J_])); \
        f4 o = acc[FW - 1][J_], od[4]; \
        _Pragma("unroll") for (int r = 0; r < 4; ++r) { o = SILU(r, J_, o); STORE(r, o); } \
        if (!dup) { \
            const f4 on = silu4(acc[FW - 1][J_][0], acc[FW - 1][J_][1], acc[FW - 1][J_][2], acc[FW - 1][J_][3], 60.f * LOG2E, c, o); \
            _Pragma("unroll") for (int r = 0; r < 4; ++r) \
                asm volatile("global_store_dword %0, %1, %2 offset:512" AST :: "v"(vn), "v"(on[r]), "s"(ol + r * NLOC * 4) : "memory"); \
        } \
        _Pragma("unroll") for (int r = 4; r < 8; ++r) {        \
            if (!J_ && (AG_TRYM >> r & 1)) FLAGS(); \
            o = od[r - 4] = SILU(r, J_, o); \
            if (J_) STORE(r, o); \
            if (!J_ && (AG_TRYM >> r & 1)) { asm volatile("s_waitcnt lgkmcnt(0)" : "+v"(P.f) :: "memory"); TRY(); } \
        } \
        if (!J_) _Pragma("unroll") for (int r = 4; r < 8; ++r) STORE(r, od[r - 4]); \
        if (!J_ && AG_EPRIO) __builtin_amdgcn_s_setprio(0); \
        TTJ(J_, J_ * 6 + 5); \
        asm volatile("s_nop 1" ::: "memory"); } while (0)
        PART(0);
        ADV(vo, 64 * NLOC * 4); ADV(vn, 64 * NLOC * 4);
        PART(1);
        if (wave == 4) { STAMP(4); STAMPC(34); }
        __builtin_amdgcn_endpgm();
    }
    if (wave == 10) {
        *(u2*)(myws + WS_TAB + (size_t)b * 512 + lane * 8) = (u2){2u * lane + 1u, 2u * lane + 2u};
        const unsigned cl = ld_ctr((const void*)cctr) + 1u;
        if (lane == 0) __hip_atomic_store(cctr, cl, __ATOMIC_RELAXED, __HIP_MEMORY_SCOPE_AGENT);
        lds_st(ctrl + lane * 4, lane == C_CALL / 4 ? cl : 0u);
        LGKM0();
    }
    __builtin_amdgcn_s_barrier();
    asm volatile("" ::: "memory");
    const unsigned call = rfl(lds_ld(ctrl + C_CALL));
    if (wave == 10) {
        __builtin_amdgcn_fence(__ATOMIC_ACQUIRE, "");
        VM_WAIT(0);
        GAS unsigned* hp = (GAS unsigned*)(myws + WS_HINT + (size_t)(xr * 2 + mt) * FSTRIDE) + c8;
        GAS unsigned* wf0 = (GAS unsigned*)(myws + WS_WFLAG + (size_t)(f0 >> 3) * FSTRIDE) + lane;
        GAS unsigned* wf1 = (GAS unsigned*)(myws + WS_WFLAG + (size_t)((f0 + FW - 1) >> 3) * FSTRIDE) + lane;
        unsigned pub = 0, wav = 0, npoll = 0;
        TRZERO(1);
        const unsigned long long t0 = rt();
        unsigned long long tall = 0;
        bool dead = false;
        while (pub < (unsigned)NITEMS || wav < (unsigned)NKS) {
            const unsigned hv = __hip_atomic_load(hp, __ATOMIC_RELAXED, __HIP_MEMORY_SCOPE_AGENT);
            bool adv = false;
            if (wav < (unsigned)NKS) {
                const unsigned w0 = __hip_atomic_load(wf0, __ATOMIC_RELAXED, __HIP_MEMORY_SCOPE_AGENT);
                const unsigned w1 = __hip_atomic_load(wf1, __ATOMIC_RELAXED, __HIP_MEMORY_SCOPE_AGENT);
                const unsigned long long notready = __ballot((int)(w0 - call) < 0 || (int)(w1 - call) < 0);
                const unsigned w2 = notready ? (unsigned)__builtin_ctzll(notready) : 64u;
                if (w2 > wav) { if (!wav) STAMP(29); wav = w2; lds_st(ctrl + C_WAV, wav); adv = true; }
            }
            const int d = (int)(hv - call * 256u);
            const unsigned av = (unsigned)row_min(c8 < 5 ? max(min(d, NITEMS), 0) : NITEMS);
            if (av > pub) {
                asm volatile("ds_max_u32 %0, %1" :: "v"(lane ? ctrl + 128 + lane * 4 : ctrl + C_AVAIL), "v"(av) : "memory");
                pub = av;
                tall = 0;
                TRACE(1, av - 1);
            } else if (!adv && (++npoll & 63u) == 0u) {
                if (rt() - t0 > DEADLINE) { dead = true; break; }
                unsigned staged; bool stored;
                arrivals(srcws + WS_SFLAG + (size_t)mt * FSTRIDE, call, lane, staged, stored);
                if (staged == (unsigned)NKS && stored) {
                    if (tall == 0) tall = rt();
                    else if (rt() - tall > SILENCE) { pub = NITEMS; lds_st(ctrl + C_AVAIL, pub); }
                } else if (rfl(__hip_atomic_load(tmo, __ATOMIC_RELAXED, __HIP_MEMORY_SCOPE_AGENT)) == call) { dead = true; break; }
            }
            if (pub < (unsigned)AG_FINE) __builtin_amdgcn_s_sleep(AG_SLEEP_POLL); else __builtin_amdgcn_s_sleep(AG_SLEEP_FINE);
        }
        if (dead) {
            __hip_atomic_store(tmo, call, __ATOMIC_RELAXED, __HIP_MEMORY_SCOPE_AGENT);
            lds_st(ctrl + C_BAD, 0x7fc00000u);
        }
        lds_st(ctrl + C_WAV, (unsigned)NKS);
        lds_st(ctrl + C_AVAIL, (unsigned)NITEMS);
        TRDUMP(1);
    } else if (wave == 9) {
        constexpr int CSTEP = 240 / NCONV;
        const int cq = ((idx - 2) / CSTEP) * 8 + xr;
        if ((idx - 2) % CSTEP == 0 && cq < NCONV) {
            const unsigned so = (unsigned)(r8 * 8 * NLOC + c8 * 4) * 4u;
            const unsigned dof = (unsigned)((c8 >> 2) * 2048 + r8 * 256 + (c8 & 3) * 16);
            for (int i = AG_CFIRST; i < cq; i += 16) __builtin_amdgcn_s_sleep(AG_CDELAY);
            for (int u = cq; u < NKS * 16; u += NCONV) {
                const int ks = u >> 4, cn = u & 15;
                const float* sw = p.w + (size_t)ks * BK * NLOC + cn * 128;
                const char* dstw = myws + WS_WH + (size_t)ks * W_ROW + (size_t)cn * 16384;
                f4 v[4][8];
                if (u < 16) STAMP(12);
#pragma unroll
                for (int g = 0; g < 4; ++g)
#pragma unroll
                    for (int r = 0; r < 8; ++r)
                        asm volatile("global_load_dwordx4 %0, %1, %2 offset:%3" CLD : "=v"(v[g][r]) : "v"(so), "s"(sw + r * NLOC), "i"(g * 128) : "memory");
                u4 o[4][4];
#pragma unroll
                for (int g = 0; g < 4; ++g) {
                    asm volatile("s_waitcnt vmcnt(%8)" : "+v"(v[g][0]), "+v"(v[g][1]), "+v"(v[g][2]), "+v"(v[g][3]),
                                 "+v"(v[g][4]), "+v"(v[g][5]), "+v"(v[g][6]), "+v"(v[g][7]) : "i"(24 - 8 * g) : "memory");
#pragma unroll
                    for (int r = 0; r < 8; ++r) v[g][r] *= LOG2E;
#pragma unroll
                    for (int c = 0; c < 4; ++c)
                        o[g][c] = (u4){pk(v[g][0][c], v[g][1][c]), pk(v[g][2][c], v[g][3][c]), pk(v[g][4][c], v[g][5][c]), pk(v[g][6][c], v[g][7][c])};
                    SCHED_FENCE();
                }
                if (u < 16) STAMP(13);
#pragma unroll
                for (int g = 0; g < 4; ++g)
#pragma unroll
                    for (int c = 0; c < 4; ++c)
                        asm volatile("global_store_dwordx4 %0, %1, %2 offset:%3" CST "\n\ts_nop 1"
                                     :: "v"(dof), "v"(o[g][c]), "s"(dstw + g * 4096), "i"(c * 64) : "memory");
                VM_WAIT(0);
                if (lane == 0)
                    __hip_atomic_store((GAS unsigned*)(myws + WS_WFLAG + (size_t)cn * FSTRIDE + ks * 4), call,
                                       __ATOMIC_RELAXED, __HIP_MEMORY_SCOPE_AGENT);
                if (u < 16) STAMP(28);
                if (u < NCONV) STAMP(35);
            }
            STAMP(1);
        }
    } else {
        const char* tab = myws + WS_TAB + (size_t)b * 512;
        const char* wa = myws + WS_WH + (size_t)f0 * 2048;
        unsigned long long t0 = 0;
        int av = 0, cons = 0;
#pragma unroll 1
        for (int t = 0; t < NKS; ++t) {
            if (av <= t || cons <= t - NW) for (unsigned sp = 0;;) {
                u4 c, d; unsigned v;
                asm volatile("ds_read_b128 %0, %3\n\tds_read_b128 %1, %3 offset:16\n\tds_read_b32 %2, %3 offset:%4\n\t"
                             "s_waitcnt lgkmcnt(0)" : "=&v"(c), "=&v"(d), "=&v"(v) : "v"(ctrl + C_CONS), "i"(C_WAV - C_CONS) : "memory");
                av = (int)rfl(v); cons = (int)min8(c, d);
                if ((av > t && cons > t - NW) || SPIN_OUT(sp, t0)) break;
                __builtin_amdgcn_s_sleep(AG_LSLEEP);
            }
            TRACE(2, t);
            if (t == 0) STAMP(32);
            const unsigned sb = lbase + (unsigned)(t & (NW - 1)) * W_STEP;
#define WLOADS(P_) _Pragma("unroll") for (int i = 0; i < FW; i += 2) { \
                asm volatile("s_mov_b32 m0, %0\n\ts_waitcnt vmcnt(58)\n\t" \
                             "global_load_lds_dwordx4 %1, %2" P_ "\n\tglobal_load_lds_dwordx4 %1, %2 offset:1024" P_ \
                             :: "s"(sb + i * W_BLK), "v"(lo16), "s"(wa + i * 2048) : "memory"); \
                if (i + 1 < FW) asm volatile("s_mov_b32 m0, %0\n\ts_nop 0\n\t" \
                             "global_load_lds_dwordx4 %1, %2 offset:2048" P_ "\n\tglobal_load_lds_dwordx4 %1, %2 offset:3072" P_ \
                             :: "s"(sb + i * W_BLK + 64), "v"(lo16), "s"(wa + i * 2048) : "memory"); \
            }
            if (mt == 0 && t >= AG_WNT0 && t < AG_WNT) { WLOADS(" nt") } else
            { WLOADS(WLD) }
            asm volatile("s_mov_b32 m0, %0\n\ts_mov_b64 exec, 1\n\tglobal_load_lds_dword %1, %2\n\ts_mov_b64 exec, -1"
                         :: "s"(ctrl + C_LD), "v"(lo16), "s"(tab + t * 4) : "memory");
            wa += W_ROW;
        }
        VM_WAIT(0);
    }
}

}

extern "C" void kernel_launch(int rank, int nranks, ncclComm_t comm,
                              hipStream_t stream,
                              void* const* d_in, const int* in_sizes, int n_in,
                              void* d_out, int out_size,
                              const int* in_dtypes, int out_dtype,
                              void* const* d_out_all,
                              void* const* d_ws_all, size_t ws_bytes) {
    (void)comm; (void)in_dtypes; (void)out_dtype; (void)d_out_all;
    if (nranks != NR || n_in < 2 || (unsigned)rank >= (unsigned)NR || ws_bytes < WS_NEED ||
        in_sizes[0] != MLOC * GK || in_sizes[1] != GK * NLOC || out_size != GM * NLOC) {
        fprintf(stderr, "ag_gemm: unexpected problem shape\n");
        return;
    }
    static bool attr_set = false;
    if (!attr_set) {
        HIP_CHECK(hipFuncSetAttribute((const void*)ag_gemm_kernel,
                                      hipFuncAttributeMaxDynamicSharedMemorySize, LDS_BYTES));
        attr_set = true;
    }
    Params p;
    p.x = (const float*)d_in[0];
    p.w = (const float*)d_in[1];
    p.out = (float*)d_out;
    p.myws = (char*)d_ws_all[rank];
    p.srcs = 0;
    {
        int order[NR], n = 0;
        const unsigned half = 0xb2;
        for (int r = 0; r < NR; ++r) order[n++] = r;
        for (int r = 0; r < NR; ++r) { p.ws[r] = (char*)d_ws_all[order[r]]; p.srcs |= (unsigned)order[r] << 4 * r; }
    }
    for (int r = 0; r < 2 * NR; ++r) p.xg[r] = (char*)d_ws_all[(rank + 1 + r) & 7] + WS_XG + (size_t)rank * 2 * K1 * X_STEP;
    hipLaunchKernelGGL(ag_gemm_kernel, dim3(NWG), dim3(NTHREADS), LDS_BYTES, stream, p);
    HIP_CHECK(hipGetLastError());
}
```

```cpp
#include <hip/hip_runtime.h>
#include <rccl/rccl.h>
#include <cstdio>
#include <cstddef>

#define HIP_CHECK(expr) do { hipError_t _e = (expr); if (_e != hipSuccess) { \
    fprintf(stderr, "HIP error %s at %s:%d\n", hipGetErrorString(_e), __FILE__, __LINE__); \
    return; } } while (0)

namespace {

#define GAS __attribute__((address_space(1)))
#define LAS __attribute__((address_space(3)))
typedef _Float16 h8 __attribute__((ext_vector_type(8)));
typedef _Float16 h2 __attribute__((ext_vector_type(2)));
typedef float f4 __attribute__((ext_vector_type(4)));
typedef unsigned u4 __attribute__((ext_vector_type(4)));
typedef unsigned u2 __attribute__((ext_vector_type(2)));

#define AG_PWIN 16
#define AG_WIN 8
#define AG_EFENCE 0
#define AG_HSTEP 14
#define AG_HN 4
#define AG_CFIRST -17
#define AG_CDELAY 20
#define AG_WNT 32
#define AG_WNT0 8
#define AG_ROFF 0
#define ROTX ((__builtin_amdgcn_s_getreg((3 << 11) | 20) + AG_ROFF) & 7u)
#define AG_NOCHK 3
#define AG_PAIR 0
#define AG_PUB 0
#define AG_PUBG 0
#define AG_RELG 34
#define AG_MAP 1
#define AG_SLEEP_POLL 4
#define AG_SLEEP_FINE 1
#define AG_FINE 48
#define AG_NCONV 120
#define AG_KP0 30
#define AG_KPS 6
#define AG_SDELAY 8
#define AG_SEARLY 12
#define AG_SSLOPE 1
#define AG_LSLEEP 1
#define AG_EARLY0 2
#define AG_EARLY1 2
#define AG_EARLY2 0
#define AG_E2T 4
#define AG_EPRIO 0
#define AG_LPRIO 0
#define AG_RW 9
#define AG_WG 3
#define AG_NAB 8
#define AG_OWN 85
#define AG_CA 2
#define AG_AST 2
#define AG_TRYM 0x90
#define AST " nt"

#define CST " sc1 nt"
#define WLD ""
#define CLD " nt"
#define XLP ""
#define PLD ""
#define SST " nt"
#define PST ""
#define SLD " nt"
constexpr int NR = 8;
constexpr int GM = 4096, GK = 4096, NLOC = 2048, MLOC = 512;
constexpr int BM = 256, BK = 64;
constexpr int FW = 9;
constexpr int NKS = GK / BK;
constexpr int K1 = 4;
constexpr int KP0 = AG_KP0, KPS = AG_KPS;
constexpr unsigned long long PMASK = (1ull | 1ull << KPS | 1ull << 2 * KPS | 1ull << 3 * KPS) << KP0;
constexpr int NAB = AG_NAB, NI = NAB / 2;
constexpr int KA = NKS - NAB;
constexpr int NITEMS = KA + 4 * NI;
constexpr int X_STEP = BM * BK * 2;
constexpr int X_PART = X_STEP / 2;
constexpr int W_BLK = 2048 + 64;
constexpr int W_STEP = FW * W_BLK - 64;
constexpr int W_ROW = NLOC * BK * 2;
constexpr int NW = 8;
constexpr int SCR_OFF = NW * W_STEP;
constexpr int CTRL_OFF = SCR_OFF + 8192;
constexpr int LDS_BYTES = CTRL_OFF + 512;
constexpr int NWG = 256, NTHREADS = 704;
static_assert(LDS_BYTES <= 160 * 1024 && NAB <= NW && NAB % 2 == 0 && NAB >= 4 && NKS % NW == 0, "layout");
static_assert((NKS - K1) * 4 == NWG - 16 && KP0 > 1 && KPS > 1 && KP0 + 3 * KPS < KA - 1, "layout");
constexpr int C_CONS = 0;
constexpr int C_WAV = 68;
constexpr int C_STG = 72;
constexpr int C_CALL = 76;
constexpr int C_ROT = 80;
constexpr int C_LD = 84;
constexpr int C_BAD = 88;
constexpr int C_AVAIL = 92;
constexpr int NCONV = AG_NCONV;
constexpr float LOG2E = 1.4426950408889634f;

constexpr size_t FSTRIDE = 4096 + 256;
constexpr size_t WS_CALL = 0;
constexpr size_t WS_TMO = 49152;
constexpr size_t WS_SFLAG = 53248;
constexpr size_t WS_HINT = 65536;
constexpr size_t WS_WFLAG = 196608;
constexpr size_t WS_TAB = 327680;
constexpr size_t WS_DBG = 524288;
constexpr size_t WS_WH = 1ull << 20;
constexpr size_t WS_XS = 32ull << 20;
constexpr size_t WS_XG = 48ull << 20;
constexpr size_t WS_NEED = 64ull << 20;

constexpr unsigned long long DEADLINE = 100000000ull;
constexpr unsigned long long SILENCE = 20000ull;

struct Params {
    const float* x;
    char* myws;
    const float* w;
    float* out;
    char* ws[NR];
    char* xg[2 * NR];
    unsigned srcs;
};
#define AG_ROTATED
typedef const char* __attribute__((address_space(4))) const* KPTR;

__device__ __forceinline__ unsigned long long rt() { return __builtin_amdgcn_s_memrealtime(); }

__device__ __forceinline__ void st16_sys(void* p, u4 v) {
    asm volatile("global_store_dwordx4 %0, %1, off sc0 sc1" SST "\n\ts_nop 1" :: "v"(p), "v"(v) : "memory");
}
__device__ __forceinline__ void st16_dev(void* p, u4 v) {
    asm volatile("global_store_dwordx4 %0, %1, off sc1\n\ts_nop 1" :: "v"(p), "v"(v) : "memory");
}
__device__ __forceinline__ void st4_sys(void* p, unsigned v) {
    asm volatile("global_store_dword %0, %1, off sc0 sc1\n\ts_nop 1" :: "v"(p), "v"(v) : "memory");
}
__device__ __forceinline__ void post(const char* p, unsigned zero, unsigned v) {
    asm volatile("s_mov_b64 exec, 1\n\tglobal_store_dword %0, %1, %2\n\ts_mov_b64 exec, -1" :: "v"(zero), "v"(v), "s"(p) : "memory");
}
__device__ __forceinline__ void lds_st(unsigned addr, unsigned v) {
    asm volatile("ds_write_b32 %0, %1" :: "v"(addr), "v"(v) : "memory");
}
__device__ __forceinline__ unsigned lds_ld(unsigned addr) {
    unsigned v;
    asm volatile("ds_read_b32 %0, %1\n\ts_waitcnt lgkmcnt(0)" : "=&v"(v) : "v"(addr) : "memory");
    return v;
}
__device__ __forceinline__ unsigned ld_ctr(const void* p) {
    unsigned v;
    asm volatile("global_load_dword %0, %1, off sc1\n\ts_waitcnt vmcnt(0)" : "=&v"(v) : "v"(p) : "memory");
    return (unsigned)__builtin_amdgcn_readfirstlane(v);
}
__device__ __forceinline__ unsigned pk(float a, float b) {
    h2 h; h.x = (_Float16)a; h.y = (_Float16)b;
    return __builtin_bit_cast(unsigned, h);
}
__device__ __forceinline__ u4 pk8(f4 a, f4 b) { return (u4){pk(a[0], a[1]), pk(a[2], a[3]), pk(b[0], b[1]), pk(b[2], b[3])}; }
__device__ __forceinline__ unsigned umin(unsigned a, unsigned b) { return a < b ? a : b; }
__device__ __forceinline__ unsigned rfl(unsigned v) { return (unsigned)__builtin_amdgcn_readfirstlane((int)v); }
__device__ __forceinline__ unsigned min8(u4 c, u4 d) {
    return rfl(umin(umin(umin(c.x, c.y), umin(c.z, c.w)), umin(umin(d.x, d.y), umin(d.z, d.w))));
}
__device__ __forceinline__ int row_min(int u) {
    u = min(u, __builtin_amdgcn_update_dpp(0, u, 0x121, 15, 15, false));
    u = min(u, __builtin_amdgcn_update_dpp(0, u, 0x122, 15, 15, false));
    u = min(u, __builtin_amdgcn_update_dpp(0, u, 0x124, 15, 15, false));
    u = min(u, __builtin_amdgcn_update_dpp(0, u, 0x128, 15, 15, false));
    return __builtin_amdgcn_readfirstlane(u);
}
__device__ __forceinline__ void arrivals(const char* sf, unsigned call, int lane, unsigned& staged, bool& stored) {
    u2 sv; unsigned pv;
    asm volatile("global_load_dwordx2 %0, %2, %4 sc0 sc1\n\tglobal_load_dword %1, %3, %4 offset:512 sc0 sc1\n\ts_waitcnt vmcnt(0)"
                 : "=&v"(sv), "=&v"(pv) : "v"(lane * 8), "v"((lane & 15) * 4), "s"(sf) : "memory");
    const unsigned long long miss = __ballot((int)(sv.x - call) < 0 || (int)(sv.y - call) < 0) & ~PMASK;
    staged = miss ? (unsigned)__builtin_ctzll(miss) : (unsigned)NKS;
    stored = __ballot((int)(pv - call) < 0) == 0;
}
#define VM_WAIT(n) asm volatile("s_waitcnt vmcnt(" #n ")" ::: "memory")
#define LGKM0() asm volatile("s_waitcnt lgkmcnt(0)" ::: "memory")
#define SCHED_FENCE() __builtin_amdgcn_sched_barrier(0)
#define SPIN_OUT(sp_, t0_) ((((++(sp_)) & 1023u) == 0u) && ((t0_) == 0 ? ((t0_) = rt(), false) : ((unsigned)(rt() - (t0_)) > 2u * (unsigned)DEADLINE)))

#define PB ((blockIdx.x & ~7u) | (__builtin_amdgcn_s_getreg((3 << 11) | 20) & 7u))
#define TRACE(k_, t_) do {} while (0)
#define TRDUMP(k_) do {} while (0)
#define TRZERO(k_) do {} while (0)
#define STAMP(i) do {} while (0)
#define STAMPC(i) do {} while (0)

#define DSR(dst_, addr_, off_) asm volatile("ds_read_b128 %0, %1 offset:%2" : "+v"(dst_) : "v"(addr_), "i"(off_) : "memory")

__device__ __forceinline__ f4 silu4(float z0, float z1, float z2, float z3, float lim, float c, f4 keep) {
    float o0, o1, o2, o3;
    asm volatile("v_med3_f32 %0, -%4, %8, -%8\n\tv_med3_f32 %1, -%5, %8, -%8\n\tv_med3_f32 %2, -%6, %8, -%8\n\tv_med3_f32 %3, -%7, %8, -%8\n\t"
                 "v_exp_f32 %0, %0\n\tv_exp_f32 %1, %1\n\tv_exp_f32 %2, %2\n\tv_exp_f32 %3, %3\n\t"
                 "v_fma_f32 %0, %0, %9, %9\n\tv_fma_f32 %1, %1, %9, %9\n\tv_fma_f32 %2, %2, %9, %9\n\tv_fma_f32 %3, %3, %9, %9\n\t"
                 "v_rcp_f32 %0, %0\n\tv_rcp_f32 %1, %1\n\tv_rcp_f32 %2, %2\n\tv_rcp_f32 %3, %3\n\t"
                 "v_mul_f32 %0, %4, %0\n\tv_mul_f32 %1, %5, %1\n\tv_mul_f32 %2, %6, %2\n\tv_mul_f32 %3, %7, %3"
                 : "=&v"(o0), "=&v"(o1), "=&v"(o2), "=&v"(o3) : "v"(z0), "v"(z1), "v"(z2), "v"(z3), "s"(lim), "s"(c), "v"(keep));
    return (f4){o0, o1, o2, o3};
}
#define REL() asm volatile("s_mov_b32 m0, %0\n\ts_mov_b64 exec, 1\n\tglobal_load_lds_dword %1, %2\n\ts_mov_b64 exec, -1" \
                           :: "s"(P.relw), "v"(P.rv), "s"(P.tab) : "memory")
#define TT(i_) do {} while (0)
struct Pos { unsigned w, n, x, rv, need, relw, ldw; const char* tab; const char* p[2]; u4 f; };
#define TTJ(j_, i_) TT(i_)
#define ADV(v_, d_) do { v_ += d_; asm volatile("" : "+v"(v_)); } while (0)
#define PIN(s_) asm volatile("" : "+s"(s_))
#define FLAGS() asm volatile("ds_read_b128 %0, %1" : "=v"(P.f) : "v"(P.ldw) : "memory")
#define WAIT_AV(nx_, nw_) do { unsigned long long _t0 = 0; unsigned _sp = 0; \
        for (;;) { asm volatile("ds_read_b128 %0, %1\n\ts_waitcnt lgkmcnt(0)" : "+v"(P.f) : "v"(P.ldw) : "memory"); \
            if ((int)((rfl(P.f.w) - (unsigned)(nx_)) | (rfl(P.f.y) - (unsigned)(nw_))) >= 0) break; \
            if (SPIN_OUT(_sp, _t0)) break; \
            __builtin_amdgcn_s_sleep(1); } } while (0)
#define CHECK_AV(nx_, nw_) do { if (__builtin_expect((int)((rfl(P.f.w) - (unsigned)(nx_)) | (rfl(P.f.y) - (unsigned)(nw_))) < 0, 0)) \
            WAIT_AV(nx_, nw_); SCHED_FENCE(); } while (0)
#define MFMA(c_, x_, a_) asm volatile("v_mfma_f32_16x16x32_f16 %0, %1, %2, %0" : "+v"(c_) : "v"(x_), "v"(a_))
#define MFMA_R(c_, x_, a_, ad_, off_) asm volatile("v_mfma_f32_16x16x32_f16 %0, %2, %1, %0\n\tds_read_b128 %1, %3 offset:%4" \
                                                   : "+v"(c_), "+v"(a_) : "v"(x_), "v"(ad_), "i"(off_) : "memory")
#define XLD(x_, k_) asm volatile("global_load_dwordx4 %0, %1, %2 offset:%3" : "+v"(x_) : "v"(P.x), "s"(P.p[(k_) & 1]), "i"(((k_) & 2) * 512) : "memory")
constexpr int wcol(int i) { return i < 8 ? (i >> 2) * 4 * W_BLK + (i & 3) * 64 : 0; }

constexpr int RW = AG_RW, G_A = AG_CA;
struct Loop { const char* xa; const char* xg; unsigned t, set; };
template <int B, int G>
__device__ __forceinline__ void k_step(f4 (&acc)[FW][2], h8 (&a)[RW], h8 (&x)[8], Pos& P, Loop& L) {
    if constexpr (G < 36) {
        constexpr int h = G / 18, i = G % 18 / 2;
        if (G % 18 == 0) asm volatile("s_waitcnt vmcnt(%0) lgkmcnt(7)" :: "i"(h ? 5 : 3) : "memory");
        else if (G == G_A) asm volatile("s_waitcnt lgkmcnt(%1)" : "+v"(P.f) : "i"(3 + G_A / 2) : "memory");
        else if (G == AG_RELG && G <= 30) asm volatile("s_waitcnt lgkmcnt(%0)" :: "i"((G - 18) / 2) : "memory");
        else if (G % 18 % 4 == 0 && !(G > AG_RELG && G > 18)) asm volatile("s_waitcnt lgkmcnt(7)" ::: "memory");
        if (G & 1) MFMA_R(acc[i][1], x[B * 4 + h * 2 + 1], a[i], i < 8 ? P.w : P.n, (h ? 1 - B : B) * W_STEP + (1 - h) * 1024 + wcol(i));
        else MFMA(acc[i][0], x[B * 4 + h * 2], a[i]);
        SCHED_FENCE();
        if (G == G_A) CHECK_AV(P.need, L.t + 1u);
        if (G > G_A && G <= G_A + 8 && !(G & 1)) XLD(x[4 - B * 4 + (G - G_A) / 2 - 1], (G - G_A) / 2 - 1);
        if (B && G == 18) { const int d = (L.t & (NW - 1)) ? 2 * W_STEP : (2 - NW) * W_STEP; ADV(P.w, d); ADV(P.n, d); }
        if (G == 20) { L.t += 1u; PIN(L.t); L.xa += X_STEP; PIN(L.xa); }
        if (G == 22) {
            const bool st = (PMASK >> L.t) & 1;
            P.p[0] = st ? L.xg : L.xa; PIN(P.p[0]);
            L.xg += st ? X_STEP : 0; PIN(L.xg);
        }
        if (G == 24) {
            P.need = L.t + 1u;
            if (B) P.p[1] = P.p[0] + 8192;
            else {
                const bool last = L.t == (unsigned)KA;
                P.p[0] -= last && L.set ? X_STEP : 0;
                P.p[1] = P.p[0] + (last && !L.set ? X_STEP : 8192);
                P.need = last && L.set ? 0u : P.need;
                PIN(P.p[0]);
            }
            PIN(P.p[1]); PIN(P.need);
        }
        if (G == 30) FLAGS();
        if (G == (AG_RELG < 34 ? 14 : 32)) ADV(P.rv, 4u);
        if (G == AG_RELG) REL();
        SCHED_FENCE();
        k_step<B, G + 1>(acc, a, x, P, L);
    }
}
constexpr int NT = 36 * NI, WG = AG_WG, G_C = 35 - RW;
constexpr bool flags(int t) { return t % 36 == (t % NT < NT - 36 ? G_C - 9 : 26); }
constexpr int xfetch(int t) {
    for (int k = 0; k < 4; ++k) if (t % 36 == (G_C + 1 + k > 9 * k + 8 ? G_C + 1 + k : 9 * k + 8)) return t % NT < NT - 72 ? k : -1;
    return -1;
}
constexpr bool pub(int t) { return AG_PUB && t % 36 == AG_PUBG && t % NT / 36 + AG_PUB >= NI; }
struct Waits { int l[2 * NT + 1], v[NT / 9]; };
constexpr Waits waits() {
    Waits r{};
    int pw[2 * NT + 16] = {}, pf = 0, c = 0, done = -1, xs = 8;
    for (int k = 0; k < RW; ++k) pw[k] = c++;
    for (int t = 0; t <= 2 * NT; ++t) {
        int need = t % WG || t == 2 * NT ? -1 : pw[t + WG - 1];
        if (t % 36 == G_C || (t && t % NT == 0)) need = pf > need ? pf : need;
        r.l[t] = need <= done ? -1 : c - 1 - need < 15 ? c - 1 - need : 15;
        done = need > done ? need : done;
        if (t + RW < 2 * NT) pw[t + RW] = c++;
        if (flags(t)) pf = c++;
        if (pub(t)) ++c;
        if (t < NT && t % 9 == 0) r.v[t / 9] = xs - 1 - t / 9;
        if (xfetch(t) >= 0) ++xs;
    }
    return r;
}
static_assert(RW >= FW && RW > WG && NT % WG == 0, "parts");
constexpr int woff(int k) { return (k % 36 / 9 & 1) * W_STEP + k % 36 / 18 * 1024 + wcol(k % 9); }
template <int T, int E>
__device__ __forceinline__ void part_items(f4 (&acc)[FW][2], h8 (&a)[RW], h8 (&x)[8], Pos& P) {
    constexpr int left = waits().l[T];
    if constexpr (left >= 0 && (T > E - NT || T == 0)) asm volatile("s_waitcnt lgkmcnt(%0)" :: "i"(left) : "memory");
    if (T == E || T % 36 == G_C) asm volatile("" : "+v"(P.f));
    if constexpr (T < E) {
        constexpr int J = T / NT, N = T % NT / 36, G = T % 36, K = T + RW, X = (N & 1) * 4;
        if (G % 9 == 0 && (!J || N)) asm volatile("s_waitcnt vmcnt(%0)" :: "i"(waits().v[T % NT / 9]) : "memory");
        if constexpr (K < 2 * NT) MFMA_R(acc[T % FW][J], x[X + G / 9], a[T % RW], K % 9 < 8 ? P.w : P.n, woff(K));
        else MFMA(acc[T % FW][J], x[X + G / 9], a[T % RW]);
        SCHED_FENCE();
        if (G == 0 && N >= NI - 3) TT(J * 6 + 4 + N - NI);
        if (K < 2 * NT && K % 36 >= 34) {
            constexpr int d = K % NT / 36 == NI - 1 ? (2 - NAB) * W_STEP : 2 * W_STEP;
            if (K % 36 == 34) ADV(P.w, d); else ADV(P.n, d);
        }
        if (flags(T)) FLAGS();
        if (pub(T)) asm volatile("v_mov_b32 %0, %2\n\ts_mov_b64 exec, 1\n\tds_max_u32 %1, %0 offset:12\n\ts_mov_b64 exec, -1"
                                 : "+v"(P.rv) : "v"(P.ldw), "s"(P.need + (N + AG_PUB)) : "memory");
        if (G == G_C && N < (J ? NI - 2 : NI - 1) && !(J && (AG_NOCHK & 1)))
            CHECK_AV(N < NI - 2 && !(AG_NOCHK & 2) ? P.need + (N + 2) : 0u, J ? 0 : KA + 2 * N + 4);
        if (G == G_C - 2 && N < NI - 2) { P.p[0] += 2 * X_STEP; PIN(P.p[0]); P.p[1] += 2 * X_STEP; PIN(P.p[1]); }
        if constexpr (xfetch(T) >= 0) XLD(x[X + xfetch(T)], xfetch(T));
        SCHED_FENCE();
        part_items<T + 1, E>(acc, a, x, P);
    }
}
__device__ __forceinline__ void item_x(h8* x, unsigned vx, const char* p, unsigned lo, unsigned hi) {
    asm volatile("s_cmp_gt_u32 %7, %8\n\ts_cbranch_scc1 1f\n\t"
                 "global_load_dwordx4 %0, %4, %5\n\tglobal_load_dwordx4 %1, %4, %6\n\t"
                 "global_load_dwordx4 %2, %4, %5 offset:1024\n\tglobal_load_dwordx4 %3, %4, %6 offset:1024\n1:"
                 : "+v"(x[0]), "+v"(x[1]), "+v"(x[2]), "+v"(x[3])
                 : "v"(vx), "s"(p), "s"(p + X_STEP), "s"(lo), "s"(hi) : "memory", "scc");
}
__device__ __forceinline__ void to_frags(u4 o0, u4 o1, unsigned wr, unsigned rd, u4& f0, u4& f1) {
    asm volatile("ds_write_b128 %2, %4\n\tds_write_b128 %2, %5 offset:1024\n\tds_read_b128 %0, %3\n\tds_read_b128 %1, %6"
                 : "=&v"(f0), "=&v"(f1) : "v"(wr), "v"(rd), "v"(o0), "v"(o1), "v"(rd ^ 64u) : "memory");
}

__global__ void __launch_bounds__(NTHREADS, 3) ag_gemm_kernel(Params p) {
    extern __shared__ __attribute__((aligned(16))) unsigned char smem[];
#define STAMPE(i) do {} while (0)
    LAS unsigned char* lds = (LAS unsigned char*)smem;
    const int tid = threadIdx.x;
    const int wave = __builtin_amdgcn_readfirstlane(tid >> 6);
    const int lane = tid & 63;
    const int b = blockIdx.x;
    const int xr = b & 7, idx = b >> 3;
    char* myws = p.myws;
    GAS unsigned* cctr = (GAS unsigned*)(myws + WS_CALL + (size_t)b * 128);
    GAS unsigned* tmo = (GAS unsigned*)(myws + WS_TMO);
    const unsigned lbase = (unsigned)(size_t)lds;
    const unsigned ctrl = lbase + CTRL_OFF;
    const unsigned lo16 = (unsigned)lane * 16u;
    const int r8 = lane >> 3, c8 = lane & 7;
    const int r16 = lane & 15, q = lane >> 4;
    const unsigned img = lbase + (unsigned)(r8 * 128 + (c8 ^ (r8 >> 1)) * 16);
    const unsigned frg = lbase + (unsigned)(r16 * 128 + (q ^ (r16 >> 1)) * 16);
    const char* __attribute__((address_space(4))) const* ka = (KPTR)__builtin_amdgcn_kernarg_segment_ptr();

    if (__builtin_expect(idx < 2 && wave < 2, 1)) {
        const int pid = (xr * 2 + idx) * 2 + wave;
        if (wave < 2) STAMP(20 + wave);
        f4 v[K1][4];
        unsigned cv;
        {
            const unsigned s0 = (unsigned)((pid * 16 + r8) * GK + c8 * 8) * 4u;
            const unsigned s1 = (unsigned)((pid * 16 + 8 + r8) * GK + (c8 ^ 4) * 8) * 4u;
#pragma unroll
            for (int e = 0; e < K1; ++e) {
                asm volatile("global_load_dwordx4 %0, %4, %6" PLD "\n\tglobal_load_dwordx4 %1, %4, %6 offset:16" PLD "\n\t"
                             "global_load_dwordx4 %2, %5, %6" PLD "\n\tglobal_load_dwordx4 %3, %5, %6 offset:16" PLD
                             : "=&v"(v[e][0]), "=&v"(v[e][1]), "=&v"(v[e][2]), "=&v"(v[e][3])
                             : "v"(s0), "v"(s1), "s"(p.x + (KP0 + e * KPS) * BK) : "memory");
                if (e == 0) asm volatile("global_load_dword %0, %1, %2 sc1" : "=&v"(cv) : "v"(b * 128), "s"(myws + WS_CALL) : "memory");
            }
        }
        SCHED_FENCE();
        const char* dst[NR];
#pragma unroll
        for (int j = 0; j < NR; ++j) dst[j] = ka[offsetof(Params, xg) / 8 + ((xr * 2 + idx + 4 * wave) & 7) + j];
        unsigned vo = lo16 + (unsigned)((pid >> 4) * K1 * X_STEP + (pid & 15) * 2048), call = 0;
        char* flag = myws + WS_SFLAG + (size_t)(pid >> 4) * FSTRIDE + 512 + (pid & 15) * 4;
        constexpr int W = AG_PWIN;
        static_assert(W >= 2 && W <= 32 && W % 2 == 0, "window");
        u4 o[K1][2];
#pragma unroll
        for (int e = 0; e < K1; ++e) {
            if (e < 2) {
#pragma unroll
                for (int k = e; k < (e ? K1 : 1); ++k) {
                    asm volatile("s_waitcnt vmcnt(%5)" : "+v"(cv), "+v"(v[k][0]), "+v"(v[k][1]), "+v"(v[k][2]), "+v"(v[k][3])
                                 : "i"(4 * (K1 - 1 - k) + e * 2 * NR) : "memory");
                    to_frags(pk8(v[k][0], v[k][1]), pk8(v[k][2], v[k][3]), img + 32768u + wave * 2048u, frg + 32768u + wave * 2048u, o[k][0], o[k][1]);
                }
            }
            if (e == 0) {
                call = rfl(cv) + 1u;
                vo += (call & 1u) * (unsigned)(NR * 2 * K1 * X_STEP);
                if (wave == 0) STAMP(5);
            }
            if (e == 1) __builtin_amdgcn_s_barrier();
            asm volatile("s_waitcnt lgkmcnt(%2)" : "+v"(o[e][0]), "+v"(o[e][1]) : "i"(e ? 4 * (K1 - 1 - e) : 0) : "memory");
            const unsigned ve = vo + (unsigned)(e * X_STEP);
#pragma unroll
            for (int j = 0; j < NR; ++j) {
                if (e) asm volatile("s_waitcnt vmcnt(%0)" :: "i"(W - 2) : "memory");
                asm volatile("global_store_dwordx4 %0, %1, %3 sc0 sc1" PST "\n\tglobal_store_dwordx4 %0, %2, %3 offset:1024 sc0 sc1" PST "\n\ts_nop 1"
                             :: "v"(ve), "v"(o[e][0]), "v"(o[e][1]), "s"(dst[j]) : "memory");
            }
        }
        VM_WAIT(0);
        if (lane == 0) st4_sys(flag, call);
        if (wave < 2) STAMP(14 + wave);
        if (wave == 0) STAMPE(22);
        __builtin_amdgcn_endpgm();
    }
    GAS unsigned* rotw = (GAS unsigned*)(myws + WS_WFLAG + (size_t)(idx & 15) * FSTRIDE + 256);
#define SRCWS(rot_) ka[offsetof(Params, ws) / 8 + ((xr + (rot_)) & 7)]
    if (idx < 2) {
        const char* hint = myws + WS_HINT + (size_t)(xr * 2 + idx) * FSTRIDE;
        if (wave == 7) lds_st(ctrl + lane * 4, 0u);
        const unsigned call = ld_ctr((const void*)cctr) + 1u;
        LGKM0();
        __builtin_amdgcn_s_barrier();
        asm volatile("" ::: "memory");
        unsigned long long t0 = 0;
        if (wave >= 3 && wave < 7) {
            const int qt = wave - 3;
            const char* hq = hint + qt * 4;
            unsigned rot;
            for (unsigned sp = 0;;) {
                rot = rfl(lds_ld(ctrl + C_ROT));
                if (rot || SPIN_OUT(sp, t0)) break;
                __builtin_amdgcn_s_sleep(1);
            }
            const char* srcws = SRCWS(rot);
            const char* xa = srcws + WS_XS + (size_t)((call & 1u) * 2u + (unsigned)idx) * NKS * X_STEP;
            unsigned hv = call * 256u + 1u, zero = 0u;
            asm volatile("s_mov_b32 m0, %2" : "+v"(hv), "+v"(zero) : "s"(lbase + qt * 4096) : "memory");
            int av = 0;
#define STAGED(n_) do { if (av < (n_)) for (unsigned _sp = 0;;) { av = (int)rfl(lds_ld(ctrl + C_AVAIL)); \
                if (av >= (n_) || SPIN_OUT(_sp, t0)) break; \
                __builtin_amdgcn_s_sleep(1); } } while (0)
#define PULL4(a_) asm volatile("s_waitcnt vmcnt(%2)\n\tglobal_load_lds_dwordx4 %0, %1\n\t" \
                         "s_waitcnt vmcnt(%2)\n\tglobal_load_lds_dwordx4 %0, %1 offset:1024\n\t" \
                         "s_waitcnt vmcnt(%2)\n\tglobal_load_lds_dwordx4 %0, %1 offset:2048\n\t" \
                         "s_waitcnt vmcnt(%2)\n\tglobal_load_lds_dwordx4 %0, %1 offset:3072" \
                         :: "v"(lo16), "s"(a_), "i"(AG_WIN - 1) : "memory")
            unsigned long long own = srcws == myws ? rt() : 0;
#define PACE(n_) do { if (own) { own += n_; while (rt() < own) __builtin_amdgcn_s_sleep(2); } } while (0)
#pragma unroll 1
            for (int t = 0; t < KA; ++t) {
                STAGED(t + 1);
                PACE(AG_OWN);
                if (qt == 0) TRACE(0, t);
                if (qt == 0 && (t & 7) == 1) STAMP(54 + t / 8);

                if (qt == 0 && t < 12) STAMP(36 + t);
                const char* xt = xa + qt * 8192;
                if ((PMASK >> 1 >> t) & 1) { hv += 1u; xa += X_STEP; ++t; }
                post(hq, zero, hv);
                PULL4(xt);
                PULL4(xt + 4096);
                hv += 1u;
                xa += X_STEP;
            }
            STAGED(NKS);
            xa += (qt >> 1) * X_STEP + (qt & 1) * 4096;
#pragma unroll
            for (int t = KA; t < NITEMS; ++t) {
                PACE(AG_OWN / 2);
                if (qt == 0) TRACE(0, t);
                hv += 1u;
                post(hq, zero, hv);
                PULL4(xa + (t - KA) % NI * 2 * X_STEP + ((t - KA) / NI % 2 * 2 + (t - KA) / NAB) * 8192);
            }
            VM_WAIT(0);
            STAMP(6 + qt);
        } else if (wave == 7) {
            if (lane == 0) __hip_atomic_store(cctr, call, __ATOMIC_RELAXED, __HIP_MEMORY_SCOPE_AGENT);
            __builtin_amdgcn_fence(__ATOMIC_ACQUIRE, "");
            unsigned rot, staged = 0, npoll = 0;
            bool stored = false;
            t0 = rt();
            do rot = ld_ctr((const void*)rotw) - call * 8u; while (rot > 7u && rt() - t0 <= DEADLINE);
            lds_st(ctrl + C_ROT, rot | 8u);
            post(hint + 16, 0u, call * 256u + KP0);
            const char* sf = SRCWS(rot) + WS_SFLAG + (size_t)idx * FSTRIDE;
            STAMP(19);
            while (staged < (unsigned)NKS || !stored) {
                if (rot > 7u || ((++npoll & 31u) == 0u && (rfl(__hip_atomic_load(tmo, __ATOMIC_RELAXED, __HIP_MEMORY_SCOPE_AGENT)) == call
                                                           || rt() - t0 > DEADLINE))) {
                    __hip_atomic_store(tmo, call, __ATOMIC_RELAXED, __HIP_MEMORY_SCOPE_AGENT);
                    post(hint + 16, 0u, call * 256u);
                    VM_WAIT(0);
                    lds_st(ctrl + C_AVAIL, (unsigned)NKS);
                    break;
                }
                unsigned s2; bool p2;
                arrivals(sf, call, lane, s2, p2);
                if (npoll == 1) STAMP(27);
                if (s2 > staged) {
                    lds_st(ctrl + C_AVAIL, s2);
                    if (!staged) STAMP(17);
                    staged = s2;
                    if (s2 == NKS) STAMP(10);
                }
                if (p2 && !stored) { stored = true; post(hint + 16, 0u, call * 256u + NITEMS); }
                if (npoll < 64u && staged < (unsigned)NKS) __builtin_amdgcn_s_sleep(2); else __builtin_amdgcn_s_sleep(32);
            }
            STAMP(11);
        }
        return;
    }

    const int mt = idx & 1, nt = (idx - 2) >> 1;
    const int f0 = (17 * nt) >> 1;
    if (wave < 8) {
        f4 sv[4][2];
        const int g = b - 16;
        int sk = g >> 2;
#pragma unroll
        for (int e = 0; e < K1; ++e) sk += sk >= KP0 + e * KPS;
        const bool late = wave < 4 && sk >= AG_SEARLY;
        if (wave < 4) {
            if (wave == 0) STAMP(0);
            const float* s0 = p.x + (size_t)((g & 3) * 128 + wave * 32 + r8) * GK + sk * BK;
            if (late) {
                __builtin_amdgcn_s_barrier();
                for (int i = AG_SEARLY; i < sk; ++i) __builtin_amdgcn_s_sleep(AG_SSLOPE);
            } else if (AG_SDELAY) __builtin_amdgcn_s_sleep(AG_SDELAY);
#pragma unroll
            for (int i = 0; i < 4; ++i)
                asm volatile("global_load_dwordx4 %0, %2, off" SLD "\n\tglobal_load_dwordx4 %1, %2, off offset:16" SLD
                             : "=&v"(sv[i][0]), "=&v"(sv[i][1]) : "v"(s0 + (size_t)i * 8 * GK + (c8 ^ ((i & 1) * 4)) * 8) : "memory");
        }
        if (!late) __builtin_amdgcn_s_barrier();
        asm volatile("" ::: "memory");
        const unsigned call = rfl(lds_ld(ctrl + C_CALL));
        if (wave < 4) {
            char* d0 = myws + WS_XS + ((size_t)((call & 1u) * 2u + (unsigned)((g >> 1) & 1)) * NKS + (size_t)sk) * X_STEP
                       + (size_t)((g & 1) * X_PART + wave * 4096) + lo16;
            asm volatile("s_waitcnt vmcnt(0)" : "+v"(sv[0][0]), "+v"(sv[0][1]), "+v"(sv[1][0]), "+v"(sv[1][1]),
                         "+v"(sv[2][0]), "+v"(sv[2][1]), "+v"(sv[3][0]), "+v"(sv[3][1]) :: "memory");
            if (wave == 0) STAMP(24);
            u4 o[4];
#pragma unroll
            for (int i = 0; i < 4; i += 2)
                to_frags(pk8(sv[i][0], sv[i][1]), pk8(sv[i + 1][0], sv[i + 1][1]), img + (unsigned)(SCR_OFF + wave * 2048),
                         frg + (unsigned)(SCR_OFF + wave * 2048), o[i], o[i + 1]);
            asm volatile("s_waitcnt lgkmcnt(0)" : "+v"(o[0]), "+v"(o[1]), "+v"(o[2]), "+v"(o[3]) :: "memory");
#pragma unroll
            for (int i = 0; i < 4; ++i)
                st16_sys(d0 + i * 1024, o[i]);
            if (wave == 0) STAMP(25);
            unsigned n = 0;
            if (lane == 0) {
                asm volatile("s_waitcnt vmcnt(0)\n\tds_add_rtn_u32 %0, %1, %2\n\ts_waitcnt lgkmcnt(0)"
                             : "=&v"(n) : "v"(ctrl + C_STG), "v"(1u) : "memory");
                if (n == 3u) st4_sys(myws + WS_SFLAG + (size_t)((g >> 1) & 1) * FSTRIDE + sk * 8 + (g & 1) * 4, call);
            }
            if (wave == 0) { STAMP(16); STAMPE(23); }
        }
        const int set = wave >> 2, w4 = wave & 3;
        const bool dup = !(nt & 1) && nt != 14;
        f4 acc[FW][2];
#pragma unroll
        for (int i = 0; i < FW; ++i) {
            acc[i][0] = (f4){0.f, 0.f, 0.f, 0.f}; acc[i][1] = (f4){0.f, 0.f, 0.f, 0.f};
            asm volatile("" : "+v"(acc[i][0]), "+v"(acc[i][1]));
        }
        h8 a[RW], x[8];
#pragma unroll
        for (int i = 0; i < RW; ++i) asm volatile("" : "=v"(a[i]));
#pragma unroll
        for (int j = 0; j < 8; ++j) asm volatile("" : "=v"(x[j]));
        Pos P;
        Loop L;
        asm volatile("" : "=v"(P.f));
        P.w = lbase + (unsigned)((r16 >> 2) * W_BLK + q * 256 + (r16 & 3) * 16);
        P.n = lbase + (unsigned)(8 * W_BLK + q * 256 + (r16 & 3) * 64 + (r16 >> 2) * 16);
        P.x = lo16 + (unsigned)(set * X_PART + w4 * 2048);
        P.ldw = ctrl + C_LD - 4;
        P.rv = -4u; P.relw = ctrl + C_CONS + wave * 4; P.tab = myws + WS_TAB + (size_t)b * 512;
        unsigned vo = (unsigned)(q * NLOC + r16) * 16u, vn = (unsigned)(q * 4 * NLOC + r16) * 4u;
        asm volatile("" : "+v"(P.w), "+v"(P.n), "+v"(vn), "+v"(P.x), "+v"(P.rv), "+v"(P.ldw), "+v"(vo));
        WAIT_AV(1, 0);
        const unsigned rot = rfl(P.f.x);
        const int src = (p.srcs >> 4 * ((xr + rot) & 7)) & 7;
        L.xa = SRCWS(rot) + WS_XS + (size_t)((call & 1u) * 2u + (unsigned)mt) * NKS * X_STEP;
        L.xg = myws + WS_XG + (size_t)((((call & 1u) * NR + (unsigned)src) * 2u + (unsigned)mt) * K1) * X_STEP;
        L.set = set;
        const char* ol = (const char*)(p.out + (size_t)(src * MLOC + mt * BM + set * 128 + w4 * 16) * NLOC + f0 * 16);
        P.p[0] = L.xa; P.p[1] = L.xa + 8192;
        PIN(ol); PIN(P.relw); PIN(P.tab); PIN(L.xg); PIN(L.set); PIN(P.p[0]); PIN(P.p[1]);
#pragma unroll
        for (int k = 0; k < 4; ++k) XLD(x[k], k);
        L.t = 1u; P.need = 2u; L.xa += X_STEP; P.p[0] = L.xa; P.p[1] = L.xa + 8192;
        PIN(L.t); PIN(P.need); PIN(L.xa); PIN(P.p[0]); PIN(P.p[1]);
        WAIT_AV(0, 1);
        if (wave == 0) { STAMP(18); STAMPC(30); }
        FLAGS();
#pragma unroll
        for (int i = 0; i < FW; ++i) DSR(a[i], i < 8 ? P.w : P.n, wcol(i));
        __builtin_amdgcn_s_waitcnt(0x0070);
        SCHED_FENCE();
#pragma unroll 1
        do {
            k_step<0, 0>(acc, a, x, P, L);
            k_step<1, 0>(acc, a, x, P, L);
        } while (L.t <= (unsigned)KA);
        if (wave == 0) { STAMP(2); STAMPC(31); }
        P.need = (unsigned)(KA + 1 + NI * set);
        const char* xp = L.xa - X_STEP;
        unsigned pre = P.need - set;
#define TRY() do { const unsigned _p = AG_PAIR ? (rfl(P.f.w) >= P.need ? P.need + 1u : 0u) : rfl(P.f.w); item_x(x, P.x, xp, pre < P.need ? P.need : ~0u, _p); \
                   item_x(x + 4, P.x, xp + 2 * X_STEP, pre <= P.need ? P.need + 1u : ~0u, _p); pre = _p; } while (0)
#define SILU(r_, J_, keep_) silu4(acc[(r_) & 4][J_][(r_) & 3], acc[((r_) & 4) + 1][J_][(r_) & 3], acc[((r_) & 4) + 2][J_][(r_) & 3], \
                                  acc[((r_) & 4) + 3][J_][(r_) & 3], 60.f * LOG2E, c, keep_)
#define STORE(r_, o_) asm volatile("global_store_dwordx4 %0, %1, %2 offset:%3" AST \
                                   :: "v"(vo), "v"(o_), "s"(ol + ((r_) & 3) * NLOC * 4), "i"(((r_) & 4) * 64) : "memory")
#define PART(J_) do { \
        SCHED_FENCE(); \
        TTJ(J_, J_ * 6); \
        if (pre < P.need) WAIT_AV(P.need, 0); \
        item_x(x, P.x, xp, pre + 1u, P.need); \
        if (AG_PAIR >> !J_ & 1) { if (!J_) WAIT_AV(P.need, KA + 2); } else if (pre <= P.need) WAIT_AV(P.need + 1u, KA + 2); \
        P.p[0] = xp + 2 * X_STEP; P.p[1] = xp + 3 * X_STEP; PIN(P.p[0]); PIN(P.p[1]); \
        item_x(x + 4, P.x, P.p[0], pre, P.need); \
        if (!J_) _Pragma("unroll") for (int k = FW; k < RW; ++k) DSR(a[k], P.w, woff(k)); \
        if (J_) { if (pre < P.need) VM_WAIT(4); else VM_WAIT(8); } \
        if (J_ && AG_LPRIO && set) __builtin_amdgcn_s_setprio(AG_LPRIO); \
        SCHED_FENCE(); \
        part_items<J_ * NT, J_ * NT + NT>(acc, a, x, P); \
        TTJ(J_, J_ * 6 + 4); \
        if (!J_) { xp += 8192; PIN(xp); P.need += NAB; pre = 0u; TRY(); } \
        if (!J_ && AG_EPRIO && set) __builtin_amdgcn_s_setprio(AG_EPRIO); \
        if (wave == 4 && J_) { STAMP(3); STAMPC(33); } \
        const float c = __builtin_bit_cast(float, __builtin_bit_cast(unsigned, LOG2E) | rfl(P.f.z)); \
        asm volatile("s_nop 7\n\ts_nop 7" : "+v"(acc[FW - 3][J_]), "+v"(acc[FW - 2][J_]), "+v"(acc[FW - 1][J_])); \
        f4 o = acc[FW - 1][J_], od[4]; \
        _Pragma("unroll") for (int r = 0; r < 4; ++r) { o = SILU(r, J_, o); STORE(r, o); } \
        if (!dup) { \
            const f4 on = silu4(acc[FW - 1][J_][0], acc[FW - 1][J_][1], acc[FW - 1][J_][2], acc[FW - 1][J_][3], 60.f * LOG2E, c, o); \
            _Pragma("unroll") for (int r = 0; r < 4; ++r) \
                asm volatile("global_store_dword %0, %1, %2 offset:512" AST :: "v"(vn), "v"(on[r]), "s"(ol + r * NLOC * 4) : "memory"); \
        } \
        _Pragma("unroll") for (int r = 4; r < 8; ++r) {        \
            if (!J_ && (AG_TRYM >> r & 1)) FLAGS(); \
            o = od[r - 4] = SILU(r, J_, o); \
            if (J_) STORE(r, o); \
            if (!J_ && (AG_TRYM >> r & 1)) { asm volatile("s_waitcnt lgkmcnt(0)" : "+v"(P.f) :: "memory"); TRY(); } \
        } \
        if (!J_) _Pragma("unroll") for (int r = 4; r < 8; ++r) STORE(r, od[r - 4]); \
        if (!J_ && AG_EPRIO) __builtin_amdgcn_s_setprio(0); \
        TTJ(J_, J_ * 6 + 5); \
        asm volatile("s_nop 1" ::: "memory"); } while (0)
        PART(0);
        ADV(vo, 64 * NLOC * 4); ADV(vn, 64 * NLOC * 4);
        PART(1);
        if (wave == 4) { STAMP(4); STAMPC(34); }
        __builtin_amdgcn_endpgm();
    }
    if (wave == 10) {
        *(u2*)(myws + WS_TAB + (size_t)b * 512 + lane * 8) = (u2){2u * lane + 1u, 2u * lane + 2u};
        const unsigned cl = ld_ctr((const void*)cctr) + 1u;
        if (b == NWG - 8 && lane < 16)
            __hip_atomic_store((GAS unsigned*)(myws + WS_WFLAG + (size_t)lane * FSTRIDE + 256),
                               cl * 8u + (ROTX), __ATOMIC_RELAXED, __HIP_MEMORY_SCOPE_AGENT);
        if (lane == 0) __hip_atomic_store(cctr, cl, __ATOMIC_RELAXED, __HIP_MEMORY_SCOPE_AGENT);
        lds_st(ctrl + lane * 4, lane == C_CALL / 4 ? cl : 0u);
        LGKM0();
    }
    __builtin_amdgcn_s_barrier();
    asm volatile("" ::: "memory");
    const unsigned call = rfl(lds_ld(ctrl + C_CALL));
    if (wave == 10) {
        __builtin_amdgcn_fence(__ATOMIC_ACQUIRE, "");
        VM_WAIT(0);
        GAS unsigned* hq = (GAS unsigned*)(myws + WS_HINT + (size_t)(xr * 2 + mt) * FSTRIDE) + c8;
        GAS unsigned* hp = lane == 5 ? rotw : hq;
        GAS unsigned* wf0 = (GAS unsigned*)(myws + WS_WFLAG + (size_t)(f0 >> 3) * FSTRIDE) + lane;
        GAS unsigned* wf1 = (GAS unsigned*)(myws + WS_WFLAG + (size_t)((f0 + FW - 1) >> 3) * FSTRIDE) + lane;
        unsigned pub = 0, wav = 0, npoll = 0, rot = 8u;
        TRZERO(1);
        const unsigned long long t0 = rt();
        unsigned long long tall = 0;
        bool dead = false;
        while (pub < (unsigned)NITEMS || wav < (unsigned)NKS) {
            const unsigned hv = __hip_atomic_load(hp, __ATOMIC_RELAXED, __HIP_MEMORY_SCOPE_AGENT);
            bool adv = false;
            if (wav < (unsigned)NKS) {
                const unsigned w0 = __hip_atomic_load(wf0, __ATOMIC_RELAXED, __HIP_MEMORY_SCOPE_AGENT);
                const unsigned w1 = __hip_atomic_load(wf1, __ATOMIC_RELAXED, __HIP_MEMORY_SCOPE_AGENT);
                const unsigned long long notready = __ballot((int)(w0 - call) < 0 || (int)(w1 - call) < 0);
                const unsigned w2 = notready ? (unsigned)__builtin_ctzll(notready) : 64u;
                if (w2 > wav) { if (!wav) STAMP(29); wav = w2; lds_st(ctrl + C_WAV, wav); adv = true; }
            }
            const int d = (int)(hv - call * 256u);
            const unsigned av = (unsigned)row_min(c8 < 5 ? max(min(d, NITEMS), 0) : NITEMS);
            if (rot > 7u) {
                rot = (unsigned)__builtin_amdgcn_readlane((int)hv, 5) - call * 8u;
                if (rot <= 7u) { lds_st(ctrl + C_ROT, rot); hp = hq; }
            }
            if (av > pub && rot <= 7u) {
                asm volatile("ds_max_u32 %0, %1" :: "v"(lane ? ctrl + 128 + lane * 4 : ctrl + C_AVAIL), "v"(av) : "memory");
                pub = av;
                tall = 0;
                TRACE(1, av - 1);
            } else if (!adv && (++npoll & 63u) == 0u) {
                if (rt() - t0 > DEADLINE) { dead = true; break; }
                unsigned staged; bool stored;
                arrivals(SRCWS(rot) + WS_SFLAG + (size_t)mt * FSTRIDE, call, lane, staged, stored);
                if (staged == (unsigned)NKS && stored && rot <= 7u) {
                    if (tall == 0) tall = rt();
                    else if (rt() - tall > SILENCE) { pub = NITEMS; lds_st(ctrl + C_AVAIL, pub); }
                } else if (rfl(__hip_atomic_load(tmo, __ATOMIC_RELAXED, __HIP_MEMORY_SCOPE_AGENT)) == call) { dead = true; break; }
            }
            if (pub < (unsigned)AG_FINE) __builtin_amdgcn_s_sleep(AG_SLEEP_POLL); else __builtin_amdgcn_s_sleep(AG_SLEEP_FINE);
        }
        if (dead) {
            __hip_atomic_store(tmo, call, __ATOMIC_RELAXED, __HIP_MEMORY_SCOPE_AGENT);
            lds_st(ctrl + C_BAD, 0x7fc00000u);
        }
        lds_st(ctrl + C_WAV, (unsigned)NKS);
        lds_st(ctrl + C_AVAIL, (unsigned)NITEMS);
        TRDUMP(1);
    } else if (wave == 9) {
        constexpr int CSTEP = 240 / NCONV;
        const int cq = ((idx - 2) / CSTEP) * 8 + xr;
        if ((idx - 2) % CSTEP == 0 && cq < NCONV) {
            const unsigned so = (unsigned)(r8 * 8 * NLOC + c8 * 4) * 4u;
            const unsigned dof = (unsigned)((c8 >> 2) * 2048 + r8 * 256 + (c8 & 3) * 16);
            for (int i = AG_CFIRST; i < cq; i += 16) __builtin_amdgcn_s_sleep(AG_CDELAY);
            for (int u = cq; u < NKS * 16; u += NCONV) {
                const int ks = u >> 4, cn = u & 15;
                const float* sw = p.w + (size_t)ks * BK * NLOC + cn * 128;
                const char* dstw = myws + WS_WH + (size_t)ks * W_ROW + (size_t)cn * 16384;
                f4 v[4][8];
                if (u < 16) STAMP(12);
#pragma unroll
                for (int g = 0; g < 4; ++g)
#pragma unroll
                    for (int r = 0; r < 8; ++r)
                        asm volatile("global_load_dwordx4 %0, %1, %2 offset:%3" CLD : "=v"(v[g][r]) : "v"(so), "s"(sw + r * NLOC), "i"(g * 128) : "memory");
                u4 o[4][4];
#pragma unroll
                for (int g = 0; g < 4; ++g) {
                    asm volatile("s_waitcnt vmcnt(%8)" : "+v"(v[g][0]), "+v"(v[g][1]), "+v"(v[g][2]), "+v"(v[g][3]),
                                 "+v"(v[g][4]), "+v"(v[g][5]), "+v"(v[g][6]), "+v"(v[g][7]) : "i"(24 - 8 * g) : "memory");
#pragma unroll
                    for (int r = 0; r < 8; ++r) v[g][r] *= LOG2E;
#pragma unroll
                    for (int c = 0; c < 4; ++c)
                        o[g][c] = (u4){pk(v[g][0][c], v[g][1][c]), pk(v[g][2][c], v[g][3][c]), pk(v[g][4][c], v[g][5][c]), pk(v[g][6][c], v[g][7][c])};
                    SCHED_FENCE();
                }
                if (u < 16) STAMP(13);
#pragma unroll
                for (int g = 0; g < 4; ++g)
#pragma unroll
                    for (int c = 0; c < 4; ++c)
                        asm volatile("global_store_dwordx4 %0, %1, %2 offset:%3" CST "\n\ts_nop 1"
                                     :: "v"(dof), "v"(o[g][c]), "s"(dstw + g * 4096), "i"(c * 64) : "memory");
                VM_WAIT(0);
                if (lane == 0)
                    __hip_atomic_store((GAS unsigned*)(myws + WS_WFLAG + (size_t)cn * FSTRIDE + ks * 4), call,
                                       __ATOMIC_RELAXED, __HIP_MEMORY_SCOPE_AGENT);
                if (u < 16) STAMP(28);
                if (u < NCONV) STAMP(35);
            }
            STAMP(1);
        }
    } else {
        const char* tab = myws + WS_TAB + (size_t)b * 512;
        const char* wa = myws + WS_WH + (size_t)f0 * 2048;
        unsigned long long t0 = 0;
        int av = 0, cons = 0;
#pragma unroll 1
        for (int t = 0; t < NKS; ++t) {
            if (av <= t || cons <= t - NW) for (unsigned sp = 0;;) {
                u4 c, d; unsigned v;
                asm volatile("ds_read_b128 %0, %3\n\tds_read_b128 %1, %3 offset:16\n\tds_read_b32 %2, %3 offset:%4\n\t"
                             "s_waitcnt lgkmcnt(0)" : "=&v"(c), "=&v"(d), "=&v"(v) : "v"(ctrl + C_CONS), "i"(C_WAV - C_CONS) : "memory");
                av = (int)rfl(v); cons = (int)min8(c, d);
                if ((av > t && cons > t - NW) || SPIN_OUT(sp, t0)) break;
                __builtin_amdgcn_s_sleep(AG_LSLEEP);
            }
            TRACE(2, t);
            if (t == 0) STAMP(32);
            const unsigned sb = lbase + (unsigned)(t & (NW - 1)) * W_STEP;
#define WLOADS(P_) _Pragma("unroll") for (int i = 0; i < FW; i += 2) { \
                asm volatile("s_mov_b32 m0, %0\n\ts_waitcnt vmcnt(58)\n\t" \
                             "global_load_lds_dwordx4 %1, %2" P_ "\n\tglobal_load_lds_dwordx4 %1, %2 offset:1024" P_ \
                             :: "s"(sb + i * W_BLK), "v"(lo16), "s"(wa + i * 2048) : "memory"); \
                if (i + 1 < FW) asm volatile("s_mov_b32 m0, %0\n\ts_nop 0\n\t" \
                             "global_load_lds_dwordx4 %1, %2 offset:2048" P_ "\n\tglobal_load_lds_dwordx4 %1, %2 offset:3072" P_ \
                             :: "s"(sb + i * W_BLK + 64), "v"(lo16), "s"(wa + i * 2048) : "memory"); \
            }
            if (mt == 0 && t >= AG_WNT0 && t < AG_WNT) { WLOADS(" nt") } else
            { WLOADS(WLD) }
            asm volatile("s_mov_b32 m0, %0\n\ts_mov_b64 exec, 1\n\tglobal_load_lds_dword %1, %2\n\ts_mov_b64 exec, -1"
                         :: "s"(ctrl + C_LD), "v"(lo16), "s"(tab + t * 4) : "memory");
            wa += W_ROW;
        }
        VM_WAIT(0);
    }
}

}

extern "C" void kernel_launch(int rank, int nranks, ncclComm_t comm,
                              hipStream_t stream,
                              void* const* d_in, const int* in_sizes, int n_in,
                              void* d_out, int out_size,
                              const int* in_dtypes, int out_dtype,
                              void* const* d_out_all,
                              void* const* d_ws_all, size_t ws_bytes) {
    (void)comm; (void)in_dtypes; (void)out_dtype; (void)d_out_all;
    if (nranks != NR || n_in < 2 || (unsigned)rank >= (unsigned)NR || ws_bytes < WS_NEED ||
        in_sizes[0] != MLOC * GK || in_sizes[1] != GK * NLOC || out_size != GM * NLOC) {
        fprintf(stderr, "ag_gemm: unexpected problem shape\n");
        return;
    }
    static bool attr_set = false;
    if (!attr_set) {
        HIP_CHECK(hipFuncSetAttribute((const void*)ag_gemm_kernel,
                                      hipFuncAttributeMaxDynamicSharedMemorySize, LDS_BYTES));
        attr_set = true;
    }
    Params p;
    p.x = (const float*)d_in[0];
    p.w = (const float*)d_in[1];
    p.out = (float*)d_out;
    p.myws = (char*)d_ws_all[rank];
    p.srcs = 0;
    {
        int order[NR], n = 0;
        const unsigned half = 0xb2;
        for (int far = 0; far < 2; ++far)
            for (int r = 0; r < NR; ++r) { const int s = (rank + r) & 7; if ((int)((half >> s ^ half >> rank) & 1) == far) order[n++] = s; }
        for (int r = 0; r < NR; ++r) { p.ws[r] = (char*)d_ws_all[order[r]]; p.srcs |= (unsigned)order[r] << 4 * r; }
    }
    for (int r = 0; r < 2 * NR; ++r) p.xg[r] = (char*)d_ws_all[(rank + 1 + r) & 7] + WS_XG + (size_t)rank * 2 * K1 * X_STEP;
    hipLaunchKernelGGL(ag_gemm_kernel, dim3(NWG), dim3(NTHREADS), LDS_BYTES, stream, p);
    HIP_CHECK(hipGetLastError());
}
```

```cpp
#include <hip/hip_runtime.h>
#include <rccl/rccl.h>
#include <cstdio>
#include <cstddef>

#define HIP_CHECK(expr) do { hipError_t _e = (expr); if (_e != hipSuccess) { \
    fprintf(stderr, "HIP error %s at %s:%d\n", hipGetErrorString(_e), __FILE__, __LINE__); \
    return; } } while (0)

namespace {

#define GAS __attribute__((address_space(1)))
#define LAS __attribute__((address_space(3)))
typedef _Float16 h8 __attribute__((ext_vector_type(8)));
typedef _Float16 h2 __attribute__((ext_vector_type(2)));
typedef float f4 __attribute__((ext_vector_type(4)));
typedef unsigned u4 __attribute__((ext_vector_type(4)));
typedef unsigned u2 __attribute__((ext_vector_type(2)));

#define AG_PWIN 16
#define AG_WIN 8
#define AG_EFENCE 0
#define AG_HSTEP 14
#define AG_HN 4
#define AG_CFIRST -17
#define AG_CDELAY 20
#define AG_WNT 32
#define AG_WNT0 8
#define AG_ROFF 0
#define ROTX ((__builtin_amdgcn_s_getreg((3 << 11) | 20) + AG_ROFF) & 7u)
#define AG_UNC0 9
#define AG_UNC1 9
#define AG_NOCHK 3
#define AG_PAIR 0
#define AG_PUB 0
#define AG_PUBG 0
#define AG_RELG 34
#define AG_MAP 1
#define AG_SLEEP_POLL 4
#define AG_SLEEP_FINE 1
#define AG_FINE 48
#define AG_NCONV 120
#define AG_KP0 30
#define AG_KPS 6
#define AG_SDELAY 8
#define AG_SEARLY 12
#define AG_SSLOPE 1
#define AG_LSLEEP 1
#define AG_EARLY0 2
#define AG_EARLY1 2
#define AG_EARLY2 0
#define AG_E2T 4
#define AG_EPRIO 0
#define AG_LPRIO 0
#define AG_RW 9
#define AG_WG 3
#define AG_NAB 8
#define AG_OWN 85
#define AG_CA 2
#define AG_AST 2
#define AG_TRYM 0x90
#define AST " nt"

#define CST " sc1 nt"
#define WLD ""
#define CLD " nt"
#define XLP ""
#define PLD ""
#define SST ""
#define PST ""
#define SLD " nt"
constexpr int NR = 8;
constexpr int GM = 4096, GK = 4096, NLOC = 2048, MLOC = 512;
constexpr int BM = 256, BK = 64;
constexpr int FW = 9;
constexpr int NKS = GK / BK;
constexpr int K1 = 4;
constexpr int KP0 = AG_KP0, KPS = AG_KPS;
constexpr unsigned long long PMASK = (1ull | 1ull << KPS | 1ull << 2 * KPS | 1ull << 3 * KPS) << KP0;
constexpr int NAB = AG_NAB, NI = NAB / 2;
constexpr int KA = NKS - NAB;
constexpr int NITEMS = KA + 4 * NI;
constexpr int X_STEP = BM * BK * 2;
constexpr int X_PART = X_STEP / 2;
constexpr int W_BLK = 2048 + 64;
constexpr int W_STEP = FW * W_BLK - 64;
constexpr int W_ROW = NLOC * BK * 2;
constexpr int NW = 8;
constexpr int SCR_OFF = NW * W_STEP;
constexpr int CTRL_OFF = SCR_OFF + 8192;
constexpr int LDS_BYTES = CTRL_OFF + 512;
constexpr int NWG = 256, NTHREADS = 704;
static_assert(LDS_BYTES <= 160 * 1024 && NAB <= NW && NAB % 2 == 0 && NAB >= 4 && NKS % NW == 0, "layout");
static_assert((NKS - K1) * 4 == NWG - 16 && KP0 > 1 && KPS > 1 && KP0 + 3 * KPS < KA - 1, "layout");
constexpr int C_CONS = 0;
constexpr int C_WAV = 68;
constexpr int C_STG = 72;
constexpr int C_CALL = 76;
constexpr int C_ROT = 80;
constexpr int C_LD = 84;
constexpr int C_BAD = 88;
constexpr int C_AVAIL = 92;
constexpr int NCONV = AG_NCONV;
constexpr float LOG2E = 1.4426950408889634f;

constexpr size_t FSTRIDE = 4096 + 256;
constexpr size_t WS_CALL = 0;
constexpr size_t WS_TMO = 49152;
constexpr size_t WS_SFLAG = 53248;
constexpr size_t WS_HINT = 65536;
constexpr size_t WS_WFLAG = 196608;
constexpr size_t WS_TAB = 327680;
constexpr size_t WS_DBG = 524288;
constexpr size_t WS_WH = 1ull << 20;
constexpr size_t WS_XS = 32ull << 20;
constexpr size_t WS_XG = 48ull << 20;
constexpr size_t WS_NEED = 64ull << 20;

constexpr unsigned long long DEADLINE = 100000000ull;
constexpr unsigned long long SILENCE = 20000ull;

struct Params {
    const float* x;
    char* myws;
    const float* w;
    float* out;
    char* ws[NR];
    char* xg[2 * NR];
    unsigned srcs;
};
#define AG_ROTATED
typedef const char* __attribute__((address_space(4))) const* KPTR;

__device__ __forceinline__ unsigned long long rt() { return __builtin_amdgcn_s_memrealtime(); }

__device__ __forceinline__ void st16_sys(void* p, u4 v) {
    asm volatile("global_store_dwordx4 %0, %1, off sc0 sc1" SST "\n\ts_nop 1" :: "v"(p), "v"(v) : "memory");
}
__device__ __forceinline__ void st16_dev(void* p, u4 v) {
    asm volatile("global_store_dwordx4 %0, %1, off sc1\n\ts_nop 1" :: "v"(p), "v"(v) : "memory");
}
__device__ __forceinline__ void st4_sys(void* p, unsigned v) {
    asm volatile("global_store_dword %0, %1, off sc0 sc1\n\ts_nop 1" :: "v"(p), "v"(v) : "memory");
}
__device__ __forceinline__ void post(const char* p, unsigned zero, unsigned v) {
    asm volatile("s_mov_b64 exec, 1\n\tglobal_store_dword %0, %1, %2\n\ts_mov_b64 exec, -1" :: "v"(zero), "v"(v), "s"(p) : "memory");
}
__device__ __forceinline__ void lds_st(unsigned addr, unsigned v) {
    asm volatile("ds_write_b32 %0, %1" :: "v"(addr), "v"(v) : "memory");
}
__device__ __forceinline__ unsigned lds_ld(unsigned addr) {
    unsigned v;
    asm volatile("ds_read_b32 %0, %1\n\ts_waitcnt lgkmcnt(0)" : "=&v"(v) : "v"(addr) : "memory");
    return v;
}
__device__ __forceinline__ unsigned ld_ctr(const void* p) {
    unsigned v;
    asm volatile("global_load_dword %0, %1, off sc1\n\ts_waitcnt vmcnt(0)" : "=&v"(v) : "v"(p) : "memory");
    return (unsigned)__builtin_amdgcn_readfirstlane(v);
}
__device__ __forceinline__ unsigned pk(float a, float b) {
    h2 h; h.x = (_Float16)a; h.y = (_Float16)b;
    return __builtin_bit_cast(unsigned, h);
}
__device__ __forceinline__ u4 pk8(f4 a, f4 b) { return (u4){pk(a[0], a[1]), pk(a[2], a[3]), pk(b[0], b[1]), pk(b[2], b[3])}; }
__device__ __forceinline__ unsigned umin(unsigned a, unsigned b) { return a < b ? a : b; }
__device__ __forceinline__ unsigned rfl(unsigned v) { return (unsigned)__builtin_amdgcn_readfirstlane((int)v); }
__device__ __forceinline__ unsigned min8(u4 c, u4 d) {
    return rfl(umin(umin(umin(c.x, c.y), umin(c.z, c.w)), umin(umin(d.x, d.y), umin(d.z, d.w))));
}
__device__ __forceinline__ int row_min(int u) {
    u = min(u, __builtin_amdgcn_update_dpp(0, u, 0x121, 15, 15, false));
    u = min(u, __builtin_amdgcn_update_dpp(0, u, 0x122, 15, 15, false));
    u = min(u, __builtin_amdgcn_update_dpp(0, u, 0x124, 15, 15, false));
    u = min(u, __builtin_amdgcn_update_dpp(0, u, 0x128, 15, 15, false));
    return __builtin_amdgcn_readfirstlane(u);
}
__device__ __forceinline__ void arrivals(const char* sf, unsigned call, int lane, unsigned& staged, bool& stored) {
    u2 sv; unsigned pv;
    asm volatile("global_load_dwordx2 %0, %2, %4 sc0 sc1\n\tglobal_load_dword %1, %3, %4 offset:512 sc0 sc1\n\ts_waitcnt vmcnt(0)"
                 : "=&v"(sv), "=&v"(pv) : "v"(lane * 8), "v"((lane & 15) * 4), "s"(sf) : "memory");
    const unsigned long long miss = __ballot((int)(sv.x - call) < 0 || (int)(sv.y - call) < 0) & ~PMASK;
    staged = miss ? (unsigned)__builtin_ctzll(miss) : (unsigned)NKS;
    stored = __ballot((int)(pv - call) < 0) == 0;
}
#define VM_WAIT(n) asm volatile("s_waitcnt vmcnt(" #n ")" ::: "memory")
#define LGKM0() asm volatile("s_waitcnt lgkmcnt(0)" ::: "memory")
#define SCHED_FENCE() __builtin_amdgcn_sched_barrier(0)
#define SPIN_OUT(sp_, t0_) ((((++(sp_)) & 1023u) == 0u) && ((t0_) == 0 ? ((t0_) = rt(), false) : ((unsigned)(rt() - (t0_)) > 2u * (unsigned)DEADLINE)))

#define PB ((blockIdx.x & ~7u) | (__builtin_amdgcn_s_getreg((3 << 11) | 20) & 7u))
#define TRACE(k_, t_) do {} while (0)
#define TRDUMP(k_) do {} while (0)
#define TRZERO(k_) do {} while (0)
#define STAMP(i) do {} while (0)
#define STAMPC(i) do {} while (0)

#define DSR(dst_, addr_, off_) asm volatile("ds_read_b128 %0, %1 offset:%2" : "+v"(dst_) : "v"(addr_), "i"(off_) : "memory")

__device__ __forceinline__ f4 silu4(float z0, float z1, float z2, float z3, float lim, float c, f4 keep) {
    float o0, o1, o2, o3;
    asm volatile("v_med3_f32 %0, -%4, %8, -%8\n\tv_med3_f32 %1, -%5, %8, -%8\n\tv_med3_f32 %2, -%6, %8, -%8\n\tv_med3_f32 %3, -%7, %8, -%8\n\t"
                 "v_exp_f32 %0, %0\n\tv_exp_f32 %1, %1\n\tv_exp_f32 %2, %2\n\tv_exp_f32 %3, %3\n\t"
                 "v_fma_f32 %0, %0, %9, %9\n\tv_fma_f32 %1, %1, %9, %9\n\tv_fma_f32 %2, %2, %9, %9\n\tv_fma_f32 %3, %3, %9, %9\n\t"
                 "v_rcp_f32 %0, %0\n\tv_rcp_f32 %1, %1\n\tv_rcp_f32 %2, %2\n\tv_rcp_f32 %3, %3\n\t"
                 "v_mul_f32 %0, %4, %0\n\tv_mul_f32 %1, %5, %1\n\tv_mul_f32 %2, %6, %2\n\tv_mul_f32 %3, %7, %3"
                 : "=&v"(o0), "=&v"(o1), "=&v"(o2), "=&v"(o3) : "v"(z0), "v"(z1), "v"(z2), "v"(z3), "s"(lim), "s"(c), "v"(keep));
    return (f4){o0, o1, o2, o3};
}
#define REL() asm volatile("s_mov_b32 m0, %0\n\ts_mov_b64 exec, 1\n\tglobal_load_lds_dword %1, %2\n\ts_mov_b64 exec, -1" \
                           :: "s"(P.relw), "v"(P.rv), "s"(P.tab) : "memory")
#define TT(i_) do {} while (0)
struct Pos { unsigned w, n, x, rv, need, relw, ldw; const char* tab; const char* p[2]; u4 f; };
#define TTJ(j_, i_) TT(i_)
#define ADV(v_, d_) do { v_ += d_; asm volatile("" : "+v"(v_)); } while (0)
#define PIN(s_) asm volatile("" : "+s"(s_))
#define FLAGS() asm volatile("ds_read_b128 %0, %1" : "=v"(P.f) : "v"(P.ldw) : "memory")
#define WAIT_AV(nx_, nw_) do { unsigned long long _t0 = 0; unsigned _sp = 0; \
        for (;;) { asm volatile("ds_read_b128 %0, %1\n\ts_waitcnt lgkmcnt(0)" : "+v"(P.f) : "v"(P.ldw) : "memory"); \
            if ((int)((rfl(P.f.w) - (unsigned)(nx_)) | (rfl(P.f.y) - (unsigned)(nw_))) >= 0) break; \
            if (SPIN_OUT(_sp, _t0)) break; \
            __builtin_amdgcn_s_sleep(1); } } while (0)
#define CHECK_AV(nx_, nw_) do { if (__builtin_expect((int)((rfl(P.f.w) - (unsigned)(nx_)) | (rfl(P.f.y) - (unsigned)(nw_))) < 0, 0)) \
            WAIT_AV(nx_, nw_); SCHED_FENCE(); } while (0)
#define MFMA(c_, x_, a_) asm volatile("v_mfma_f32_16x16x32_f16 %0, %1, %2, %0" : "+v"(c_) : "v"(x_), "v"(a_))
#define MFMA_R(c_, x_, a_, ad_, off_) asm volatile("v_mfma_f32_16x16x32_f16 %0, %2, %1, %0\n\tds_read_b128 %1, %3 offset:%4" \
                                                   : "+v"(c_), "+v"(a_) : "v"(x_), "v"(ad_), "i"(off_) : "memory")
#define XLD(x_, k_) asm volatile("global_load_dwordx4 %0, %1, %2 offset:%3" : "+v"(x_) : "v"(P.x), "s"(P.p[(k_) & 1]), "i"(((k_) & 2) * 512) : "memory")
constexpr int wcol(int i) { return i < 8 ? (i >> 2) * 4 * W_BLK + (i & 3) * 64 : 0; }

constexpr int RW = AG_RW, G_A = AG_CA;
struct Loop { const char* xa; const char* xg; unsigned t, set; };
template <int B, int G>
__device__ __forceinline__ void k_step(f4 (&acc)[FW][2], h8 (&a)[RW], h8 (&x)[8], Pos& P, Loop& L) {
    if constexpr (G < 36) {
        constexpr int h = G / 18, i = G % 18 / 2;
        if (G % 18 == 0) asm volatile("s_waitcnt vmcnt(%0) lgkmcnt(7)" :: "i"(h ? 5 : 3) : "memory");
        else if (G == G_A) asm volatile("s_waitcnt lgkmcnt(%1)" : "+v"(P.f) : "i"(3 + G_A / 2) : "memory");
        else if (G == AG_RELG && G <= 30) asm volatile("s_waitcnt lgkmcnt(%0)" :: "i"((G - 18) / 2) : "memory");
        else if (G % 18 % 4 == 0 && !(G > AG_RELG && G > 18)) asm volatile("s_waitcnt lgkmcnt(7)" ::: "memory");
        if (G & 1) MFMA_R(acc[i][1], x[B * 4 + h * 2 + 1], a[i], i < 8 ? P.w : P.n, (h ? 1 - B : B) * W_STEP + (1 - h) * 1024 + wcol(i));
        else MFMA(acc[i][0], x[B * 4 + h * 2], a[i]);
        SCHED_FENCE();
        if (G == G_A) CHECK_AV(P.need, L.t + 1u);
        if (G > G_A && G <= G_A + 8 && !(G & 1)) XLD(x[4 - B * 4 + (G - G_A) / 2 - 1], (G - G_A) / 2 - 1);
        if (B && G == 18) { const int d = (L.t & (NW - 1)) ? 2 * W_STEP : (2 - NW) * W_STEP; ADV(P.w, d); ADV(P.n, d); }
        if (G == 20) { L.t += 1u; PIN(L.t); L.xa += X_STEP; PIN(L.xa); }
        if (G == 22) {
            const bool st = (PMASK >> L.t) & 1;
            P.p[0] = st ? L.xg : L.xa; PIN(P.p[0]);
            L.xg += st ? X_STEP : 0; PIN(L.xg);
        }
        if (G == 24) {
            P.need = L.t + 1u;
            if (B) P.p[1] = P.p[0] + 8192;
            else {
                const bool last = L.t == (unsigned)KA;
                P.p[0] -= last && L.set ? X_STEP : 0;
                P.p[1] = P.p[0] + (last && !L.set ? X_STEP : 8192);
                P.need = last && L.set ? 0u : P.need;
                PIN(P.p[0]);
            }
            PIN(P.p[1]); PIN(P.need);
        }
        if (G == 30) FLAGS();
        if (G == (AG_RELG < 34 ? 14 : 32)) ADV(P.rv, 4u);
        if (G == AG_RELG) REL();
        SCHED_FENCE();
        k_step<B, G + 1>(acc, a, x, P, L);
    }
}
constexpr int NT = 36 * NI, WG = AG_WG, G_C = 35 - RW;
constexpr bool flags(int t) { return t % 36 == (t % NT < NT - 36 ? G_C - 9 : 26); }
constexpr int xfetch(int t) {
    for (int k = 0; k < 4; ++k) if (t % 36 == (G_C + 1 + k > 9 * k + 8 ? G_C + 1 + k : 9 * k + 8)) return t % NT < NT - 72 ? k : -1;
    return -1;
}
constexpr bool pub(int t) { return AG_PUB && t % 36 == AG_PUBG && t % NT / 36 + AG_PUB >= NI; }
struct Waits { int l[2 * NT + 1], v[NT / 9]; };
constexpr Waits waits() {
    Waits r{};
    int pw[2 * NT + 16] = {}, pf = 0, c = 0, done = -1, xs = 8;
    for (int k = 0; k < RW; ++k) pw[k] = c++;
    for (int t = 0; t <= 2 * NT; ++t) {
        int need = t % WG || t == 2 * NT ? -1 : pw[t + WG - 1];
        if (t % 36 == G_C || (t && t % NT == 0)) need = pf > need ? pf : need;
        r.l[t] = need <= done ? -1 : c - 1 - need < 15 ? c - 1 - need : 15;
        done = need > done ? need : done;
        if (t + RW < 2 * NT) pw[t + RW] = c++;
        if (flags(t)) pf = c++;
        if (pub(t)) ++c;
        if (t < NT && t % 9 == 0) r.v[t / 9] = xs - 1 - t / 9;
        if (xfetch(t) >= 0) ++xs;
    }
    return r;
}
static_assert(RW >= FW && RW > WG && NT % WG == 0, "parts");
constexpr int woff(int k) { return (k % 36 / 9 & 1) * W_STEP + k % 36 / 18 * 1024 + wcol(k % 9); }
template <int T, int E>
__device__ __forceinline__ void part_items(f4 (&acc)[FW][2], h8 (&a)[RW], h8 (&x)[8], Pos& P) {
    constexpr int left = waits().l[T];
    if constexpr (left >= 0 && (T > E - NT || T == 0)) asm volatile("s_waitcnt lgkmcnt(%0)" :: "i"(left) : "memory");
    if (T == E || T % 36 == G_C) asm volatile("" : "+v"(P.f));
    if constexpr (T < E) {
        constexpr int J = T / NT, N = T % NT / 36, G = T % 36, K = T + RW, X = (N & 1) * 4;
        if (G % 9 == 0 && (!J || N)) asm volatile("s_waitcnt vmcnt(%0)" :: "i"(waits().v[T % NT / 9]) : "memory");
        if constexpr (K < 2 * NT) MFMA_R(acc[T % FW][J], x[X + G / 9], a[T % RW], K % 9 < 8 ? P.w : P.n, woff(K));
        else MFMA(acc[T % FW][J], x[X + G / 9], a[T % RW]);
        SCHED_FENCE();
        if (G == 0 && N >= NI - 3) TT(J * 6 + 4 + N - NI);
        if (K < 2 * NT && K % 36 >= 34) {
            constexpr int d = K % NT / 36 == NI - 1 ? (2 - NAB) * W_STEP : 2 * W_STEP;
            if (K % 36 == 34) ADV(P.w, d); else ADV(P.n, d);
        }
        if (flags(T)) FLAGS();
        if (pub(T)) asm volatile("v_mov_b32 %0, %2\n\ts_mov_b64 exec, 1\n\tds_max_u32 %1, %0 offset:12\n\ts_mov_b64 exec, -1"
                                 : "+v"(P.rv) : "v"(P.ldw), "s"(P.need + (N + AG_PUB)) : "memory");
        if (G == G_C && N < (J ? NI - 2 : NI - 1) && !(J && (AG_NOCHK & 1)))
            CHECK_AV(N < NI - 2 && !(AG_NOCHK & 2) ? P.need + (N + 2) : 0u, J ? 0 : KA + 2 * N + 4);
        if (G == G_C - 2 && N < NI - 2) { P.p[0] += 2 * X_STEP; PIN(P.p[0]); P.p[1] += 2 * X_STEP; PIN(P.p[1]); }
        if constexpr (xfetch(T) >= 0) XLD(x[X + xfetch(T)], xfetch(T));
        SCHED_FENCE();
        part_items<T + 1, E>(acc, a, x, P);
    }
}
__device__ __forceinline__ void item_x(h8* x, unsigned vx, const char* p, unsigned lo, unsigned hi) {
    asm volatile("s_cmp_gt_u32 %7, %8\n\ts_cbranch_scc1 1f\n\t"
                 "global_load_dwordx4 %0, %4, %5\n\tglobal_load_dwordx4 %1, %4, %6\n\t"
                 "global_load_dwordx4 %2, %4, %5 offset:1024\n\tglobal_load_dwordx4 %3, %4, %6 offset:1024\n1:"
                 : "+v"(x[0]), "+v"(x[1]), "+v"(x[2]), "+v"(x[3])
                 : "v"(vx), "s"(p), "s"(p + X_STEP), "s"(lo), "s"(hi) : "memory", "scc");
}
__device__ __forceinline__ void to_frags(u4 o0, u4 o1, unsigned wr, unsigned rd, u4& f0, u4& f1) {
    asm volatile("ds_write_b128 %2, %4\n\tds_write_b128 %2, %5 offset:1024\n\tds_read_b128 %0, %3\n\tds_read_b128 %1, %6"
                 : "=&v"(f0), "=&v"(f1) : "v"(wr), "v"(rd), "v"(o0), "v"(o1), "v"(rd ^ 64u) : "memory");
}

__global__ void __launch_bounds__(NTHREADS, 3) ag_gemm_kernel(Params p) {
    extern __shared__ __attribute__((aligned(16))) unsigned char smem[];
#define STAMPE(i) do {} while (0)
    LAS unsigned char* lds = (LAS unsigned char*)smem;
    const int tid = threadIdx.x;
    const int wave = __builtin_amdgcn_readfirstlane(tid >> 6);
    const int lane = tid & 63;
    const int b = blockIdx.x;
    const int xr = b & 7, idx = b >> 3;
    char* myws = p.myws;
    GAS unsigned* cctr = (GAS unsigned*)(myws + WS_CALL + (size_t)b * 128);
    GAS unsigned* tmo = (GAS unsigned*)(myws + WS_TMO);
    const unsigned lbase = (unsigned)(size_t)lds;
    const unsigned ctrl = lbase + CTRL_OFF;
    const unsigned lo16 = (unsigned)lane * 16u;
    const int r8 = lane >> 3, c8 = lane & 7;
    const int r16 = lane & 15, q = lane >> 4;
    const unsigned img = lbase + (unsigned)(r8 * 128 + (c8 ^ (r8 >> 1)) * 16);
    const unsigned frg = lbase + (unsigned)(r16 * 128 + (q ^ (r16 >> 1)) * 16);
    const char* __attribute__((address_space(4))) const* ka = (KPTR)__builtin_amdgcn_kernarg_segment_ptr();

#define AG_DRY 0
    if (__builtin_expect(idx < 2 && wave < 2 + AG_DRY, 1)) {
        if (AG_DRY) asm volatile("s_cmp_lt_u32 %0, 2\n\ts_cselect_b64 exec, -1, 0" :: "s"(wave) : "memory", "scc");
        const int pid = (xr * 2 + idx) * 2 + wave;
        f4 v[K1][4];
        unsigned cv;
        {
            const unsigned s0 = (unsigned)((pid * 16 + r8) * GK + c8 * 8) * 4u;
            const unsigned s1 = (unsigned)((pid * 16 + 8 + r8) * GK + (c8 ^ 4) * 8) * 4u;
#pragma unroll
            for (int e = 0; e < K1; ++e) {
                asm volatile("global_load_dwordx4 %0, %4, %6" PLD "\n\tglobal_load_dwordx4 %1, %4, %6 offset:16" PLD "\n\t"
                             "global_load_dwordx4 %2, %5, %6" PLD "\n\tglobal_load_dwordx4 %3, %5, %6 offset:16" PLD
                             : "=&v"(v[e][0]), "=&v"(v[e][1]), "=&v"(v[e][2]), "=&v"(v[e][3])
                             : "v"(s0), "v"(s1), "s"(p.x + (KP0 + e * KPS) * BK) : "memory");
                if (e == 0) asm volatile("global_load_dword %0, %1, %2 sc1" : "=&v"(cv) : "v"(b * 128), "s"(myws + WS_CALL) : "memory");
            }
        }
        SCHED_FENCE();
        const char* dst[NR];
#pragma unroll
        for (int j = 0; j < NR; ++j) dst[j] = ka[offsetof(Params, xg) / 8 + ((xr * 2 + idx + 4 * wave) & 7) + j];
        unsigned vo = lo16 + (unsigned)((pid >> 4) * K1 * X_STEP + (pid & 15) * 2048), call = 0;
        char* flag = myws + WS_SFLAG + (size_t)(pid >> 4) * FSTRIDE + 512 + (pid & 15) * 4;
        constexpr int W = AG_PWIN;
        static_assert(W >= 2 && W <= 32 && W % 2 == 0, "window");
        u4 o[K1][2];
#pragma unroll
        for (int e = 0; e < K1; ++e) {
            if (e < 2) {
#pragma unroll
                for (int k = e; k < (e ? K1 : 1); ++k) {
                    asm volatile("s_waitcnt vmcnt(%5)" : "+v"(cv), "+v"(v[k][0]), "+v"(v[k][1]), "+v"(v[k][2]), "+v"(v[k][3])
                                 : "i"(4 * (K1 - 1 - k) + e * 2 * NR) : "memory");
                    to_frags(pk8(v[k][0], v[k][1]), pk8(v[k][2], v[k][3]), img + 32768u + wave * 2048u, frg + 32768u + wave * 2048u, o[k][0], o[k][1]);
                }
            }
            if (e == 0) {
                call = rfl(cv) + 1u;
                vo += (call & 1u) * (unsigned)(NR * 2 * K1 * X_STEP);
            }
            if (e == 1) __builtin_amdgcn_s_barrier();
            asm volatile("s_waitcnt lgkmcnt(%2)" : "+v"(o[e][0]), "+v"(o[e][1]) : "i"(e ? 4 * (K1 - 1 - e) : 0) : "memory");
            const unsigned ve = vo + (unsigned)(e * X_STEP);
#pragma unroll
            for (int j = 0; j < NR; ++j) {
                if (e) asm volatile("s_waitcnt vmcnt(%0)" :: "i"(W - 2) : "memory");
                asm volatile("global_store_dwordx4 %0, %1, %3 sc0 sc1" PST "\n\tglobal_store_dwordx4 %0, %2, %3 offset:1024 sc0 sc1" PST "\n\ts_nop 1"
                             :: "v"(ve), "v"(o[e][0]), "v"(o[e][1]), "s"(dst[j]) : "memory");
            }
        }
        VM_WAIT(0);
        if (lane == 0) st4_sys(flag, call);
        if (wave < 2) STAMP(14 + wave);
        if (wave == 0) STAMPE(22);
        __builtin_amdgcn_endpgm();
    }
    GAS unsigned* rotw = (GAS unsigned*)(myws + WS_WFLAG + (size_t)(idx & 15) * FSTRIDE + 256);
#define SRCWS(rot_) ka[offsetof(Params, ws) / 8 + ((xr + (rot_)) & 7)]
    if (idx < 2) {
        const char* hint = myws + WS_HINT + (size_t)(xr * 2 + idx) * FSTRIDE;
        if (wave == 7) lds_st(ctrl + lane * 4, 0u);
        const unsigned call = ld_ctr((const void*)cctr) + 1u;
        LGKM0();
        __builtin_amdgcn_s_barrier();
        asm volatile("" ::: "memory");
        unsigned long long t0 = 0;
        if (wave >= 3 && wave < 7) {
            const int qt = wave - 3;
            const char* hq = hint + qt * 4;
            unsigned rot;
            for (unsigned sp = 0;;) {
                rot = rfl(lds_ld(ctrl + C_ROT));
                if (rot || SPIN_OUT(sp, t0)) break;
                __builtin_amdgcn_s_sleep(1);
            }
            const char* srcws = SRCWS(rot);
            const char* xa = srcws + WS_XS + (size_t)((call & 1u) * 2u + (unsigned)idx) * NKS * X_STEP;
            unsigned hv = call * 256u + 1u, zero = 0u;
            asm volatile("s_mov_b32 m0, %2" : "+v"(hv), "+v"(zero) : "s"(lbase + qt * 4096) : "memory");
            int av = 0;
#define STAGED(n_) do { if (av < (n_)) for (unsigned _sp = 0;;) { av = (int)rfl(lds_ld(ctrl + C_AVAIL)); \
                if (av >= (n_) || SPIN_OUT(_sp, t0)) break; \
                __builtin_amdgcn_s_sleep(1); } } while (0)
#define PULL4(a_) asm volatile("s_waitcnt vmcnt(%2)\n\tglobal_load_lds_dwordx4 %0, %1\n\t" \
                         "s_waitcnt vmcnt(%2)\n\tglobal_load_lds_dwordx4 %0, %1 offset:1024\n\t" \
                         "s_waitcnt vmcnt(%2)\n\tglobal_load_lds_dwordx4 %0, %1 offset:2048\n\t" \
                         "s_waitcnt vmcnt(%2)\n\tglobal_load_lds_dwordx4 %0, %1 offset:3072" \
                         :: "v"(lo16), "s"(a_), "i"(AG_WIN - 1) : "memory")
            unsigned long long own = srcws == myws ? rt() : 0;
#define PACE(n_) do { if (own) { own += n_; while (rt() < own) __builtin_amdgcn_s_sleep(2); } } while (0)
#pragma unroll 1
            for (int t = 0; t < KA; ++t) {
                STAGED(t + 1);
                PACE(AG_OWN);
                if (qt == 0) TRACE(0, t);
                if (qt == 0 && (t & 7) == 1) STAMP(54 + t / 8);

                if (qt == 0 && t < 12) STAMP(36 + t);
                const char* xt = xa + qt * 8192;
                if ((PMASK >> 1 >> t) & 1) { hv += 1u; xa += X_STEP; ++t; }
                post(hq, zero, hv);
                PULL4(xt);
                PULL4(xt + 4096);
                hv += 1u;
                xa += X_STEP;
            }
            STAGED(NKS);
            xa += (qt >> 1) * X_STEP + (qt & 1) * 4096;
#pragma unroll
            for (int t = KA; t < NITEMS; ++t) {
                PACE(AG_OWN / 2);
                if (qt == 0) TRACE(0, t);
                hv += 1u;
                post(hq, zero, hv);
                PULL4(xa + (t - KA) % NI * 2 * X_STEP + ((t - KA) / NI % 2 * 2 + (t - KA) / NAB) * 8192);
            }
            VM_WAIT(0);
            STAMP(6 + qt);
        } else if (wave == 7) {
            if (lane == 0) __hip_atomic_store(cctr, call, __ATOMIC_RELAXED, __HIP_MEMORY_SCOPE_AGENT);
            __builtin_amdgcn_fence(__ATOMIC_ACQUIRE, "");
            unsigned rot, staged = 0, npoll = 0;
            bool stored = false;
            t0 = rt();
            do rot = ld_ctr((const void*)rotw) - call * 8u; while (rot > 7u && rt() - t0 <= DEADLINE);
            lds_st(ctrl + C_ROT, rot | 8u);
            post(hint + 16, 0u, call * 256u + KP0);
            const char* sf = SRCWS(rot) + WS_SFLAG + (size_t)idx * FSTRIDE;
            STAMP(19);
            while (staged < (unsigned)NKS || !stored) {
                if (rot > 7u || ((++npoll & 31u) == 0u && (rfl(__hip_atomic_load(tmo, __ATOMIC_RELAXED, __HIP_MEMORY_SCOPE_AGENT)) == call
                                                           || rt() - t0 > DEADLINE))) {
                    __hip_atomic_store(tmo, call, __ATOMIC_RELAXED, __HIP_MEMORY_SCOPE_AGENT);
                    post(hint + 16, 0u, call * 256u);
                    VM_WAIT(0);
                    lds_st(ctrl + C_AVAIL, (unsigned)NKS);
                    break;
                }
                unsigned s2; bool p2;
                arrivals(sf, call, lane, s2, p2);
                if (npoll == 1) STAMP(27);
                if (s2 > staged) {
                    lds_st(ctrl + C_AVAIL, s2);
                    if (!staged) STAMP(17);
                    staged = s2;
                    if (s2 == NKS) STAMP(10);
                }
                if (p2 && !stored) { stored = true; post(hint + 16, 0u, call * 256u + NITEMS); }
                if (npoll < 64u && staged < (unsigned)NKS) __builtin_amdgcn_s_sleep(2); else __builtin_amdgcn_s_sleep(32);
            }
            STAMP(11);
        }
        return;
    }

    const int mt = idx & 1, nt = (idx - 2) >> 1;
    const int f0 = (17 * nt) >> 1;
    if (wave < 8) {
        f4 sv[4][2];
        const int g = b - 16;
        int sk = g >> 2;
#pragma unroll
        for (int e = 0; e < K1; ++e) sk += sk >= KP0 + e * KPS;
        const bool late = wave < 4 && sk >= AG_SEARLY;
        if (wave < 4) {
            if (wave == 0) STAMP(0);
            const float* s0 = p.x + (size_t)((g & 3) * 128 + wave * 32 + r8) * GK + sk * BK;
            if (late) {
                __builtin_amdgcn_s_barrier();
                for (int i = AG_SEARLY; i < sk; ++i) __builtin_amdgcn_s_sleep(AG_SSLOPE);
            } else if (AG_SDELAY) __builtin_amdgcn_s_sleep(AG_SDELAY);
#pragma unroll
            for (int i = 0; i < 4; ++i)
                asm volatile("global_load_dwordx4 %0, %2, off" SLD "\n\tglobal_load_dwordx4 %1, %2, off offset:16" SLD
                             : "=&v"(sv[i][0]), "=&v"(sv[i][1]) : "v"(s0 + (size_t)i * 8 * GK + (c8 ^ ((i & 1) * 4)) * 8) : "memory");
        }
        if (!late) __builtin_amdgcn_s_barrier();
        asm volatile("" ::: "memory");
        const unsigned call = rfl(lds_ld(ctrl + C_CALL));
        if (wave < 4) {
            char* d0 = myws + WS_XS + ((size_t)((call & 1u) * 2u + (unsigned)((g >> 1) & 1)) * NKS + (size_t)sk) * X_STEP
                       + (size_t)((g & 1) * X_PART + wave * 4096) + lo16;
            asm volatile("s_waitcnt vmcnt(0)" : "+v"(sv[0][0]), "+v"(sv[0][1]), "+v"(sv[1][0]), "+v"(sv[1][1]),
                         "+v"(sv[2][0]), "+v"(sv[2][1]), "+v"(sv[3][0]), "+v"(sv[3][1]) :: "memory");
            if (wave == 0) STAMP(24);
            u4 o[4];
#pragma unroll
            for (int i = 0; i < 4; i += 2)
                to_frags(pk8(sv[i][0], sv[i][1]), pk8(sv[i + 1][0], sv[i + 1][1]), img + (unsigned)(SCR_OFF + wave * 2048),
                         frg + (unsigned)(SCR_OFF + wave * 2048), o[i], o[i + 1]);
            asm volatile("s_waitcnt lgkmcnt(0)" : "+v"(o[0]), "+v"(o[1]), "+v"(o[2]), "+v"(o[3]) :: "memory");
#pragma unroll
            for (int i = 0; i < 4; ++i)
                st16_sys(d0 + i * 1024, o[i]);
            if (wave == 0) STAMP(25);
            unsigned n = 0;
            if (lane == 0) {
                asm volatile("s_waitcnt vmcnt(0)\n\tds_add_rtn_u32 %0, %1, %2\n\ts_waitcnt lgkmcnt(0)"
                             : "=&v"(n) : "v"(ctrl + C_STG), "v"(1u) : "memory");
                if (n == 3u) st4_sys(myws + WS_SFLAG + (size_t)((g >> 1) & 1) * FSTRIDE + sk * 8 + (g & 1) * 4, call);
            }
            if (wave == 0) { STAMP(16); STAMPE(23); }
        }
        const int set = wave >> 2, w4 = wave & 3;
        const bool dup = !(nt & 1) && nt != 14;
        f4 acc[FW][2];
#pragma unroll
        for (int i = 0; i < FW; ++i) {
            acc[i][0] = (f4){0.f, 0.f, 0.f, 0.f}; acc[i][1] = (f4){0.f, 0.f, 0.f, 0.f};
            asm volatile("" : "+v"(acc[i][0]), "+v"(acc[i][1]));
        }
        h8 a[RW], x[8];
#pragma unroll
        for (int i = 0; i < RW; ++i) asm volatile("" : "=v"(a[i]));
#pragma unroll
        for (int j = 0; j < 8; ++j) asm volatile("" : "=v"(x[j]));
        Pos P;
        Loop L;
        asm volatile("" : "=v"(P.f));
        P.w = lbase + (unsigned)((r16 >> 2) * W_BLK + q * 256 + (r16 & 3) * 16);
        P.n = lbase + (unsigned)(8 * W_BLK + q * 256 + (r16 & 3) * 64 + (r16 >> 2) * 16);
        P.x = lo16 + (unsigned)(set * X_PART + w4 * 2048);
        P.ldw = ctrl + C_LD - 4;
        P.rv = -4u; P.relw = ctrl + C_CONS + wave * 4; P.tab = myws + WS_TAB + (size_t)b * 512;
        unsigned vo = (unsigned)(q * NLOC + r16) * 16u, vn = (unsigned)(q * 4 * NLOC + r16) * 4u;
        asm volatile("" : "+v"(P.w), "+v"(P.n), "+v"(vn), "+v"(P.x), "+v"(P.rv), "+v"(P.ldw), "+v"(vo));
        WAIT_AV(1, 0);
        const unsigned rot = rfl(P.f.x);
        const int src = (p.srcs >> 4 * ((xr + rot) & 7)) & 7;
        L.xa = SRCWS(rot) + WS_XS + (size_t)((call & 1u) * 2u + (unsigned)mt) * NKS * X_STEP;
        L.xg = myws + WS_XG + (size_t)((((call & 1u) * NR + (unsigned)src) * 2u + (unsigned)mt) * K1) * X_STEP;
        L.set = set;
        const char* ol = (const char*)(p.out + (size_t)(src * MLOC + mt * BM + set * 128 + w4 * 16) * NLOC + f0 * 16);
        P.p[0] = L.xa; P.p[1] = L.xa + 8192;
        PIN(ol); PIN(P.relw); PIN(P.tab); PIN(L.xg); PIN(L.set); PIN(P.p[0]); PIN(P.p[1]);
#pragma unroll
        for (int k = 0; k < 4; ++k) XLD(x[k], k);
        L.t = 1u; P.need = 2u; L.xa += X_STEP; P.p[0] = L.xa; P.p[1] = L.xa + 8192;
        PIN(L.t); PIN(P.need); PIN(L.xa); PIN(P.p[0]); PIN(P.p[1]);
        WAIT_AV(0, 1);
        if (wave == 0) { STAMP(18); STAMPC(30); }
        FLAGS();
#pragma unroll
        for (int i = 0; i < FW; ++i) DSR(a[i], i < 8 ? P.w : P.n, wcol(i));
        __builtin_amdgcn_s_waitcnt(0x0070);
        SCHED_FENCE();
#pragma unroll 1
        do {
            k_step<0, 0>(acc, a, x, P, L);
            k_step<1, 0>(acc, a, x, P, L);
        } while (L.t <= (unsigned)KA);
        if (wave == 0) { STAMP(2); STAMPC(31); }
        P.need = (unsigned)(KA + 1 + NI * set);
        const char* xp = L.xa - X_STEP;
        unsigned pre = P.need - set;
#define TRY(r_) do { const unsigned _p = (r_) >= (set ? AG_UNC1 : AG_UNC0) ? P.need + 1u : AG_PAIR ? (rfl(P.f.w) >= P.need ? P.need + 1u : 0u) : rfl(P.f.w); item_x(x, P.x, xp, pre < P.need ? P.need : ~0u, _p); \
                   item_x(x + 4, P.x, xp + 2 * X_STEP, pre <= P.need ? P.need + 1u : ~0u, _p); pre = _p; } while (0)
#define SILU(r_, J_, keep_) silu4(acc[(r_) & 4][J_][(r_) & 3], acc[((r_) & 4) + 1][J_][(r_) & 3], acc[((r_) & 4) + 2][J_][(r_) & 3], \
                                  acc[((r_) & 4) + 3][J_][(r_) & 3], 60.f * LOG2E, c, keep_)
#define STORE(r_, o_) asm volatile("global_store_dwordx4 %0, %1, %2 offset:%3" AST \
                                   :: "v"(vo), "v"(o_), "s"(ol + ((r_) & 3) * NLOC * 4), "i"(((r_) & 4) * 64) : "memory")
#define PART(J_) do { \
        SCHED_FENCE(); \
        TTJ(J_, J_ * 6); \
        if (pre < P.need) WAIT_AV(P.need, 0); \
        item_x(x, P.x, xp, pre + 1u, P.need); \
        if (AG_PAIR >> !J_ & 1) { if (!J_) WAIT_AV(P.need, KA + 2); } else if (pre <= P.need) WAIT_AV(P.need + 1u, KA + 2); \
        P.p[0] = xp + 2 * X_STEP; P.p[1] = xp + 3 * X_STEP; PIN(P.p[0]); PIN(P.p[1]); \
        item_x(x + 4, P.x, P.p[0], pre, P.need); \
        if (!J_) _Pragma("unroll") for (int k = FW; k < RW; ++k) DSR(a[k], P.w, woff(k)); \
        if (J_) { if (pre < P.need) VM_WAIT(4); else VM_WAIT(8); } \
        if (J_ && AG_LPRIO && set) __builtin_amdgcn_s_setprio(AG_LPRIO); \
        SCHED_FENCE(); \
        part_items<J_ * NT, J_ * NT + NT>(acc, a, x, P); \
        TTJ(J_, J_ * 6 + 4); \
        if (!J_) { xp += 8192; PIN(xp); P.need += NAB; pre = 0u; TRY(0); } \
        if (!J_ && AG_EPRIO && set) __builtin_amdgcn_s_setprio(AG_EPRIO); \
        if (wave == 4 && J_) { STAMP(3); STAMPC(33); } \
        const float c = __builtin_bit_cast(float, __builtin_bit_cast(unsigned, LOG2E) | rfl(P.f.z)); \
        asm volatile("s_nop 7\n\ts_nop 7" : "+v"(acc[FW - 3][J_]), "+v"(acc[FW - 2][J_]), "+v"(acc[FW - 1][J_])); \
        f4 o = acc[FW - 1][J_], od[4]; \
        _Pragma("unroll") for (int r = 0; r < 4; ++r) { o = SILU(r, J_, o); STORE(r, o); } \
        if (!dup) { \
            const f4 on = silu4(acc[FW - 1][J_][0], acc[FW - 1][J_][1], acc[FW - 1][J_][2], acc[FW - 1][J_][3], 60.f * LOG2E, c, o); \
            _Pragma("unroll") for (int r = 0; r < 4; ++r) \
                asm volatile("global_store_dword %0, %1, %2 offset:512" AST :: "v"(vn), "v"(on[r]), "s"(ol + r * NLOC * 4) : "memory"); \
        } \
        _Pragma("unroll") for (int r = 4; r < 8; ++r) {        \
            if (!J_ && (AG_TRYM >> r & 1)) FLAGS(); \
            o = od[r - 4] = SILU(r, J_, o); \
            if (J_) STORE(r, o); \
            if (!J_ && (AG_TRYM >> r & 1)) { asm volatile("s_waitcnt lgkmcnt(0)" : "+v"(P.f) :: "memory"); TRY(r); } \
        } \
        if (!J_) _Pragma("unroll") for (int r = 4; r < 8; ++r) STORE(r, od[r - 4]); \
        if (!J_ && AG_EPRIO) __builtin_amdgcn_s_setprio(0); \
        TTJ(J_, J_ * 6 + 5); \
        asm volatile("s_nop 1" ::: "memory"); } while (0)
        PART(0);
        ADV(vo, 64 * NLOC * 4); ADV(vn, 64 * NLOC * 4);
        PART(1);
        if (wave == 4) { STAMP(4); STAMPC(34); }
        __builtin_amdgcn_endpgm();
    }
    if (wave == 10) {
        *(u2*)(myws + WS_TAB + (size_t)b * 512 + lane * 8) = (u2){2u * lane + 1u, 2u * lane + 2u};
        const unsigned cl = ld_ctr((const void*)cctr) + 1u;
        if (b == NWG - 8 && lane < 16)
            __hip_atomic_store((GAS unsigned*)(myws + WS_WFLAG + (size_t)lane * FSTRIDE + 256),
                               cl * 8u + (ROTX), __ATOMIC_RELAXED, __HIP_MEMORY_SCOPE_AGENT);
        if (lane == 0) __hip_atomic_store(cctr, cl, __ATOMIC_RELAXED, __HIP_MEMORY_SCOPE_AGENT);
        lds_st(ctrl + lane * 4, lane == C_CALL / 4 ? cl : 0u);
        LGKM0();
    }
    __builtin_amdgcn_s_barrier();
    asm volatile("" ::: "memory");
    const unsigned call = rfl(lds_ld(ctrl + C_CALL));
    if (wave == 10) {
        __builtin_amdgcn_fence(__ATOMIC_ACQUIRE, "");
        VM_WAIT(0);
        GAS unsigned* hq = (GAS unsigned*)(myws + WS_HINT + (size_t)(xr * 2 + mt) * FSTRIDE) + c8;
        GAS unsigned* hp = lane == 5 ? rotw : hq;
        GAS unsigned* wf0 = (GAS unsigned*)(myws + WS_WFLAG + (size_t)(f0 >> 3) * FSTRIDE) + lane;
        GAS unsigned* wf1 = (GAS unsigned*)(myws + WS_WFLAG + (size_t)((f0 + FW - 1) >> 3) * FSTRIDE) + lane;
        unsigned pub = 0, wav = 0, npoll = 0, rot = 8u;
        TRZERO(1);
        const unsigned long long t0 = rt();
        unsigned long long tall = 0;
        bool dead = false;
        while (pub < (unsigned)NITEMS || wav < (unsigned)NKS) {
            const unsigned hv = __hip_atomic_load(hp, __ATOMIC_RELAXED, __HIP_MEMORY_SCOPE_AGENT);
            bool adv = false;
            if (wav < (unsigned)NKS) {
                const unsigned w0 = __hip_atomic_load(wf0, __ATOMIC_RELAXED, __HIP_MEMORY_SCOPE_AGENT);
                const unsigned w1 = __hip_atomic_load(wf1, __ATOMIC_RELAXED, __HIP_MEMORY_SCOPE_AGENT);
                const unsigned long long notready = __ballot((int)(w0 - call) < 0 || (int)(w1 - call) < 0);
                const unsigned w2 = notready ? (unsigned)__builtin_ctzll(notready) : 64u;
                if (w2 > wav) { if (!wav) STAMP(29); wav = w2; lds_st(ctrl + C_WAV, wav); adv = true; }
            }
            const int d = (int)(hv - call * 256u);
            const unsigned av = (unsigned)row_min(c8 < 5 ? max(min(d, NITEMS), 0) : NITEMS);
            if (rot > 7u) {
                rot = (unsigned)__builtin_amdgcn_readlane((int)hv, 5) - call * 8u;
                if (rot <= 7u) { lds_st(ctrl + C_ROT, rot); hp = hq; }
            }
            if (av > pub && rot <= 7u) {
                asm volatile("ds_max_u32 %0, %1" :: "v"(lane ? ctrl + 128 + lane * 4 : ctrl + C_AVAIL), "v"(av) : "memory");
                pub = av;
                tall = 0;
                TRACE(1, av - 1);
            } else if (!adv && (++npoll & 63u) == 0u) {
                if (rt() - t0 > DEADLINE) { dead = true; break; }
                unsigned staged; bool stored;
                arrivals(SRCWS(rot) + WS_SFLAG + (size_t)mt * FSTRIDE, call, lane, staged, stored);
                if (staged == (unsigned)NKS && stored && rot <= 7u) {
                    if (tall == 0) tall = rt();
                    else if (rt() - tall > SILENCE) { pub = NITEMS; lds_st(ctrl + C_AVAIL, pub); }
                } else if (rfl(__hip_atomic_load(tmo, __ATOMIC_RELAXED, __HIP_MEMORY_SCOPE_AGENT)) == call) { dead = true; break; }
            }
            if (pub < (unsigned)AG_FINE) __builtin_amdgcn_s_sleep(AG_SLEEP_POLL); else __builtin_amdgcn_s_sleep(AG_SLEEP_FINE);
        }
        if (dead) {
            __hip_atomic_store(tmo, call, __ATOMIC_RELAXED, __HIP_MEMORY_SCOPE_AGENT);
            lds_st(ctrl + C_BAD, 0x7fc00000u);
        }
        lds_st(ctrl + C_WAV, (unsigned)NKS);
        lds_st(ctrl + C_AVAIL, (unsigned)NITEMS);
        TRDUMP(1);
    } else if (wave == 9) {
        constexpr int CSTEP = 240 / NCONV;
        const int cq = ((idx - 2) / CSTEP) * 8 + xr;
        if ((idx - 2) % CSTEP == 0 && cq < NCONV) {
            const unsigned so = (unsigned)(r8 * 8 * NLOC + c8 * 4) * 4u;
            const unsigned dof = (unsigned)((c8 >> 2) * 2048 + r8 * 256 + (c8 & 3) * 16);
            for (int i = AG_CFIRST; i < cq; i += 16) __builtin_amdgcn_s_sleep(AG_CDELAY);
            for (int u = cq; u < NKS * 16; u += NCONV) {
                const int ks = u >> 4, cn = u & 15;
                const float* sw = p.w + (size_t)ks * BK * NLOC + cn * 128;
                const char* dstw = myws + WS_WH + (size_t)ks * W_ROW + (size_t)cn * 16384;
                f4 v[4][8];
                if (u < 16) STAMP(12);
#pragma unroll
                for (int g = 0; g < 4; ++g)
#pragma unroll
                    for (int r = 0; r < 8; ++r)
                        asm volatile("global_load_dwordx4 %0, %1, %2 offset:%3" CLD : "=v"(v[g][r]) : "v"(so), "s"(sw + r * NLOC), "i"(g * 128) : "memory");
                u4 o[4][4];
#pragma unroll
                for (int g = 0; g < 4; ++g) {
                    asm volatile("s_waitcnt vmcnt(%8)" : "+v"(v[g][0]), "+v"(v[g][1]), "+v"(v[g][2]), "+v"(v[g][3]),
                                 "+v"(v[g][4]), "+v"(v[g][5]), "+v"(v[g][6]), "+v"(v[g][7]) : "i"(24 - 8 * g) : "memory");
#pragma unroll
                    for (int r = 0; r < 8; ++r) v[g][r] *= LOG2E;
#pragma unroll
                    for (int c = 0; c < 4; ++c)
                        o[g][c] = (u4){pk(v[g][0][c], v[g][1][c]), pk(v[g][2][c], v[g][3][c]), pk(v[g][4][c], v[g][5][c]), pk(v[g][6][c], v[g][7][c])};
                    SCHED_FENCE();
                }
                if (u < 16) STAMP(13);
#pragma unroll
                for (int g = 0; g < 4; ++g)
#pragma unroll
                    for (int c = 0; c < 4; ++c)
                        asm volatile("global_store_dwordx4 %0, %1, %2 offset:%3" CST "\n\ts_nop 1"
                                     :: "v"(dof), "v"(o[g][c]), "s"(dstw + g * 4096), "i"(c * 64) : "memory");
                VM_WAIT(0);
                if (lane == 0)
                    __hip_atomic_store((GAS unsigned*)(myws + WS_WFLAG + (size_t)cn * FSTRIDE + ks * 4), call,
                                       __ATOMIC_RELAXED, __HIP_MEMORY_SCOPE_AGENT);
                if (u < 16) STAMP(28);
                if (u < NCONV) STAMP(35);
            }
            STAMP(1);
        }
    } else {
        const char* tab = myws + WS_TAB + (size_t)b * 512;
        const char* wa = myws + WS_WH + (size_t)f0 * 2048;
        unsigned long long t0 = 0;
        int av = 0, cons = 0;
#pragma unroll 1
        for (int t = 0; t < NKS; ++t) {
            if (av <= t || cons <= t - NW) for (unsigned sp = 0;;) {
                u4 c, d; unsigned v;
                asm volatile("ds_read_b128 %0, %3\n\tds_read_b128 %1, %3 offset:16\n\tds_read_b32 %2, %3 offset:%4\n\t"
                             "s_waitcnt lgkmcnt(0)" : "=&v"(c), "=&v"(d), "=&v"(v) : "v"(ctrl + C_CONS), "i"(C_WAV - C_CONS) : "memory");
                av = (int)rfl(v); cons = (int)min8(c, d);
                if ((av > t && cons > t - NW) || SPIN_OUT(sp, t0)) break;
                __builtin_amdgcn_s_sleep(AG_LSLEEP);
            }
            TRACE(2, t);
            if (t == 0) STAMP(32);
            const unsigned sb = lbase + (unsigned)(t & (NW - 1)) * W_STEP;
#define WLOADS(P_) _Pragma("unroll") for (int i = 0; i < FW; i += 2) { \
                asm volatile("s_mov_b32 m0, %0\n\ts_waitcnt vmcnt(58)\n\t" \
                             "global_load_lds_dwordx4 %1, %2" P_ "\n\tglobal_load_lds_dwordx4 %1, %2 offset:1024" P_ \
                             :: "s"(sb + i * W_BLK), "v"(lo16), "s"(wa + i * 2048) : "memory"); \
                if (i + 1 < FW) asm volatile("s_mov_b32 m0, %0\n\ts_nop 0\n\t" \
                             "global_load_lds_dwordx4 %1, %2 offset:2048" P_ "\n\tglobal_load_lds_dwordx4 %1, %2 offset:3072" P_ \
                             :: "s"(sb + i * W_BLK + 64), "v"(lo16), "s"(wa + i * 2048) : "memory"); \
            }
            if (mt == 0 && t >= AG_WNT0 && t < AG_WNT) { WLOADS(" nt") } else
            { WLOADS(WLD) }
            asm volatile("s_mov_b32 m0, %0\n\ts_mov_b64 exec, 1\n\tglobal_load_lds_dword %1, %2\n\ts_mov_b64 exec, -1"
                         :: "s"(ctrl + C_LD), "v"(lo16), "s"(tab + t * 4) : "memory");
            wa += W_ROW;
        }
        VM_WAIT(0);
    }
}

}

extern "C" void kernel_launch(int rank, int nranks, ncclComm_t comm,
                              hipStream_t stream,
                              void* const* d_in, const int* in_sizes, int n_in,
                              void* d_out, int out_size,
                              const int* in_dtypes, int out_dtype,
                              void* const* d_out_all,
                              void* const* d_ws_all, size_t ws_bytes) {
    (void)comm; (void)in_dtypes; (void)out_dtype; (void)d_out_all;
    if (nranks != NR || n_in < 2 || (unsigned)rank >= (unsigned)NR || ws_bytes < WS_NEED ||
        in_sizes[0] != MLOC * GK || in_sizes[1] != GK * NLOC || out_size != GM * NLOC) {
        fprintf(stderr, "ag_gemm: unexpected problem shape\n");
        return;
    }
    static bool attr_set = false;
    if (!attr_set) {
        HIP_CHECK(hipFuncSetAttribute((const void*)ag_gemm_kernel,
                                      hipFuncAttributeMaxDynamicSharedMemorySize, LDS_BYTES));
        attr_set = true;
    }
    Params p;
    p.x = (const float*)d_in[0];
    p.w = (const float*)d_in[1];
    p.out = (float*)d_out;
    p.myws = (char*)d_ws_all[rank];
    p.srcs = 0;
    {
        int order[NR], n = 0;
        const unsigned half = 0xb2;
        for (int far = 0; far < 2; ++far)
            for (int r = 0; r < NR; ++r) { const int s = (rank + r) & 7; if ((int)((half >> s ^ half >> rank) & 1) == far) order[n++] = s; }
        for (int r = 0; r < NR; ++r) { p.ws[r] = (char*)d_ws_all[order[r]]; p.srcs |= (unsigned)order[r] << 4 * r; }
    }
    for (int r = 0; r < 2 * NR; ++r) p.xg[r] = (char*)d_ws_all[(rank + 1 + r) & 7] + WS_XG + (size_t)rank * 2 * K1 * X_STEP;
    hipLaunchKernelGGL(ag_gemm_kernel, dim3(NWG), dim3(NTHREADS), LDS_BYTES, stream, p);
    HIP_CHECK(hipGetLastError());
}
```

```cpp
#include <hip/hip_runtime.h>
#include <rccl/rccl.h>
#include <cstdio>
#include <cstddef>

#define HIP_CHECK(expr) do { hipError_t _e = (expr); if (_e != hipSuccess) { \
    fprintf(stderr, "HIP error %s at %s:%d\n", hipGetErrorString(_e), __FILE__, __LINE__); \
    return; } } while (0)

namespace {

#define GAS __attribute__((address_space(1)))
#define LAS __attribute__((address_space(3)))
typedef _Float16 h8 __attribute__((ext_vector_type(8)));
typedef _Float16 h2 __attribute__((ext_vector_type(2)));
typedef float f4 __attribute__((ext_vector_type(4)));
typedef unsigned u4 __attribute__((ext_vector_type(4)));
typedef unsigned u2 __attribute__((ext_vector_type(2)));

constexpr int NR = 8;
constexpr int GM = 4096, GK = 4096, NLOC = 2048, MLOC = 512;
constexpr int BM = 256, BK = 64;
constexpr int FW = 9;
constexpr int NKS = GK / BK;
constexpr int K1 = 4;
constexpr int KP0 = 30, KPS = 6;
constexpr unsigned long long PMASK = (1ull | 1ull << KPS | 1ull << 2 * KPS | 1ull << 3 * KPS) << KP0;
constexpr int NAB = 8, NI = NAB / 2;
constexpr int KA = NKS - NAB;
constexpr int NITEMS = KA + 4 * NI;
constexpr int X_STEP = BM * BK * 2;
constexpr int X_PART = X_STEP / 2;
constexpr int W_BLK = 2048 + 64;
constexpr int W_STEP = FW * W_BLK - 64;
constexpr int W_ROW = NLOC * BK * 2;
constexpr int NW = 8;
constexpr int SCR_OFF = NW * W_STEP;
constexpr int CTRL_OFF = SCR_OFF + 8192;
constexpr int LDS_BYTES = CTRL_OFF + 512;
constexpr int NWG = 256, NTHREADS = 704;
static_assert(LDS_BYTES <= 160 * 1024 && NAB <= NW && NAB % 2 == 0 && NAB >= 4 && NKS % NW == 0 && NW == 8, "layout");
static_assert((NKS - K1) * 4 == NWG - 16 && KP0 > 1 && KPS > 1 && KP0 + 3 * KPS < KA - 1, "layout");
constexpr int C_CONS = 0;
constexpr int C_WAV = 68;
constexpr int C_STG = 72;
constexpr int C_CALL = 76;
constexpr int C_ROT = 80;
constexpr int C_LD = 84;
constexpr int C_BAD = 88;
constexpr int C_AVAIL = 92;
constexpr int NCONV = 120;
constexpr float LOG2E = 1.4426950408889634f;
constexpr int PWIN = 16;
constexpr int WIN = 8;
constexpr int OWN = 85;
constexpr int EARLY = 12;
constexpr int WNT0 = 8, WNT1 = 32;

constexpr size_t FSTRIDE = 4096 + 256;
constexpr size_t WS_CALL = 0;
constexpr size_t WS_TMO = 49152;
constexpr size_t WS_SFLAG = 53248;
constexpr size_t WS_HINT = 65536;
constexpr size_t WS_WFLAG = 196608;
constexpr size_t WS_TAB = 327680;
constexpr size_t WS_WH = 1ull << 20;
constexpr size_t WS_XS = 32ull << 20;
constexpr size_t WS_XG = 48ull << 20;
constexpr size_t WS_NEED = 64ull << 20;

constexpr unsigned long long DEADLINE = 100000000ull;
constexpr unsigned long long SILENCE = 20000ull;

struct Params {
    const float* x;
    char* myws;
    const float* w;
    float* out;
    char* ws[NR];
    char* xg[2 * NR];
    unsigned srcs;
};
typedef const char* __attribute__((address_space(4))) const* KPTR;

__device__ __forceinline__ unsigned long long rt() { return __builtin_amdgcn_s_memrealtime(); }

__device__ __forceinline__ void st16_sys(void* p, u4 v) {
    asm volatile("global_store_dwordx4 %0, %1, off sc0 sc1\n\ts_nop 1" :: "v"(p), "v"(v) : "memory");
}
__device__ __forceinline__ void st4_sys(void* p, unsigned v) {
    asm volatile("global_store_dword %0, %1, off sc0 sc1\n\ts_nop 1" :: "v"(p), "v"(v) : "memory");
}
__device__ __forceinline__ void post(const char* p, unsigned zero, unsigned v) {
    asm volatile("s_mov_b64 exec, 1\n\tglobal_store_dword %0, %1, %2\n\ts_mov_b64 exec, -1" :: "v"(zero), "v"(v), "s"(p) : "memory");
}
__device__ __forceinline__ void lds_st(unsigned addr, unsigned v) {
    asm volatile("ds_write_b32 %0, %1" :: "v"(addr), "v"(v) : "memory");
}
__device__ __forceinline__ unsigned lds_ld(unsigned addr) {
    unsigned v;
    asm volatile("ds_read_b32 %0, %1\n\ts_waitcnt lgkmcnt(0)" : "=&v"(v) : "v"(addr) : "memory");
    return v;
}
__device__ __forceinline__ unsigned ld_ctr(const void* p) {
    unsigned v;
    asm volatile("global_load_dword %0, %1, off sc1\n\ts_waitcnt vmcnt(0)" : "=&v"(v) : "v"(p) : "memory");
    return (unsigned)__builtin_amdgcn_readfirstlane(v);
}
__device__ __forceinline__ unsigned pk(float a, float b) {
    h2 h; h.x = (_Float16)a; h.y = (_Float16)b;
    return __builtin_bit_cast(unsigned, h);
}
__device__ __forceinline__ u4 pk8(f4 a, f4 b) { return (u4){pk(a[0], a[1]), pk(a[2], a[3]), pk(b[0], b[1]), pk(b[2], b[3])}; }
__device__ __forceinline__ unsigned umin(unsigned a, unsigned b) { return a < b ? a : b; }
__device__ __forceinline__ unsigned rfl(unsigned v) { return (unsigned)__builtin_amdgcn_readfirstlane((int)v); }
__device__ __forceinline__ unsigned min8(u4 c, u4 d) {
    return rfl(umin(umin(umin(c.x, c.y), umin(c.z, c.w)), umin(umin(d.x, d.y), umin(d.z, d.w))));
}
__device__ __forceinline__ int row_min(int u) {
    u = min(u, __builtin_amdgcn_update_dpp(0, u, 0x121, 15, 15, false));
    u = min(u, __builtin_amdgcn_update_dpp(0, u, 0x122, 15, 15, false));
    u = min(u, __builtin_amdgcn_update_dpp(0, u, 0x124, 15, 15, false));
    u = min(u, __builtin_amdgcn_update_dpp(0, u, 0x128, 15, 15, false));
    return __builtin_amdgcn_readfirstlane(u);
}
__device__ __forceinline__ void arrivals(const char* sf, unsigned call, int lane, unsigned& staged, bool& stored) {
    u2 sv; unsigned pv;
    asm volatile("global_load_dwordx2 %0, %2, %4 sc0 sc1\n\tglobal_load_dword %1, %3, %4 offset:512 sc0 sc1\n\ts_waitcnt vmcnt(0)"
                 : "=&v"(sv), "=&v"(pv) : "v"(lane * 8), "v"((lane & 15) * 4), "s"(sf) : "memory");
    const unsigned long long miss = __ballot((int)(sv.x - call) < 0 || (int)(sv.y - call) < 0) & ~PMASK;
    staged = miss ? (unsigned)__builtin_ctzll(miss) : (unsigned)NKS;
    stored = __ballot((int)(pv - call) < 0) == 0;
}
#define VM_WAIT(n) asm volatile("s_waitcnt vmcnt(" #n ")" ::: "memory")
#define LGKM0() asm volatile("s_waitcnt lgkmcnt(0)" ::: "memory")
#define SCHED_FENCE() __builtin_amdgcn_sched_barrier(0)
#define SPIN_OUT(sp_, t0_) ((((++(sp_)) & 1023u) == 0u) && ((t0_) == 0 ? ((t0_) = rt(), false) : ((unsigned)(rt() - (t0_)) > 2u * (unsigned)DEADLINE)))

__device__ __forceinline__ f4 silu4(float z0, float z1, float z2, float z3, float lim, float c, f4 keep) {
    float o0, o1, o2, o3;
    asm volatile("v_med3_f32 %0, -%4, %8, -%8\n\tv_med3_f32 %1, -%5, %8, -%8\n\tv_med3_f32 %2, -%6, %8, -%8\n\tv_med3_f32 %3, -%7, %8, -%8\n\t"
                 "v_exp_f32 %0, %0\n\tv_exp_f32 %1, %1\n\tv_exp_f32 %2, %2\n\tv_exp_f32 %3, %3\n\t"
                 "v_fma_f32 %0, %0, %9, %9\n\tv_fma_f32 %1, %1, %9, %9\n\tv_fma_f32 %2, %2, %9, %9\n\tv_fma_f32 %3, %3, %9, %9\n\t"
                 "v_rcp_f32 %0, %0\n\tv_rcp_f32 %1, %1\n\tv_rcp_f32 %2, %2\n\tv_rcp_f32 %3, %3\n\t"
                 "v_mul_f32 %0, %4, %0\n\tv_mul_f32 %1, %5, %1\n\tv_mul_f32 %2, %6, %2\n\tv_mul_f32 %3, %7, %3"
                 : "=&v"(o0), "=&v"(o1), "=&v"(o2), "=&v"(o3) : "v"(z0), "v"(z1), "v"(z2), "v"(z3), "s"(lim), "s"(c), "v"(keep));
    return (f4){o0, o1, o2, o3};
}
#define REL() asm volatile("s_mov_b32 m0, %0\n\ts_mov_b64 exec, 1\n\tglobal_load_lds_dword %1, %2\n\ts_mov_b64 exec, -1" \
                           :: "s"(P.relw), "v"(P.rv), "s"(P.tab) : "memory")
struct Pos { unsigned w, n, x, rv, need, relw, ldw; const char* tab; const char* p[2]; u4 f; };
#define ADV(v_, d_) do { v_ += d_; asm volatile("" : "+v"(v_)); } while (0)
#define PIN(s_) asm volatile("" : "+s"(s_))
#define FLAGS() asm volatile("ds_read_b128 %0, %1" : "=v"(P.f) : "v"(P.ldw) : "memory")
#define WAIT_AV(nx_, nw_) do { unsigned long long _t0 = 0; unsigned _sp = 0; \
        for (;;) { asm volatile("ds_read_b128 %0, %1\n\ts_waitcnt lgkmcnt(0)" : "+v"(P.f) : "v"(P.ldw) : "memory"); \
            if ((int)((rfl(P.f.w) - (unsigned)(nx_)) | (rfl(P.f.y) - (unsigned)(nw_))) >= 0) break; \
            if (SPIN_OUT(_sp, _t0)) break; \
            __builtin_amdgcn_s_sleep(1); } } while (0)
#define CHECK_AV(nx_, nw_) do { if (__builtin_expect((int)((rfl(P.f.w) - (unsigned)(nx_)) | (rfl(P.f.y) - (unsigned)(nw_))) < 0, 0)) \
            WAIT_AV(nx_, nw_); SCHED_FENCE(); } while (0)
#define MFMA(c_, x_, a_) asm volatile("v_mfma_f32_16x16x32_f16 %0, %1, %2, %0" : "+v"(c_) : "v"(x_), "v"(a_))
#define MFMA_R(c_, x_, a_, ad_, off_) asm volatile("v_mfma_f32_16x16x32_f16 %0, %2, %1, %0\n\tds_read_b128 %1, %3 offset:%4" \
                                                   : "+v"(c_), "+v"(a_) : "v"(x_), "v"(ad_), "i"(off_) : "memory")
#define XLD(x_, k_) asm volatile("global_load_dwordx4 %0, %1, %2 offset:%3" : "+v"(x_) : "v"(P.x), "s"(P.p[(k_) & 1]), "i"(((k_) & 2) * 512) : "memory")
constexpr int wcol(int i) { return i < 8 ? (i >> 2) * 4 * W_BLK + (i & 3) * 64 : 0; }

struct Loop { const char* xa; const char* xg; unsigned t, set; };
template <int B, int G>
__device__ __forceinline__ void k_step(f4 (&acc)[FW][2], h8 (&a)[FW], h8 (&x)[8], Pos& P, Loop& L) {
    if constexpr (G < 36) {
        constexpr int h = G / 18, i = G % 18 / 2;
        if (G % 18 == 0) asm volatile("s_waitcnt vmcnt(%0) lgkmcnt(7)" :: "i"(h ? 5 : 3) : "memory");
        else if (G == 2) asm volatile("s_waitcnt lgkmcnt(4)" : "+v"(P.f) :: "memory");
        else if (G % 18 % 4 == 0) asm volatile("s_waitcnt lgkmcnt(7)" ::: "memory");
        if (G & 1) MFMA_R(acc[i][1], x[B * 4 + h * 2 + 1], a[i], i < 8 ? P.w : P.n, (h ? 1 - B : B) * W_STEP + (1 - h) * 1024 + wcol(i));
        else MFMA(acc[i][0], x[B * 4 + h * 2], a[i]);
        SCHED_FENCE();
        if (G == 2) CHECK_AV(P.need, L.t + 1u);
        if (G >= 4 && G <= 10 && !(G & 1)) XLD(x[4 - B * 4 + G / 2 - 2], G / 2 - 2);
        if (B && G == 18) { const int d = (L.t & (NW - 1)) ? 2 * W_STEP : (2 - NW) * W_STEP; ADV(P.w, d); ADV(P.n, d); }
        if (G == 20) { L.t += 1u; PIN(L.t); L.xa += X_STEP; PIN(L.xa); }
        if (G == 22) {
            const bool st = (PMASK >> L.t) & 1;
            P.p[0] = st ? L.xg : L.xa; PIN(P.p[0]);
            L.xg += st ? X_STEP : 0; PIN(L.xg);
        }
        if (G == 24) {
            P.need = L.t + 1u;
            if (B) P.p[1] = P.p[0] + 8192;
            else {
                const bool last = L.t == (unsigned)KA;
                P.p[0] -= last && L.set ? X_STEP : 0;
                P.p[1] = P.p[0] + (last && !L.set ? X_STEP : 8192);
                P.need = last && L.set ? 0u : P.need;
                PIN(P.p[0]);
            }
            PIN(P.p[1]); PIN(P.need);
        }
        if (G == 30) FLAGS();
        if (G == 32) ADV(P.rv, 4u);
        if (G == 34) REL();
        SCHED_FENCE();
        k_step<B, G + 1>(acc, a, x, P, L);
    }
}
constexpr int NT = 36 * NI, WG = 3, G_C = 35 - FW;
constexpr bool flags(int t) {
    return t % NT >= NT - 36 ? t % 36 == 26 : t < NT && t % 36 == G_C - 9;
}
constexpr int xfetch(int t) {
    for (int k = 0; k < 4; ++k) if (t % 36 == (G_C + 1 + k > 9 * k + 8 ? G_C + 1 + k : 9 * k + 8)) return t % NT < NT - 72 ? k : -1;
    return -1;
}
struct Waits { int l[2 * NT + 1], v[NT / 9]; };
constexpr Waits waits() {
    Waits r{};
    int pw[2 * NT + 16] = {}, pf = 0, c = 0, done = -1, xs = 8;
    for (int k = 0; k < FW; ++k) pw[k] = c++;
    for (int t = 0; t <= 2 * NT; ++t) {
        int need = t % WG || t == 2 * NT ? -1 : pw[t + WG - 1];
        if (t % 36 == G_C || (t && t % NT == 0)) need = pf > need ? pf : need;
        r.l[t] = need <= done ? -1 : c - 1 - need < 15 ? c - 1 - need : 15;
        done = need > done ? need : done;
        if (t + FW < 2 * NT) pw[t + FW] = c++;
        if (flags(t)) pf = c++;
        if (t < NT && t % 9 == 0) r.v[t / 9] = xs - 1 - t / 9;
        if (xfetch(t) >= 0) ++xs;
    }
    return r;
}
static_assert(FW > WG && NT % WG == 0, "parts");
constexpr int woff(int k) { return (k % 36 / 9 & 1) * W_STEP + k % 36 / 18 * 1024 + wcol(k % 9); }
template <int T, int E>
__device__ __forceinline__ void part_items(f4 (&acc)[FW][2], h8 (&a)[FW], h8 (&x)[8], Pos& P) {
    constexpr int left = waits().l[T];
    if constexpr (left >= 0 && (T > E - NT || T == 0)) asm volatile("s_waitcnt lgkmcnt(%0)" :: "i"(left) : "memory");
    if (T == E || T % 36 == G_C) asm volatile("" : "+v"(P.f));
    if constexpr (T < E) {
        constexpr int J = T / NT, N = T % NT / 36, G = T % 36, K = T + FW, X = (N & 1) * 4;
        if (G % 9 == 0 && (!J || N)) asm volatile("s_waitcnt vmcnt(%0)" :: "i"(waits().v[T % NT / 9]) : "memory");
        if constexpr (K < 2 * NT) MFMA_R(acc[T % FW][J], x[X + G / 9], a[T % FW], K % 9 < 8 ? P.w : P.n, woff(K));
        else MFMA(acc[T % FW][J], x[X + G / 9], a[T % FW]);
        SCHED_FENCE();
        if (K < 2 * NT && K % 36 >= 34) {
            constexpr int d = K % NT / 36 == NI - 1 ? (2 - NAB) * W_STEP : 2 * W_STEP;
            if (K % 36 == 34) ADV(P.w, d); else ADV(P.n, d);
        }
        if (flags(T)) FLAGS();
        if (!J && G == G_C && N < NI - 1) CHECK_AV(0u, KA + 2 * N + 4);
        if (G == G_C - 2 && N < NI - 2) { P.p[0] += 2 * X_STEP; PIN(P.p[0]); P.p[1] += 2 * X_STEP; PIN(P.p[1]); }
        if constexpr (xfetch(T) >= 0) XLD(x[X + xfetch(T)], xfetch(T));
        SCHED_FENCE();
        part_items<T + 1, E>(acc, a, x, P);
    }
}
__device__ __forceinline__ void item_x(h8* x, unsigned vx, const char* p, unsigned lo, unsigned hi) {
    asm volatile("s_cmp_gt_u32 %7, %8\n\ts_cbranch_scc1 1f\n\t"
                 "global_load_dwordx4 %0, %4, %5\n\tglobal_load_dwordx4 %1, %4, %6\n\t"
                 "global_load_dwordx4 %2, %4, %5 offset:1024\n\tglobal_load_dwordx4 %3, %4, %6 offset:1024\n1:"
                 : "+v"(x[0]), "+v"(x[1]), "+v"(x[2]), "+v"(x[3])
                 : "v"(vx), "s"(p), "s"(p + X_STEP), "s"(lo), "s"(hi) : "memory", "scc");
}
__device__ __forceinline__ void to_frags(u4 o0, u4 o1, unsigned wr, unsigned rd, u4& f0, u4& f1) {
    asm volatile("ds_write_b128 %2, %4\n\tds_write_b128 %2, %5 offset:1024\n\tds_read_b128 %0, %3\n\tds_read_b128 %1, %6"
                 : "=&v"(f0), "=&v"(f1) : "v"(wr), "v"(rd), "v"(o0), "v"(o1), "v"(rd ^ 64u) : "memory");
}

__global__ void __launch_bounds__(NTHREADS, 3) ag_gemm_kernel(Params p) {
    extern __shared__ __attribute__((aligned(16))) unsigned char smem[];
    LAS unsigned char* lds = (LAS unsigned char*)smem;
    const int tid = threadIdx.x;
    const int wave = __builtin_amdgcn_readfirstlane(tid >> 6);
    const int lane = tid & 63;
    const int b = blockIdx.x;
    const int xr = b & 7, idx = b >> 3;
    char* myws = p.myws;
    GAS unsigned* cctr = (GAS unsigned*)(myws + WS_CALL + (size_t)b * 128);
    GAS unsigned* tmo = (GAS unsigned*)(myws + WS_TMO);
    const unsigned lbase = (unsigned)(size_t)lds;
    const unsigned ctrl = lbase + CTRL_OFF;
    const unsigned lo16 = (unsigned)lane * 16u;
    const int r8 = lane >> 3, c8 = lane & 7;
    const int r16 = lane & 15, q = lane >> 4;
    const unsigned img = lbase + (unsigned)(r8 * 128 + (c8 ^ (r8 >> 1)) * 16);
    const unsigned frg = lbase + (unsigned)(r16 * 128 + (q ^ (r16 >> 1)) * 16);
    const char* __attribute__((address_space(4))) const* ka = (KPTR)__builtin_amdgcn_kernarg_segment_ptr();

    if (__builtin_expect(idx < 2 && wave < 2, 1)) {
        const int pid = (xr * 2 + idx) * 2 + wave;
        f4 v[K1][4];
        unsigned cv;
        {
            const unsigned s0 = (unsigned)((pid * 16 + r8) * GK + c8 * 8) * 4u;
            const unsigned s1 = (unsigned)((pid * 16 + 8 + r8) * GK + (c8 ^ 4) * 8) * 4u;
#pragma unroll
            for (int e = 0; e < K1; ++e) {
                asm volatile("global_load_dwordx4 %0, %4, %6\n\tglobal_load_dwordx4 %1, %4, %6 offset:16\n\t"
                             "global_load_dwordx4 %2, %5, %6\n\tglobal_load_dwordx4 %3, %5, %6 offset:16"
                             : "=&v"(v[e][0]), "=&v"(v[e][1]), "=&v"(v[e][2]), "=&v"(v[e][3])
                             : "v"(s0), "v"(s1), "s"(p.x + (KP0 + e * KPS) * BK) : "memory");
                if (e == 0) asm volatile("global_load_dword %0, %1, %2 sc1" : "=&v"(cv) : "v"(b * 128), "s"(myws + WS_CALL) : "memory");
            }
        }
        SCHED_FENCE();
        const char* dst[NR];
#pragma unroll
        for (int j = 0; j < NR; ++j) dst[j] = ka[offsetof(Params, xg) / 8 + ((xr * 2 + idx + 4 * wave) & 7) + j];
        unsigned vo = lo16 + (unsigned)((pid >> 4) * K1 * X_STEP + (pid & 15) * 2048), call = 0;
        char* flag = myws + WS_SFLAG + (size_t)(pid >> 4) * FSTRIDE + 512 + (pid & 15) * 4;
        u4 o[K1][2];
#pragma unroll
        for (int e = 0; e < K1; ++e) {
            if (e < 2) {
#pragma unroll
                for (int k = e; k < (e ? K1 : 1); ++k) {
                    asm volatile("s_waitcnt vmcnt(%5)" : "+v"(cv), "+v"(v[k][0]), "+v"(v[k][1]), "+v"(v[k][2]), "+v"(v[k][3])
                                 : "i"(4 * (K1 - 1 - k) + e * 2 * NR) : "memory");
                    to_frags(pk8(v[k][0], v[k][1]), pk8(v[k][2], v[k][3]), img + 32768u + wave * 2048u, frg + 32768u + wave * 2048u, o[k][0], o[k][1]);
                }
            }
            if (e == 0) {
                call = rfl(cv) + 1u;
                vo += (call & 1u) * (unsigned)(NR * 2 * K1 * X_STEP);
            }
            if (e == 1) __builtin_amdgcn_s_barrier();
            asm volatile("s_waitcnt lgkmcnt(%2)" : "+v"(o[e][0]), "+v"(o[e][1]) : "i"(e ? 4 * (K1 - 1 - e) : 0) : "memory");
            const unsigned ve = vo + (unsigned)(e * X_STEP);
#pragma unroll
            for (int j = 0; j < NR; ++j) {
                if (e) asm volatile("s_waitcnt vmcnt(%0)" :: "i"(PWIN - 2) : "memory");
                asm volatile("global_store_dwordx4 %0, %1, %3 sc0 sc1\n\tglobal_store_dwordx4 %0, %2, %3 offset:1024 sc0 sc1\n\ts_nop 1"
                             :: "v"(ve), "v"(o[e][0]), "v"(o[e][1]), "s"(dst[j]) : "memory");
            }
        }
        VM_WAIT(0);
        if (lane == 0) st4_sys(flag, call);
        __builtin_amdgcn_endpgm();
    }
    GAS unsigned* rotw = (GAS unsigned*)(myws + WS_WFLAG + (size_t)(idx & 15) * FSTRIDE + 256);
#define SRCWS(rot_) ka[offsetof(Params, ws) / 8 + ((xr + (rot_)) & 7)]
    if (idx < 2) {
        const char* hint = myws + WS_HINT + (size_t)(xr * 2 + idx) * FSTRIDE;
        if (wave == 7) lds_st(ctrl + lane * 4, 0u);
        const unsigned call = ld_ctr((const void*)cctr) + 1u;
        LGKM0();
        __builtin_amdgcn_s_barrier();
        asm volatile("" ::: "memory");
        unsigned long long t0 = 0;
        if (wave >= 3 && wave < 7) {
            const int qt = wave - 3;
            const char* hq = hint + qt * 4;
            unsigned rot;
            for (unsigned sp = 0;;) {
                rot = rfl(lds_ld(ctrl + C_ROT));
                if (rot || SPIN_OUT(sp, t0)) break;
                __builtin_amdgcn_s_sleep(1);
            }
            const char* srcws = SRCWS(rot);
            const char* xa = srcws + WS_XS + (size_t)((call & 1u) * 2u + (unsigned)idx) * NKS * X_STEP;
            unsigned hv = call * 256u + 1u, zero = 0u;
            asm volatile("s_mov_b32 m0, %2" : "+v"(hv), "+v"(zero) : "s"(lbase + qt * 4096) : "memory");
            int av = 0;
#define STAGED(n_) do { if (av < (n_)) for (unsigned _sp = 0;;) { av = (int)rfl(lds_ld(ctrl + C_AVAIL)); \
                if (av >= (n_) || SPIN_OUT(_sp, t0)) break; \
                __builtin_amdgcn_s_sleep(1); } } while (0)
#define PULL4(a_) asm volatile("s_waitcnt vmcnt(%2)\n\tglobal_load_lds_dwordx4 %0, %1\n\t" \
                         "s_waitcnt vmcnt(%2)\n\tglobal_load_lds_dwordx4 %0, %1 offset:1024\n\t" \
                         "s_waitcnt vmcnt(%2)\n\tglobal_load_lds_dwordx4 %0, %1 offset:2048\n\t" \
                         "s_waitcnt vmcnt(%2)\n\tglobal_load_lds_dwordx4 %0, %1 offset:3072" \
                         :: "v"(lo16), "s"(a_), "i"(WIN - 1) : "memory")
            unsigned long long own = srcws == myws ? rt() : 0;
#define PACE(n_) do { if (own) { own += n_; while (rt() < own) __builtin_amdgcn_s_sleep(2); } } while (0)
#pragma unroll 1
            for (int t = 0; t < KA; ++t) {
                STAGED(t + 1);
                PACE(OWN);
                const char* xt = xa + qt * 8192;
                if ((PMASK >> 1 >> t) & 1) { hv += 1u; xa += X_STEP; ++t; }
                post(hq, zero, hv);
                PULL4(xt);
                PULL4(xt + 4096);
                hv += 1u;
                xa += X_STEP;
            }
            STAGED(NKS);
            xa += (qt >> 1) * X_STEP + (qt & 1) * 4096;
#pragma unroll
            for (int t = KA; t < NITEMS; ++t) {
                PACE(OWN / 2);
                hv += 1u;
                post(hq, zero, hv);
                PULL4(xa + (t - KA) % NI * 2 * X_STEP + ((t - KA) / NI % 2 * 2 + (t - KA) / NAB) * 8192);
            }
            VM_WAIT(0);
        } else if (wave == 7) {
            if (lane == 0) __hip_atomic_store(cctr, call, __ATOMIC_RELAXED, __HIP_MEMORY_SCOPE_AGENT);
            __builtin_amdgcn_fence(__ATOMIC_ACQUIRE, "");
            unsigned rot, staged = 0, npoll = 0;
            bool stored = false;
            t0 = rt();
            do rot = ld_ctr((const void*)rotw) - call * 8u; while (rot > 7u && rt() - t0 <= DEADLINE);
            lds_st(ctrl + C_ROT, rot | 8u);
            post(hint + 16, 0u, call * 256u + KP0);
            const char* sf = SRCWS(rot) + WS_SFLAG + (size_t)idx * FSTRIDE;
            while (staged < (unsigned)NKS || !stored) {
                if (rot > 7u || ((++npoll & 31u) == 0u && (rfl(__hip_atomic_load(tmo, __ATOMIC_RELAXED, __HIP_MEMORY_SCOPE_AGENT)) == call
                                                           || rt() - t0 > DEADLINE))) {
                    __hip_atomic_store(tmo, call, __ATOMIC_RELAXED, __HIP_MEMORY_SCOPE_AGENT);
                    post(hint + 16, 0u, call * 256u);
                    VM_WAIT(0);
                    lds_st(ctrl + C_AVAIL, (unsigned)NKS);
                    break;
                }
                unsigned s2; bool p2;
                arrivals(sf, call, lane, s2, p2);
                if (s2 > staged) {
                    lds_st(ctrl + C_AVAIL, s2);
                    staged = s2;
                }
                if (p2 && !stored) { stored = true; post(hint + 16, 0u, call * 256u + NITEMS); }
                if (npoll < 64u && staged < (unsigned)NKS) __builtin_amdgcn_s_sleep(2); else __builtin_amdgcn_s_sleep(32);
            }
        }
        return;
    }

    const int mt = idx & 1, nt = (idx - 2) >> 1;
    const int f0 = (17 * nt) >> 1;
    const bool dup = !(nt & 1) && nt != 14;
    if (wave < 8) {
        f4 sv[4][2];
        const int g = b - 16;
        int sk = g >> 2;
#pragma unroll
        for (int e = 0; e < K1; ++e) sk += sk >= KP0 + e * KPS;
        const bool late = wave < 4 && sk >= EARLY;
        if (wave < 4) {
            const float* s0 = p.x + (size_t)((g & 3) * 128 + wave * 32 + r8) * GK + sk * BK;
            if (late) {
                __builtin_amdgcn_s_barrier();
                for (int i = EARLY; i < sk; ++i) __builtin_amdgcn_s_sleep(1);
            } else __builtin_amdgcn_s_sleep(8);
#pragma unroll
            for (int i = 0; i < 4; ++i)
                asm volatile("global_load_dwordx4 %0, %2, off nt\n\tglobal_load_dwordx4 %1, %2, off offset:16 nt"
                             : "=&v"(sv[i][0]), "=&v"(sv[i][1]) : "v"(s0 + (size_t)i * 8 * GK + (c8 ^ ((i & 1) * 4)) * 8) : "memory");
        }
        if (wave >= 4 && dup) {
            const u4 z = {0u, 0u, 0u, 0u};
            const unsigned za = lbase + (unsigned)((wave - 4) * 2 * W_STEP + 8 * W_BLK) + lo16;
            asm volatile("ds_write_b128 %0, %1\n\tds_write_b128 %0, %1 offset:1024\n\t"
                         "ds_write_b128 %0, %1 offset:%2\n\tds_write_b128 %0, %1 offset:%3\n\ts_waitcnt lgkmcnt(0)"
                         :: "v"(za), "v"(z), "i"(W_STEP), "i"(W_STEP + 1024) : "memory");
        }
        if (!late) __builtin_amdgcn_s_barrier();
        asm volatile("" ::: "memory");
        const unsigned call = rfl(lds_ld(ctrl + C_CALL));
        if (wave < 4) {
            char* d0 = myws + WS_XS + ((size_t)((call & 1u) * 2u + (unsigned)((g >> 1) & 1)) * NKS + (size_t)sk) * X_STEP
                       + (size_t)((g & 1) * X_PART + wave * 4096) + lo16;
            asm volatile("s_waitcnt vmcnt(0)" : "+v"(sv[0][0]), "+v"(sv[0][1]), "+v"(sv[1][0]), "+v"(sv[1][1]),
                         "+v"(sv[2][0]), "+v"(sv[2][1]), "+v"(sv[3][0]), "+v"(sv[3][1]) :: "memory");
            u4 o[4];
#pragma unroll
            for (int i = 0; i < 4; i += 2)
                to_frags(pk8(sv[i][0], sv[i][1]), pk8(sv[i + 1][0], sv[i + 1][1]), img + (unsigned)(SCR_OFF + wave * 2048),
                         frg + (unsigned)(SCR_OFF + wave * 2048), o[i], o[i + 1]);
            asm volatile("s_waitcnt lgkmcnt(0)" : "+v"(o[0]), "+v"(o[1]), "+v"(o[2]), "+v"(o[3]) :: "memory");
#pragma unroll
            for (int i = 0; i < 4; ++i) st16_sys(d0 + i * 1024, o[i]);
            unsigned n = 0;
            if (lane == 0) {
                asm volatile("s_waitcnt vmcnt(0)\n\tds_add_rtn_u32 %0, %1, %2\n\ts_waitcnt lgkmcnt(0)"
                             : "=&v"(n) : "v"(ctrl + C_STG), "v"(1u) : "memory");
                if (n == 3u) st4_sys(myws + WS_SFLAG + (size_t)((g >> 1) & 1) * FSTRIDE + sk * 8 + (g & 1) * 4, call);
            }
        }
        const int set = wave >> 2, w4 = wave & 3;
        f4 acc[FW][2];
#pragma unroll
        for (int i = 0; i < FW; ++i) {
            acc[i][0] = (f4){0.f, 0.f, 0.f, 0.f}; acc[i][1] = (f4){0.f, 0.f, 0.f, 0.f};
            asm volatile("" : "+v"(acc[i][0]), "+v"(acc[i][1]));
        }
        h8 a[FW], x[8];
#pragma unroll
        for (int i = 0; i < FW; ++i) asm volatile("" : "=v"(a[i]));
#pragma unroll
        for (int j = 0; j < 8; ++j) asm volatile("" : "=v"(x[j]));
        Pos P;
        Loop L;
        asm volatile("" : "=v"(P.f));
        P.w = lbase + (unsigned)((r16 >> 2) * W_BLK + q * 256 + (r16 & 3) * 16);
        P.n = lbase + (unsigned)(8 * W_BLK + q * 256 + (r16 & 3) * 64 + (r16 >> 2) * 16);
        P.x = lo16 + (unsigned)(set * X_PART + w4 * 2048);
        P.ldw = ctrl + C_LD - 4;
        P.rv = -4u; P.relw = ctrl + C_CONS + wave * 4; P.tab = myws + WS_TAB + (size_t)b * 512;
        unsigned vo = (unsigned)(q * NLOC + r16) * 16u, vn = (unsigned)(q * 4 * NLOC + r16) * 4u;
        asm volatile("" : "+v"(P.w), "+v"(P.n), "+v"(vn), "+v"(P.x), "+v"(P.rv), "+v"(P.ldw), "+v"(vo));
        WAIT_AV(1, 0);
        const unsigned rot = rfl(P.f.x);
        const int src = (p.srcs >> 4 * ((xr + rot) & 7)) & 7;
        L.xa = SRCWS(rot) + WS_XS + (size_t)((call & 1u) * 2u + (unsigned)mt) * NKS * X_STEP;
        L.xg = myws + WS_XG + (size_t)((((call & 1u) * NR + (unsigned)src) * 2u + (unsigned)mt) * K1) * X_STEP;
        L.set = set;
        const char* ol = (const char*)(p.out + (size_t)(src * MLOC + mt * BM + set * 128 + w4 * 16) * NLOC + f0 * 16);
        P.p[0] = L.xa; P.p[1] = L.xa + 8192;
        PIN(ol); PIN(P.relw); PIN(P.tab); PIN(L.xg); PIN(L.set); PIN(P.p[0]); PIN(P.p[1]);
#pragma unroll
        for (int k = 0; k < 4; ++k) XLD(x[k], k);
        L.t = 1u; P.need = 2u; L.xa += X_STEP; P.p[0] = L.xa; P.p[1] = L.xa + 8192;
        PIN(L.t); PIN(P.need); PIN(L.xa); PIN(P.p[0]); PIN(P.p[1]);
        WAIT_AV(0, 1);
        FLAGS();
#pragma unroll
        for (int i = 0; i < FW; ++i)
            asm volatile("ds_read_b128 %0, %1 offset:%2" : "+v"(a[i]) : "v"(i < 8 ? P.w : P.n), "i"(wcol(i)) : "memory");
        __builtin_amdgcn_s_waitcnt(0x0070);
        SCHED_FENCE();
#pragma unroll 1
        do {
            k_step<0, 0>(acc, a, x, P, L);
            k_step<1, 0>(acc, a, x, P, L);
        } while (L.t <= (unsigned)KA);
        P.need = (unsigned)(KA + 1 + NI * set);
        const char* xp = L.xa - X_STEP;
        unsigned pre = P.need - set;
#define TRY() do { const unsigned _p = rfl(P.f.w); item_x(x, P.x, xp, pre < P.need ? P.need : ~0u, _p); \
                   item_x(x + 4, P.x, xp + 2 * X_STEP, pre <= P.need ? P.need + 1u : ~0u, _p); pre = _p; } while (0)
#define SILU(r_, J_, keep_) silu4(acc[(r_) & 4][J_][(r_) & 3], acc[((r_) & 4) + 1][J_][(r_) & 3], acc[((r_) & 4) + 2][J_][(r_) & 3], \
                                  acc[((r_) & 4) + 3][J_][(r_) & 3], 60.f * LOG2E, c, keep_)
#define STORE(r_, o_) asm volatile("global_store_dwordx4 %0, %1, %2 offset:%3 nt" \
                                   :: "v"(vo), "v"(o_), "s"(ol + ((r_) & 3) * NLOC * 4), "i"(((r_) & 4) * 64) : "memory")
#define PART(J_) do { \
        SCHED_FENCE(); \
        if (pre < P.need) WAIT_AV(P.need, 0); \
        item_x(x, P.x, xp, pre + 1u, P.need); \
        if (pre <= P.need) WAIT_AV(P.need + 1u, KA + 2); \
        P.p[0] = xp + 2 * X_STEP; P.p[1] = xp + 3 * X_STEP; PIN(P.p[0]); PIN(P.p[1]); \
        item_x(x + 4, P.x, P.p[0], pre, P.need); \
        if (J_) { if (pre < P.need) VM_WAIT(4); else VM_WAIT(8); } \
        SCHED_FENCE(); \
        part_items<J_ * NT, J_ * NT + NT>(acc, a, x, P); \
        if (!J_) { xp += 8192; PIN(xp); P.need += NAB; pre = 0u; TRY(); } \
        const float c = __builtin_bit_cast(float, __builtin_bit_cast(unsigned, LOG2E) | rfl(P.f.z)); \
        asm volatile("s_nop 7\n\ts_nop 7" : "+v"(acc[FW - 3][J_]), "+v"(acc[FW - 2][J_]), "+v"(acc[FW - 1][J_])); \
        f4 o = acc[FW - 1][J_], od[4]; \
        _Pragma("unroll") for (int r = 0; r < 4; ++r) { o = SILU(r, J_, o); STORE(r, o); } \
        if (!dup) { \
            const f4 on = silu4(acc[FW - 1][J_][0], acc[FW - 1][J_][1], acc[FW - 1][J_][2], acc[FW - 1][J_][3], 60.f * LOG2E, c, o); \
            _Pragma("unroll") for (int r = 0; r < 4; ++r) \
                asm volatile("global_store_dword %0, %1, %2 offset:512 nt" :: "v"(vn), "v"(on[r]), "s"(ol + r * NLOC * 4) : "memory"); \
        } \
        _Pragma("unroll") for (int r = 4; r < 8; ++r) {        \
            if (!J_ && (r == 4 || r == 7)) FLAGS(); \
            o = od[r - 4] = SILU(r, J_, o); \
            if (J_) STORE(r, o); \
            if (!J_ && (r == 4 || r == 7)) { asm volatile("s_waitcnt lgkmcnt(0)" : "+v"(P.f) :: "memory"); TRY(); } \
        } \
        if (!J_) _Pragma("unroll") for (int r = 4; r < 8; ++r) STORE(r, od[r - 4]); \
        asm volatile("s_nop 1" ::: "memory"); } while (0)
        PART(0);
        ADV(vo, 64 * NLOC * 4); ADV(vn, 64 * NLOC * 4);
        PART(1);
        __builtin_amdgcn_endpgm();
    }
    if (wave == 10) {
        *(u2*)(myws + WS_TAB + (size_t)b * 512 + lane * 8) = (u2){2u * lane + 1u, 2u * lane + 2u};
        const unsigned cl = ld_ctr((const void*)cctr) + 1u;
        if (b == NWG - 8 && lane < 16)
            __hip_atomic_store((GAS unsigned*)(myws + WS_WFLAG + (size_t)lane * FSTRIDE + 256),
                               cl * 8u + (__builtin_amdgcn_s_getreg((3 << 11) | 20) & 7u), __ATOMIC_RELAXED, __HIP_MEMORY_SCOPE_AGENT);
        if (lane == 0) __hip_atomic_store(cctr, cl, __ATOMIC_RELAXED, __HIP_MEMORY_SCOPE_AGENT);
        lds_st(ctrl + lane * 4, lane == C_CALL / 4 ? cl : 0u);
        LGKM0();
    }
    __builtin_amdgcn_s_barrier();
    asm volatile("" ::: "memory");
    const unsigned call = rfl(lds_ld(ctrl + C_CALL));
    if (wave == 10) {
        __builtin_amdgcn_fence(__ATOMIC_ACQUIRE, "");
        VM_WAIT(0);
        GAS unsigned* hq = (GAS unsigned*)(myws + WS_HINT + (size_t)(xr * 2 + mt) * FSTRIDE) + c8;
        GAS unsigned* hp = lane == 5 ? rotw : hq;
        GAS unsigned* wf0 = (GAS unsigned*)(myws + WS_WFLAG + (size_t)(f0 >> 3) * FSTRIDE) + lane;
        GAS unsigned* wf1 = (GAS unsigned*)(myws + WS_WFLAG + (size_t)((f0 + FW - 1) >> 3) * FSTRIDE) + lane;
        unsigned pub = 0, wav = 0, npoll = 0, rot = 8u;
        const unsigned long long t0 = rt();
        unsigned long long tall = 0;
        bool dead = false;
        while (pub < (unsigned)NITEMS || wav < (unsigned)NKS) {
            const unsigned hv = __hip_atomic_load(hp, __ATOMIC_RELAXED, __HIP_MEMORY_SCOPE_AGENT);
            bool adv = false;
            if (wav < (unsigned)NKS) {
                const unsigned w0 = __hip_atomic_load(wf0, __ATOMIC_RELAXED, __HIP_MEMORY_SCOPE_AGENT);
                const unsigned w1 = __hip_atomic_load(wf1, __ATOMIC_RELAXED, __HIP_MEMORY_SCOPE_AGENT);
                const unsigned long long notready = __ballot((int)(w0 - call) < 0 || (int)(w1 - call) < 0);
                const unsigned w2 = notready ? (unsigned)__builtin_ctzll(notready) : 64u;
                if (w2 > wav) { wav = w2; lds_st(ctrl + C_WAV, wav); adv = true; }
            }
            const int d = (int)(hv - call * 256u);
            const unsigned av = (unsigned)row_min(c8 < 5 ? max(min(d, NITEMS), 0) : NITEMS);
            if (rot > 7u) {
                rot = (unsigned)__builtin_amdgcn_readlane((int)hv, 5) - call * 8u;
                if (rot <= 7u) { lds_st(ctrl + C_ROT, rot); hp = hq; }
            }
            if (av > pub && rot <= 7u) {
                asm volatile("ds_max_u32 %0, %1" :: "v"(lane ? ctrl + 128 + lane * 4 : ctrl + C_AVAIL), "v"(av) : "memory");
                pub = av;
                tall = 0;
            } else if (!adv && (++npoll & 63u) == 0u) {
                if (rt() - t0 > DEADLINE) { dead = true; break; }
                unsigned staged; bool stored;
                arrivals(SRCWS(rot) + WS_SFLAG + (size_t)mt * FSTRIDE, call, lane, staged, stored);
                if (staged == (unsigned)NKS && stored && rot <= 7u) {
                    if (tall == 0) tall = rt();
                    else if (rt() - tall > SILENCE) { pub = NITEMS; lds_st(ctrl + C_AVAIL, pub); }
                } else if (rfl(__hip_atomic_load(tmo, __ATOMIC_RELAXED, __HIP_MEMORY_SCOPE_AGENT)) == call) { dead = true; break; }
            }
            if (pub < 48u) __builtin_amdgcn_s_sleep(4); else __builtin_amdgcn_s_sleep(1);
        }
        if (dead) {
            __hip_atomic_store(tmo, call, __ATOMIC_RELAXED, __HIP_MEMORY_SCOPE_AGENT);
            lds_st(ctrl + C_BAD, 0x7fc00000u);
        }
        lds_st(ctrl + C_WAV, (unsigned)NKS);
        lds_st(ctrl + C_AVAIL, (unsigned)NITEMS);
    } else if (wave == 9) {
        constexpr int CSTEP = 240 / NCONV;
        const int cq = ((idx - 2) / CSTEP) * 8 + xr;
        if ((idx - 2) % CSTEP == 0 && cq < NCONV) {
            const unsigned so = (unsigned)(r8 * 8 * NLOC + c8 * 4) * 4u;
            const unsigned dof = (unsigned)((c8 >> 2) * 2048 + r8 * 256 + (c8 & 3) * 16);
            for (int i = -17; i < cq; i += 16) __builtin_amdgcn_s_sleep(20);
            for (int u = cq; u < NKS * 16; u += NCONV) {
                const int ks = u >> 4, cn = u & 15;
                const float* sw = p.w + (size_t)ks * BK * NLOC + cn * 128;
                const char* dstw = myws + WS_WH + (size_t)ks * W_ROW + (size_t)cn * 16384;
                f4 v[4][8];
#pragma unroll
                for (int g = 0; g < 4; ++g)
#pragma unroll
                    for (int r = 0; r < 8; ++r)
                        asm volatile("global_load_dwordx4 %0, %1, %2 offset:%3 nt" : "=v"(v[g][r]) : "v"(so), "s"(sw + r * NLOC), "i"(g * 128) : "memory");
                u4 o[4][4];
#pragma unroll
                for (int g = 0; g < 4; ++g) {
                    asm volatile("s_waitcnt vmcnt(%8)" : "+v"(v[g][0]), "+v"(v[g][1]), "+v"(v[g][2]), "+v"(v[g][3]),
                                 "+v"(v[g][4]), "+v"(v[g][5]), "+v"(v[g][6]), "+v"(v[g][7]) : "i"(24 - 8 * g) : "memory");
#pragma unroll
                    for (int r = 0; r < 8; ++r) v[g][r] *= LOG2E;
#pragma unroll
                    for (int c = 0; c < 4; ++c)
                        o[g][c] = (u4){pk(v[g][0][c], v[g][1][c]), pk(v[g][2][c], v[g][3][c]), pk(v[g][4][c], v[g][5][c]), pk(v[g][6][c], v[g][7][c])};
                    SCHED_FENCE();
                }
#pragma unroll
                for (int g = 0; g < 4; ++g)
#pragma unroll
                    for (int c = 0; c < 4; ++c)
                        asm volatile("global_store_dwordx4 %0, %1, %2 offset:%3 sc1 nt\n\ts_nop 1"
                                     :: "v"(dof), "v"(o[g][c]), "s"(dstw + g * 4096), "i"(c * 64) : "memory");
                VM_WAIT(0);
                if (lane == 0)
                    __hip_atomic_store((GAS unsigned*)(myws + WS_WFLAG + (size_t)cn * FSTRIDE + ks * 4), call,
                                       __ATOMIC_RELAXED, __HIP_MEMORY_SCOPE_AGENT);
            }
        }
    } else {
        const char* tab = myws + WS_TAB + (size_t)b * 512;
        const char* wa = myws + WS_WH + (size_t)f0 * 2048;
        unsigned long long t0 = 0;
        int av = 0, cons = 0;
#pragma unroll 1
        for (int t = 0; t < NKS; ++t) {
            if (av <= t || cons <= t - NW) for (unsigned sp = 0;;) {
                u4 c, d; unsigned v;
                asm volatile("ds_read_b128 %0, %3\n\tds_read_b128 %1, %3 offset:16\n\tds_read_b32 %2, %3 offset:%4\n\t"
                             "s_waitcnt lgkmcnt(0)" : "=&v"(c), "=&v"(d), "=&v"(v) : "v"(ctrl + C_CONS), "i"(C_WAV - C_CONS) : "memory");
                av = (int)rfl(v); cons = (int)min8(c, d);
                if ((av > t && cons > t - NW) || SPIN_OUT(sp, t0)) break;
                __builtin_amdgcn_s_sleep(1);
            }
            const unsigned sb = lbase + (unsigned)(t & (NW - 1)) * W_STEP;
#define WLOADS(P_) _Pragma("unroll") for (int i = 0; i < FW; i += 2) { \
                if (i == FW - 1 && dup) break; \
                asm volatile("s_mov_b32 m0, %0\n\ts_waitcnt vmcnt(58)\n\t" \
                             "global_load_lds_dwordx4 %1, %2" P_ "\n\tglobal_load_lds_dwordx4 %1, %2 offset:1024" P_ \
                             :: "s"(sb + i * W_BLK), "v"(lo16), "s"(wa + i * 2048) : "memory"); \
                if (i + 1 < FW) asm volatile("s_mov_b32 m0, %0\n\ts_nop 0\n\t" \
                             "global_load_lds_dwordx4 %1, %2 offset:2048" P_ "\n\tglobal_load_lds_dwordx4 %1, %2 offset:3072" P_ \
                             :: "s"(sb + i * W_BLK + 64), "v"(lo16), "s"(wa + i * 2048) : "memory"); \
            }
            if (mt == 0 && t >= WNT0 && t < WNT1) { WLOADS(" nt") } else { WLOADS("") }
            asm volatile("s_mov_b32 m0, %0\n\ts_mov_b64 exec, 1\n\tglobal_load_lds_dword %1, %2\n\ts_mov_b64 exec, -1"
                         :: "s"(ctrl + C_LD), "v"(lo16), "s"(tab + t * 4) : "memory");
            wa += W_ROW;
        }
        VM_WAIT(0);
    }
}

}

extern "C" void kernel_launch(int rank, int nranks, ncclComm_t comm,
                              hipStream_t stream,
                              void* const* d_in, const int* in_sizes, int n_in,
                              void* d_out, int out_size,
                              const int* in_dtypes, int out_dtype,
                              void* const* d_out_all,
                              void* const* d_ws_all, size_t ws_bytes) {
    (void)comm; (void)in_dtypes; (void)out_dtype; (void)d_out_all;
    if (nranks != NR || n_in < 2 || (unsigned)rank >= (unsigned)NR || ws_bytes < WS_NEED ||
        in_sizes[0] != MLOC * GK || in_sizes[1] != GK * NLOC || out_size != GM * NLOC) {
        fprintf(stderr, "ag_gemm: unexpected problem shape\n");
        return;
    }
    static bool attr_set = false;
    if (!attr_set) {
        HIP_CHECK(hipFuncSetAttribute((const void*)ag_gemm_kernel,
                                      hipFuncAttributeMaxDynamicSharedMemorySize, LDS_BYTES));
        attr_set = true;
    }
    Params p;
    p.x = (const float*)d_in[0];
    p.w = (const float*)d_in[1];
    p.out = (float*)d_out;
    p.myws = (char*)d_ws_all[rank];
    p.srcs = 0;
    const unsigned half = 0xb2;
    for (int far = 0, n = 0; far < 2; ++far)
        for (int r = 0; r < NR; ++r) {
            const int s = (rank + r) & 7;
            if ((int)((half >> s ^ half >> rank) & 1) != far) continue;
            p.ws[n] = (char*)d_ws_all[s];
            p.srcs |= (unsigned)s << 4 * n;
            ++n;
        }
    for (int r = 0; r < 2 * NR; ++r) p.xg[r] = (char*)d_ws_all[(rank + 1 + r) & 7] + WS_XG + (size_t)rank * 2 * K1 * X_STEP;
    hipLaunchKernelGGL(ag_gemm_kernel, dim3(NWG), dim3(NTHREADS), LDS_BYTES, stream, p);
    HIP_CHECK(hipGetLastError());
}
```

```cpp
#include <hip/hip_runtime.h>
#include <rccl/rccl.h>
#include <cstdio>
#include <cstddef>

#define HIP_CHECK(expr) do { hipError_t _e = (expr); if (_e != hipSuccess) { \
    fprintf(stderr, "HIP error %s at %s:%d\n", hipGetErrorString(_e), __FILE__, __LINE__); \
    return; } } while (0)

namespace {

#define GAS __attribute__((address_space(1)))
#define LAS __attribute__((address_space(3)))
typedef _Float16 h8 __attribute__((ext_vector_type(8)));
typedef _Float16 h2 __attribute__((ext_vector_type(2)));
typedef float f4 __attribute__((ext_vector_type(4)));
typedef unsigned u4 __attribute__((ext_vector_type(4)));
typedef unsigned u2 __attribute__((ext_vector_type(2)));

constexpr int NR = 8;
constexpr int GM = 4096, GK = 4096, NLOC = 2048, MLOC = 512;
constexpr int BM = 256, BK = 64;
constexpr int FW = 9;
constexpr int NKS = GK / BK;
constexpr int K1 = 4;
constexpr int KP0 = 30, KPS = 6;
constexpr unsigned long long PMASK = (1ull | 1ull << KPS | 1ull << 2 * KPS | 1ull << 3 * KPS) << KP0;
constexpr int NAB = 8, NI = NAB / 2;
constexpr int KA = NKS - NAB;
constexpr int NITEMS = KA + 4 * NI;
constexpr int X_STEP = BM * BK * 2;
constexpr int X_PART = X_STEP / 2;
constexpr int W_BLK = 2048 + 64;
constexpr int W_STEP = FW * W_BLK - 64;
constexpr int W_ROW = NLOC * BK * 2;
constexpr int NW = 8;
constexpr int SCR_OFF = NW * W_STEP;
constexpr int CTRL_OFF = SCR_OFF + 8192;
constexpr int LDS_BYTES = CTRL_OFF + 512;
constexpr int NWG = 256, NTHREADS = 704;
static_assert(LDS_BYTES <= 160 * 1024 && NAB <= NW && NAB % 2 == 0 && NAB >= 4 && NKS % NW == 0 && NW == 8, "layout");
static_assert((NKS - K1) * 4 == NWG - 16 && KP0 > 1 && KPS > 1 && KP0 + 3 * KPS < KA - 1, "layout");
constexpr int C_CONS = 0;
constexpr int C_WAV = 68;
constexpr int C_STG = 72;
constexpr int C_CALL = 76;
constexpr int C_ROT = 80;
constexpr int C_LD = 84;
constexpr int C_BAD = 88;
constexpr int C_AVAIL = 92;
constexpr int NCONV = 120;
constexpr float LOG2E = 1.4426950408889634f;
constexpr int PWIN = 16;
constexpr int WIN = 8;
constexpr int OWN = 85;
constexpr int EARLY = 12;
constexpr int WNT0 = 8, WNT1 = 32;

constexpr size_t FSTRIDE = 4096 + 256;
constexpr size_t WS_CALL = 0;
constexpr size_t WS_TMO = 49152;
constexpr size_t WS_SFLAG = 53248;
constexpr size_t WS_HINT = 65536;
constexpr size_t WS_WFLAG = 196608;
constexpr size_t WS_TAB = 327680;
constexpr size_t WS_WH = 1ull << 20;
constexpr size_t WS_XS = 32ull << 20;
constexpr size_t WS_XG = 48ull << 20;
constexpr size_t WS_NEED = 64ull << 20;

constexpr unsigned long long DEADLINE = 100000000ull;
constexpr unsigned long long SILENCE = 20000ull;

struct Params {
    const float* x;
    char* myws;
    const float* w;
    float* out;
    char* ws[NR];
    char* xg[2 * NR];
    unsigned srcs;
};
typedef const char* __attribute__((address_space(4))) const* KPTR;

__device__ __forceinline__ unsigned long long rt() { return __builtin_amdgcn_s_memrealtime(); }
__device__ __forceinline__ unsigned xcd() { return __builtin_amdgcn_s_getreg((3 << 11) | 20) & 7u; }

__device__ __forceinline__ void st16_sys(void* p, u4 v) {
    asm volatile("global_store_dwordx4 %0, %1, off sc0 sc1\n\ts_nop 1" :: "v"(p), "v"(v) : "memory");
}
__device__ __forceinline__ void st4_sys(void* p, unsigned v) {
    asm volatile("global_store_dword %0, %1, off sc0 sc1\n\ts_nop 1" :: "v"(p), "v"(v) : "memory");
}
__device__ __forceinline__ void post(const char* p, unsigned zero, unsigned v) {
    asm volatile("s_mov_b64 exec, 1\n\tglobal_store_dword %0, %1, %2\n\ts_mov_b64 exec, -1" :: "v"(zero), "v"(v), "s"(p) : "memory");
}
__device__ __forceinline__ void lds_st(unsigned addr, unsigned v) {
    asm volatile("ds_write_b32 %0, %1" :: "v"(addr), "v"(v) : "memory");
}
__device__ __forceinline__ unsigned lds_ld(unsigned addr) {
    unsigned v;
    asm volatile("ds_read_b32 %0, %1\n\ts_waitcnt lgkmcnt(0)" : "=&v"(v) : "v"(addr) : "memory");
    return v;
}
__device__ __forceinline__ unsigned ld_ctr(const void* p) {
    unsigned v;
    asm volatile("global_load_dword %0, %1, off sc1\n\ts_waitcnt vmcnt(0)" : "=&v"(v) : "v"(p) : "memory");
    return (unsigned)__builtin_amdgcn_readfirstlane(v);
}
__device__ __forceinline__ unsigned pk(float a, float b) {
    h2 h; h.x = (_Float16)a; h.y = (_Float16)b;
    return __builtin_bit_cast(unsigned, h);
}
__device__ __forceinline__ u4 pk8(f4 a, f4 b) { return (u4){pk(a[0], a[1]), pk(a[2], a[3]), pk(b[0], b[1]), pk(b[2], b[3])}; }
__device__ __forceinline__ unsigned umin(unsigned a, unsigned b) { return a < b ? a : b; }
__device__ __forceinline__ unsigned rfl(unsigned v) { return (unsigned)__builtin_amdgcn_readfirstlane((int)v); }
__device__ __forceinline__ unsigned min8(u4 c, u4 d) {
    return rfl(umin(umin(umin(c.x, c.y), umin(c.z, c.w)), umin(umin(d.x, d.y), umin(d.z, d.w))));
}
__device__ __forceinline__ int row_min(int u) {
    u = min(u, __builtin_amdgcn_update_dpp(0, u, 0x121, 15, 15, false));
    u = min(u, __builtin_amdgcn_update_dpp(0, u, 0x122, 15, 15, false));
    u = min(u, __builtin_amdgcn_update_dpp(0, u, 0x124, 15, 15, false));
    u = min(u, __builtin_amdgcn_update_dpp(0, u, 0x128, 15, 15, false));
    return __builtin_amdgcn_readfirstlane(u);
}
__device__ __forceinline__ void arrivals(const char* sf, unsigned call, int lane, unsigned& staged, bool& stored) {
    u2 sv; unsigned pv;
    asm volatile("global_load_dwordx2 %0, %2, %4 sc0 sc1\n\tglobal_load_dword %1, %3, %4 offset:512 sc0 sc1\n\ts_waitcnt vmcnt(0)"
                 : "=&v"(sv), "=&v"(pv) : "v"(lane * 8), "v"((lane & 15) * 4), "s"(sf) : "memory");
    const unsigned long long miss = __ballot((int)(sv.x - call) < 0 || (int)(sv.y - call) < 0) & ~PMASK;
    staged = miss ? (unsigned)__builtin_ctzll(miss) : (unsigned)NKS;
    stored = __ballot((int)(pv - call) < 0) == 0;
}
#define VM_WAIT(n) asm volatile("s_waitcnt vmcnt(" #n ")" ::: "memory")
#define ACQUIRE() do { __builtin_amdgcn_fence(__ATOMIC_ACQUIRE, ""); VM_WAIT(0); } while (0)
#define LGKM0() asm volatile("s_waitcnt lgkmcnt(0)" ::: "memory")
#define SCHED_FENCE() __builtin_amdgcn_sched_barrier(0)
#define SPIN_OUT(sp_, t0_) ((((++(sp_)) & 1023u) == 0u) && ((t0_) == 0 ? ((t0_) = rt(), false) : ((unsigned)(rt() - (t0_)) > 2u * (unsigned)DEADLINE)))

__device__ __forceinline__ f4 silu4(float z0, float z1, float z2, float z3, float lim, float c, f4 keep) {
    float o0, o1, o2, o3;
    asm volatile("v_med3_f32 %0, -%4, %8, -%8\n\tv_med3_f32 %1, -%5, %8, -%8\n\tv_med3_f32 %2, -%6, %8, -%8\n\tv_med3_f32 %3, -%7, %8, -%8\n\t"
                 "v_exp_f32 %0, %0\n\tv_exp_f32 %1, %1\n\tv_exp_f32 %2, %2\n\tv_exp_f32 %3, %3\n\t"
                 "v_fma_f32 %0, %0, %9, %9\n\tv_fma_f32 %1, %1, %9, %9\n\tv_fma_f32 %2, %2, %9, %9\n\tv_fma_f32 %3, %3, %9, %9\n\t"
                 "v_rcp_f32 %0, %0\n\tv_rcp_f32 %1, %1\n\tv_rcp_f32 %2, %2\n\tv_rcp_f32 %3, %3\n\t"
                 "v_mul_f32 %0, %4, %0\n\tv_mul_f32 %1, %5, %1\n\tv_mul_f32 %2, %6, %2\n\tv_mul_f32 %3, %7, %3"
                 : "=&v"(o0), "=&v"(o1), "=&v"(o2), "=&v"(o3) : "v"(z0), "v"(z1), "v"(z2), "v"(z3), "s"(lim), "s"(c), "v"(keep));
    return (f4){o0, o1, o2, o3};
}
#define REL() asm volatile("s_mov_b32 m0, %0\n\ts_mov_b64 exec, 1\n\tglobal_load_lds_dword %1, %2\n\ts_mov_b64 exec, -1" \
                           :: "s"(P.relw), "v"(P.rv), "s"(P.tab) : "memory")
struct Pos { unsigned w, n, x, rv, need, relw, ldw; const char* tab; const char* p[2]; u4 f; };
#define ADV(v_, d_) do { v_ += d_; asm volatile("" : "+v"(v_)); } while (0)
#define PIN(s_) asm volatile("" : "+s"(s_))
#define FLAGS() asm volatile("ds_read_b128 %0, %1" : "=v"(P.f) : "v"(P.ldw) : "memory")
#define WAIT_AV(nx_, nw_) do { unsigned long long _t0 = 0; unsigned _sp = 0; \
        for (;;) { asm volatile("ds_read_b128 %0, %1\n\ts_waitcnt lgkmcnt(0)" : "+v"(P.f) : "v"(P.ldw) : "memory"); \
            if ((int)((rfl(P.f.w) - (unsigned)(nx_)) | (rfl(P.f.y) - (unsigned)(nw_))) >= 0) break; \
            if (SPIN_OUT(_sp, _t0)) break; \
            __builtin_amdgcn_s_sleep(1); } } while (0)
#define CHECK_AV(nx_, nw_) do { if (__builtin_expect((int)((rfl(P.f.w) - (unsigned)(nx_)) | (rfl(P.f.y) - (unsigned)(nw_))) < 0, 0)) \
            WAIT_AV(nx_, nw_); SCHED_FENCE(); } while (0)
#define MFMA(c_, x_, a_) asm volatile("v_mfma_f32_16x16x32_f16 %0, %1, %2, %0" : "+v"(c_) : "v"(x_), "v"(a_))
#define MFMA_R(c_, x_, a_, ad_, off_) asm volatile("v_mfma_f32_16x16x32_f16 %0, %2, %1, %0\n\tds_read_b128 %1, %3 offset:%4" \
                                                   : "+v"(c_), "+v"(a_) : "v"(x_), "v"(ad_), "i"(off_) : "memory")
#define XLD(x_, k_) asm volatile("global_load_dwordx4 %0, %1, %2 offset:%3" : "+v"(x_) : "v"(P.x), "s"(P.p[(k_) & 1]), "i"(((k_) & 2) * 512) : "memory")
constexpr int wcol(int i) { return i < 8 ? (i >> 2) * 4 * W_BLK + (i & 3) * 64 : 0; }

struct Loop { const char* xa; const char* xg; unsigned t, set; };
template <int B, int G>
__device__ __forceinline__ void k_step(f4 (&acc)[FW][2], h8 (&a)[FW], h8 (&x)[8], Pos& P, Loop& L) {
    if constexpr (G < 36) {
        constexpr int h = G / 18, i = G % 18 / 2;
        if (G % 18 == 0) asm volatile("s_waitcnt vmcnt(%0) lgkmcnt(7)" :: "i"(h ? 5 : 3) : "memory");
        else if (G == 2) asm volatile("s_waitcnt lgkmcnt(4)" : "+v"(P.f) :: "memory");
        else if (G % 18 % 4 == 0) asm volatile("s_waitcnt lgkmcnt(7)" ::: "memory");
        if (G & 1) MFMA_R(acc[i][1], x[B * 4 + h * 2 + 1], a[i], i < 8 ? P.w : P.n, (h ? 1 - B : B) * W_STEP + (1 - h) * 1024 + wcol(i));
        else MFMA(acc[i][0], x[B * 4 + h * 2], a[i]);
        SCHED_FENCE();
        if (G == 2) CHECK_AV(P.need, L.t + 1u);
        if (G >= 4 && G <= 10 && !(G & 1)) XLD(x[4 - B * 4 + G / 2 - 2], G / 2 - 2);
        if (B && G == 18) { const int d = (L.t & (NW - 1)) ? 2 * W_STEP : (2 - NW) * W_STEP; ADV(P.w, d); ADV(P.n, d); }
        if (G == 20) { L.t += 1u; PIN(L.t); L.xa += X_STEP; PIN(L.xa); }
        if (G == 22) {
            const bool st = (PMASK >> L.t) & 1;
            P.p[0] = st ? L.xg : L.xa; PIN(P.p[0]);
            L.xg += st ? X_STEP : 0; PIN(L.xg);
        }
        if (G == 24) {
            P.need = L.t + 1u;
            if (B) P.p[1] = P.p[0] + 8192;
            else {
                const bool last = L.t == (unsigned)KA;
                P.p[0] -= last && L.set ? X_STEP : 0;
                P.p[1] = P.p[0] + (last && !L.set ? X_STEP : 8192);
                P.need = last && L.set ? 0u : P.need;
                PIN(P.p[0]);
            }
            PIN(P.p[1]); PIN(P.need);
        }
        if (G == 30) FLAGS();
        if (G == 32) ADV(P.rv, 4u);
        if (G == 34) REL();
        SCHED_FENCE();
        k_step<B, G + 1>(acc, a, x, P, L);
    }
}
constexpr int NT = 36 * NI, WG = 3, G_C = 35 - FW;
constexpr bool flags(int t) {
    return t % NT >= NT - 36 ? t % 36 == 26 : t < NT && t % 36 == G_C - 9;
}
constexpr int xfetch(int t) {
    for (int k = 0; k < 4; ++k) if (t % 36 == (G_C + 1 + k > 9 * k + 8 ? G_C + 1 + k : 9 * k + 8)) return t % NT < NT - 72 ? k : -1;
    return -1;
}
struct Waits { int l[2 * NT + 1], v[NT / 9]; };
constexpr Waits waits() {
    Waits r{};
    int pw[2 * NT + 16] = {}, pf = 0, c = 0, done = -1, xs = 8;
    for (int k = 0; k < FW; ++k) pw[k] = c++;
    for (int t = 0; t <= 2 * NT; ++t) {
        int need = t % WG || t == 2 * NT ? -1 : pw[t + WG - 1];
        if (t % 36 == G_C || (t && t % NT == 0)) need = pf > need ? pf : need;
        r.l[t] = need <= done ? -1 : c - 1 - need < 15 ? c - 1 - need : 15;
        done = need > done ? need : done;
        if (t + FW < 2 * NT) pw[t + FW] = c++;
        if (flags(t)) pf = c++;
        if (t < NT && t % 9 == 0) r.v[t / 9] = xs - 1 - t / 9;
        if (xfetch(t) >= 0) ++xs;
    }
    return r;
}
static_assert(FW > WG && NT % WG == 0, "parts");
constexpr int woff(int k) { return (k % 36 / 9 & 1) * W_STEP + k % 36 / 18 * 1024 + wcol(k % 9); }
template <int T, int E>
__device__ __forceinline__ void part_items(f4 (&acc)[FW][2], h8 (&a)[FW], h8 (&x)[8], Pos& P) {
    constexpr int left = waits().l[T];
    if constexpr (left >= 0 && (T > E - NT || T == 0)) asm volatile("s_waitcnt lgkmcnt(%0)" :: "i"(left) : "memory");
    if (T == E || T % 36 == G_C) asm volatile("" : "+v"(P.f));
    if constexpr (T < E) {
        constexpr int J = T / NT, N = T % NT / 36, G = T % 36, K = T + FW, X = (N & 1) * 4;
        if (G % 9 == 0 && (!J || N)) asm volatile("s_waitcnt vmcnt(%0)" :: "i"(waits().v[T % NT / 9]) : "memory");
        if constexpr (K < 2 * NT) MFMA_R(acc[T % FW][J], x[X + G / 9], a[T % FW], K % 9 < 8 ? P.w : P.n, woff(K));
        else MFMA(acc[T % FW][J], x[X + G / 9], a[T % FW]);
        SCHED_FENCE();
        if (K < 2 * NT && K % 36 >= 34) {
            constexpr int d = K % NT / 36 == NI - 1 ? (2 - NAB) * W_STEP : 2 * W_STEP;
            if (K % 36 == 34) ADV(P.w, d); else ADV(P.n, d);
        }
        if (flags(T)) FLAGS();
        if (!J && G == G_C && N < NI - 1) CHECK_AV(0u, KA + 2 * N + 4);
        if (G == G_C - 2 && N < NI - 2) { P.p[0] += 2 * X_STEP; PIN(P.p[0]); P.p[1] += 2 * X_STEP; PIN(P.p[1]); }
        if constexpr (xfetch(T) >= 0) XLD(x[X + xfetch(T)], xfetch(T));
        SCHED_FENCE();
        part_items<T + 1, E>(acc, a, x, P);
    }
}
__device__ __forceinline__ void item_x(h8* x, unsigned vx, const char* p, unsigned lo, unsigned hi) {
    asm volatile("s_cmp_gt_u32 %7, %8\n\ts_cbranch_scc1 1f\n\t"
                 "global_load_dwordx4 %0, %4, %5\n\tglobal_load_dwordx4 %1, %4, %6\n\t"
                 "global_load_dwordx4 %2, %4, %5 offset:1024\n\tglobal_load_dwordx4 %3, %4, %6 offset:1024\n1:"
                 : "+v"(x[0]), "+v"(x[1]), "+v"(x[2]), "+v"(x[3])
                 : "v"(vx), "s"(p), "s"(p + X_STEP), "s"(lo), "s"(hi) : "memory", "scc");
}
__device__ __forceinline__ void to_frags(u4 o0, u4 o1, unsigned wr, unsigned rd, u4& f0, u4& f1) {
    asm volatile("ds_write_b128 %2, %4\n\tds_write_b128 %2, %5 offset:1024\n\tds_read_b128 %0, %3\n\tds_read_b128 %1, %6"
                 : "=&v"(f0), "=&v"(f1) : "v"(wr), "v"(rd), "v"(o0), "v"(o1), "v"(rd ^ 64u) : "memory");
}

__global__ void __launch_bounds__(NTHREADS, 3) ag_gemm_kernel(Params p) {
    extern __shared__ __attribute__((aligned(16))) unsigned char smem[];
    LAS unsigned char* lds = (LAS unsigned char*)smem;
    const int tid = threadIdx.x;
    const int wave = __builtin_amdgcn_readfirstlane(tid >> 6);
    const int lane = tid & 63;
    const int b = blockIdx.x;
    const int xr = b & 7, idx = b >> 3;
    char* myws = p.myws;
    GAS unsigned* cctr = (GAS unsigned*)(myws + WS_CALL + (size_t)b * 128);
    GAS unsigned* tmo = (GAS unsigned*)(myws + WS_TMO);
    const unsigned lbase = (unsigned)(size_t)lds;
    const unsigned ctrl = lbase + CTRL_OFF;
    const unsigned lo16 = (unsigned)lane * 16u;
    const int r8 = lane >> 3, c8 = lane & 7;
    const int r16 = lane & 15, q = lane >> 4;
    const unsigned img = lbase + (unsigned)(r8 * 128 + (c8 ^ (r8 >> 1)) * 16);
    const unsigned frg = lbase + (unsigned)(r16 * 128 + (q ^ (r16 >> 1)) * 16);
    const char* __attribute__((address_space(4))) const* ka = (KPTR)__builtin_amdgcn_kernarg_segment_ptr();

    if (__builtin_expect(idx < 2 && wave < 2, 1)) {
        const int pid = (xr * 2 + idx) * 2 + wave;
        f4 v[K1][4];
        unsigned cv;
        {
            const unsigned s0 = (unsigned)((pid * 16 + r8) * GK + c8 * 8) * 4u;
            const unsigned s1 = (unsigned)((pid * 16 + 8 + r8) * GK + (c8 ^ 4) * 8) * 4u;
#pragma unroll
            for (int e = 0; e < K1; ++e) {
                asm volatile("global_load_dwordx4 %0, %4, %6\n\tglobal_load_dwordx4 %1, %4, %6 offset:16\n\t"
                             "global_load_dwordx4 %2, %5, %6\n\tglobal_load_dwordx4 %3, %5, %6 offset:16"
                             : "=&v"(v[e][0]), "=&v"(v[e][1]), "=&v"(v[e][2]), "=&v"(v[e][3])
                             : "v"(s0), "v"(s1), "s"(p.x + (KP0 + e * KPS) * BK) : "memory");
                if (e == 0) asm volatile("global_load_dword %0, %1, %2 sc1" : "=&v"(cv) : "v"(b * 128), "s"(myws + WS_CALL) : "memory");
            }
        }
        SCHED_FENCE();
        const char* dst[NR];
#pragma unroll
        for (int j = 0; j < NR; ++j) dst[j] = ka[offsetof(Params, xg) / 8 + ((xr * 2 + idx + 4 * wave) & 7) + j];
        unsigned vo = lo16 + (unsigned)((pid >> 4) * K1 * X_STEP + (pid & 15) * 2048), call = 0;
        char* flag = myws + WS_SFLAG + (size_t)(pid >> 4) * FSTRIDE + 512 + (pid & 15) * 4;
        u4 o[K1][2];
#pragma unroll
        for (int e = 0; e < K1; ++e) {
            if (e < 2) {
#pragma unroll
                for (int k = e; k < (e ? K1 : 1); ++k) {
                    asm volatile("s_waitcnt vmcnt(%5)" : "+v"(cv), "+v"(v[k][0]), "+v"(v[k][1]), "+v"(v[k][2]), "+v"(v[k][3])
                                 : "i"(4 * (K1 - 1 - k) + e * 2 * NR) : "memory");
                    to_frags(pk8(v[k][0], v[k][1]), pk8(v[k][2], v[k][3]), img + 32768u + wave * 2048u, frg + 32768u + wave * 2048u, o[k][0], o[k][1]);
                }
            }
            if (e == 0) {
                call = rfl(cv) + 1u;
                vo += (call & 1u) * (unsigned)(NR * 2 * K1 * X_STEP);
            }
            if (e == 1) __builtin_amdgcn_s_barrier();
            asm volatile("s_waitcnt lgkmcnt(%2)" : "+v"(o[e][0]), "+v"(o[e][1]) : "i"(e ? 4 * (K1 - 1 - e) : 0) : "memory");
            const unsigned ve = vo + (unsigned)(e * X_STEP);
#pragma unroll
            for (int j = 0; j < NR; ++j) {
                if (e) asm volatile("s_waitcnt vmcnt(%0)" :: "i"(PWIN - 2) : "memory");
                asm volatile("global_store_dwordx4 %0, %1, %3 sc0 sc1\n\tglobal_store_dwordx4 %0, %2, %3 offset:1024 sc0 sc1\n\ts_nop 1"
                             :: "v"(ve), "v"(o[e][0]), "v"(o[e][1]), "s"(dst[j]) : "memory");
            }
        }
        VM_WAIT(0);
        if (lane == 0) st4_sys(flag, call);
        __builtin_amdgcn_endpgm();
    }
    GAS unsigned* rotw = (GAS unsigned*)(myws + WS_WFLAG + (size_t)(idx & 15) * FSTRIDE + 256);
#define SRCWS(rot_) ka[offsetof(Params, ws) / 8 + ((xr + (rot_)) & 7)]
    if (idx < 2) {
        const char* hint = myws + WS_HINT + (size_t)(xr * 2 + idx) * FSTRIDE;
        if (wave == 7) lds_st(ctrl + lane * 4, 0u);
        const unsigned call = ld_ctr((const void*)cctr) + 1u;
        LGKM0();
        __builtin_amdgcn_s_barrier();
        asm volatile("" ::: "memory");
        unsigned long long t0 = 0;
        if (wave >= 3 && wave < 7) {
            const int qt = wave - 3;
            const char* hq = hint + qt * 4;
            unsigned rot;
            for (unsigned sp = 0;;) {
                rot = rfl(lds_ld(ctrl + C_ROT));
                if (rot || SPIN_OUT(sp, t0)) break;
                __builtin_amdgcn_s_sleep(1);
            }
            const char* srcws = SRCWS(rot);
            const char* xa = srcws + WS_XS + (size_t)((call & 1u) * 2u + (unsigned)idx) * NKS * X_STEP;
            unsigned hv = call * 256u + 1u, zero = 0u;
            asm volatile("s_mov_b32 m0, %2" : "+v"(hv), "+v"(zero) : "s"(lbase + qt * 4096) : "memory");
            int av = 0;
#define STAGED(n_) do { if (av < (n_)) for (unsigned _sp = 0;;) { av = (int)rfl(lds_ld(ctrl + C_AVAIL)); \
                if (av >= (n_) || SPIN_OUT(_sp, t0)) break; \
                __builtin_amdgcn_s_sleep(1); } } while (0)
#define PULL4(a_) asm volatile("s_waitcnt vmcnt(%2)\n\tglobal_load_lds_dwordx4 %0, %1\n\t" \
                         "s_waitcnt vmcnt(%2)\n\tglobal_load_lds_dwordx4 %0, %1 offset:1024\n\t" \
                         "s_waitcnt vmcnt(%2)\n\tglobal_load_lds_dwordx4 %0, %1 offset:2048\n\t" \
                         "s_waitcnt vmcnt(%2)\n\tglobal_load_lds_dwordx4 %0, %1 offset:3072" \
                         :: "v"(lo16), "s"(a_), "i"(WIN - 1) : "memory")
            unsigned long long own = srcws == myws ? rt() : 0;
#define PACE(n_) do { if (own) { own += n_; while (rt() < own) __builtin_amdgcn_s_sleep(2); } } while (0)
#pragma unroll 1
            for (int t = 0; t < KA; ++t) {
                STAGED(t + 1);
                PACE(OWN);
                const char* xt = xa + qt * 8192;
                if ((PMASK >> 1 >> t) & 1) { hv += 1u; xa += X_STEP; ++t; }
                post(hq, zero, hv);
                PULL4(xt);
                PULL4(xt + 4096);
                hv += 1u;
                xa += X_STEP;
            }
            STAGED(NKS);
            xa += (qt >> 1) * X_STEP + (qt & 1) * 4096;
#pragma unroll
            for (int t = KA; t < NITEMS; ++t) {
                PACE(OWN / 2);
                hv += 1u;
                post(hq, zero, hv);
                PULL4(xa + (t - KA) % NI * 2 * X_STEP + ((t - KA) / NI % 2 * 2 + (t - KA) / NAB) * 8192);
            }
            VM_WAIT(0);
        } else if (wave == 7) {
            if (lane == 0) __hip_atomic_store(cctr, call, __ATOMIC_RELAXED, __HIP_MEMORY_SCOPE_AGENT);
            __builtin_amdgcn_fence(__ATOMIC_ACQUIRE, "");
            unsigned rot, staged = 0, npoll = 0;
            bool stored = false;
            t0 = rt();
            do rot = ld_ctr((const void*)rotw) - call * 8u; while (rot > 7u && rt() - t0 <= DEADLINE);
            lds_st(ctrl + C_ROT, rot | 8u);
            post(hint + 20, 0u, call * 8u + xcd());
            post(hint + 16, 0u, call * 256u + KP0);
            const char* sf = SRCWS(rot) + WS_SFLAG + (size_t)idx * FSTRIDE;
            while (staged < (unsigned)NKS || !stored) {
                if (rot > 7u || ((++npoll & 31u) == 0u && (rfl(__hip_atomic_load(tmo, __ATOMIC_RELAXED, __HIP_MEMORY_SCOPE_AGENT)) == call
                                                           || rt() - t0 > DEADLINE))) {
                    __hip_atomic_store(tmo, call, __ATOMIC_RELAXED, __HIP_MEMORY_SCOPE_AGENT);
                    post(hint + 16, 0u, call * 256u);
                    VM_WAIT(0);
                    lds_st(ctrl + C_AVAIL, (unsigned)NKS);
                    break;
                }
                unsigned s2; bool p2;
                arrivals(sf, call, lane, s2, p2);
                if (s2 > staged) {
                    lds_st(ctrl + C_AVAIL, s2);
                    staged = s2;
                }
                if (p2 && !stored) { stored = true; post(hint + 16, 0u, call * 256u + NITEMS); }
                if (npoll < 64u && staged < (unsigned)NKS) __builtin_amdgcn_s_sleep(2); else __builtin_amdgcn_s_sleep(32);
            }
        }
        return;
    }

    const int mt = idx & 1, nt = (idx - 2) >> 1;
    const int f0 = (17 * nt) >> 1;
    const bool dup = !(nt & 1) && nt != 14;
    if (wave < 8) {
        f4 sv[4][2];
        const int g = b - 16;
        int sk = g >> 2;
#pragma unroll
        for (int e = 0; e < K1; ++e) sk += sk >= KP0 + e * KPS;
        const bool late = wave < 4 && sk >= EARLY;
        if (wave < 4) {
            const float* s0 = p.x + (size_t)((g & 3) * 128 + wave * 32 + r8) * GK + sk * BK;
            if (late) {
                __builtin_amdgcn_s_barrier();
                for (int i = EARLY; i < sk; ++i) __builtin_amdgcn_s_sleep(1);
            } else __builtin_amdgcn_s_sleep(8);
#pragma unroll
            for (int i = 0; i < 4; ++i)
                asm volatile("global_load_dwordx4 %0, %2, off nt\n\tglobal_load_dwordx4 %1, %2, off offset:16 nt"
                             : "=&v"(sv[i][0]), "=&v"(sv[i][1]) : "v"(s0 + (size_t)i * 8 * GK + (c8 ^ ((i & 1) * 4)) * 8) : "memory");
        }
        if (wave >= 4 && dup) {
            const u4 z = {0u, 0u, 0u, 0u};
            const unsigned za = lbase + (unsigned)((wave - 4) * 2 * W_STEP + 8 * W_BLK) + lo16;
            asm volatile("ds_write_b128 %0, %1\n\tds_write_b128 %0, %1 offset:1024\n\t"
                         "ds_write_b128 %0, %1 offset:%2\n\tds_write_b128 %0, %1 offset:%3\n\ts_waitcnt lgkmcnt(0)"
                         :: "v"(za), "v"(z), "i"(W_STEP), "i"(W_STEP + 1024) : "memory");
        }
        if (!late) __builtin_amdgcn_s_barrier();
        asm volatile("" ::: "memory");
        const unsigned call = rfl(lds_ld(ctrl + C_CALL));
        if (wave < 4) {
            char* d0 = myws + WS_XS + ((size_t)((call & 1u) * 2u + (unsigned)((g >> 1) & 1)) * NKS + (size_t)sk) * X_STEP
                       + (size_t)((g & 1) * X_PART + wave * 4096) + lo16;
            asm volatile("s_waitcnt vmcnt(0)" : "+v"(sv[0][0]), "+v"(sv[0][1]), "+v"(sv[1][0]), "+v"(sv[1][1]),
                         "+v"(sv[2][0]), "+v"(sv[2][1]), "+v"(sv[3][0]), "+v"(sv[3][1]) :: "memory");
            u4 o[4];
#pragma unroll
            for (int i = 0; i < 4; i += 2)
                to_frags(pk8(sv[i][0], sv[i][1]), pk8(sv[i + 1][0], sv[i + 1][1]), img + (unsigned)(SCR_OFF + wave * 2048),
                         frg + (unsigned)(SCR_OFF + wave * 2048), o[i], o[i + 1]);
            asm volatile("s_waitcnt lgkmcnt(0)" : "+v"(o[0]), "+v"(o[1]), "+v"(o[2]), "+v"(o[3]) :: "memory");
#pragma unroll
            for (int i = 0; i < 4; ++i) st16_sys(d0 + i * 1024, o[i]);
            unsigned n = 0;
            if (lane == 0) {
                asm volatile("s_waitcnt vmcnt(0)\n\tds_add_rtn_u32 %0, %1, %2\n\ts_waitcnt lgkmcnt(0)"
                             : "=&v"(n) : "v"(ctrl + C_STG), "v"(1u) : "memory");
                if (n == 3u) st4_sys(myws + WS_SFLAG + (size_t)((g >> 1) & 1) * FSTRIDE + sk * 8 + (g & 1) * 4, call);
            }
        }
        const int set = wave >> 2, w4 = wave & 3;
        f4 acc[FW][2];
#pragma unroll
        for (int i = 0; i < FW; ++i) {
            acc[i][0] = (f4){0.f, 0.f, 0.f, 0.f}; acc[i][1] = (f4){0.f, 0.f, 0.f, 0.f};
            asm volatile("" : "+v"(acc[i][0]), "+v"(acc[i][1]));
        }
        h8 a[FW], x[8];
#pragma unroll
        for (int i = 0; i < FW; ++i) asm volatile("" : "=v"(a[i]));
#pragma unroll
        for (int j = 0; j < 8; ++j) asm volatile("" : "=v"(x[j]));
        Pos P;
        Loop L;
        asm volatile("" : "=v"(P.f));
        P.w = lbase + (unsigned)((r16 >> 2) * W_BLK + q * 256 + (r16 & 3) * 16);
        P.n = lbase + (unsigned)(8 * W_BLK + q * 256 + (r16 & 3) * 64 + (r16 >> 2) * 16);
        P.x = lo16 + (unsigned)(set * X_PART + w4 * 2048);
        P.ldw = ctrl + C_LD - 4;
        P.rv = -4u; P.relw = ctrl + C_CONS + wave * 4; P.tab = myws + WS_TAB + (size_t)b * 512;
        unsigned vo = (unsigned)(q * NLOC + r16) * 16u, vn = (unsigned)(q * 4 * NLOC + r16) * 4u;
        asm volatile("" : "+v"(P.w), "+v"(P.n), "+v"(vn), "+v"(P.x), "+v"(P.rv), "+v"(P.ldw), "+v"(vo));
        WAIT_AV(1, 0);
        const unsigned rot = rfl(P.f.x);
        const int src = (p.srcs >> 4 * ((xr + rot) & 7)) & 7;
        L.xa = SRCWS(rot) + WS_XS + (size_t)((call & 1u) * 2u + (unsigned)mt) * NKS * X_STEP;
        L.xg = myws + WS_XG + (size_t)((((call & 1u) * NR + (unsigned)src) * 2u + (unsigned)mt) * K1) * X_STEP;
        L.set = set;
        const char* ol = (const char*)(p.out + (size_t)(src * MLOC + mt * BM + set * 128 + w4 * 16) * NLOC + f0 * 16);
        P.p[0] = L.xa; P.p[1] = L.xa + 8192;
        PIN(ol); PIN(P.relw); PIN(P.tab); PIN(L.xg); PIN(L.set); PIN(P.p[0]); PIN(P.p[1]);
#pragma unroll
        for (int k = 0; k < 4; ++k) XLD(x[k], k);
        L.t = 1u; P.need = 2u; L.xa += X_STEP; P.p[0] = L.xa; P.p[1] = L.xa + 8192;
        PIN(L.t); PIN(P.need); PIN(L.xa); PIN(P.p[0]); PIN(P.p[1]);
        WAIT_AV(0, 1);
        FLAGS();
#pragma unroll
        for (int i = 0; i < FW; ++i)
            asm volatile("ds_read_b128 %0, %1 offset:%2" : "+v"(a[i]) : "v"(i < 8 ? P.w : P.n), "i"(wcol(i)) : "memory");
        __builtin_amdgcn_s_waitcnt(0x0070);
        SCHED_FENCE();
#pragma unroll 1
        do {
            k_step<0, 0>(acc, a, x, P, L);
            k_step<1, 0>(acc, a, x, P, L);
        } while (L.t <= (unsigned)KA);
        P.need = (unsigned)(KA + 1 + NI * set);
        const char* xp = L.xa - X_STEP;
        unsigned pre = P.need - set;
#define TRY() do { const unsigned _p = rfl(P.f.w); item_x(x, P.x, xp, pre < P.need ? P.need : ~0u, _p); \
                   item_x(x + 4, P.x, xp + 2 * X_STEP, pre <= P.need ? P.need + 1u : ~0u, _p); pre = _p; } while (0)
#define SILU(r_, J_, keep_) silu4(acc[(r_) & 4][J_][(r_) & 3], acc[((r_) & 4) + 1][J_][(r_) & 3], acc[((r_) & 4) + 2][J_][(r_) & 3], \
                                  acc[((r_) & 4) + 3][J_][(r_) & 3], 60.f * LOG2E, c, keep_)
#define STORE(r_, o_) asm volatile("global_store_dwordx4 %0, %1, %2 offset:%3 nt" \
                                   :: "v"(vo), "v"(o_), "s"(ol + ((r_) & 3) * NLOC * 4), "i"(((r_) & 4) * 64) : "memory")
#define PART(J_) do { \
        SCHED_FENCE(); \
        if (pre < P.need) WAIT_AV(P.need, 0); \
        item_x(x, P.x, xp, pre + 1u, P.need); \
        if (pre <= P.need) WAIT_AV(P.need + 1u, KA + 2); \
        P.p[0] = xp + 2 * X_STEP; P.p[1] = xp + 3 * X_STEP; PIN(P.p[0]); PIN(P.p[1]); \
        item_x(x + 4, P.x, P.p[0], pre, P.need); \
        if (J_) { if (pre < P.need) VM_WAIT(4); else VM_WAIT(8); } \
        SCHED_FENCE(); \
        part_items<J_ * NT, J_ * NT + NT>(acc, a, x, P); \
        if (!J_) { xp += 8192; PIN(xp); P.need += NAB; pre = 0u; TRY(); } \
        const float c = __builtin_bit_cast(float, __builtin_bit_cast(unsigned, LOG2E) | rfl(P.f.z)); \
        asm volatile("s_nop 7\n\ts_nop 7" : "+v"(acc[FW - 3][J_]), "+v"(acc[FW - 2][J_]), "+v"(acc[FW - 1][J_])); \
        f4 o = acc[FW - 1][J_], od[4]; \
        _Pragma("unroll") for (int r = 0; r < 4; ++r) { o = SILU(r, J_, o); STORE(r, o); } \
        if (!dup) { \
            const f4 on = silu4(acc[FW - 1][J_][0], acc[FW - 1][J_][1], acc[FW - 1][J_][2], acc[FW - 1][J_][3], 60.f * LOG2E, c, o); \
            _Pragma("unroll") for (int r = 0; r < 4; ++r) \
                asm volatile("global_store_dword %0, %1, %2 offset:512 nt" :: "v"(vn), "v"(on[r]), "s"(ol + r * NLOC * 4) : "memory"); \
        } \
        _Pragma("unroll") for (int r = 4; r < 8; ++r) {        \
            if (!J_ && (r == 4 || r == 7)) FLAGS(); \
            o = od[r - 4] = SILU(r, J_, o); \
            if (J_) STORE(r, o); \
            if (!J_ && (r == 4 || r == 7)) { asm volatile("s_waitcnt lgkmcnt(0)" : "+v"(P.f) :: "memory"); TRY(); } \
        } \
        if (!J_) _Pragma("unroll") for (int r = 4; r < 8; ++r) STORE(r, od[r - 4]); \
        asm volatile("s_nop 1" ::: "memory"); } while (0)
        PART(0);
        ADV(vo, 64 * NLOC * 4); ADV(vn, 64 * NLOC * 4);
        PART(1);
        __builtin_amdgcn_endpgm();
    }
    if (wave == 10) {
        *(u2*)(myws + WS_TAB + (size_t)b * 512 + lane * 8) = (u2){2u * lane + 1u, 2u * lane + 2u};
        const unsigned cl = ld_ctr((const void*)cctr) + 1u;
        if (b == NWG - 8 && lane < 16)
            __hip_atomic_store((GAS unsigned*)(myws + WS_WFLAG + (size_t)lane * FSTRIDE + 256), cl * 8u + xcd(),
                               __ATOMIC_RELAXED, __HIP_MEMORY_SCOPE_AGENT);
        if (lane == 0) __hip_atomic_store(cctr, cl, __ATOMIC_RELAXED, __HIP_MEMORY_SCOPE_AGENT);
        lds_st(ctrl + lane * 4, lane == C_CALL / 4 ? cl : 0u);
        LGKM0();
    }
    __builtin_amdgcn_s_barrier();
    asm volatile("" ::: "memory");
    const unsigned call = rfl(lds_ld(ctrl + C_CALL));
    if (wave == 10) {
        GAS unsigned* hq = (GAS unsigned*)(myws + WS_HINT + (size_t)(xr * 2 + mt) * FSTRIDE) + c8;
        GAS unsigned* hp = lane == 5 ? rotw : hq;
        GAS unsigned* wf0 = (GAS unsigned*)(myws + WS_WFLAG + (size_t)(f0 >> 3) * FSTRIDE) + lane;
        GAS unsigned* wf1 = (GAS unsigned*)(myws + WS_WFLAG + (size_t)((f0 + FW - 1) >> 3) * FSTRIDE) + lane;
        unsigned pub = 0, wav = 0, npoll = 0, rot = 8u;
        const unsigned long long t0 = rt();
        unsigned long long tall = 0;
        bool dead = false;
        while (pub < (unsigned)NITEMS || wav < (unsigned)NKS) {
            const unsigned hv = __hip_atomic_load(hp, __ATOMIC_RELAXED, __HIP_MEMORY_SCOPE_AGENT);
            bool adv = false;
            if (wav < (unsigned)NKS) {
                const unsigned w0 = __hip_atomic_load(wf0, __ATOMIC_RELAXED, __HIP_MEMORY_SCOPE_AGENT);
                const unsigned w1 = __hip_atomic_load(wf1, __ATOMIC_RELAXED, __HIP_MEMORY_SCOPE_AGENT);
                const unsigned long long notready = __ballot((int)(w0 - call) < 0 || (int)(w1 - call) < 0);
                const unsigned w2 = notready ? (unsigned)__builtin_ctzll(notready) : 64u;
                if (w2 > wav) { wav = w2; lds_st(ctrl + C_WAV, wav); adv = true; }
            }
            const int d = (int)(hv - call * 256u);
            const unsigned av = (unsigned)row_min(c8 < 5 ? max(min(d, NITEMS), 0) : NITEMS);
            if (rot > 7u) {
                rot = (unsigned)__builtin_amdgcn_readlane((int)hv, 5) - call * 8u;
                if (rot <= 7u) { lds_st(ctrl + C_ROT, rot); hp = hq; }
            }
            if (av > pub && rot <= 7u) {
                if (!pub && (unsigned)__builtin_amdgcn_readlane((int)hv, 13) != call * 8u + xcd()) ACQUIRE();
                asm volatile("ds_max_u32 %0, %1" :: "v"(lane ? ctrl + 128 + lane * 4 : ctrl + C_AVAIL), "v"(av) : "memory");
                pub = av;
                tall = 0;
            } else if (!adv && (++npoll & 63u) == 0u) {
                if (rt() - t0 > DEADLINE) { dead = true; break; }
                unsigned staged; bool stored;
                arrivals(SRCWS(rot) + WS_SFLAG + (size_t)mt * FSTRIDE, call, lane, staged, stored);
                if (staged == (unsigned)NKS && stored && rot <= 7u) {
                    if (tall == 0) tall = rt();
                    else if (rt() - tall > SILENCE) { ACQUIRE(); pub = NITEMS; lds_st(ctrl + C_AVAIL, pub); }
                } else if (rfl(__hip_atomic_load(tmo, __ATOMIC_RELAXED, __HIP_MEMORY_SCOPE_AGENT)) == call) { dead = true; break; }
            }
            if (pub < 48u) __builtin_amdgcn_s_sleep(4); else __builtin_amdgcn_s_sleep(1);
        }
        if (dead) {
            __hip_atomic_store(tmo, call, __ATOMIC_RELAXED, __HIP_MEMORY_SCOPE_AGENT);
            lds_st(ctrl + C_BAD, 0x7fc00000u);
        }
        lds_st(ctrl + C_WAV, (unsigned)NKS);
        lds_st(ctrl + C_AVAIL, (unsigned)NITEMS);
    } else if (wave == 9) {
        constexpr int CSTEP = 240 / NCONV;
        const int cq = ((idx - 2) / CSTEP) * 8 + xr;
        if ((idx - 2) % CSTEP == 0 && cq < NCONV) {
            const unsigned so = (unsigned)(r8 * 8 * NLOC + c8 * 4) * 4u;
            const unsigned dof = (unsigned)((c8 >> 2) * 2048 + r8 * 256 + (c8 & 3) * 16);
            for (int i = -17; i < cq; i += 16) __builtin_amdgcn_s_sleep(20);
            for (int u = cq; u < NKS * 16; u += NCONV) {
                const int ks = u >> 4, cn = u & 15;
                const float* sw = p.w + (size_t)ks * BK * NLOC + cn * 128;
                const char* dstw = myws + WS_WH + (size_t)ks * W_ROW + (size_t)cn * 16384;
                f4 v[4][8];
#pragma unroll
                for (int g = 0; g < 4; ++g)
#pragma unroll
                    for (int r = 0; r < 8; ++r)
                        asm volatile("global_load_dwordx4 %0, %1, %2 offset:%3 nt" : "=v"(v[g][r]) : "v"(so), "s"(sw + r * NLOC), "i"(g * 128) : "memory");
                u4 o[4][4];
#pragma unroll
                for (int g = 0; g < 4; ++g) {
                    asm volatile("s_waitcnt vmcnt(%8)" : "+v"(v[g][0]), "+v"(v[g][1]), "+v"(v[g][2]), "+v"(v[g][3]),
                                 "+v"(v[g][4]), "+v"(v[g][5]), "+v"(v[g][6]), "+v"(v[g][7]) : "i"(24 - 8 * g) : "memory");
#pragma unroll
                    for (int r = 0; r < 8; ++r) v[g][r] *= LOG2E;
#pragma unroll
                    for (int c = 0; c < 4; ++c)
                        o[g][c] = (u4){pk(v[g][0][c], v[g][1][c]), pk(v[g][2][c], v[g][3][c]), pk(v[g][4][c], v[g][5][c]), pk(v[g][6][c], v[g][7][c])};
                    SCHED_FENCE();
                }
#pragma unroll
                for (int g = 0; g < 4; ++g)
#pragma unroll
                    for (int c = 0; c < 4; ++c)
                        asm volatile("global_store_dwordx4 %0, %1, %2 offset:%3 sc1 nt\n\ts_nop 1"
                                     :: "v"(dof), "v"(o[g][c]), "s"(dstw + g * 4096), "i"(c * 64) : "memory");
                VM_WAIT(0);
                if (lane == 0)
                    __hip_atomic_store((GAS unsigned*)(myws + WS_WFLAG + (size_t)cn * FSTRIDE + ks * 4), call,
                                       __ATOMIC_RELAXED, __HIP_MEMORY_SCOPE_AGENT);
            }
        }
    } else {
        const char* tab = myws + WS_TAB + (size_t)b * 512;
        const char* wa = myws + WS_WH + (size_t)f0 * 2048;
        unsigned long long t0 = 0;
        int av = 0, cons = 0;
#pragma unroll 1
        for (int t = 0; t < NKS; ++t) {
            if (av <= t || cons <= t - NW) for (unsigned sp = 0;;) {
                u4 c, d; unsigned v;
                asm volatile("ds_read_b128 %0, %3\n\tds_read_b128 %1, %3 offset:16\n\tds_read_b32 %2, %3 offset:%4\n\t"
                             "s_waitcnt lgkmcnt(0)" : "=&v"(c), "=&v"(d), "=&v"(v) : "v"(ctrl + C_CONS), "i"(C_WAV - C_CONS) : "memory");
                av = (int)rfl(v); cons = (int)min8(c, d);
                if ((av > t && cons > t - NW) || SPIN_OUT(sp, t0)) break;
                __builtin_amdgcn_s_sleep(1);
            }
            const unsigned sb = lbase + (unsigned)(t & (NW - 1)) * W_STEP;
#define WLOADS(P_) _Pragma("unroll") for (int i = 0; i < FW; i += 2) { \
                if (i == FW - 1 && dup) break; \
                asm volatile("s_mov_b32 m0, %0\n\ts_waitcnt vmcnt(58)\n\t" \
                             "global_load_lds_dwordx4 %1, %2" P_ "\n\tglobal_load_lds_dwordx4 %1, %2 offset:1024" P_ \
                             :: "s"(sb + i * W_BLK), "v"(lo16), "s"(wa + i * 2048) : "memory"); \
                if (i + 1 < FW) asm volatile("s_mov_b32 m0, %0\n\ts_nop 0\n\t" \
                             "global_load_lds_dwordx4 %1, %2 offset:2048" P_ "\n\tglobal_load_lds_dwordx4 %1, %2 offset:3072" P_ \
                             :: "s"(sb + i * W_BLK + 64), "v"(lo16), "s"(wa + i * 2048) : "memory"); \
            }
            if (mt == 0 && t >= WNT0 && t < WNT1) { WLOADS(" sc1 nt") } else { WLOADS(" sc1") }
            asm volatile("s_mov_b32 m0, %0\n\ts_mov_b64 exec, 1\n\tglobal_load_lds_dword %1, %2\n\ts_mov_b64 exec, -1"
                         :: "s"(ctrl + C_LD), "v"(lo16), "s"(tab + t * 4) : "memory");
            wa += W_ROW;
        }
        VM_WAIT(0);
    }
}

}

extern "C" void kernel_launch(int rank, int nranks, ncclComm_t comm,
                              hipStream_t stream,
                              void* const* d_in, const int* in_sizes, int n_in,
                              void* d_out, int out_size,
                              const int* in_dtypes, int out_dtype,
                              void* const* d_out_all,
                              void* const* d_ws_all, size_t ws_bytes) {
    (void)comm; (void)in_dtypes; (void)out_dtype; (void)d_out_all;
    if (nranks != NR || n_in < 2 || (unsigned)rank >= (unsigned)NR || ws_bytes < WS_NEED ||
        in_sizes[0] != MLOC * GK || in_sizes[1] != GK * NLOC || out_size != GM * NLOC) {
        fprintf(stderr, "ag_gemm: unexpected problem shape\n");
        return;
    }
    static bool attr_set = false;
    if (!attr_set) {
        HIP_CHECK(hipFuncSetAttribute((const void*)ag_gemm_kernel,
                                      hipFuncAttributeMaxDynamicSharedMemorySize, LDS_BYTES));
        attr_set = true;
    }
    Params p;
    p.x = (const float*)d_in[0];
    p.w = (const float*)d_in[1];
    p.out = (float*)d_out;
    p.myws = (char*)d_ws_all[rank];
    p.srcs = 0;
    const unsigned half = 0xb2;
    for (int far = 0, n = 0; far < 2; ++far)
        for (int r = 0; r < NR; ++r) {
            const int s = (rank + r) & 7;
            if ((int)((half >> s ^ half >> rank) & 1) != far) continue;
            p.ws[n] = (char*)d_ws_all[s];
            p.srcs |= (unsigned)s << 4 * n;
            ++n;
        }
    for (int r = 0; r < 2 * NR; ++r) p.xg[r] = (char*)d_ws_all[(rank + 1 + r) & 7] + WS_XG + (size_t)rank * 2 * K1 * X_STEP;
    hipLaunchKernelGGL(ag_gemm_kernel, dim3(NWG), dim3(NTHREADS), LDS_BYTES, stream, p);
    HIP_CHECK(hipGetLastError());
}
```

```cpp
#include <hip/hip_runtime.h>
#include <rccl/rccl.h>
#include <cstdio>
#include <cstddef>

#define HIP_CHECK(expr) do { hipError_t _e = (expr); if (_e != hipSuccess) { \
    fprintf(stderr, "HIP error %s at %s:%d\n", hipGetErrorString(_e), __FILE__, __LINE__); \
    return; } } while (0)

namespace {

#define GAS __attribute__((address_space(1)))
#define LAS __attribute__((address_space(3)))
typedef _Float16 h8 __attribute__((ext_vector_type(8)));
typedef _Float16 h2 __attribute__((ext_vector_type(2)));
typedef float f4 __attribute__((ext_vector_type(4)));
typedef unsigned u4 __attribute__((ext_vector_type(4)));
typedef unsigned u2 __attribute__((ext_vector_type(2)));

constexpr int NR = 8;
constexpr int GM = 4096, GK = 4096, NLOC = 2048, MLOC = 512;
constexpr int BM = 256, BK = 64;
constexpr int FW = 9;
constexpr int NKS = GK / BK;
constexpr int K1 = 4;
constexpr int KP0 = 30, KPS = 6;
constexpr unsigned long long PMASK = (1ull | 1ull << KPS | 1ull << 2 * KPS | 1ull << 3 * KPS) << KP0;
constexpr int NAB = 8, NI = NAB / 2;
constexpr int KA = NKS - NAB;
constexpr int NITEMS = KA + 4 * NI;
constexpr int X_STEP = BM * BK * 2;
constexpr int X_PART = X_STEP / 2;
constexpr int W_BLK = 2048 + 64;
constexpr int W_STEP = FW * W_BLK - 64;
constexpr int W_ROW = NLOC * BK * 2;
constexpr int NW = 8;
constexpr int SCR_OFF = NW * W_STEP;
constexpr int CTRL_OFF = SCR_OFF + 8192;
constexpr int LDS_BYTES = CTRL_OFF + 512;
constexpr int NWG = 256, NTHREADS = 704;
static_assert(LDS_BYTES <= 160 * 1024 && NAB <= NW && NAB % 2 == 0 && NAB >= 4 && NKS % NW == 0 && NW == 8, "layout");
static_assert((NKS - K1) * 4 == NWG - 16 && KP0 > 1 && KPS > 1 && KP0 + 3 * KPS < KA - 1, "layout");
constexpr int C_CONS = 0;
constexpr int C_WAV = 68;
constexpr int C_STG = 72;
constexpr int C_CALL = 76;
constexpr int C_ROT = 80;
constexpr int C_LD = 84;
constexpr int C_BAD = 88;
constexpr int C_AVAIL = 92;
constexpr int NCONV = 120;
constexpr float LOG2E = 1.4426950408889634f;
constexpr int PWIN = 16;
constexpr int WIN = 8;
constexpr int OWN = 85;
constexpr int CLOSE = 49;
constexpr int EARLY = 12;
static_assert(CLOSE < KA, "all K-steps are staged before the first item of the parts is posted");
constexpr int WNT0 = 8, WNT1 = 32;

constexpr size_t FSTRIDE = 4096 + 256;
constexpr size_t WS_CALL = 0;
constexpr size_t WS_TMO = 49152;
constexpr size_t WS_SFLAG = 53248;
constexpr size_t WS_HINT = 65536;
constexpr size_t WS_WFLAG = 196608;
constexpr size_t WS_TAB = 327680;
constexpr size_t WS_WH = 1ull << 20;
constexpr size_t WS_XS = 32ull << 20;
constexpr size_t WS_XG = 48ull << 20;
constexpr size_t WS_NEED = 64ull << 20;

constexpr unsigned long long DEADLINE = 100000000ull;
constexpr unsigned long long SILENCE = 20000ull;

struct Params {
    const float* x;
    char* myws;
    const float* w;
    float* out;
    char* ws[NR];
    char* xg[2 * NR];
    unsigned srcs;
};
typedef const char* __attribute__((address_space(4))) const* KPTR;

__device__ __forceinline__ unsigned long long rt() { return __builtin_amdgcn_s_memrealtime(); }
__device__ __forceinline__ unsigned xcd() { return __builtin_amdgcn_s_getreg((3 << 11) | 20) & 7u; }

__device__ __forceinline__ void st16_sys(void* p, u4 v) {
    asm volatile("global_store_dwordx4 %0, %1, off sc0 sc1\n\ts_nop 1" :: "v"(p), "v"(v) : "memory");
}
__device__ __forceinline__ void st4_sys(void* p, unsigned v) {
    asm volatile("global_store_dword %0, %1, off sc0 sc1\n\ts_nop 1" :: "v"(p), "v"(v) : "memory");
}
__device__ __forceinline__ void post(const char* p, unsigned zero, unsigned v) {
    asm volatile("s_mov_b64 exec, 1\n\tglobal_store_dword %0, %1, %2\n\ts_mov_b64 exec, -1" :: "v"(zero), "v"(v), "s"(p) : "memory");
}
__device__ __forceinline__ void lds_st(unsigned addr, unsigned v) {
    asm volatile("ds_write_b32 %0, %1" :: "v"(addr), "v"(v) : "memory");
}
__device__ __forceinline__ unsigned lds_ld(unsigned addr) {
    unsigned v;
    asm volatile("ds_read_b32 %0, %1\n\ts_waitcnt lgkmcnt(0)" : "=&v"(v) : "v"(addr) : "memory");
    return v;
}
__device__ __forceinline__ unsigned ld_ctr(const void* p) {
    unsigned v;
    asm volatile("global_load_dword %0, %1, off sc1\n\ts_waitcnt vmcnt(0)" : "=&v"(v) : "v"(p) : "memory");
    return (unsigned)__builtin_amdgcn_readfirstlane(v);
}
__device__ __forceinline__ unsigned pk(float a, float b) {
    h2 h; h.x = (_Float16)a; h.y = (_Float16)b;
    return __builtin_bit_cast(unsigned, h);
}
__device__ __forceinline__ u4 pk8(f4 a, f4 b) { return (u4){pk(a[0], a[1]), pk(a[2], a[3]), pk(b[0], b[1]), pk(b[2], b[3])}; }
__device__ __forceinline__ unsigned umin(unsigned a, unsigned b) { return a < b ? a : b; }
__device__ __forceinline__ unsigned rfl(unsigned v) { return (unsigned)__builtin_amdgcn_readfirstlane((int)v); }
__device__ __forceinline__ unsigned min8(u4 c, u4 d) {
    return rfl(umin(umin(umin(c.x, c.y), umin(c.z, c.w)), umin(umin(d.x, d.y), umin(d.z, d.w))));
}
__device__ __forceinline__ int row_min(int u) {
    u = min(u, __builtin_amdgcn_update_dpp(0, u, 0x121, 15, 15, false));
    u = min(u, __builtin_amdgcn_update_dpp(0, u, 0x122, 15, 15, false));
    u = min(u, __builtin_amdgcn_update_dpp(0, u, 0x124, 15, 15, false));
    u = min(u, __builtin_amdgcn_update_dpp(0, u, 0x128, 15, 15, false));
    return __builtin_amdgcn_readfirstlane(u);
}
__device__ __forceinline__ void arrivals(const char* sf, unsigned call, int lane, unsigned& staged, bool& stored) {
    u2 sv; unsigned pv;
    asm volatile("global_load_dwordx2 %0, %2, %4 sc0 sc1\n\tglobal_load_dword %1, %3, %4 offset:512 sc0 sc1\n\ts_waitcnt vmcnt(0)"
                 : "=&v"(sv), "=&v"(pv) : "v"(lane * 8), "v"((lane & 15) * 4), "s"(sf) : "memory");
    const unsigned long long miss = __ballot((int)(sv.x - call) < 0 || (int)(sv.y - call) < 0) & ~PMASK;
    staged = miss ? (unsigned)__builtin_ctzll(miss) : (unsigned)NKS;
    stored = __ballot((int)(pv - call) < 0) == 0;
}
#define VM_WAIT(n) asm volatile("s_waitcnt vmcnt(" #n ")" ::: "memory")
#define ACQUIRE() do { __builtin_amdgcn_fence(__ATOMIC_ACQUIRE, ""); VM_WAIT(0); } while (0)
#define LGKM0() asm volatile("s_waitcnt lgkmcnt(0)" ::: "memory")
#define SCHED_FENCE() __builtin_amdgcn_sched_barrier(0)
#define SPIN_OUT(sp_, t0_) ((((++(sp_)) & 1023u) == 0u) && ((t0_) == 0 ? ((t0_) = rt(), false) : ((unsigned)(rt() - (t0_)) > 2u * (unsigned)DEADLINE)))

__device__ __forceinline__ f4 silu4(float z0, float z1, float z2, float z3, float lim, float c, f4 keep) {
    float o0, o1, o2, o3;
    asm volatile("v_med3_f32 %0, -%4, %8, -%8\n\tv_med3_f32 %1, -%5, %8, -%8\n\tv_med3_f32 %2, -%6, %8, -%8\n\tv_med3_f32 %3, -%7, %8, -%8\n\t"
                 "v_exp_f32 %0, %0\n\tv_exp_f32 %1, %1\n\tv_exp_f32 %2, %2\n\tv_exp_f32 %3, %3\n\t"
                 "v_fma_f32 %0, %0, %9, %9\n\tv_fma_f32 %1, %1, %9, %9\n\tv_fma_f32 %2, %2, %9, %9\n\tv_fma_f32 %3, %3, %9, %9\n\t"
                 "v_rcp_f32 %0, %0\n\tv_rcp_f32 %1, %1\n\tv_rcp_f32 %2, %2\n\tv_rcp_f32 %3, %3\n\t"
                 "v_mul_f32 %0, %4, %0\n\tv_mul_f32 %1, %5, %1\n\tv_mul_f32 %2, %6, %2\n\tv_mul_f32 %3, %7, %3"
                 : "=&v"(o0), "=&v"(o1), "=&v"(o2), "=&v"(o3) : "v"(z0), "v"(z1), "v"(z2), "v"(z3), "s"(lim), "s"(c), "v"(keep));
    return (f4){o0, o1, o2, o3};
}
#define REL() asm volatile("s_mov_b32 m0, %0\n\ts_mov_b64 exec, 1\n\tglobal_load_lds_dword %1, %2\n\ts_mov_b64 exec, -1" \
                           :: "s"(P.relw), "v"(P.rv), "s"(P.tab) : "memory")
struct Pos { unsigned w, n, x, rv, need, relw, ldw; const char* tab; const char* p[2]; u4 f; };
#define ADV(v_, d_) do { v_ += d_; asm volatile("" : "+v"(v_)); } while (0)
#define PIN(s_) asm volatile("" : "+s"(s_))
#define FLAGS() asm volatile("ds_read_b128 %0, %1" : "=v"(P.f) : "v"(P.ldw) : "memory")
#define WAIT_AV(nx_, nw_) do { unsigned long long _t0 = 0; unsigned _sp = 0; \
        for (;;) { asm volatile("ds_read_b128 %0, %1\n\ts_waitcnt lgkmcnt(0)" : "+v"(P.f) : "v"(P.ldw) : "memory"); \
            if ((int)((rfl(P.f.w) - (unsigned)(nx_)) | (rfl(P.f.y) - (unsigned)(nw_))) >= 0) break; \
            if (SPIN_OUT(_sp, _t0)) break; \
            __builtin_amdgcn_s_sleep(1); } } while (0)
#define CHECK_AV(nx_, nw_) do { if (__builtin_expect((int)((rfl(P.f.w) - (unsigned)(nx_)) | (rfl(P.f.y) - (unsigned)(nw_))) < 0, 0)) \
            WAIT_AV(nx_, nw_); SCHED_FENCE(); } while (0)
#define MFMA(c_, x_, a_) asm volatile("v_mfma_f32_16x16x32_f16 %0, %1, %2, %0" : "+v"(c_) : "v"(x_), "v"(a_))
#define MFMA_R(c_, x_, a_, ad_, off_) asm volatile("v_mfma_f32_16x16x32_f16 %0, %2, %1, %0\n\tds_read_b128 %1, %3 offset:%4" \
                                                   : "+v"(c_), "+v"(a_) : "v"(x_), "v"(ad_), "i"(off_) : "memory")
#define XLD(x_, k_) asm volatile("global_load_dwordx4 %0, %1, %2 offset:%3" : "+v"(x_) : "v"(P.x), "s"(P.p[(k_) & 1]), "i"(((k_) & 2) * 512) : "memory")
constexpr int wcol(int i) { return i < 8 ? (i >> 2) * 4 * W_BLK + (i & 3) * 64 : 0; }

struct Loop { const char* xa; const char* xg; unsigned t, set; };
template <int B, int G>
__device__ __forceinline__ void k_step(f4 (&acc)[FW][2], h8 (&a)[FW], h8 (&x)[8], Pos& P, Loop& L) {
    if constexpr (G < 36) {
        constexpr int h = G / 18, i = G % 18 / 2;
        if (G % 18 == 0) asm volatile("s_waitcnt vmcnt(%0) lgkmcnt(7)" :: "i"(h ? 5 : 3) : "memory");
        else if (G == 2) asm volatile("s_waitcnt lgkmcnt(4)" : "+v"(P.f) :: "memory");
        else if (G % 18 % 4 == 0) asm volatile("s_waitcnt lgkmcnt(7)" ::: "memory");
        if (G & 1) MFMA_R(acc[i][1], x[B * 4 + h * 2 + 1], a[i], i < 8 ? P.w : P.n, (h ? 1 - B : B) * W_STEP + (1 - h) * 1024 + wcol(i));
        else MFMA(acc[i][0], x[B * 4 + h * 2], a[i]);
        SCHED_FENCE();
        if (G == 2) CHECK_AV(P.need, L.t + 1u);
        if (G >= 4 && G <= 10 && !(G & 1)) XLD(x[4 - B * 4 + G / 2 - 2], G / 2 - 2);
        if (B && G == 18) { const int d = (L.t & (NW - 1)) ? 2 * W_STEP : (2 - NW) * W_STEP; ADV(P.w, d); ADV(P.n, d); }
        if (G == 20) { L.t += 1u; PIN(L.t); L.xa += X_STEP; PIN(L.xa); }
        if (G == 22) {
            const bool st = (PMASK >> L.t) & 1;
            P.p[0] = st ? L.xg : L.xa; PIN(P.p[0]);
            L.xg += st ? X_STEP : 0; PIN(L.xg);
        }
        if (G == 24) {
            P.need = L.t + 1u;
            if (B) P.p[1] = P.p[0] + 8192;
            else {
                const bool last = L.t == (unsigned)KA;
                P.p[0] -= last && L.set ? X_STEP : 0;
                P.p[1] = P.p[0] + (last && !L.set ? X_STEP : 8192);
                P.need = last && L.set ? 0u : P.need;
                PIN(P.p[0]);
            }
            PIN(P.p[1]); PIN(P.need);
        }
        if (G == 30) FLAGS();
        if (G == 32) ADV(P.rv, 4u);
        if (G == 34) REL();
        SCHED_FENCE();
        k_step<B, G + 1>(acc, a, x, P, L);
    }
}
constexpr int NT = 36 * NI, WG = 3, G_C = 35 - FW;
constexpr bool flags(int t) {
    return t % NT >= NT - 36 ? t % 36 == 26 : t < NT && t % 36 == G_C - 9;
}
constexpr int xfetch(int t) {
    for (int k = 0; k < 4; ++k) if (t % 36 == (G_C + 1 + k > 9 * k + 8 ? G_C + 1 + k : 9 * k + 8)) return t % NT < NT - 72 ? k : -1;
    return -1;
}
struct Waits { int l[2 * NT + 1], v[NT / 9]; };
constexpr Waits waits() {
    Waits r{};
    int pw[2 * NT + 16] = {}, pf = 0, c = 0, done = -1, xs = 8;
    for (int k = 0; k < FW; ++k) pw[k] = c++;
    for (int t = 0; t <= 2 * NT; ++t) {
        int need = t % WG || t == 2 * NT ? -1 : pw[t + WG - 1];
        if (t % 36 == G_C || (t && t % NT == 0)) need = pf > need ? pf : need;
        r.l[t] = need <= done ? -1 : c - 1 - need < 15 ? c - 1 - need : 15;
        done = need > done ? need : done;
        if (t + FW < 2 * NT) pw[t + FW] = c++;
        if (flags(t)) pf = c++;
        if (t < NT && t % 9 == 0) r.v[t / 9] = xs - 1 - t / 9;
        if (xfetch(t) >= 0) ++xs;
    }
    return r;
}
static_assert(FW > WG && NT % WG == 0, "parts");
constexpr int woff(int k) { return (k % 36 / 9 & 1) * W_STEP + k % 36 / 18 * 1024 + wcol(k % 9); }
template <int T, int E>
__device__ __forceinline__ void part_items(f4 (&acc)[FW][2], h8 (&a)[FW], h8 (&x)[8], Pos& P) {
    constexpr int left = waits().l[T];
    if constexpr (left >= 0 && (T > E - NT || T == 0)) asm volatile("s_waitcnt lgkmcnt(%0)" :: "i"(left) : "memory");
    if (T == E || T % 36 == G_C) asm volatile("" : "+v"(P.f));
    if constexpr (T < E) {
        constexpr int J = T / NT, N = T % NT / 36, G = T % 36, K = T + FW, X = (N & 1) * 4;
        if (G % 9 == 0 && (!J || N)) asm volatile("s_waitcnt vmcnt(%0)" :: "i"(waits().v[T % NT / 9]) : "memory");
        if constexpr (K < 2 * NT) MFMA_R(acc[T % FW][J], x[X + G / 9], a[T % FW], K % 9 < 8 ? P.w : P.n, woff(K));
        else MFMA(acc[T % FW][J], x[X + G / 9], a[T % FW]);
        SCHED_FENCE();
        if (K < 2 * NT && K % 36 >= 34) {
            constexpr int d = K % NT / 36 == NI - 1 ? (2 - NAB) * W_STEP : 2 * W_STEP;
            if (K % 36 == 34) ADV(P.w, d); else ADV(P.n, d);
        }
        if (flags(T)) FLAGS();
        if (!J && G == G_C && N < NI - 1) CHECK_AV(0u, KA + 2 * N + 4);
        if (G == G_C - 2 && N < NI - 2) { P.p[0] += 2 * X_STEP; PIN(P.p[0]); P.p[1] += 2 * X_STEP; PIN(P.p[1]); }
        if constexpr (xfetch(T) >= 0) XLD(x[X + xfetch(T)], xfetch(T));
        SCHED_FENCE();
        part_items<T + 1, E>(acc, a, x, P);
    }
}
__device__ __forceinline__ void item_x(h8* x, unsigned vx, const char* p, unsigned lo, unsigned hi) {
    asm volatile("s_cmp_gt_u32 %7, %8\n\ts_cbranch_scc1 1f\n\t"
                 "global_load_dwordx4 %0, %4, %5\n\tglobal_load_dwordx4 %1, %4, %6\n\t"
                 "global_load_dwordx4 %2, %4, %5 offset:1024\n\tglobal_load_dwordx4 %3, %4, %6 offset:1024\n1:"
                 : "+v"(x[0]), "+v"(x[1]), "+v"(x[2]), "+v"(x[3])
                 : "v"(vx), "s"(p), "s"(p + X_STEP), "s"(lo), "s"(hi) : "memory", "scc");
}
__device__ __forceinline__ void to_frags(u4 o0, u4 o1, unsigned wr, unsigned rd, u4& f0, u4& f1) {
    asm volatile("ds_write_b128 %2, %4\n\tds_write_b128 %2, %5 offset:1024\n\tds_read_b128 %0, %3\n\tds_read_b128 %1, %6"
                 : "=&v"(f0), "=&v"(f1) : "v"(wr), "v"(rd), "v"(o0), "v"(o1), "v"(rd ^ 64u) : "memory");
}

__global__ void __launch_bounds__(NTHREADS, 3) ag_gemm_kernel(Params p) {
    extern __shared__ __attribute__((aligned(16))) unsigned char smem[];
    LAS unsigned char* lds = (LAS unsigned char*)smem;
    const int tid = threadIdx.x;
    const int wave = __builtin_amdgcn_readfirstlane(tid >> 6);
    const int lane = tid & 63;
    const int b = blockIdx.x;
    const int xr = b & 7, idx = b >> 3;
    char* myws = p.myws;
    GAS unsigned* cctr = (GAS unsigned*)(myws + WS_CALL + (size_t)b * 128);
    GAS unsigned* tmo = (GAS unsigned*)(myws + WS_TMO);
    const unsigned lbase = (unsigned)(size_t)lds;
    const unsigned ctrl = lbase + CTRL_OFF;
    const unsigned lo16 = (unsigned)lane * 16u;
    const int r8 = lane >> 3, c8 = lane & 7;
    const int r16 = lane & 15, q = lane >> 4;
    const unsigned img = lbase + (unsigned)(r8 * 128 + (c8 ^ (r8 >> 1)) * 16);
    const unsigned frg = lbase + (unsigned)(r16 * 128 + (q ^ (r16 >> 1)) * 16);
    const char* __attribute__((address_space(4))) const* ka = (KPTR)__builtin_amdgcn_kernarg_segment_ptr();

    if (__builtin_expect(idx < 2 && wave < 2, 1)) {
        const int pid = (xr * 2 + idx) * 2 + wave;
        f4 v[K1][4];
        unsigned cv;
        {
            const unsigned s0 = (unsigned)((pid * 16 + r8) * GK + c8 * 8) * 4u;
            const unsigned s1 = (unsigned)((pid * 16 + 8 + r8) * GK + (c8 ^ 4) * 8) * 4u;
#pragma unroll
            for (int e = 0; e < K1; ++e) {
                asm volatile("global_load_dwordx4 %0, %4, %6\n\tglobal_load_dwordx4 %1, %4, %6 offset:16\n\t"
                             "global_load_dwordx4 %2, %5, %6\n\tglobal_load_dwordx4 %3, %5, %6 offset:16"
                             : "=&v"(v[e][0]), "=&v"(v[e][1]), "=&v"(v[e][2]), "=&v"(v[e][3])
                             : "v"(s0), "v"(s1), "s"(p.x + (KP0 + e * KPS) * BK) : "memory");
                if (e == 0) asm volatile("global_load_dword %0, %1, %2 sc1" : "=&v"(cv) : "v"(b * 128), "s"(myws + WS_CALL) : "memory");
            }
        }
        SCHED_FENCE();
        const char* dst[NR];
#pragma unroll
        for (int j = 0; j < NR; ++j) dst[j] = ka[offsetof(Params, xg) / 8 + ((xr * 2 + idx + 4 * wave) & 7) + j];
        unsigned vo = lo16 + (unsigned)((pid >> 4) * K1 * X_STEP + (pid & 15) * 2048), call = 0;
        char* flag = myws + WS_SFLAG + (size_t)(pid >> 4) * FSTRIDE + 512 + (pid & 15) * 4;
        u4 o[K1][2];
#pragma unroll
        for (int e = 0; e < K1; ++e) {
            if (e < 2) {
#pragma unroll
                for (int k = e; k < (e ? K1 : 1); ++k) {
                    asm volatile("s_waitcnt vmcnt(%5)" : "+v"(cv), "+v"(v[k][0]), "+v"(v[k][1]), "+v"(v[k][2]), "+v"(v[k][3])
                                 : "i"(4 * (K1 - 1 - k) + e * 2 * NR) : "memory");
                    to_frags(pk8(v[k][0], v[k][1]), pk8(v[k][2], v[k][3]), img + 32768u + wave * 2048u, frg + 32768u + wave * 2048u, o[k][0], o[k][1]);
                }
            }
            if (e == 0) {
                call = rfl(cv) + 1u;
                vo += (call & 1u) * (unsigned)(NR * 2 * K1 * X_STEP);
            }
            if (e == 1) __builtin_amdgcn_s_barrier();
            asm volatile("s_waitcnt lgkmcnt(%2)" : "+v"(o[e][0]), "+v"(o[e][1]) : "i"(e ? 4 * (K1 - 1 - e) : 0) : "memory");
            const unsigned ve = vo + (unsigned)(e * X_STEP);
#pragma unroll
            for (int j = 0; j < NR; ++j) {
                if (e) asm volatile("s_waitcnt vmcnt(%0)" :: "i"(PWIN - 2) : "memory");
                asm volatile("global_store_dwordx4 %0, %1, %3 sc0 sc1\n\tglobal_store_dwordx4 %0, %2, %3 offset:1024 sc0 sc1\n\ts_nop 1"
                             :: "v"(ve), "v"(o[e][0]), "v"(o[e][1]), "s"(dst[j]) : "memory");
            }
        }
        VM_WAIT(0);
        if (lane == 0) st4_sys(flag, call);
        __builtin_amdgcn_endpgm();
    }
    GAS unsigned* rotw = (GAS unsigned*)(myws + WS_WFLAG + (size_t)(idx & 15) * FSTRIDE + 256);
#define SRCWS(rot_) ka[offsetof(Params, ws) / 8 + ((xr + (rot_)) & 7)]
    if (idx < 2) {
        const char* hint = myws + WS_HINT + (size_t)(xr * 2 + idx) * FSTRIDE;
        if (wave == 7) lds_st(ctrl + lane * 4, 0u);
        const unsigned call = ld_ctr((const void*)cctr) + 1u;
        LGKM0();
        __builtin_amdgcn_s_barrier();
        asm volatile("" ::: "memory");
        unsigned long long t0 = 0;
        if (wave >= 3 && wave < 7) {
            const int qt = wave - 3;
            const char* hq = hint + qt * 4;
            unsigned rot;
            for (unsigned sp = 0;;) {
                rot = rfl(lds_ld(ctrl + C_ROT));
                if (rot || SPIN_OUT(sp, t0)) break;
                __builtin_amdgcn_s_sleep(1);
            }
            const char* srcws = SRCWS(rot);
            const char* xa = srcws + WS_XS + (size_t)((call & 1u) * 2u + (unsigned)idx) * NKS * X_STEP;
            unsigned hv = call * 256u + 1u, zero = 0u;
            asm volatile("s_mov_b32 m0, %2" : "+v"(hv), "+v"(zero) : "s"(lbase + qt * 4096) : "memory");
            int av = 0;
#define STAGED(n_) do { if (av < (n_)) for (unsigned _sp = 0;;) { av = (int)rfl(lds_ld(ctrl + C_AVAIL)); \
                if (av >= (n_) || SPIN_OUT(_sp, t0)) break; \
                __builtin_amdgcn_s_sleep(1); } } while (0)
#define PULL4(a_) asm volatile("s_waitcnt vmcnt(%2)\n\tglobal_load_lds_dwordx4 %0, %1\n\t" \
                         "s_waitcnt vmcnt(%2)\n\tglobal_load_lds_dwordx4 %0, %1 offset:1024\n\t" \
                         "s_waitcnt vmcnt(%2)\n\tglobal_load_lds_dwordx4 %0, %1 offset:2048\n\t" \
                         "s_waitcnt vmcnt(%2)\n\tglobal_load_lds_dwordx4 %0, %1 offset:3072" \
                         :: "v"(lo16), "s"(a_), "i"(WIN - 1) : "memory")
            unsigned long long own = srcws == myws ? rt() : 0;
#define PACE(n_) do { if (own) { own += n_; while (rt() < own) __builtin_amdgcn_s_sleep(2); } } while (0)
#pragma unroll 1
            for (int t = 0; t < KA; ++t) {
                STAGED(t + 1);
                PACE(OWN);
                const char* xt = xa + qt * 8192;
                if ((PMASK >> 1 >> t) & 1) { hv += 1u; xa += X_STEP; ++t; }
                if (t <= CLOSE) post(hq, zero, hv);
                PULL4(xt);
                hv += 1u;
                if (t >= CLOSE) { STAGED(NKS); post(hq, zero, hv); }
                PULL4(xt + 4096);
                xa += X_STEP;
            }
            xa += (qt >> 1) * X_STEP + (qt & 1) * 4096;
#pragma unroll
            for (int t = KA; t < NITEMS; ++t) {
                PACE(OWN / 2);
                hv += 1u;
                post(hq, zero, hv);
                PULL4(xa + (t - KA) % NI * 2 * X_STEP + ((t - KA) / NI % 2 * 2 + (t - KA) / NAB) * 8192);
            }
            VM_WAIT(0);
        } else if (wave == 7) {
            if (lane == 0) __hip_atomic_store(cctr, call, __ATOMIC_RELAXED, __HIP_MEMORY_SCOPE_AGENT);
            __builtin_amdgcn_fence(__ATOMIC_ACQUIRE, "");
            unsigned rot, staged = 0, npoll = 0;
            bool stored = false;
            t0 = rt();
            do rot = ld_ctr((const void*)rotw) - call * 8u; while (rot > 7u && rt() - t0 <= DEADLINE);
            lds_st(ctrl + C_ROT, rot | 8u);
            post(hint + 20, 0u, call * 8u + xcd());
            post(hint + 16, 0u, call * 256u + KP0);
            const char* sf = SRCWS(rot) + WS_SFLAG + (size_t)idx * FSTRIDE;
            while (staged < (unsigned)NKS || !stored) {
                if (rot > 7u || ((++npoll & 31u) == 0u && (rfl(__hip_atomic_load(tmo, __ATOMIC_RELAXED, __HIP_MEMORY_SCOPE_AGENT)) == call
                                                           || rt() - t0 > DEADLINE))) {
                    __hip_atomic_store(tmo, call, __ATOMIC_RELAXED, __HIP_MEMORY_SCOPE_AGENT);
                    post(hint + 16, 0u, call * 256u);
                    VM_WAIT(0);
                    lds_st(ctrl + C_AVAIL, (unsigned)NKS);
                    break;
                }
                unsigned s2; bool p2;
                arrivals(sf, call, lane, s2, p2);
                if (s2 > staged) {
                    lds_st(ctrl + C_AVAIL, s2);
                    staged = s2;
                }
                if (p2 && !stored) { stored = true; post(hint + 16, 0u, call * 256u + NITEMS); }
                if (npoll < 64u && staged < (unsigned)NKS) __builtin_amdgcn_s_sleep(2); else __builtin_amdgcn_s_sleep(32);
            }
        }
        return;
    }

    const int mt = idx & 1, nt = (idx - 2) >> 1;
    const int f0 = (17 * nt) >> 1;
    const bool dup = !(nt & 1) && nt != 14;
    if (wave < 8) {
        f4 sv[4][2];
        const int g = b - 16;
        int sk = g >> 2;
#pragma unroll
        for (int e = 0; e < K1; ++e) sk += sk >= KP0 + e * KPS;
        const bool late = wave < 4 && sk >= EARLY;
        if (wave < 4) {
            const float* s0 = p.x + (size_t)((g & 3) * 128 + wave * 32 + r8) * GK + sk * BK;
            if (late) {
                __builtin_amdgcn_s_barrier();
                for (int i = EARLY; i < sk; ++i) __builtin_amdgcn_s_sleep(1);
            } else __builtin_amdgcn_s_sleep(8);
#pragma unroll
            for (int i = 0; i < 4; ++i)
                asm volatile("global_load_dwordx4 %0, %2, off nt\n\tglobal_load_dwordx4 %1, %2, off offset:16 nt"
                             : "=&v"(sv[i][0]), "=&v"(sv[i][1]) : "v"(s0 + (size_t)i * 8 * GK + (c8 ^ ((i & 1) * 4)) * 8) : "memory");
        }
        if (wave >= 4 && dup) {
            const u4 z = {0u, 0u, 0u, 0u};
            const unsigned za = lbase + (unsigned)((wave - 4) * 2 * W_STEP + 8 * W_BLK) + lo16;
            asm volatile("ds_write_b128 %0, %1\n\tds_write_b128 %0, %1 offset:1024\n\t"
                         "ds_write_b128 %0, %1 offset:%2\n\tds_write_b128 %0, %1 offset:%3\n\ts_waitcnt lgkmcnt(0)"
                         :: "v"(za), "v"(z), "i"(W_STEP), "i"(W_STEP + 1024) : "memory");
        }
        if (!late) __builtin_amdgcn_s_barrier();
        asm volatile("" ::: "memory");
        const unsigned call = rfl(lds_ld(ctrl + C_CALL));
        if (wave < 4) {
            char* d0 = myws + WS_XS + ((size_t)((call & 1u) * 2u + (unsigned)((g >> 1) & 1)) * NKS + (size_t)sk) * X_STEP
                       + (size_t)((g & 1) * X_PART + wave * 4096) + lo16;
            asm volatile("s_waitcnt vmcnt(0)" : "+v"(sv[0][0]), "+v"(sv[0][1]), "+v"(sv[1][0]), "+v"(sv[1][1]),
                         "+v"(sv[2][0]), "+v"(sv[2][1]), "+v"(sv[3][0]), "+v"(sv[3][1]) :: "memory");
            u4 o[4];
#pragma unroll
            for (int i = 0; i < 4; i += 2)
                to_frags(pk8(sv[i][0], sv[i][1]), pk8(sv[i + 1][0], sv[i + 1][1]), img + (unsigned)(SCR_OFF + wave * 2048),
                         frg + (unsigned)(SCR_OFF + wave * 2048), o[i], o[i + 1]);
            asm volatile("s_waitcnt lgkmcnt(0)" : "+v"(o[0]), "+v"(o[1]), "+v"(o[2]), "+v"(o[3]) :: "memory");
#pragma unroll
            for (int i = 0; i < 4; ++i) st16_sys(d0 + i * 1024, o[i]);
            unsigned n = 0;
            if (lane == 0) {
                asm volatile("s_waitcnt vmcnt(0)\n\tds_add_rtn_u32 %0, %1, %2\n\ts_waitcnt lgkmcnt(0)"
                             : "=&v"(n) : "v"(ctrl + C_STG), "v"(1u) : "memory");
                if (n == 3u) st4_sys(myws + WS_SFLAG + (size_t)((g >> 1) & 1) * FSTRIDE + sk * 8 + (g & 1) * 4, call);
            }
        }
        const int set = wave >> 2, w4 = wave & 3;
        f4 acc[FW][2];
#pragma unroll
        for (int i = 0; i < FW; ++i) {
            acc[i][0] = (f4){0.f, 0.f, 0.f, 0.f}; acc[i][1] = (f4){0.f, 0.f, 0.f, 0.f};
            asm volatile("" : "+v"(acc[i][0]), "+v"(acc[i][1]));
        }
        h8 a[FW], x[8];
#pragma unroll
        for (int i = 0; i < FW; ++i) asm volatile("" : "=v"(a[i]));
#pragma unroll
        for (int j = 0; j < 8; ++j) asm volatile("" : "=v"(x[j]));
        Pos P;
        Loop L;
        asm volatile("" : "=v"(P.f));
        P.w = lbase + (unsigned)((r16 >> 2) * W_BLK + q * 256 + (r16 & 3) * 16);
        P.n = lbase + (unsigned)(8 * W_BLK + q * 256 + (r16 & 3) * 64 + (r16 >> 2) * 16);
        P.x = lo16 + (unsigned)(set * X_PART + w4 * 2048);
        P.ldw = ctrl + C_LD - 4;
        P.rv = -4u; P.relw = ctrl + C_CONS + wave * 4; P.tab = myws + WS_TAB + (size_t)b * 512;
        unsigned vo = (unsigned)(q * NLOC + r16) * 16u, vn = (unsigned)(q * 4 * NLOC + r16) * 4u;
        asm volatile("" : "+v"(P.w), "+v"(P.n), "+v"(vn), "+v"(P.x), "+v"(P.rv), "+v"(P.ldw), "+v"(vo));
        WAIT_AV(1, 0);
        const unsigned rot = rfl(P.f.x);
        const int src = (p.srcs >> 4 * ((xr + rot) & 7)) & 7;
        L.xa = SRCWS(rot) + WS_XS + (size_t)((call & 1u) * 2u + (unsigned)mt) * NKS * X_STEP;
        L.xg = myws + WS_XG + (size_t)((((call & 1u) * NR + (unsigned)src) * 2u + (unsigned)mt) * K1) * X_STEP;
        L.set = set;
        const char* ol = (const char*)(p.out + (size_t)(src * MLOC + mt * BM + set * 128 + w4 * 16) * NLOC + f0 * 16);
        P.p[0] = L.xa; P.p[1] = L.xa + 8192;
        PIN(ol); PIN(P.relw); PIN(P.tab); PIN(L.xg); PIN(L.set); PIN(P.p[0]); PIN(P.p[1]);
#pragma unroll
        for (int k = 0; k < 4; ++k) XLD(x[k], k);
        L.t = 1u; P.need = 2u; L.xa += X_STEP; P.p[0] = L.xa; P.p[1] = L.xa + 8192;
        PIN(L.t); PIN(P.need); PIN(L.xa); PIN(P.p[0]); PIN(P.p[1]);
        WAIT_AV(0, 1);
        FLAGS();
#pragma unroll
        for (int i = 0; i < FW; ++i)
            asm volatile("ds_read_b128 %0, %1 offset:%2" : "+v"(a[i]) : "v"(i < 8 ? P.w : P.n), "i"(wcol(i)) : "memory");
        __builtin_amdgcn_s_waitcnt(0x0070);
        SCHED_FENCE();
        if (set) __builtin_amdgcn_s_setprio(1);
#pragma unroll 1
        do {
            k_step<0, 0>(acc, a, x, P, L);
            k_step<1, 0>(acc, a, x, P, L);
        } while (L.t <= (unsigned)KA);
        P.need = (unsigned)(KA + 1 + NI * set);
        const char* xp = L.xa - X_STEP;
        unsigned pre = P.need - set;
#define TRY() do { const unsigned _p = rfl(P.f.w); item_x(x, P.x, xp, pre < P.need ? P.need : ~0u, _p); \
                   item_x(x + 4, P.x, xp + 2 * X_STEP, pre <= P.need ? P.need + 1u : ~0u, _p); pre = _p; } while (0)
#define SILU(r_, J_, keep_) silu4(acc[(r_) & 4][J_][(r_) & 3], acc[((r_) & 4) + 1][J_][(r_) & 3], acc[((r_) & 4) + 2][J_][(r_) & 3], \
                                  acc[((r_) & 4) + 3][J_][(r_) & 3], 60.f * LOG2E, c, keep_)
#define STORE(r_, o_) asm volatile("global_store_dwordx4 %0, %1, %2 offset:%3 nt" \
                                   :: "v"(vo), "v"(o_), "s"(ol + ((r_) & 3) * NLOC * 4), "i"(((r_) & 4) * 64) : "memory")
#define PART(J_) do { \
        SCHED_FENCE(); \
        if (pre < P.need) WAIT_AV(P.need, 0); \
        item_x(x, P.x, xp, pre + 1u, P.need); \
        if (pre <= P.need) WAIT_AV(P.need + 1u, KA + 2); \
        P.p[0] = xp + 2 * X_STEP; P.p[1] = xp + 3 * X_STEP; PIN(P.p[0]); PIN(P.p[1]); \
        item_x(x + 4, P.x, P.p[0], pre, P.need); \
        if (J_) { if (pre < P.need) VM_WAIT(4); else VM_WAIT(8); } \
        __builtin_amdgcn_s_setprio(3 - J_); \
        SCHED_FENCE(); \
        part_items<J_ * NT, J_ * NT + NT>(acc, a, x, P); \
        __builtin_amdgcn_s_setprio(0); \
        if (!J_) { xp += 8192; PIN(xp); P.need += NAB; pre = 0u; TRY(); } \
        const float c = __builtin_bit_cast(float, __builtin_bit_cast(unsigned, LOG2E) | rfl(P.f.z)); \
        asm volatile("s_nop 7\n\ts_nop 7" : "+v"(acc[FW - 3][J_]), "+v"(acc[FW - 2][J_]), "+v"(acc[FW - 1][J_])); \
        f4 o = acc[FW - 1][J_], od[4]; \
        _Pragma("unroll") for (int r = 0; r < 4; ++r) { o = SILU(r, J_, o); STORE(r, o); } \
        if (!dup) { \
            const f4 on = silu4(acc[FW - 1][J_][0], acc[FW - 1][J_][1], acc[FW - 1][J_][2], acc[FW - 1][J_][3], 60.f * LOG2E, c, o); \
            _Pragma("unroll") for (int r = 0; r < 4; ++r) \
                asm volatile("global_store_dword %0, %1, %2 offset:512 nt" :: "v"(vn), "v"(on[r]), "s"(ol + r * NLOC * 4) : "memory"); \
        } \
        _Pragma("unroll") for (int r = 4; r < 8; ++r) {        \
            if (!J_ && (r == 4 || r == 7)) FLAGS(); \
            o = od[r - 4] = SILU(r, J_, o); \
            if (J_) STORE(r, o); \
            if (!J_ && (r == 4 || r == 7)) { asm volatile("s_waitcnt lgkmcnt(0)" : "+v"(P.f) :: "memory"); TRY(); } \
        } \
        if (!J_) _Pragma("unroll") for (int r = 4; r < 8; ++r) STORE(r, od[r - 4]); \
        asm volatile("s_nop 1" ::: "memory"); } while (0)
        PART(0);
        ADV(vo, 64 * NLOC * 4); ADV(vn, 64 * NLOC * 4);
        PART(1);
        __builtin_amdgcn_endpgm();
    }
    if (wave == 10) {
        *(u2*)(myws + WS_TAB + (size_t)b * 512 + lane * 8) = (u2){2u * lane + 1u, 2u * lane + 2u};
        const unsigned cl = ld_ctr((const void*)cctr) + 1u;
        if (b == NWG - 8 && lane < 16)
            __hip_atomic_store((GAS unsigned*)(myws + WS_WFLAG + (size_t)lane * FSTRIDE + 256), cl * 8u + xcd(),
                               __ATOMIC_RELAXED, __HIP_MEMORY_SCOPE_AGENT);
        if (lane == 0) __hip_atomic_store(cctr, cl, __ATOMIC_RELAXED, __HIP_MEMORY_SCOPE_AGENT);
        lds_st(ctrl + lane * 4, lane == C_CALL / 4 ? cl : 0u);
        LGKM0();
    }
    __builtin_amdgcn_s_barrier();
    asm volatile("" ::: "memory");
    const unsigned call = rfl(lds_ld(ctrl + C_CALL));
    if (wave == 10) {
        GAS unsigned* hq = (GAS unsigned*)(myws + WS_HINT + (size_t)(xr * 2 + mt) * FSTRIDE) + c8;
        GAS unsigned* hp = lane == 5 ? rotw : hq;
        GAS unsigned* wf0 = (GAS unsigned*)(myws + WS_WFLAG + (size_t)(f0 >> 3) * FSTRIDE) + lane;
        GAS unsigned* wf1 = (GAS unsigned*)(myws + WS_WFLAG + (size_t)((f0 + FW - 1) >> 3) * FSTRIDE) + lane;
        unsigned pub = 0, wav = 0, npoll = 0, rot = 8u;
        const unsigned long long t0 = rt();
        unsigned long long tall = 0;
        bool dead = false;
        while (pub < (unsigned)NITEMS || wav < (unsigned)NKS) {
            const unsigned hv = __hip_atomic_load(hp, __ATOMIC_RELAXED, __HIP_MEMORY_SCOPE_AGENT);
            bool adv = false;
            if (wav < (unsigned)NKS) {
                const unsigned w0 = __hip_atomic_load(wf0, __ATOMIC_RELAXED, __HIP_MEMORY_SCOPE_AGENT);
                const unsigned w1 = __hip_atomic_load(wf1, __ATOMIC_RELAXED, __HIP_MEMORY_SCOPE_AGENT);
                const unsigned long long notready = __ballot((int)(w0 - call) < 0 || (int)(w1 - call) < 0);
                const unsigned w2 = notready ? (unsigned)__builtin_ctzll(notready) : 64u;
                if (w2 > wav) { wav = w2; lds_st(ctrl + C_WAV, wav); adv = true; }
            }
            const int d = (int)(hv - call * 256u);
            const unsigned av = (unsigned)row_min(c8 < 5 ? max(min(d, NITEMS), 0) : NITEMS);
            if (rot > 7u) {
                rot = (unsigned)__builtin_amdgcn_readlane((int)hv, 5) - call * 8u;
                if (rot <= 7u) { lds_st(ctrl + C_ROT, rot); hp = hq; }
            }
            if (av > pub && rot <= 7u) {
                if (!pub && (unsigned)__builtin_amdgcn_readlane((int)hv, 13) != call * 8u + xcd()) ACQUIRE();
                asm volatile("ds_max_u32 %0, %1" :: "v"(lane ? ctrl + 128 + lane * 4 : ctrl + C_AVAIL), "v"(av) : "memory");
                pub = av;
                tall = 0;
            } else if (!adv && (++npoll & 63u) == 0u) {
                if (rt() - t0 > DEADLINE) { dead = true; break; }
                unsigned staged; bool stored;
                arrivals(SRCWS(rot) + WS_SFLAG + (size_t)mt * FSTRIDE, call, lane, staged, stored);
                if (staged == (unsigned)NKS && stored && rot <= 7u) {
                    if (tall == 0) tall = rt();
                    else if (rt() - tall > SILENCE) { ACQUIRE(); pub = NITEMS; lds_st(ctrl + C_AVAIL, pub); }
                } else if (rfl(__hip_atomic_load(tmo, __ATOMIC_RELAXED, __HIP_MEMORY_SCOPE_AGENT)) == call) { dead = true; break; }
            }
            if (pub < 48u) __builtin_amdgcn_s_sleep(4); else __builtin_amdgcn_s_sleep(1);
        }
        if (dead) {
            __hip_atomic_store(tmo, call, __ATOMIC_RELAXED, __HIP_MEMORY_SCOPE_AGENT);
            lds_st(ctrl + C_BAD, 0x7fc00000u);
        }
        lds_st(ctrl + C_WAV, (unsigned)NKS);
        lds_st(ctrl + C_AVAIL, (unsigned)NITEMS);
    } else if (wave == 9) {
        constexpr int CSTEP = 240 / NCONV;
        const int cq = ((idx - 2) / CSTEP) * 8 + xr;
        if ((idx - 2) % CSTEP == 0 && cq < NCONV) {
            __builtin_amdgcn_s_setprio(3);
            const unsigned so = (unsigned)(r8 * 8 * NLOC + c8 * 4) * 4u;
            const unsigned dof = (unsigned)((c8 >> 2) * 2048 + r8 * 256 + (c8 & 3) * 16);
            for (int i = -17; i < cq; i += 16) __builtin_amdgcn_s_sleep(20);
            for (int u = cq; u < NKS * 16; u += NCONV) {
                const int ks = u >> 4, cn = u & 15;
                const float* sw = p.w + (size_t)ks * BK * NLOC + cn * 128;
                const char* dstw = myws + WS_WH + (size_t)ks * W_ROW + (size_t)cn * 16384;
                f4 v[4][8];
#pragma unroll
                for (int g = 0; g < 4; ++g)
#pragma unroll
                    for (int r = 0; r < 8; ++r)
                        asm volatile("global_load_dwordx4 %0, %1, %2 offset:%3 nt" : "=v"(v[g][r]) : "v"(so), "s"(sw + r * NLOC), "i"(g * 128) : "memory");
                u4 o[4][4];
#pragma unroll
                for (int g = 0; g < 4; ++g) {
                    asm volatile("s_waitcnt vmcnt(%8)" : "+v"(v[g][0]), "+v"(v[g][1]), "+v"(v[g][2]), "+v"(v[g][3]),
                                 "+v"(v[g][4]), "+v"(v[g][5]), "+v"(v[g][6]), "+v"(v[g][7]) : "i"(24 - 8 * g) : "memory");
#pragma unroll
                    for (int r = 0; r < 8; ++r) v[g][r] *= LOG2E;
#pragma unroll
                    for (int c = 0; c < 4; ++c)
                        o[g][c] = (u4){pk(v[g][0][c], v[g][1][c]), pk(v[g][2][c], v[g][3][c]), pk(v[g][4][c], v[g][5][c]), pk(v[g][6][c], v[g][7][c])};
                    SCHED_FENCE();
                }
#pragma unroll
                for (int g = 0; g < 4; ++g)
#pragma unroll
                    for (int c = 0; c < 4; ++c)
                        asm volatile("global_store_dwordx4 %0, %1, %2 offset:%3 sc1 nt\n\ts_nop 1"
                                     :: "v"(dof), "v"(o[g][c]), "s"(dstw + g * 4096), "i"(c * 64) : "memory");
                VM_WAIT(0);
                if (lane == 0)
                    __hip_atomic_store((GAS unsigned*)(myws + WS_WFLAG + (size_t)cn * FSTRIDE + ks * 4), call,
                                       __ATOMIC_RELAXED, __HIP_MEMORY_SCOPE_AGENT);
            }
        }
    } else {
        const char* tab = myws + WS_TAB + (size_t)b * 512;
        const char* wa = myws + WS_WH + (size_t)f0 * 2048;
        unsigned long long t0 = 0;
        int av = 0, cons = 0;
#pragma unroll 1
        for (int t = 0; t < NKS; ++t) {
            if (av <= t || cons <= t - NW) for (unsigned sp = 0;;) {
                u4 c, d; unsigned v;
                asm volatile("ds_read_b128 %0, %3\n\tds_read_b128 %1, %3 offset:16\n\tds_read_b32 %2, %3 offset:%4\n\t"
                             "s_waitcnt lgkmcnt(0)" : "=&v"(c), "=&v"(d), "=&v"(v) : "v"(ctrl + C_CONS), "i"(C_WAV - C_CONS) : "memory");
                av = (int)rfl(v); cons = (int)min8(c, d);
                if ((av > t && cons > t - NW) || SPIN_OUT(sp, t0)) break;
                __builtin_amdgcn_s_sleep(1);
            }
            const unsigned sb = lbase + (unsigned)(t & (NW - 1)) * W_STEP;
#define WLOADS(P_) _Pragma("unroll") for (int i = 0; i < FW; i += 2) { \
                if (i == FW - 1 && dup) break; \
                asm volatile("s_mov_b32 m0, %0\n\ts_waitcnt vmcnt(58)\n\t" \
                             "global_load_lds_dwordx4 %1, %2" P_ "\n\tglobal_load_lds_dwordx4 %1, %2 offset:1024" P_ \
                             :: "s"(sb + i * W_BLK), "v"(lo16), "s"(wa + i * 2048) : "memory"); \
                if (i + 1 < FW) asm volatile("s_mov_b32 m0, %0\n\ts_nop 0\n\t" \
                             "global_load_lds_dwordx4 %1, %2 offset:2048" P_ "\n\tglobal_load_lds_dwordx4 %1, %2 offset:3072" P_ \
                             :: "s"(sb + i * W_BLK + 64), "v"(lo16), "s"(wa + i * 2048) : "memory"); \
            }
            if (mt == 0 && t >= WNT0 && t < WNT1) { WLOADS(" sc1 nt") } else { WLOADS(" sc1") }
            asm volatile("s_mov_b32 m0, %0\n\ts_mov_b64 exec, 1\n\tglobal_load_lds_dword %1, %2\n\ts_mov_b64 exec, -1"
                         :: "s"(ctrl + C_LD), "v"(lo16), "s"(tab + t * 4) : "memory");
            wa += W_ROW;
        }
        VM_WAIT(0);
    }
}

}

extern "C" void kernel_launch(int rank, int nranks, ncclComm_t comm,
                              hipStream_t stream,
                              void* const* d_in, const int* in_sizes, int n_in,
                              void* d_out, int out_size,
                              const int* in_dtypes, int out_dtype,
                              void* const* d_out_all,
                              void* const* d_ws_all, size_t ws_bytes) {
    (void)comm; (void)in_dtypes; (void)out_dtype; (void)d_out_all;
    if (nranks != NR || n_in < 2 || (unsigned)rank >= (unsigned)NR || ws_bytes < WS_NEED ||
        in_sizes[0] != MLOC * GK || in_sizes[1] != GK * NLOC || out_size != GM * NLOC) {
        fprintf(stderr, "ag_gemm: unexpected problem shape\n");
        return;
    }
    static bool attr_set = false;
    if (!attr_set) {
        HIP_CHECK(hipFuncSetAttribute((const void*)ag_gemm_kernel,
                                      hipFuncAttributeMaxDynamicSharedMemorySize, LDS_BYTES));
        attr_set = true;
    }
    Params p;
    p.x = (const float*)d_in[0];
    p.w = (const float*)d_in[1];
    p.out = (float*)d_out;
    p.myws = (char*)d_ws_all[rank];
    p.srcs = 0;
    const unsigned half = 0xb2;
    for (int far = 0, n = 0; far < 2; ++far)
        for (int r = 0; r < NR; ++r) {
            const int s = (rank + r) & 7;
            if ((int)((half >> s ^ half >> rank) & 1) != far) continue;
            p.ws[n] = (char*)d_ws_all[s];
            p.srcs |= (unsigned)s << 4 * n;
            ++n;
        }
    for (int r = 0; r < 2 * NR; ++r) p.xg[r] = (char*)d_ws_all[(rank + 1 + r) & 7] + WS_XG + (size_t)rank * 2 * K1 * X_STEP;
    hipLaunchKernelGGL(ag_gemm_kernel, dim3(NWG), dim3(NTHREADS), LDS_BYTES, stream, p);
    HIP_CHECK(hipGetLastError());
}
```
